```python
import numpy as np

kernel_name = "gpu_a2a_gemm_m4096_k4096_n32768_bf16_silu"


ABSMAX_RTOL = 0.008

COMPUTE_DTYPE = "bfloat16"
EPILOGUE = "silu"
IN_QUANT = None
OUT_QUANT = None
IN_DTYPES = {"x": "bfloat16", "w_mat": "bfloat16"}
WORLD_SIZE = 8

def _to_bf16(x):
    u = np.ascontiguousarray(x, dtype=np.float32).view(np.uint32)
    return ((u + 0x7FFF + ((u >> 16) & 1)) & 0xFFFF0000).view(np.float32)

def setup_inputs():

    x = _to_bf16(np.random.randn(4096, 4096).astype(np.float32))
    w_mat = _to_bf16((np.random.randn(4096, 32768) / np.sqrt(4096)).astype(np.float32))
    return {"x": x, "w_mat": w_mat}

def _epilogue(y):
    return (y / (1.0 + np.exp(-np.clip(y, -60.0, 60.0)))).astype(np.float32)

def reference(x, w_mat):
    y = (x @ w_mat).astype(np.float32)
    return _epilogue(y)
```

```cpp
#include <hip/hip_runtime.h>
#include <rccl/rccl.h>
#include <cstdio>

#define HIP_CHECK(expr) do { hipError_t _e = (expr); if (_e != hipSuccess) { \
    fprintf(stderr, "HIP error %s at %s:%d\n", hipGetErrorString(_e), __FILE__, __LINE__); \
    return; } } while (0)

#define LAS __attribute__((address_space(3)))
typedef short bf16x8 __attribute__((ext_vector_type(8)));
typedef short bf16x4 __attribute__((ext_vector_type(4)));
typedef __bf16 bf16v8 __attribute__((ext_vector_type(8)));
typedef float f32x4 __attribute__((ext_vector_type(4)));
typedef unsigned u32x4 __attribute__((ext_vector_type(4)));

constexpr int NR = 8;
constexpr int HTB = 16384, SHM = 8 * HTB;
constexpr size_t SLOT = 524288;
constexpr size_t WS_STAGE = 0;
constexpr size_t WS_FLAGS = 2 * 8 * SLOT;
constexpr size_t WS_CTR = WS_FLAGS + 4096;
constexpr size_t WS_TMO = WS_CTR + 1024;
constexpr size_t WS_END = WS_TMO + 128;
constexpr unsigned long long DEADLINE_TICKS = 50000000ull;

struct Params {
  const char* x;
  const char* W;
  char* out;
  char* ws[NR];
  int rank;
  int phase;
};

__device__ __forceinline__ unsigned cvt_pk_bf16(float lo, float hi) { unsigned r; asm volatile("v_cvt_pk_bf16_f32 %0, %1, %2" : "=v"(r) : "v"(lo), "v"(hi)); return r; }
__device__ __forceinline__ float silu(float y) {
  float c = __builtin_amdgcn_fmed3f(y, -60.f, 60.f);
  float e = __builtin_amdgcn_exp2f(c * -1.44269504089f);
  return y * __builtin_amdgcn_rcpf(1.0f + e);
}
__device__ __forceinline__ void ktile(int t, int rank, const char* Aloc, const char* Arem, const char* W, const char*& a, const char*& w) {
  int p, j;
  if (t < 8) { p = rank; j = t; a = Aloc + j * 128; }
  else { int u = t - 8; j = u / 7; p = (rank + 1 + (u - j * 7)) & 7; a = Arem + p * 524288 + j * 128; }
  w = W + (size_t)(p * 512 + j * 64) * 65536;
}

__global__ __launch_bounds__(512, 2) void fused(Params g) {
  extern __shared__ __attribute__((aligned(16))) unsigned char shm_[];
  LAS unsigned char* lds = (LAS unsigned char*)shm_;
  const int tid = threadIdx.x;
  char* myws = g.ws[g.rank];
  unsigned call;
  {
    LAS unsigned* bc = (LAS unsigned*)lds;
    if (tid == 0) {
      unsigned tk = __hip_atomic_fetch_add((unsigned*)(myws + WS_CTR + g.phase * 128), 1u, __ATOMIC_RELAXED, __HIP_MEMORY_SCOPE_AGENT);
      bc[0] = tk / gridDim.x + 1;
    }
    __syncthreads();
    call = __builtin_amdgcn_readfirstlane(bc[0]);
    __syncthreads();
  }
  const unsigned par = call & 1;
  if (g.phase == 0) {
    const int b = blockIdx.x, i = b % 7, c = b / 7;
    const unsigned d = (unsigned)(g.rank + 1 + i) & 7u;
    if (d < (unsigned)NR) {
      const u32x4* src = (const u32x4*)(g.x + (size_t)d * SLOT + c * 16384);
      u32x4* dst = (u32x4*)(g.ws[d] + WS_STAGE + (par * 8 + g.rank) * SLOT + c * 16384);
      u32x4 v0 = src[tid], v1 = src[tid + 512];
      dst[tid] = v0; dst[tid + 512] = v1;
      asm volatile("s_waitcnt vmcnt(0)" ::: "memory");
      __syncthreads();
      if (tid == 0) {
        __builtin_amdgcn_fence(__ATOMIC_RELEASE, "");
        asm volatile("s_waitcnt vmcnt(0)" ::: "memory");
        __hip_atomic_fetch_add((unsigned*)(g.ws[d] + WS_FLAGS + g.rank * 128), 1u, __ATOMIC_RELAXED, __HIP_MEMORY_SCOPE_SYSTEM);
      }
    }
    return;
  }
  unsigned* tmo = (unsigned*)(myws + WS_TMO);
  if (tid < 64) {
    if (tid < NR && tid != g.rank) {
      unsigned* flag = (unsigned*)(myws + WS_FLAGS + tid * 128);
      unsigned long long t0 = __builtin_amdgcn_s_memrealtime();
      while (__hip_atomic_load(flag, __ATOMIC_RELAXED, __HIP_MEMORY_SCOPE_SYSTEM) < call * 32u) {
        if (__hip_atomic_load(tmo, __ATOMIC_RELAXED, __HIP_MEMORY_SCOPE_AGENT) == call
            || __builtin_amdgcn_s_memrealtime() - t0 > DEADLINE_TICKS) {
          __hip_atomic_store(tmo, call, __ATOMIC_RELAXED, __HIP_MEMORY_SCOPE_AGENT);
          break;
        }
        __builtin_amdgcn_s_sleep(2);
      }
    }
    __builtin_amdgcn_fence(__ATOMIC_ACQUIRE, "");
    asm volatile("s_waitcnt vmcnt(0)" ::: "memory");
  }
  __syncthreads();
  const bool bad = __hip_atomic_load(tmo, __ATOMIC_RELAXED, __HIP_MEMORY_SCOPE_AGENT) == call;
  asm volatile("s_waitcnt vmcnt(0)" ::: "memory");

  const int wid = __builtin_amdgcn_readfirstlane(tid >> 6), lane = tid & 63, wr = wid >> 2, wc = wid & 3, fr = lane & 15, fq = lane >> 4;
  const int b = blockIdx.x, xcd = b & 7, idx = b >> 3, pn = xcd * 16 + (idx >> 1), pm = idx & 1;
  unsigned voffA[2], voffB[2];
  for (int i = 0; i < 2; ++i) {
    { int row = (i * 8 + wid) * 8 + (lane >> 3), c = (lane & 7) ^ ((row >> 1) & 7); voffA[i] = row * 1024 + c * 16; }
    { int row = (i * 8 + wid) * 4 + (lane >> 4), c = (lane & 15) ^ (((row & 3) << 2) | ((row >> 3) & 1)); voffB[i] = row * 65536 + c * 16; }
  }
  const char* Aloc = g.x + (size_t)g.rank * SLOT + pm * 256 * 1024;
  const char* Arem = myws + WS_STAGE + par * 8 * SLOT + pm * 256 * 1024;
  const char* W = g.W + pn * 512;
  const unsigned ldsw = wid * 1024;
  int aoff[2], boff[2];
  for (int ks = 0; ks < 2; ++ks) aoff[ks] = (wr * 64 + fr) * 128 + 16 * ((4 * ks + fq) ^ (fr >> 1));
  { int q = (lane & 15) >> 2, p = lane & 3, gq = lane >> 4;
    for (int n = 0; n < 2; ++n) boff[n] = 256 * (8 * gq + q) + 16 * ((wc * 4 + p) ^ ((q << 2) | (gq & 1))) + 8 * ((p & 1) ^ n); }
#define SA(b, h) (((b) * 2 + (h)) * HTB)
#define SB(b, h) ((4 + (b) * 2 + (h)) * HTB)
#define STAGE(bufoff, gbase, voff) do { _Pragma("unroll") for (int _i = 0; _i < 2; ++_i) \
    __builtin_amdgcn_global_load_lds((const unsigned*)((const char*)(gbase) + (voff)[_i]), (LAS unsigned*)(lds + (bufoff) + ldsw + _i * 8192), 16, 0, 0); } while (0)
#define LDA(dst, b, h) do { _Pragma("unroll") for (int m = 0; m < 4; ++m) _Pragma("unroll") for (int k = 0; k < 2; ++k) dst[m][k] = *(const LAS bf16x8*)(lds + SA(b, h) + aoff[k] + m * 2048); } while (0)
#define LDB(dst, b, h) do { _Pragma("unroll") for (int n = 0; n < 2; ++n) _Pragma("unroll") for (int k = 0; k < 2; ++k) { \
    bf16x4 lo, hi; \
    asm volatile("ds_read_b64_tr_b16 %0, %1 offset:%2" : "=v"(lo) : "v"(boff[n] + 65536), "i"(SB(b, h) - 65536 + k * 8192) : "memory"); \
    asm volatile("ds_read_b64_tr_b16 %0, %1 offset:%2" : "=v"(hi) : "v"(boff[n] + 65536), "i"(SB(b, h) - 65536 + k * 8192 + 1024) : "memory"); \
    dst[n][k] = __builtin_shufflevector(lo, hi, 0, 1, 2, 3, 4, 5, 6, 7); } } while (0)
#define MMA(ai, bj, At, Bt) do { __builtin_amdgcn_s_setprio(1); _Pragma("unroll") for (int m = 0; m < 4; ++m) _Pragma("unroll") for (int n = 0; n < 2; ++n) _Pragma("unroll") for (int k = 0; k < 2; ++k) \
    acc[ai][bj][m][n] = __builtin_amdgcn_mfma_f32_16x16x32_bf16(__builtin_bit_cast(bf16v8, Bt[n][k]), __builtin_bit_cast(bf16v8, At[m][k]), acc[ai][bj][m][n], 0, 0, 0); __builtin_amdgcn_s_setprio(0); } while (0)
#define WAIT_V(n) asm volatile("s_waitcnt vmcnt(" #n ")" ::: "memory")
#define WAIT_L(n) do { asm volatile("s_waitcnt lgkmcnt(" #n ")" ::: "memory"); __builtin_amdgcn_sched_barrier(0); } while (0)
#define BAR __builtin_amdgcn_s_barrier()
#define SCHED __builtin_amdgcn_sched_barrier(0)
  f32x4 acc[2][2][4][2];
#pragma unroll
  for (int a = 0; a < 2; ++a)
#pragma unroll
    for (int bb = 0; bb < 2; ++bb)
#pragma unroll
      for (int m = 0; m < 4; ++m)
#pragma unroll
        for (int n = 0; n < 2; ++n) acc[a][bb][m][n] = (f32x4){0.f, 0.f, 0.f, 0.f};
  bf16x8 At[4][2], B0[2][2], B1[2][2];
  constexpr int hA = 128 * 1024, hB = 256;
  const char *a0, *w0, *a1, *w1, *a2, *w2, *a3, *w3;
  ktile(0, g.rank, Aloc, Arem, W, a0, w0); ktile(1, g.rank, Aloc, Arem, W, a1, w1);
  STAGE(SB(0, 0), w0, voffB); STAGE(SA(0, 0), a0, voffA); STAGE(SB(0, 1), w0 + hB, voffB); STAGE(SA(0, 1), a0 + hA, voffA);
  if (wr == 1) BAR;
  WAIT_V(4); BAR;
  STAGE(SB(1, 0), w1, voffB); STAGE(SA(1, 0), a1, voffA); STAGE(SB(1, 1), w1 + hB, voffB);
  WAIT_V(6); BAR;
  for (int t = 0; t < 64; t += 2) {
    int t2 = t + 2 < 64 ? t + 2 : 62, t3 = t2 + 1;
    ktile(t2, g.rank, Aloc, Arem, W, a2, w2); ktile(t3, g.rank, Aloc, Arem, W, a3, w3);
    LDB(B0, 0, 0); SCHED; LDA(At, 0, 0); STAGE(SA(1, 1), a1 + hA, voffA);
    WAIT_L(8); BAR; WAIT_L(0); MMA(0, 0, At, B0); BAR; SCHED;
    LDB(B1, 0, 1); STAGE(SB(0, 0), w2, voffB);
    BAR; WAIT_L(0); MMA(0, 1, At, B1); BAR;
    LDA(At, 0, 1); STAGE(SA(0, 0), a2, voffA);
    BAR; WAIT_L(0); MMA(1, 0, At, B0); BAR; SCHED;
    STAGE(SB(0, 1), w2 + hB, voffB);
    WAIT_V(6); BAR; MMA(1, 1, At, B1); BAR;
    LDB(B0, 1, 0); SCHED; LDA(At, 1, 0); STAGE(SA(0, 1), a2 + hA, voffA);
    WAIT_L(8); BAR; WAIT_L(0); MMA(0, 0, At, B0); BAR; SCHED;
    LDB(B1, 1, 1); STAGE(SB(1, 0), w3, voffB);
    BAR; WAIT_L(0); MMA(0, 1, At, B1); BAR;
    LDA(At, 1, 1); STAGE(SA(1, 0), a3, voffA);
    BAR; WAIT_L(0); MMA(1, 0, At, B0); BAR; SCHED;
    STAGE(SB(1, 1), w3 + hB, voffB);
    WAIT_V(6); BAR; MMA(1, 1, At, B1); BAR;
    a1 = a3;
  }
  if (wr == 0) BAR;
  {
    int tz = threadIdx.x; asm volatile("" : "+v"(tz));
    const int ln = tz & 63, fr2 = ln & 15, fq2 = ln >> 4;
    char* op = g.out + (size_t)(pm * 256 + wr * 64 + fr2) * 65536 + (pn * 256 + wc * 32 + 8 * fq2) * 2;
    const bool odd = fq2 & 1;
#pragma unroll
    for (int ai = 0; ai < 2; ++ai)
#pragma unroll
      for (int m = 0; m < 4; ++m)
#pragma unroll
        for (int bj = 0; bj < 2; ++bj) {
          f32x4 v0 = acc[ai][bj][m][0], v1 = acc[ai][bj][m][1];
          unsigned x0 = cvt_pk_bf16(silu(v0[0]), silu(v0[1])), x1 = cvt_pk_bf16(silu(v0[2]), silu(v0[3]));
          unsigned y0 = cvt_pk_bf16(silu(v1[0]), silu(v1[1])), y1 = cvt_pk_bf16(silu(v1[2]), silu(v1[3]));
          u32x4 w; w.x = odd ? y0 : x0; w.y = odd ? y1 : x1; w.z = odd ? x0 : y0; w.w = odd ? x1 : y1;
          if (bad) w = (u32x4){0x7fc07fc0u, 0x7fc07fc0u, 0x7fc07fc0u, 0x7fc07fc0u};
          *(u32x4*)(op + (size_t)(ai * 128 + m * 16) * 65536 + bj * 256) = w;
        }
  }
  WAIT_V(0);
}

extern "C" void kernel_launch(int rank, int nranks, ncclComm_t comm,
                              hipStream_t stream,
                              void* const* d_in, const int* in_sizes, int n_in,
                              void* d_out, int out_size,
                              const int* in_dtypes, int out_dtype,
                              void* const* d_out_all,
                              void* const* d_ws_all, size_t ws_bytes) {
  if (nranks != NR || ws_bytes < WS_END || n_in < 2) { fprintf(stderr, "unexpected configuration\n"); return; }
  static bool init = false;
  if (!init) { HIP_CHECK(hipFuncSetAttribute((const void*)fused, hipFuncAttributeMaxDynamicSharedMemorySize, SHM)); init = true; }
  Params p;
  p.x = (const char*)d_in[0]; p.W = (const char*)d_in[1]; p.out = (char*)d_out; p.rank = rank;
  for (int i = 0; i < NR; ++i) p.ws[i] = (char*)d_ws_all[i];
  p.phase = 0;
  fused<<<224, 512, 1024, stream>>>(p);
  p.phase = 1;
  fused<<<256, 512, SHM, stream>>>(p);
  HIP_CHECK(hipGetLastError());
}
```

```cpp
#include <hip/hip_runtime.h>
#include <rccl/rccl.h>
#include <cstdio>

#define HIP_CHECK(expr) do { hipError_t _e = (expr); if (_e != hipSuccess) { \
    fprintf(stderr, "HIP error %s at %s:%d\n", hipGetErrorString(_e), __FILE__, __LINE__); \
    return; } } while (0)

typedef unsigned u32x8 __attribute__((ext_vector_type(8)));
typedef unsigned u32x4 __attribute__((ext_vector_type(4)));
typedef unsigned u32x3 __attribute__((ext_vector_type(3)));
typedef unsigned u32x2 __attribute__((ext_vector_type(2)));

constexpr int NR = 8;
constexpr unsigned SLOT = 524288;
constexpr unsigned WS_TMO = 0;
constexpr unsigned WS_CTR = 128;
constexpr unsigned WS_FLAGS = 8192;
constexpr unsigned WS_STAGE = 65536;
constexpr unsigned WS_END = WS_STAGE + 2 * NR * SLOT;
constexpr unsigned OOB = 0x7fff0000u;
constexpr int LDS_BYTES = 163840;

struct Params {
  const char* x;
  const char* W;
  char* out;
  char* myws;
  char* peer[NR - 1];
  int rank;
};

__device__ __forceinline__ u32x4 make_srd(const void* p, unsigned bytes) {
  return (u32x4){(unsigned)(size_t)p, (unsigned)((size_t)p >> 32) & 0xffffu, bytes, 0x00020000u};
}

__global__ __launch_bounds__(256, 1) void fused(Params g) {

  unsigned tent = (unsigned)__builtin_amdgcn_s_memrealtime();
  {
    unsigned po = threadIdx.x * 128;
    asm volatile("s_getpc_b64 s[96:97]\n s_and_b32 s97, s97, 0xffff\n s_mov_b32 s98, 32768\n s_mov_b32 s99, 0x20000\n buffer_load_dword a225, %0, s[96:99], 0 offen"
                 :: "v"(po) : "s96", "s97", "s98", "s99", "a225", "scc", "memory"); }
  const int tid = threadIdx.x, wid = __builtin_amdgcn_readfirstlane(tid >> 6), lane = tid & 63, wr = wid >> 1, wc = wid & 1, fr = lane & 15, fq = lane >> 4;
  const int b = blockIdx.x, xcd = b & 7, idx = b >> 3, pn = xcd * 16 + (idx >> 1), pm = idx & 1;
  const int sid = (idx >> 1) * 4 + (xcd >> 1), sj = sid / 7;
  const bool sender = !(xcd & 1) && sid < 56;
  const unsigned si = (unsigned)(sid - sj * 7) < 7u ? (unsigned)(sid - sj * 7) : 0u;
  const char* px = g.x; const char* pW = g.W; char* pout = g.out; char* myws = g.myws; char* pdst = g.peer[si];
  int rank_ = g.rank;
  u32x8 voffA, voffB, misc;
  for (int i = 0; i < 8; ++i) {
    { int I = wid * 8 + i, row = I * 8 + (lane >> 3), c = (lane & 7) ^ ((row >> 1) & 7); voffA[i] = row * 1024 + c * 16; }
    { int R = 4 * (i >> 1) + 2 * (wid & 1) + (i & 1), half = wid >> 1, ii = lane >> 4, q = (lane >> 2) & 3, pp = lane & 3;
       voffB[i] = (4 * R + q) * 65536 + half * 256 + (4 * ii + (pp ^ (R & 2))) * 16; }
  }
  for (int ks = 0; ks < 2; ++ks) misc[ks] = (wr * 128 + fr) * 128 + 16 * ((4 * ks + fq) ^ (fr >> 1));
  { int q = (lane & 15) >> 2, p = lane & 3;
    for (int n = 0; n < 2; ++n) misc[2 + n] = 65536 + wc * 16384 + 2048 * fq + 64 * q + 16 * ((2 * n + (p >> 1)) ^ (2 * (fq & 1))) + 8 * (p & 1); }
  const int tu = lane < 8 ? 0 : lane - 8, tq = (tu * 37) >> 8, tj = lane < 8 ? lane : tq;
  int tdp = lane < 8 ? 0 : 1 + tu - tq * 7;
  misc[4] = pm * 262144 + tj * 128;
  misc[5] = (unsigned)(tj * 64) * 65536u + pn * 512;
  const int fl = lane < 56 ? lane : 0, fj = (fl * 37) >> 8;
  int fdp = 1 + fl - fj * 7;
  misc[6] = WS_FLAGS + (pm * 8 + fj) * 4;
  misc[7] = lane < 3 ? 0 : (lane - 3 < 56 ? lane - 3 : 56);
  u32x3 ev;
  ev[0] = (unsigned)(pm * 256 + wr * 128 + (lane >> 3)) * 65536u + (pn * 256 + wc * 128 + (lane & 7) * 8) * 2;
  ev[1] = ((wid >> 1) * 98304 + (wid & 1) * 16384) + fr * 144 + (fq & 1) * 32 + (fq >> 1) * 16;
  ev[2] = ((wid >> 1) * 98304 + (wid & 1) * 16384) + (lane >> 3) * 144 + (lane & 7) * 16;
  const bool oddx = __builtin_amdgcn_s_getreg(20 | (0 << 6) | (3 << 11)) & 1;
  const unsigned nosend = sender ? 0u : OOB;
  unsigned sendt = (2 * sj) | nosend, ctroff = WS_CTR + b * 16;
  unsigned m0a = 0 + wid * 8192, bwr = 65536 + (wid >> 1) * 16384 + (wid & 1) * 2048;
  u32x3 offs;
  offs[0] = (pm * 262144 + (sj & 7) * 128) | nosend;
  offs[1] = (WS_STAGE + pm * 262144 + (sj & 7) * 128) | nosend;
  offs[2] = (WS_FLAGS + (pm * 8 + (sj & 7)) * 4) | nosend | (wid ? OOB : 0u);
  asm volatile("" : "+v"(voffA), "+v"(voffB), "+v"(misc), "+v"(ev), "+v"(tdp), "+v"(fdp), "+s"(offs), "+s"(sendt), "+s"(ctroff), "+s"(m0a), "+s"(bwr));
  asm volatile("" : "+s"(px), "+s"(pW), "+s"(pout), "+s"(myws), "+s"(pdst), "+s"(rank_));
  const int rank = rank_ & 7;
  offs[0] += ((unsigned)(rank + 1 + si) & 7u) * SLOT; offs[1] += rank * SLOT; offs[2] += rank * 128;
  { const unsigned p = (rank + tdp) & 7; misc[4] += p * SLOT; misc[5] += p * (512u * 65536u); }
  misc[6] += ((rank + fdp) & 7) * 128;
  const unsigned pace = 158 + (oddx ? -(0) : (0)), pacen = 0;
  u32x2 pace2; pace2[0] = 158 + (oddx ? -(8) : (8)); pace2[1] = 18;
  u32x2 rc; rc[0] = 200; rc[1] = (61 - (18)) * 124 + (oddx ? 1 : -1) * ((0) * (0) + ((18) - (0)) * (8));
  unsigned capt = rc[1], cred = rc[0];
  u32x4 srdW = make_srd(pW, 268435456u), srdA = make_srd(px, NR * SLOT), srdR = make_srd(myws + WS_STAGE, NR * SLOT);
  u32x4 srdX = make_srd(px, NR * SLOT);
  const u32x4 srdD = make_srd(pdst, WS_END), srdF = make_srd(myws, WS_STAGE), srdO = make_srd(pout, 33554432u);
  asm volatile(
    "s_nop 4\n"
    "v_readlane_b32 s53, v244, 0\n"
    "v_readlane_b32 s54, v245, 0\n"
    "s_add_u32 s67, s58, 98304\n"
    "s_mov_b32 s57, 0\n"
    "s_mov_b32 s68, 32768\n"
    "buffer_load_dwordx2 v[252:253], off, s[88:91], s69 sc1\n"
    "v_mbcnt_lo_u32_b32 v0, -1, 0\n"
    "v_and_b32 v0, 7, v0\n"
    "v_lshlrev_b32 v0, 4, v0\n"
    "buffer_load_dword v254, v0, s[88:91], 0 offen offset:3976 sc1\n"
    "s_add_u32 m0, s55, 0\n"
    "s_nop 0\n"
    "buffer_load_dwordx4 v224, s[44:47], s53 offen sc1 lds\n"
    "s_add_u32 m0, s55, 1024\n"
    "s_nop 0\n"
    "buffer_load_dwordx4 v225, s[44:47], s53 offen sc1 lds\n"
    "s_add_u32 m0, s55, 2048\n"
    "s_nop 0\n"
    "buffer_load_dwordx4 v226, s[44:47], s53 offen sc1 lds\n"
    "s_add_u32 m0, s55, 3072\n"
    "s_nop 0\n"
    "buffer_load_dwordx4 v227, s[44:47], s53 offen sc1 lds\n"
    "s_add_u32 m0, s55, 4096\n"
    "s_nop 0\n"
    "buffer_load_dwordx4 v228, s[44:47], s53 offen sc1 lds\n"
    "s_add_u32 m0, s55, 5120\n"
    "s_nop 0\n"
    "buffer_load_dwordx4 v229, s[44:47], s53 offen sc1 lds\n"
    "s_add_u32 m0, s55, 6144\n"
    "s_nop 0\n"
    "buffer_load_dwordx4 v230, s[44:47], s53 offen sc1 lds\n"
    "s_add_u32 m0, s55, 7168\n"
    "s_nop 0\n"
    "buffer_load_dwordx4 v231, s[44:47], s53 offen sc1 lds\n"
    "v_readlane_b32 s53, v244, 1\n"
    "s_add_u32 m0, s58, 0\n"
    "s_nop 0\n"
    "buffer_load_dwordx4 v232, s[40:43], s54 offen nt lds\n"
    "s_add_u32 m0, s58, 1024\n"
    "s_nop 0\n"
    "buffer_load_dwordx4 v233, s[40:43], s54 offen nt lds\n"
    "s_add_u32 m0, s58, 4096\n"
    "s_nop 0\n"
    "buffer_load_dwordx4 v234, s[40:43], s54 offen nt lds\n"
    "s_add_u32 m0, s58, 5120\n"
    "s_nop 0\n"
    "buffer_load_dwordx4 v235, s[40:43], s54 offen nt lds\n"
    "s_add_u32 m0, s58, 8192\n"
    "s_nop 0\n"
    "buffer_load_dwordx4 v236, s[40:43], s54 offen nt lds\n"
    "s_add_u32 m0, s58, 9216\n"
    "s_nop 0\n"
    "buffer_load_dwordx4 v237, s[40:43], s54 offen nt lds\n"
    "s_add_u32 m0, s58, 12288\n"
    "s_nop 0\n"
    "buffer_load_dwordx4 v238, s[40:43], s54 offen nt lds\n"
    "s_add_u32 m0, s58, 13312\n"
    "s_nop 0\n"
    "buffer_load_dwordx4 v239, s[40:43], s54 offen nt lds\n"
    "s_add_u32 s58, s58, 32768\n"
    "s_sub_u32 s59, s58, 98304\n"
    "s_cmp_ge_u32 s58, s67\n"
    "s_cselect_b32 s58, s59, s58\n"
    "v_readlane_b32 s54, v245, 1\n"
    "s_add_u32 m0, s55, 32768\n"
    "s_nop 0\n"
    "buffer_load_dwordx4 v224, s[44:47], s53 offen sc1 lds\n"
    "s_add_u32 m0, s55, 33792\n"
    "s_nop 0\n"
    "buffer_load_dwordx4 v225, s[44:47], s53 offen sc1 lds\n"
    "s_add_u32 m0, s55, 34816\n"
    "s_nop 0\n"
    "buffer_load_dwordx4 v226, s[44:47], s53 offen sc1 lds\n"
    "s_add_u32 m0, s55, 35840\n"
    "s_nop 0\n"
    "buffer_load_dwordx4 v227, s[44:47], s53 offen sc1 lds\n"
    "s_add_u32 m0, s55, 36864\n"
    "s_nop 0\n"
    "buffer_load_dwordx4 v228, s[44:47], s53 offen sc1 lds\n"
    "s_add_u32 m0, s55, 37888\n"
    "s_nop 0\n"
    "buffer_load_dwordx4 v229, s[44:47], s53 offen sc1 lds\n"
    "s_add_u32 m0, s55, 38912\n"
    "s_nop 0\n"
    "buffer_load_dwordx4 v230, s[44:47], s53 offen sc1 lds\n"
    "s_add_u32 m0, s55, 39936\n"
    "s_nop 0\n"
    "buffer_load_dwordx4 v231, s[44:47], s53 offen sc1 lds\n"
    "v_readlane_b32 s53, v244, 2\n"
    "v_mov_b32 v251, 0x40000000\n"
    "v_writelane_b32 v251, 0, 0\n"
    "s_memrealtime s[34:35]\n"
    "s_waitcnt vmcnt(16) lgkmcnt(0)\n"
    "s_barrier\n"
    "ds_read_b128 a[0:3], v240 offset:0\n"
    "ds_read_b128 a[4:7], v241 offset:0\n"
    "ds_read_b128 a[8:11], v240 offset:2048\n"
    "ds_read_b128 a[12:15], v241 offset:2048\n"
    "ds_read_b128 a[16:19], v240 offset:4096\n"
    "ds_read_b128 a[20:23], v241 offset:4096\n"
    "ds_read_b128 a[24:27], v240 offset:6144\n"
    "ds_read_b128 a[28:31], v241 offset:6144\n"
    "ds_read_b128 a[32:35], v240 offset:8192\n"
    "ds_read_b128 a[36:39], v241 offset:8192\n"
    "ds_read_b128 a[40:43], v240 offset:10240\n"
    "ds_read_b128 a[44:47], v241 offset:10240\n"
    "ds_read_b128 a[48:51], v240 offset:12288\n"
    "ds_read_b128 a[52:55], v241 offset:12288\n"
    "ds_read_b128 a[56:59], v240 offset:14336\n"
    "ds_read_b128 a[60:63], v241 offset:14336\n"
    "s_add_u32 m0, s58, 0\n"
    "s_nop 0\n"
    "buffer_load_dwordx4 v232, s[40:43], s54 offen nt lds\n"
    "s_add_u32 m0, s58, 1024\n"
    "s_nop 0\n"
    "buffer_load_dwordx4 v233, s[40:43], s54 offen nt lds\n"
    "s_add_u32 m0, s58, 4096\n"
    "s_nop 0\n"
    "buffer_load_dwordx4 v234, s[40:43], s54 offen nt lds\n"
    "s_add_u32 m0, s58, 5120\n"
    "s_nop 0\n"
    "buffer_load_dwordx4 v235, s[40:43], s54 offen nt lds\n"
    "s_add_u32 m0, s58, 8192\n"
    "s_nop 0\n"
    "buffer_load_dwordx4 v236, s[40:43], s54 offen nt lds\n"
    "s_add_u32 m0, s58, 9216\n"
    "s_nop 0\n"
    "buffer_load_dwordx4 v237, s[40:43], s54 offen nt lds\n"
    "s_add_u32 m0, s58, 12288\n"
    "s_nop 0\n"
    "buffer_load_dwordx4 v238, s[40:43], s54 offen nt lds\n"
    "s_add_u32 m0, s58, 13312\n"
    "s_nop 0\n"
    "buffer_load_dwordx4 v239, s[40:43], s54 offen nt lds\n"
    "s_add_u32 s58, s58, 32768\n"
    "s_sub_u32 s59, s58, 98304\n"
    "s_cmp_ge_u32 s58, s67\n"
    "s_cselect_b32 s58, s59, s58\n"
    "v_readfirstlane_b32 s63, v252\n"
    "v_readfirstlane_b32 s59, v253\n"
    "s_sub_u32 s59, s34, s59\n"
    "s_cmp_lt_u32 s59, 10000\n"
    "s_cselect_b32 s39, s63, 0\n"
    "s_add_u32 s63, s63, 1\n"
    "s_and_b32 s59, s63, 1\n"
    "s_lshl_b32 s59, s59, 22\n"
    "s_add_u32 s65, s65, s59\n"
    "s_add_u32 s48, s48, s59\n"
    "s_addc_u32 s49, s49, 0\n"
    "s_bfe_u32 s59, s69, 0x30004\n"
    "v_readlane_b32 s31, v254, s59\n"
    "v_readlane_b32 s28, v254, 0\n"
    "s_mov_b32 s94, s28\n"
    "s_mov_b32 s95, s28\n"
    "v_readlane_b32 s59, v254, 1\n"
    "s_add_u32 s28, s28, s59\n"
    "s_min_u32 s94, s94, s59\n"
    "s_max_u32 s95, s95, s59\n"
    "v_readlane_b32 s59, v254, 2\n"
    "s_add_u32 s28, s28, s59\n"
    "s_min_u32 s94, s94, s59\n"
    "s_max_u32 s95, s95, s59\n"
    "v_readlane_b32 s59, v254, 3\n"
    "s_add_u32 s28, s28, s59\n"
    "s_min_u32 s94, s94, s59\n"
    "s_max_u32 s95, s95, s59\n"
    "v_readlane_b32 s59, v254, 4\n"
    "s_add_u32 s28, s28, s59\n"
    "s_min_u32 s94, s94, s59\n"
    "s_max_u32 s95, s95, s59\n"
    "v_readlane_b32 s59, v254, 5\n"
    "s_add_u32 s28, s28, s59\n"
    "s_min_u32 s94, s94, s59\n"
    "s_max_u32 s95, s95, s59\n"
    "v_readlane_b32 s59, v254, 6\n"
    "s_add_u32 s28, s28, s59\n"
    "s_min_u32 s94, s94, s59\n"
    "s_max_u32 s95, s95, s59\n"
    "v_readlane_b32 s59, v254, 7\n"
    "s_add_u32 s28, s28, s59\n"
    "s_min_u32 s94, s94, s59\n"
    "s_max_u32 s95, s95, s59\n"
    "s_lshr_b32 s28, s28, 3\n"
    "s_sub_u32 s59, s28, s31\n"
    "s_mul_i32 s59, s59, 3640\n"
    "s_ashr_i32 s59, s59, 16\n"
    "s_max_i32 s59, s59, -24\n"
    "s_min_i32 s59, s59, 24\n"
    "s_mul_i32 s60, s59, 18\n"
    "s_sub_u32 s28, s28, s60\n"
    "s_add_u32 s59, s59, 158\n"
    "s_cmp_gt_u32 s95, 6400\n"
    "s_cselect_b32 s94, 0, s94\n"
    "s_cmp_gt_u32 s94, 4000\n"
    "s_cselect_b32 s32, s59, s32\n"
    "s_cselect_b32 s30, s28, s30\n"
    "s_cmp_gt_u32 s31, 4000\n"
    "s_cselect_b32 s31, s31, 30000\n"
    "v_mov_b32 v254, s63\n"
    "s_waitcnt vmcnt(8) lgkmcnt(0)\n"
    "s_barrier\n"
    "s_memrealtime s[34:35]\n"
    "s_waitcnt lgkmcnt(0)\n"
    "s_add_u32 s34, s92, 240\n"
    "s_lshr_b32 s59, s37, 2\n"
    "s_sub_u32 s36, s34, s59\n"
    "ds_read_b64_tr_b16 a[128:129], v242 offset:0\n"
    "ds_read_b64_tr_b16 a[130:131], v242 offset:1024\n"
    "ds_read_b64_tr_b16 a[132:133], v242 offset:8192\n"
    "ds_read_b64_tr_b16 a[134:135], v242 offset:9216\n"
    "ds_read_b64_tr_b16 a[136:137], v243 offset:0\n"
    "ds_read_b64_tr_b16 a[138:139], v243 offset:1024\n"
    "ds_read_b64_tr_b16 a[140:141], v243 offset:8192\n"
    "ds_read_b64_tr_b16 a[142:143], v243 offset:9216\n"
    "s_add_u32 m0, s55, 0\n"
    "s_nop 0\n"
    "buffer_load_dwordx4 v224, s[44:47], s53 offen sc1 lds\n"
    "s_add_u32 m0, s55, 1024\n"
    "s_nop 0\n"
    "buffer_load_dwordx4 v225, s[44:47], s53 offen sc1 lds\n"
    "s_add_u32 m0, s55, 2048\n"
    "s_nop 0\n"
    "buffer_load_dwordx4 v226, s[44:47], s53 offen sc1 lds\n"
    "s_add_u32 m0, s55, 3072\n"
    "s_nop 0\n"
    "buffer_load_dwordx4 v227, s[44:47], s53 offen sc1 lds\n"
    "s_add_u32 m0, s55, 4096\n"
    "s_nop 0\n"
    "buffer_load_dwordx4 v228, s[44:47], s53 offen sc1 lds\n"
    "s_add_u32 m0, s55, 5120\n"
    "s_nop 0\n"
    "buffer_load_dwordx4 v229, s[44:47], s53 offen sc1 lds\n"
    "s_add_u32 m0, s55, 6144\n"
    "s_nop 0\n"
    "buffer_load_dwordx4 v230, s[44:47], s53 offen sc1 lds\n"
    "s_add_u32 m0, s55, 7168\n"
    "s_nop 0\n"
    "buffer_load_dwordx4 v231, s[44:47], s53 offen sc1 lds\n"
    "s_cmp_lg_u32 s39, 0\n"
    "s_cbranch_scc1 .Lg2_%=\n"
    "s_mov_b32 s52, 0\n"
    "s_cmp_eq_u32 s52, s56\n"
    "s_cbranch_scc1 .Lg3_%=\n"
    "s_waitcnt lgkmcnt(4)\n"
    "v_mfma_f32_16x16x32_bf16 v[0:3], a[128:131], a[0:3], 0\n"
    "s_add_u32 s59, s52, 2\n"
    "v_mfma_f32_16x16x32_bf16 v[0:3], a[132:135], a[4:7], v[0:3]\n"
    "ds_read_b64_tr_b16 a[144:145], v242 offset:256\n"
    "v_readlane_b32 s54, v245, s59\n"
    "v_mfma_f32_16x16x32_bf16 v[4:7], a[128:131], a[8:11], 0\n"
    "v_mfma_f32_16x16x32_bf16 v[4:7], a[132:135], a[12:15], v[4:7]\n"
    "ds_read_b64_tr_b16 a[146:147], v242 offset:1280\n"
    "v_mfma_f32_16x16x32_bf16 v[8:11], a[128:131], a[16:19], 0\n"
    "v_mfma_f32_16x16x32_bf16 v[8:11], a[132:135], a[20:23], v[8:11]\n"
    "ds_read_b64_tr_b16 a[148:149], v242 offset:8448\n"
    "v_mfma_f32_16x16x32_bf16 v[12:15], a[128:131], a[24:27], 0\n"
    "v_mfma_f32_16x16x32_bf16 v[12:15], a[132:135], a[28:31], v[12:15]\n"
    "ds_read_b64_tr_b16 a[150:151], v242 offset:9472\n"
    "v_mfma_f32_16x16x32_bf16 v[16:19], a[128:131], a[32:35], 0\n"
    "s_add_u32 m0, s58, 0\n"
    "v_mfma_f32_16x16x32_bf16 v[16:19], a[132:135], a[36:39], v[16:19]\n"
    "buffer_load_dwordx4 v232, s[40:43], s54 offen nt lds\n"
    "v_mfma_f32_16x16x32_bf16 v[20:23], a[128:131], a[40:43], 0\n"
    "v_mfma_f32_16x16x32_bf16 v[20:23], a[132:135], a[44:47], v[20:23]\n"
    "ds_read_b128 a[64:67], v240 offset:32768\n"
    "v_mfma_f32_16x16x32_bf16 v[24:27], a[128:131], a[48:51], 0\n"
    "s_add_u32 m0, s58, 1024\n"
    "v_mfma_f32_16x16x32_bf16 v[24:27], a[132:135], a[52:55], v[24:27]\n"
    "buffer_load_dwordx4 v233, s[40:43], s54 offen nt lds\n"
    "v_mfma_f32_16x16x32_bf16 v[28:31], a[128:131], a[56:59], 0\n"
    "v_mfma_f32_16x16x32_bf16 v[28:31], a[132:135], a[60:63], v[28:31]\n"
    "ds_read_b128 a[68:71], v241 offset:32768\n"
    "s_waitcnt lgkmcnt(6)\n"
    "v_mfma_f32_16x16x32_bf16 v[32:35], a[136:139], a[0:3], 0\n"
    "v_mfma_f32_16x16x32_bf16 v[32:35], a[140:143], a[4:7], v[32:35]\n"
    "ds_read_b64_tr_b16 a[152:153], v243 offset:256\n"
    "v_mfma_f32_16x16x32_bf16 v[36:39], a[136:139], a[8:11], 0\n"
    "v_mfma_f32_16x16x32_bf16 v[36:39], a[140:143], a[12:15], v[36:39]\n"
    "ds_read_b64_tr_b16 a[154:155], v243 offset:1280\n"
    "s_cmp_lt_u32 s52, s33\n"
    "s_cselect_b32 s59, s32, 0\n"
    "v_mfma_f32_16x16x32_bf16 v[40:43], a[136:139], a[16:19], 0\n"
    "s_cmp_lt_u32 s52, s38\n"
    "s_cselect_b32 s59, s37, s59\n"
    "v_mfma_f32_16x16x32_bf16 v[40:43], a[140:143], a[20:23], v[40:43]\n"
    "ds_read_b64_tr_b16 a[156:157], v243 offset:8448\n"
    "v_mfma_f32_16x16x32_bf16 v[44:47], a[136:139], a[24:27], 0\n"
    "s_add_u32 s36, s36, s59\n"
    "v_mfma_f32_16x16x32_bf16 v[44:47], a[140:143], a[28:31], v[44:47]\n"
    "ds_read_b64_tr_b16 a[158:159], v243 offset:9472\n"
    "v_mfma_f32_16x16x32_bf16 v[48:51], a[136:139], a[32:35], 0\n"
    "s_add_u32 m0, s58, 4096\n"
    "v_mfma_f32_16x16x32_bf16 v[48:51], a[140:143], a[36:39], v[48:51]\n"
    "buffer_load_dwordx4 v234, s[40:43], s54 offen nt lds\n"
    "v_mfma_f32_16x16x32_bf16 v[52:55], a[136:139], a[40:43], 0\n"
    "v_mfma_f32_16x16x32_bf16 v[52:55], a[140:143], a[44:47], v[52:55]\n"
    "ds_read_b128 a[72:75], v240 offset:34816\n"
    "v_mfma_f32_16x16x32_bf16 v[56:59], a[136:139], a[48:51], 0\n"
    "s_add_u32 m0, s58, 5120\n"
    "v_mfma_f32_16x16x32_bf16 v[56:59], a[140:143], a[52:55], v[56:59]\n"
    "buffer_load_dwordx4 v235, s[40:43], s54 offen nt lds\n"
    "v_mfma_f32_16x16x32_bf16 v[60:63], a[136:139], a[56:59], 0\n"
    "v_mfma_f32_16x16x32_bf16 v[60:63], a[140:143], a[60:63], v[60:63]\n"
    "ds_read_b128 a[76:79], v241 offset:34816\n"
    "s_waitcnt lgkmcnt(8)\n"
    "v_mfma_f32_16x16x32_bf16 v[64:67], a[144:147], a[0:3], 0\n"
    "v_mfma_f32_16x16x32_bf16 v[64:67], a[148:151], a[4:7], v[64:67]\n"
    "ds_read_b64_tr_b16 a[128:129], v242 offset:512\n"
    "v_mfma_f32_16x16x32_bf16 v[68:71], a[144:147], a[8:11], 0\n"
    "v_mfma_f32_16x16x32_bf16 v[68:71], a[148:151], a[12:15], v[68:71]\n"
    "ds_read_b64_tr_b16 a[130:131], v242 offset:1536\n"
    "v_mfma_f32_16x16x32_bf16 v[72:75], a[144:147], a[16:19], 0\n"
    "v_mfma_f32_16x16x32_bf16 v[72:75], a[148:151], a[20:23], v[72:75]\n"
    "ds_read_b64_tr_b16 a[132:133], v242 offset:8704\n"
    "v_mfma_f32_16x16x32_bf16 v[76:79], a[144:147], a[24:27], 0\n"
    "v_mfma_f32_16x16x32_bf16 v[76:79], a[148:151], a[28:31], v[76:79]\n"
    "ds_read_b64_tr_b16 a[134:135], v242 offset:9728\n"
    "v_mfma_f32_16x16x32_bf16 v[80:83], a[144:147], a[32:35], 0\n"
    "s_add_u32 m0, s58, 8192\n"
    "v_mfma_f32_16x16x32_bf16 v[80:83], a[148:151], a[36:39], v[80:83]\n"
    "buffer_load_dwordx4 v236, s[40:43], s54 offen nt lds\n"
    "v_mfma_f32_16x16x32_bf16 v[84:87], a[144:147], a[40:43], 0\n"
    "v_mfma_f32_16x16x32_bf16 v[84:87], a[148:151], a[44:47], v[84:87]\n"
    "ds_read_b128 a[80:83], v240 offset:36864\n"
    "v_mfma_f32_16x16x32_bf16 v[88:91], a[144:147], a[48:51], 0\n"
    "s_add_u32 m0, s58, 9216\n"
    "v_mfma_f32_16x16x32_bf16 v[88:91], a[148:151], a[52:55], v[88:91]\n"
    "buffer_load_dwordx4 v237, s[40:43], s54 offen nt lds\n"
    "v_mfma_f32_16x16x32_bf16 v[92:95], a[144:147], a[56:59], 0\n"
    "v_mfma_f32_16x16x32_bf16 v[92:95], a[148:151], a[60:63], v[92:95]\n"
    "ds_read_b128 a[84:87], v241 offset:36864\n"
    "s_waitcnt lgkmcnt(8)\n"
    "v_mfma_f32_16x16x32_bf16 v[96:99], a[152:155], a[0:3], 0\n"
    "v_mfma_f32_16x16x32_bf16 v[96:99], a[156:159], a[4:7], v[96:99]\n"
    "ds_read_b64_tr_b16 a[136:137], v243 offset:512\n"
    "v_mfma_f32_16x16x32_bf16 v[100:103], a[152:155], a[8:11], 0\n"
    "v_mfma_f32_16x16x32_bf16 v[100:103], a[156:159], a[12:15], v[100:103]\n"
    "ds_read_b64_tr_b16 a[138:139], v243 offset:1536\n"
    "v_mfma_f32_16x16x32_bf16 v[104:107], a[152:155], a[16:19], 0\n"
    "v_mfma_f32_16x16x32_bf16 v[104:107], a[156:159], a[20:23], v[104:107]\n"
    "ds_read_b64_tr_b16 a[140:141], v243 offset:8704\n"
    "v_mfma_f32_16x16x32_bf16 v[108:111], a[152:155], a[24:27], 0\n"
    "v_mfma_f32_16x16x32_bf16 v[108:111], a[156:159], a[28:31], v[108:111]\n"
    "ds_read_b64_tr_b16 a[142:143], v243 offset:9728\n"
    "v_mfma_f32_16x16x32_bf16 v[112:115], a[152:155], a[32:35], 0\n"
    "s_add_u32 m0, s58, 12288\n"
    "v_mfma_f32_16x16x32_bf16 v[112:115], a[156:159], a[36:39], v[112:115]\n"
    "buffer_load_dwordx4 v238, s[40:43], s54 offen nt lds\n"
    "v_mfma_f32_16x16x32_bf16 v[116:119], a[152:155], a[40:43], 0\n"
    "v_mfma_f32_16x16x32_bf16 v[116:119], a[156:159], a[44:47], v[116:119]\n"
    "ds_read_b128 a[88:91], v240 offset:38912\n"
    "v_mfma_f32_16x16x32_bf16 v[120:123], a[152:155], a[48:51], 0\n"
    "s_add_u32 m0, s58, 13312\n"
    "v_mfma_f32_16x16x32_bf16 v[120:123], a[156:159], a[52:55], v[120:123]\n"
    "buffer_load_dwordx4 v239, s[40:43], s54 offen nt lds\n"
    "v_mfma_f32_16x16x32_bf16 v[124:127], a[152:155], a[56:59], 0\n"
    "v_mfma_f32_16x16x32_bf16 v[124:127], a[156:159], a[60:63], v[124:127]\n"
    "ds_read_b128 a[92:95], v241 offset:38912\n"
    "s_waitcnt lgkmcnt(8)\n"
    "v_mfma_f32_16x16x32_bf16 v[128:131], a[128:131], a[0:3], 0\n"
    "v_mfma_f32_16x16x32_bf16 v[128:131], a[132:135], a[4:7], v[128:131]\n"
    "ds_read_b64_tr_b16 a[144:145], v242 offset:768\n"
    "v_mfma_f32_16x16x32_bf16 v[132:135], a[128:131], a[8:11], 0\n"
    "s_add_u32 s58, s58, 32768\n"
    "v_mfma_f32_16x16x32_bf16 v[132:135], a[132:135], a[12:15], v[132:135]\n"
    "ds_read_b64_tr_b16 a[146:147], v242 offset:1792\n"
    "v_mfma_f32_16x16x32_bf16 v[136:139], a[128:131], a[16:19], 0\n"
    "s_sub_u32 s59, s58, 98304\n"
    "v_mfma_f32_16x16x32_bf16 v[136:139], a[132:135], a[20:23], v[136:139]\n"
    "ds_read_b64_tr_b16 a[148:149], v242 offset:8960\n"
    "v_mfma_f32_16x16x32_bf16 v[140:143], a[128:131], a[24:27], 0\n"
    "s_cmp_ge_u32 s58, s67\n"
    "v_mfma_f32_16x16x32_bf16 v[140:143], a[132:135], a[28:31], v[140:143]\n"
    "ds_read_b64_tr_b16 a[150:151], v242 offset:9984\n"
    "v_mfma_f32_16x16x32_bf16 v[144:147], a[128:131], a[32:35], 0\n"
    "s_cselect_b32 s58, s59, s58\n"
    "v_mfma_f32_16x16x32_bf16 v[144:147], a[132:135], a[36:39], v[144:147]\n"
    "ds_read_b128 a[96:99], v240 offset:40960\n"
    "v_mfma_f32_16x16x32_bf16 v[148:151], a[128:131], a[40:43], 0\n"
    "v_mfma_f32_16x16x32_bf16 v[148:151], a[132:135], a[44:47], v[148:151]\n"
    "ds_read_b128 a[100:103], v241 offset:40960\n"
    "v_mfma_f32_16x16x32_bf16 v[152:155], a[128:131], a[48:51], 0\n"
    "v_mfma_f32_16x16x32_bf16 v[152:155], a[132:135], a[52:55], v[152:155]\n"
    "ds_read_b128 a[104:107], v240 offset:43008\n"
    "v_mfma_f32_16x16x32_bf16 v[156:159], a[128:131], a[56:59], 0\n"
    "v_mfma_f32_16x16x32_bf16 v[156:159], a[132:135], a[60:63], v[156:159]\n"
    "ds_read_b128 a[108:111], v241 offset:43008\n"
    "s_waitcnt lgkmcnt(10)\n"
    "v_mfma_f32_16x16x32_bf16 v[160:163], a[136:139], a[0:3], 0\n"
    "v_mfma_f32_16x16x32_bf16 v[160:163], a[140:143], a[4:7], v[160:163]\n"
    "ds_read_b64_tr_b16 a[152:153], v243 offset:768\n"
    "ds_read_b128 a[112:115], v240 offset:45056\n"
    "v_mfma_f32_16x16x32_bf16 v[164:167], a[136:139], a[8:11], 0\n"
    "s_add_u32 s59, s52, 3\n"
    "v_mfma_f32_16x16x32_bf16 v[164:167], a[140:143], a[12:15], v[164:167]\n"
    "ds_read_b64_tr_b16 a[154:155], v243 offset:1792\n"
    "ds_read_b128 a[116:119], v241 offset:45056\n"
    "v_readlane_b32 s53, v244, s59\n"
    "v_mfma_f32_16x16x32_bf16 v[168:171], a[136:139], a[16:19], 0\n"
    "v_mfma_f32_16x16x32_bf16 v[168:171], a[140:143], a[20:23], v[168:171]\n"
    "ds_read_b64_tr_b16 a[156:157], v243 offset:8960\n"
    "ds_read_b128 a[120:123], v240 offset:47104\n"
    "v_mfma_f32_16x16x32_bf16 v[172:175], a[136:139], a[24:27], 0\n"
    "s_memrealtime s[34:35]\n"
    "v_mfma_f32_16x16x32_bf16 v[172:175], a[140:143], a[28:31], v[172:175]\n"
    "ds_read_b64_tr_b16 a[158:159], v243 offset:9984\n"
    "ds_read_b128 a[124:127], v241 offset:47104\n"
    "v_mfma_f32_16x16x32_bf16 v[176:179], a[136:139], a[32:35], 0\n"
    "s_add_u32 s57, s57, 1\n"
    "v_mfma_f32_16x16x32_bf16 v[176:179], a[140:143], a[36:39], v[176:179]\n"
    "s_cmp_eq_u32 s57, 3\n"
    "v_mfma_f32_16x16x32_bf16 v[180:183], a[136:139], a[40:43], 0\n"
    "s_cselect_b32 s57, 0, s57\n"
    "s_cselect_b32 s59, 4294901760, s68\n"
    "v_mfma_f32_16x16x32_bf16 v[180:183], a[140:143], a[44:47], v[180:183]\n"
    "v_mfma_f32_16x16x32_bf16 v[184:187], a[136:139], a[48:51], 0\n"
    "v_add_u32 v242, s59, v242\n"
    "v_mfma_f32_16x16x32_bf16 v[184:187], a[140:143], a[52:55], v[184:187]\n"
    "v_add_u32 v243, s59, v243\n"
    "v_mfma_f32_16x16x32_bf16 v[188:191], a[136:139], a[56:59], 0\n"
    "v_mfma_f32_16x16x32_bf16 v[188:191], a[140:143], a[60:63], v[188:191]\n"
    "s_waitcnt vmcnt(8) lgkmcnt(0)\n"
    "s_sub_u32 s34, s34, s36\n"
    "s_cmp_lt_i32 s34, -8\n"
    "s_cbranch_scc1 .Lg7_%=\n"
    ".Lg8_%=:\n"
    "s_barrier\n"
    "v_mfma_f32_16x16x32_bf16 v[192:195], a[144:147], a[0:3], 0\n"
    "v_mfma_f32_16x16x32_bf16 v[192:195], a[148:151], a[4:7], v[192:195]\n"
    "ds_read_b64_tr_b16 a[128:129], v242 offset:0\n"
    "v_mfma_f32_16x16x32_bf16 v[196:199], a[144:147], a[8:11], 0\n"
    "v_mfma_f32_16x16x32_bf16 v[196:199], a[148:151], a[12:15], v[196:199]\n"
    "ds_read_b64_tr_b16 a[130:131], v242 offset:1024\n"
    "v_mfma_f32_16x16x32_bf16 v[200:203], a[144:147], a[16:19], 0\n"
    "v_mfma_f32_16x16x32_bf16 v[200:203], a[148:151], a[20:23], v[200:203]\n"
    "ds_read_b64_tr_b16 a[132:133], v242 offset:8192\n"
    "v_mfma_f32_16x16x32_bf16 v[204:207], a[144:147], a[24:27], 0\n"
    "v_mfma_f32_16x16x32_bf16 v[204:207], a[148:151], a[28:31], v[204:207]\n"
    "ds_read_b64_tr_b16 a[134:135], v242 offset:9216\n"
    "v_mfma_f32_16x16x32_bf16 v[208:211], a[144:147], a[32:35], 0\n"
    "s_add_u32 m0, s55, 32768\n"
    "v_mfma_f32_16x16x32_bf16 v[208:211], a[148:151], a[36:39], v[208:211]\n"
    "buffer_load_dwordx4 v224, s[44:47], s53 offen sc1 lds\n"
    "v_mfma_f32_16x16x32_bf16 v[212:215], a[144:147], a[40:43], 0\n"
    "s_add_u32 m0, s55, 33792\n"
    "v_mfma_f32_16x16x32_bf16 v[212:215], a[148:151], a[44:47], v[212:215]\n"
    "buffer_load_dwordx4 v225, s[44:47], s53 offen sc1 lds\n"
    "v_mfma_f32_16x16x32_bf16 v[216:219], a[144:147], a[48:51], 0\n"
    "s_add_u32 m0, s55, 34816\n"
    "v_mfma_f32_16x16x32_bf16 v[216:219], a[148:151], a[52:55], v[216:219]\n"
    "buffer_load_dwordx4 v226, s[44:47], s53 offen sc1 lds\n"
    "v_mfma_f32_16x16x32_bf16 v[220:223], a[144:147], a[56:59], 0\n"
    "s_add_u32 m0, s55, 35840\n"
    "v_mfma_f32_16x16x32_bf16 v[220:223], a[148:151], a[60:63], v[220:223]\n"
    "buffer_load_dwordx4 v227, s[44:47], s53 offen sc1 lds\n"
    "s_waitcnt lgkmcnt(5)\n"
    "v_mfma_f32_16x16x32_bf16 a[160:163], a[152:155], a[0:3], 0\n"
    "v_mfma_f32_16x16x32_bf16 a[160:163], a[156:159], a[4:7], a[160:163]\n"
    "ds_read_b64_tr_b16 a[136:137], v243 offset:0\n"
    "v_mfma_f32_16x16x32_bf16 a[164:167], a[152:155], a[8:11], 0\n"
    "v_mfma_f32_16x16x32_bf16 a[164:167], a[156:159], a[12:15], a[164:167]\n"
    "ds_read_b64_tr_b16 a[138:139], v243 offset:1024\n"
    "v_mfma_f32_16x16x32_bf16 a[168:171], a[152:155], a[16:19], 0\n"
    "v_mfma_f32_16x16x32_bf16 a[168:171], a[156:159], a[20:23], a[168:171]\n"
    "ds_read_b64_tr_b16 a[140:141], v243 offset:8192\n"
    "v_mfma_f32_16x16x32_bf16 a[172:175], a[152:155], a[24:27], 0\n"
    "v_mfma_f32_16x16x32_bf16 a[172:175], a[156:159], a[28:31], a[172:175]\n"
    "ds_read_b64_tr_b16 a[142:143], v243 offset:9216\n"
    "v_mfma_f32_16x16x32_bf16 a[176:179], a[152:155], a[32:35], 0\n"
    "s_add_u32 m0, s55, 36864\n"
    "v_mfma_f32_16x16x32_bf16 a[176:179], a[156:159], a[36:39], a[176:179]\n"
    "buffer_load_dwordx4 v228, s[44:47], s53 offen sc1 lds\n"
    "v_mfma_f32_16x16x32_bf16 a[180:183], a[152:155], a[40:43], 0\n"
    "s_add_u32 m0, s55, 37888\n"
    "v_mfma_f32_16x16x32_bf16 a[180:183], a[156:159], a[44:47], a[180:183]\n"
    "buffer_load_dwordx4 v229, s[44:47], s53 offen sc1 lds\n"
    "v_mfma_f32_16x16x32_bf16 a[184:187], a[152:155], a[48:51], 0\n"
    "s_add_u32 m0, s55, 38912\n"
    "v_mfma_f32_16x16x32_bf16 a[184:187], a[156:159], a[52:55], a[184:187]\n"
    "buffer_load_dwordx4 v230, s[44:47], s53 offen sc1 lds\n"
    "v_mfma_f32_16x16x32_bf16 a[188:191], a[152:155], a[56:59], 0\n"
    "s_add_u32 m0, s55, 39936\n"
    "v_mfma_f32_16x16x32_bf16 a[188:191], a[156:159], a[60:63], a[188:191]\n"
    "buffer_load_dwordx4 v231, s[44:47], s53 offen sc1 lds\n"
    "s_add_u32 s52, s52, 1\n"
    ".Lg4_%=:\n"
    "s_waitcnt lgkmcnt(4)\n"
    "v_mfma_f32_16x16x32_bf16 v[0:3], a[128:131], a[64:67], v[0:3]\n"
    "s_add_u32 s59, s52, 2\n"
    "v_mfma_f32_16x16x32_bf16 v[0:3], a[132:135], a[68:71], v[0:3]\n"
    "ds_read_b64_tr_b16 a[144:145], v242 offset:256\n"
    "v_readlane_b32 s54, v245, s59\n"
    "v_mfma_f32_16x16x32_bf16 v[4:7], a[128:131], a[72:75], v[4:7]\n"
    "v_mfma_f32_16x16x32_bf16 v[4:7], a[132:135], a[76:79], v[4:7]\n"
    "ds_read_b64_tr_b16 a[146:147], v242 offset:1280\n"
    "v_mfma_f32_16x16x32_bf16 v[8:11], a[128:131], a[80:83], v[8:11]\n"
    "buffer_load_dword v252, v246, s[88:91], 0 offen sc1\n"
    "v_mfma_f32_16x16x32_bf16 v[8:11], a[132:135], a[84:87], v[8:11]\n"
    "ds_read_b64_tr_b16 a[148:149], v242 offset:8448\n"
    "v_mfma_f32_16x16x32_bf16 v[12:15], a[128:131], a[88:91], v[12:15]\n"
    "v_mfma_f32_16x16x32_bf16 v[12:15], a[132:135], a[92:95], v[12:15]\n"
    "ds_read_b64_tr_b16 a[150:151], v242 offset:9472\n"
    "v_mfma_f32_16x16x32_bf16 v[16:19], a[128:131], a[96:99], v[16:19]\n"
    "s_add_u32 m0, s58, 0\n"
    "v_mfma_f32_16x16x32_bf16 v[16:19], a[132:135], a[100:103], v[16:19]\n"
    "buffer_load_dwordx4 v232, s[40:43], s54 offen nt lds\n"
    "v_mfma_f32_16x16x32_bf16 v[20:23], a[128:131], a[104:107], v[20:23]\n"
    "v_readlane_b32 s62, v247, s52\n"
    "v_mfma_f32_16x16x32_bf16 v[20:23], a[132:135], a[108:111], v[20:23]\n"
    "ds_read_b128 a[0:3], v240 offset:0\n"
    "v_mfma_f32_16x16x32_bf16 v[24:27], a[128:131], a[112:115], v[24:27]\n"
    "s_lshl_b64 s[78:79], -1, s62\n"
    "s_add_u32 m0, s58, 1024\n"
    "v_mfma_f32_16x16x32_bf16 v[24:27], a[132:135], a[116:119], v[24:27]\n"
    "buffer_load_dwordx4 v233, s[40:43], s54 offen nt lds\n"
    "v_mfma_f32_16x16x32_bf16 v[28:31], a[128:131], a[120:123], v[28:31]\n"
    "v_mfma_f32_16x16x32_bf16 v[28:31], a[132:135], a[124:127], v[28:31]\n"
    "ds_read_b128 a[4:7], v241 offset:0\n"
    "s_waitcnt lgkmcnt(6)\n"
    "v_mfma_f32_16x16x32_bf16 v[32:35], a[136:139], a[64:67], v[32:35]\n"
    "s_sub_u32 s61, s52, 1\n"
    "v_mfma_f32_16x16x32_bf16 v[32:35], a[140:143], a[68:71], v[32:35]\n"
    "ds_read_b64_tr_b16 a[152:153], v243 offset:256\n"
    "v_mfma_f32_16x16x32_bf16 v[36:39], a[136:139], a[72:75], v[36:39]\n"
    "s_cmp_eq_u32 s61, s56\n"
    "s_cselect_b32 s61, s66, 2147418112\n"
    "v_mfma_f32_16x16x32_bf16 v[36:39], a[140:143], a[76:79], v[36:39]\n"
    "ds_read_b64_tr_b16 a[154:155], v243 offset:1280\n"
    "s_cmp_lt_u32 s52, s33\n"
    "s_cselect_b32 s59, s32, 0\n"
    "v_mfma_f32_16x16x32_bf16 v[40:43], a[136:139], a[80:83], v[40:43]\n"
    "s_cmp_lt_u32 s52, s38\n"
    "s_cselect_b32 s59, s37, s59\n"
    "v_mfma_f32_16x16x32_bf16 v[40:43], a[140:143], a[84:87], v[40:43]\n"
    "ds_read_b64_tr_b16 a[156:157], v243 offset:8448\n"
    "v_mfma_f32_16x16x32_bf16 v[44:47], a[136:139], a[88:91], v[44:47]\n"
    "s_add_u32 s36, s36, s59\n"
    "v_mfma_f32_16x16x32_bf16 v[44:47], a[140:143], a[92:95], v[44:47]\n"
    "ds_read_b64_tr_b16 a[158:159], v243 offset:9472\n"
    "v_mfma_f32_16x16x32_bf16 v[48:51], a[136:139], a[96:99], v[48:51]\n"
    "s_add_u32 m0, s58, 4096\n"
    "v_mfma_f32_16x16x32_bf16 v[48:51], a[140:143], a[100:103], v[48:51]\n"
    "buffer_load_dwordx4 v234, s[40:43], s54 offen nt lds\n"
    "v_mfma_f32_16x16x32_bf16 v[52:55], a[136:139], a[104:107], v[52:55]\n"
    "v_mfma_f32_16x16x32_bf16 v[52:55], a[140:143], a[108:111], v[52:55]\n"
    "ds_read_b128 a[8:11], v240 offset:2048\n"
    "v_mfma_f32_16x16x32_bf16 v[56:59], a[136:139], a[112:115], v[56:59]\n"
    "s_add_u32 m0, s58, 5120\n"
    "v_mfma_f32_16x16x32_bf16 v[56:59], a[140:143], a[116:119], v[56:59]\n"
    "buffer_load_dwordx4 v235, s[40:43], s54 offen nt lds\n"
    "v_mfma_f32_16x16x32_bf16 v[60:63], a[136:139], a[120:123], v[60:63]\n"
    "v_mfma_f32_16x16x32_bf16 v[60:63], a[140:143], a[124:127], v[60:63]\n"
    "ds_read_b128 a[12:15], v241 offset:2048\n"
    "s_waitcnt lgkmcnt(8)\n"
    "v_mfma_f32_16x16x32_bf16 v[64:67], a[144:147], a[64:67], v[64:67]\n"
    "v_mfma_f32_16x16x32_bf16 v[64:67], a[148:151], a[68:71], v[64:67]\n"
    "ds_read_b64_tr_b16 a[128:129], v242 offset:512\n"
    "v_mfma_f32_16x16x32_bf16 v[68:71], a[144:147], a[72:75], v[68:71]\n"
    "v_mfma_f32_16x16x32_bf16 v[68:71], a[148:151], a[76:79], v[68:71]\n"
    "ds_read_b64_tr_b16 a[130:131], v242 offset:1536\n"
    "v_mfma_f32_16x16x32_bf16 v[72:75], a[144:147], a[80:83], v[72:75]\n"
    "v_mfma_f32_16x16x32_bf16 v[72:75], a[148:151], a[84:87], v[72:75]\n"
    "ds_read_b64_tr_b16 a[132:133], v242 offset:8704\n"
    "v_mfma_f32_16x16x32_bf16 v[76:79], a[144:147], a[88:91], v[76:79]\n"
    "v_mfma_f32_16x16x32_bf16 v[76:79], a[148:151], a[92:95], v[76:79]\n"
    "ds_read_b64_tr_b16 a[134:135], v242 offset:9728\n"
    "v_mfma_f32_16x16x32_bf16 v[80:83], a[144:147], a[96:99], v[80:83]\n"
    "s_add_u32 m0, s58, 8192\n"
    "v_mfma_f32_16x16x32_bf16 v[80:83], a[148:151], a[100:103], v[80:83]\n"
    "buffer_load_dwordx4 v236, s[40:43], s54 offen nt lds\n"
    "v_mfma_f32_16x16x32_bf16 v[84:87], a[144:147], a[104:107], v[84:87]\n"
    "v_mfma_f32_16x16x32_bf16 v[84:87], a[148:151], a[108:111], v[84:87]\n"
    "ds_read_b128 a[16:19], v240 offset:4096\n"
    "v_mfma_f32_16x16x32_bf16 v[88:91], a[144:147], a[112:115], v[88:91]\n"
    "s_add_u32 m0, s58, 9216\n"
    "v_mfma_f32_16x16x32_bf16 v[88:91], a[148:151], a[116:119], v[88:91]\n"
    "buffer_load_dwordx4 v237, s[40:43], s54 offen nt lds\n"
    "v_mfma_f32_16x16x32_bf16 v[92:95], a[144:147], a[120:123], v[92:95]\n"
    "v_mfma_f32_16x16x32_bf16 v[92:95], a[148:151], a[124:127], v[92:95]\n"
    "ds_read_b128 a[20:23], v241 offset:4096\n"
    "s_waitcnt lgkmcnt(8)\n"
    "v_mfma_f32_16x16x32_bf16 v[96:99], a[152:155], a[64:67], v[96:99]\n"
    "v_mfma_f32_16x16x32_bf16 v[96:99], a[156:159], a[68:71], v[96:99]\n"
    "ds_read_b64_tr_b16 a[136:137], v243 offset:512\n"
    "v_mfma_f32_16x16x32_bf16 v[100:103], a[152:155], a[72:75], v[100:103]\n"
    "v_mfma_f32_16x16x32_bf16 v[100:103], a[156:159], a[76:79], v[100:103]\n"
    "ds_read_b64_tr_b16 a[138:139], v243 offset:1536\n"
    "v_mfma_f32_16x16x32_bf16 v[104:107], a[152:155], a[80:83], v[104:107]\n"
    "v_mfma_f32_16x16x32_bf16 v[104:107], a[156:159], a[84:87], v[104:107]\n"
    "ds_read_b64_tr_b16 a[140:141], v243 offset:8704\n"
    "v_mfma_f32_16x16x32_bf16 v[108:111], a[152:155], a[88:91], v[108:111]\n"
    "v_mfma_f32_16x16x32_bf16 v[108:111], a[156:159], a[92:95], v[108:111]\n"
    "ds_read_b64_tr_b16 a[142:143], v243 offset:9728\n"
    "v_mfma_f32_16x16x32_bf16 v[112:115], a[152:155], a[96:99], v[112:115]\n"
    "s_add_u32 m0, s58, 12288\n"
    "v_mfma_f32_16x16x32_bf16 v[112:115], a[156:159], a[100:103], v[112:115]\n"
    "buffer_load_dwordx4 v238, s[40:43], s54 offen nt lds\n"
    "v_mfma_f32_16x16x32_bf16 v[116:119], a[152:155], a[104:107], v[116:119]\n"
    "v_mfma_f32_16x16x32_bf16 v[116:119], a[156:159], a[108:111], v[116:119]\n"
    "ds_read_b128 a[24:27], v240 offset:6144\n"
    "v_mfma_f32_16x16x32_bf16 v[120:123], a[152:155], a[112:115], v[120:123]\n"
    "s_add_u32 m0, s58, 13312\n"
    "v_mfma_f32_16x16x32_bf16 v[120:123], a[156:159], a[116:119], v[120:123]\n"
    "buffer_load_dwordx4 v239, s[40:43], s54 offen nt lds\n"
    "v_mfma_f32_16x16x32_bf16 v[124:127], a[152:155], a[120:123], v[124:127]\n"
    "v_mfma_f32_16x16x32_bf16 v[124:127], a[156:159], a[124:127], v[124:127]\n"
    "ds_read_b128 a[28:31], v241 offset:6144\n"
    "s_waitcnt lgkmcnt(8)\n"
    "v_mfma_f32_16x16x32_bf16 v[128:131], a[128:131], a[64:67], v[128:131]\n"
    "v_mfma_f32_16x16x32_bf16 v[128:131], a[132:135], a[68:71], v[128:131]\n"
    "ds_read_b64_tr_b16 a[144:145], v242 offset:768\n"
    "v_mfma_f32_16x16x32_bf16 v[132:135], a[128:131], a[72:75], v[132:135]\n"
    "s_add_u32 s58, s58, 32768\n"
    "v_mfma_f32_16x16x32_bf16 v[132:135], a[132:135], a[76:79], v[132:135]\n"
    "ds_read_b64_tr_b16 a[146:147], v242 offset:1792\n"
    "v_mfma_f32_16x16x32_bf16 v[136:139], a[128:131], a[80:83], v[136:139]\n"
    "s_sub_u32 s59, s58, 98304\n"
    "v_mfma_f32_16x16x32_bf16 v[136:139], a[132:135], a[84:87], v[136:139]\n"
    "ds_read_b64_tr_b16 a[148:149], v242 offset:8960\n"
    "v_mfma_f32_16x16x32_bf16 v[140:143], a[128:131], a[88:91], v[140:143]\n"
    "s_cmp_ge_u32 s58, s67\n"
    "v_mfma_f32_16x16x32_bf16 v[140:143], a[132:135], a[92:95], v[140:143]\n"
    "ds_read_b64_tr_b16 a[150:151], v242 offset:9984\n"
    "v_mfma_f32_16x16x32_bf16 v[144:147], a[128:131], a[96:99], v[144:147]\n"
    "s_cselect_b32 s58, s59, s58\n"
    "v_mfma_f32_16x16x32_bf16 v[144:147], a[132:135], a[100:103], v[144:147]\n"
    "ds_read_b128 a[32:35], v240 offset:8192\n"
    "v_mfma_f32_16x16x32_bf16 v[148:151], a[128:131], a[104:107], v[148:151]\n"
    "v_mfma_f32_16x16x32_bf16 v[148:151], a[132:135], a[108:111], v[148:151]\n"
    "ds_read_b128 a[36:39], v241 offset:8192\n"
    "v_mfma_f32_16x16x32_bf16 v[152:155], a[128:131], a[112:115], v[152:155]\n"
    "v_mfma_f32_16x16x32_bf16 v[152:155], a[132:135], a[116:119], v[152:155]\n"
    "ds_read_b128 a[40:43], v240 offset:10240\n"
    "v_mfma_f32_16x16x32_bf16 v[156:159], a[128:131], a[120:123], v[156:159]\n"
    "v_mfma_f32_16x16x32_bf16 v[156:159], a[132:135], a[124:127], v[156:159]\n"
    "ds_read_b128 a[44:47], v241 offset:10240\n"
    "s_waitcnt lgkmcnt(10)\n"
    "v_mfma_f32_16x16x32_bf16 v[160:163], a[136:139], a[64:67], v[160:163]\n"
    "v_mfma_f32_16x16x32_bf16 v[160:163], a[140:143], a[68:71], v[160:163]\n"
    "ds_read_b64_tr_b16 a[152:153], v243 offset:768\n"
    "ds_read_b128 a[48:51], v240 offset:12288\n"
    "v_mfma_f32_16x16x32_bf16 v[164:167], a[136:139], a[72:75], v[164:167]\n"
    "s_add_u32 s59, s52, 3\n"
    "v_mfma_f32_16x16x32_bf16 v[164:167], a[140:143], a[76:79], v[164:167]\n"
    "ds_read_b64_tr_b16 a[154:155], v243 offset:1792\n"
    "ds_read_b128 a[52:55], v241 offset:12288\n"
    "v_readlane_b32 s53, v244, s59\n"
    "v_mfma_f32_16x16x32_bf16 v[168:171], a[136:139], a[80:83], v[168:171]\n"
    "s_cmp_eq_u32 s59, 8\n"
    "s_cselect_b32 s44, s48, s44\n"
    "v_mfma_f32_16x16x32_bf16 v[168:171], a[140:143], a[84:87], v[168:171]\n"
    "ds_read_b64_tr_b16 a[156:157], v243 offset:8960\n"
    "ds_read_b128 a[56:59], v240 offset:14336\n"
    "s_cselect_b32 s45, s49, s45\n"
    "v_mfma_f32_16x16x32_bf16 v[172:175], a[136:139], a[88:91], v[172:175]\n"
    "s_memrealtime s[34:35]\n"
    "v_mfma_f32_16x16x32_bf16 v[172:175], a[140:143], a[92:95], v[172:175]\n"
    "ds_read_b64_tr_b16 a[158:159], v243 offset:9984\n"
    "ds_read_b128 a[60:63], v241 offset:14336\n"
    "v_mfma_f32_16x16x32_bf16 v[176:179], a[136:139], a[96:99], v[176:179]\n"
    "s_add_u32 s57, s57, 1\n"
    "v_mfma_f32_16x16x32_bf16 v[176:179], a[140:143], a[100:103], v[176:179]\n"
    "s_cmp_eq_u32 s57, 3\n"
    "v_mfma_f32_16x16x32_bf16 v[180:183], a[136:139], a[104:107], v[180:183]\n"
    "s_cselect_b32 s57, 0, s57\n"
    "s_cselect_b32 s59, 4294901760, s68\n"
    "v_mfma_f32_16x16x32_bf16 v[180:183], a[140:143], a[108:111], v[180:183]\n"
    "v_mfma_f32_16x16x32_bf16 v[184:187], a[136:139], a[112:115], v[184:187]\n"
    "v_add_u32 v242, s59, v242\n"
    "v_mfma_f32_16x16x32_bf16 v[184:187], a[140:143], a[116:119], v[184:187]\n"
    "v_add_u32 v243, s59, v243\n"
    "v_mfma_f32_16x16x32_bf16 v[188:191], a[136:139], a[120:123], v[188:191]\n"
    "v_mfma_f32_16x16x32_bf16 v[188:191], a[140:143], a[124:127], v[188:191]\n"
    "s_waitcnt vmcnt(8) lgkmcnt(0)\n"
    "s_sub_u32 s34, s34, s36\n"
    "s_cmp_lt_i32 s34, -8\n"
    "s_cbranch_scc1 .Lg12_%=\n"
    ".Lg13_%=:\n"
    "s_barrier\n"
    "v_mfma_f32_16x16x32_bf16 v[192:195], a[144:147], a[64:67], v[192:195]\n"
    "v_cmp_le_u32 vcc, s63, v252\n"
    "v_mfma_f32_16x16x32_bf16 v[192:195], a[148:151], a[68:71], v[192:195]\n"
    "buffer_store_dword v254, v251, s[84:87], s61 offen sc0 sc1\n"
    "ds_read_b64_tr_b16 a[128:129], v242 offset:0\n"
    "v_mfma_f32_16x16x32_bf16 v[196:199], a[144:147], a[72:75], v[196:199]\n"
    "s_or_b64 vcc, vcc, s[78:79]\n"
    "s_cmp_eq_u64 vcc, -1\n"
    "v_mfma_f32_16x16x32_bf16 v[196:199], a[148:151], a[76:79], v[196:199]\n"
    "s_cbranch_scc0 .Lg9_%=\n"
    ".Lg10_%=:\n"
    "ds_read_b64_tr_b16 a[130:131], v242 offset:1024\n"
    "v_mfma_f32_16x16x32_bf16 v[200:203], a[144:147], a[80:83], v[200:203]\n"
    "v_mfma_f32_16x16x32_bf16 v[200:203], a[148:151], a[84:87], v[200:203]\n"
    "ds_read_b64_tr_b16 a[132:133], v242 offset:8192\n"
    "v_mfma_f32_16x16x32_bf16 v[204:207], a[144:147], a[88:91], v[204:207]\n"
    "v_mfma_f32_16x16x32_bf16 v[204:207], a[148:151], a[92:95], v[204:207]\n"
    "ds_read_b64_tr_b16 a[134:135], v242 offset:9216\n"
    "v_mfma_f32_16x16x32_bf16 v[208:211], a[144:147], a[96:99], v[208:211]\n"
    "s_add_u32 m0, s55, 0\n"
    "v_mfma_f32_16x16x32_bf16 v[208:211], a[148:151], a[100:103], v[208:211]\n"
    "buffer_load_dwordx4 v224, s[44:47], s53 offen sc1 lds\n"
    "v_mfma_f32_16x16x32_bf16 v[212:215], a[144:147], a[104:107], v[212:215]\n"
    "s_add_u32 m0, s55, 1024\n"
    "v_mfma_f32_16x16x32_bf16 v[212:215], a[148:151], a[108:111], v[212:215]\n"
    "buffer_load_dwordx4 v225, s[44:47], s53 offen sc1 lds\n"
    "v_mfma_f32_16x16x32_bf16 v[216:219], a[144:147], a[112:115], v[216:219]\n"
    "s_add_u32 m0, s55, 2048\n"
    "v_mfma_f32_16x16x32_bf16 v[216:219], a[148:151], a[116:119], v[216:219]\n"
    "buffer_load_dwordx4 v226, s[44:47], s53 offen sc1 lds\n"
    "v_mfma_f32_16x16x32_bf16 v[220:223], a[144:147], a[120:123], v[220:223]\n"
    "s_add_u32 m0, s55, 3072\n"
    "v_mfma_f32_16x16x32_bf16 v[220:223], a[148:151], a[124:127], v[220:223]\n"
    "buffer_load_dwordx4 v227, s[44:47], s53 offen sc1 lds\n"
    "s_waitcnt lgkmcnt(5)\n"
    "v_mfma_f32_16x16x32_bf16 a[160:163], a[152:155], a[64:67], a[160:163]\n"
    "v_mfma_f32_16x16x32_bf16 a[160:163], a[156:159], a[68:71], a[160:163]\n"
    "ds_read_b64_tr_b16 a[136:137], v243 offset:0\n"
    "v_mfma_f32_16x16x32_bf16 a[164:167], a[152:155], a[72:75], a[164:167]\n"
    "v_mfma_f32_16x16x32_bf16 a[164:167], a[156:159], a[76:79], a[164:167]\n"
    "ds_read_b64_tr_b16 a[138:139], v243 offset:1024\n"
    "v_mfma_f32_16x16x32_bf16 a[168:171], a[152:155], a[80:83], a[168:171]\n"
    "v_mfma_f32_16x16x32_bf16 a[168:171], a[156:159], a[84:87], a[168:171]\n"
    "ds_read_b64_tr_b16 a[140:141], v243 offset:8192\n"
    "v_mfma_f32_16x16x32_bf16 a[172:175], a[152:155], a[88:91], a[172:175]\n"
    "v_mfma_f32_16x16x32_bf16 a[172:175], a[156:159], a[92:95], a[172:175]\n"
    "ds_read_b64_tr_b16 a[142:143], v243 offset:9216\n"
    "v_mfma_f32_16x16x32_bf16 a[176:179], a[152:155], a[96:99], a[176:179]\n"
    "s_add_u32 m0, s55, 4096\n"
    "v_mfma_f32_16x16x32_bf16 a[176:179], a[156:159], a[100:103], a[176:179]\n"
    "buffer_load_dwordx4 v228, s[44:47], s53 offen sc1 lds\n"
    "v_mfma_f32_16x16x32_bf16 a[180:183], a[152:155], a[104:107], a[180:183]\n"
    "s_add_u32 m0, s55, 5120\n"
    "v_mfma_f32_16x16x32_bf16 a[180:183], a[156:159], a[108:111], a[180:183]\n"
    "buffer_load_dwordx4 v229, s[44:47], s53 offen sc1 lds\n"
    "v_mfma_f32_16x16x32_bf16 a[184:187], a[152:155], a[112:115], a[184:187]\n"
    "s_add_u32 m0, s55, 6144\n"
    "v_mfma_f32_16x16x32_bf16 a[184:187], a[156:159], a[116:119], a[184:187]\n"
    "buffer_load_dwordx4 v230, s[44:47], s53 offen sc1 lds\n"
    "v_mfma_f32_16x16x32_bf16 a[188:191], a[152:155], a[120:123], a[188:191]\n"
    "s_add_u32 m0, s55, 7168\n"
    "v_mfma_f32_16x16x32_bf16 a[188:191], a[156:159], a[124:127], a[188:191]\n"
    "buffer_load_dwordx4 v231, s[44:47], s53 offen sc1 lds\n"
    "s_add_u32 s52, s52, 1\n"
    "s_mov_b32 s60, 8\n"
    ".Lg14_%=:\n"
    "s_cmp_eq_u32 s52, s56\n"
    "s_cbranch_scc1 .Lg15_%=\n"
    "s_waitcnt lgkmcnt(4)\n"
    "v_mfma_f32_16x16x32_bf16 v[0:3], a[128:131], a[0:3], v[0:3]\n"
    "s_add_u32 s59, s52, 2\n"
    "v_mfma_f32_16x16x32_bf16 v[0:3], a[132:135], a[4:7], v[0:3]\n"
    "ds_read_b64_tr_b16 a[144:145], v242 offset:256\n"
    "v_readlane_b32 s54, v245, s59\n"
    "v_mfma_f32_16x16x32_bf16 v[4:7], a[128:131], a[8:11], v[4:7]\n"
    "v_mfma_f32_16x16x32_bf16 v[4:7], a[132:135], a[12:15], v[4:7]\n"
    "ds_read_b64_tr_b16 a[146:147], v242 offset:1280\n"
    "v_mfma_f32_16x16x32_bf16 v[8:11], a[128:131], a[16:19], v[8:11]\n"
    "v_mfma_f32_16x16x32_bf16 v[8:11], a[132:135], a[20:23], v[8:11]\n"
    "ds_read_b64_tr_b16 a[148:149], v242 offset:8448\n"
    "v_mfma_f32_16x16x32_bf16 v[12:15], a[128:131], a[24:27], v[12:15]\n"
    "v_mfma_f32_16x16x32_bf16 v[12:15], a[132:135], a[28:31], v[12:15]\n"
    "ds_read_b64_tr_b16 a[150:151], v242 offset:9472\n"
    "v_mfma_f32_16x16x32_bf16 v[16:19], a[128:131], a[32:35], v[16:19]\n"
    "s_add_u32 m0, s58, 0\n"
    "v_mfma_f32_16x16x32_bf16 v[16:19], a[132:135], a[36:39], v[16:19]\n"
    "buffer_load_dwordx4 v232, s[40:43], s54 offen nt lds\n"
    "v_mfma_f32_16x16x32_bf16 v[20:23], a[128:131], a[40:43], v[20:23]\n"
    "v_mfma_f32_16x16x32_bf16 v[20:23], a[132:135], a[44:47], v[20:23]\n"
    "ds_read_b128 a[64:67], v240 offset:32768\n"
    "v_mfma_f32_16x16x32_bf16 v[24:27], a[128:131], a[48:51], v[24:27]\n"
    "s_add_u32 m0, s58, 1024\n"
    "v_mfma_f32_16x16x32_bf16 v[24:27], a[132:135], a[52:55], v[24:27]\n"
    "buffer_load_dwordx4 v233, s[40:43], s54 offen nt lds\n"
    "v_mfma_f32_16x16x32_bf16 v[28:31], a[128:131], a[56:59], v[28:31]\n"
    "v_mfma_f32_16x16x32_bf16 v[28:31], a[132:135], a[60:63], v[28:31]\n"
    "ds_read_b128 a[68:71], v241 offset:32768\n"
    "s_waitcnt lgkmcnt(6)\n"
    "v_mfma_f32_16x16x32_bf16 v[32:35], a[136:139], a[0:3], v[32:35]\n"
    "v_mfma_f32_16x16x32_bf16 v[32:35], a[140:143], a[4:7], v[32:35]\n"
    "ds_read_b64_tr_b16 a[152:153], v243 offset:256\n"
    "v_mfma_f32_16x16x32_bf16 v[36:39], a[136:139], a[8:11], v[36:39]\n"
    "v_mfma_f32_16x16x32_bf16 v[36:39], a[140:143], a[12:15], v[36:39]\n"
    "ds_read_b64_tr_b16 a[154:155], v243 offset:1280\n"
    "s_cmp_lt_u32 s52, s33\n"
    "s_cselect_b32 s59, s32, 0\n"
    "v_mfma_f32_16x16x32_bf16 v[40:43], a[136:139], a[16:19], v[40:43]\n"
    "s_cmp_lt_u32 s52, s38\n"
    "s_cselect_b32 s59, s37, s59\n"
    "v_mfma_f32_16x16x32_bf16 v[40:43], a[140:143], a[20:23], v[40:43]\n"
    "ds_read_b64_tr_b16 a[156:157], v243 offset:8448\n"
    "v_mfma_f32_16x16x32_bf16 v[44:47], a[136:139], a[24:27], v[44:47]\n"
    "s_add_u32 s36, s36, s59\n"
    "v_mfma_f32_16x16x32_bf16 v[44:47], a[140:143], a[28:31], v[44:47]\n"
    "ds_read_b64_tr_b16 a[158:159], v243 offset:9472\n"
    "v_mfma_f32_16x16x32_bf16 v[48:51], a[136:139], a[32:35], v[48:51]\n"
    "s_add_u32 m0, s58, 4096\n"
    "v_mfma_f32_16x16x32_bf16 v[48:51], a[140:143], a[36:39], v[48:51]\n"
    "buffer_load_dwordx4 v234, s[40:43], s54 offen nt lds\n"
    "v_mfma_f32_16x16x32_bf16 v[52:55], a[136:139], a[40:43], v[52:55]\n"
    "v_mfma_f32_16x16x32_bf16 v[52:55], a[140:143], a[44:47], v[52:55]\n"
    "ds_read_b128 a[72:75], v240 offset:34816\n"
    "v_mfma_f32_16x16x32_bf16 v[56:59], a[136:139], a[48:51], v[56:59]\n"
    "s_add_u32 m0, s58, 5120\n"
    "v_mfma_f32_16x16x32_bf16 v[56:59], a[140:143], a[52:55], v[56:59]\n"
    "buffer_load_dwordx4 v235, s[40:43], s54 offen nt lds\n"
    "v_mfma_f32_16x16x32_bf16 v[60:63], a[136:139], a[56:59], v[60:63]\n"
    "v_mfma_f32_16x16x32_bf16 v[60:63], a[140:143], a[60:63], v[60:63]\n"
    "ds_read_b128 a[76:79], v241 offset:34816\n"
    "s_waitcnt lgkmcnt(8)\n"
    "v_mfma_f32_16x16x32_bf16 v[64:67], a[144:147], a[0:3], v[64:67]\n"
    "v_mfma_f32_16x16x32_bf16 v[64:67], a[148:151], a[4:7], v[64:67]\n"
    "ds_read_b64_tr_b16 a[128:129], v242 offset:512\n"
    "v_mfma_f32_16x16x32_bf16 v[68:71], a[144:147], a[8:11], v[68:71]\n"
    "v_mfma_f32_16x16x32_bf16 v[68:71], a[148:151], a[12:15], v[68:71]\n"
    "ds_read_b64_tr_b16 a[130:131], v242 offset:1536\n"
    "v_mfma_f32_16x16x32_bf16 v[72:75], a[144:147], a[16:19], v[72:75]\n"
    "v_mfma_f32_16x16x32_bf16 v[72:75], a[148:151], a[20:23], v[72:75]\n"
    "ds_read_b64_tr_b16 a[132:133], v242 offset:8704\n"
    "v_mfma_f32_16x16x32_bf16 v[76:79], a[144:147], a[24:27], v[76:79]\n"
    "v_mfma_f32_16x16x32_bf16 v[76:79], a[148:151], a[28:31], v[76:79]\n"
    "ds_read_b64_tr_b16 a[134:135], v242 offset:9728\n"
    "v_mfma_f32_16x16x32_bf16 v[80:83], a[144:147], a[32:35], v[80:83]\n"
    "s_add_u32 m0, s58, 8192\n"
    "v_mfma_f32_16x16x32_bf16 v[80:83], a[148:151], a[36:39], v[80:83]\n"
    "buffer_load_dwordx4 v236, s[40:43], s54 offen nt lds\n"
    "v_mfma_f32_16x16x32_bf16 v[84:87], a[144:147], a[40:43], v[84:87]\n"
    "v_mfma_f32_16x16x32_bf16 v[84:87], a[148:151], a[44:47], v[84:87]\n"
    "ds_read_b128 a[80:83], v240 offset:36864\n"
    "v_mfma_f32_16x16x32_bf16 v[88:91], a[144:147], a[48:51], v[88:91]\n"
    "s_add_u32 m0, s58, 9216\n"
    "v_mfma_f32_16x16x32_bf16 v[88:91], a[148:151], a[52:55], v[88:91]\n"
    "buffer_load_dwordx4 v237, s[40:43], s54 offen nt lds\n"
    "v_mfma_f32_16x16x32_bf16 v[92:95], a[144:147], a[56:59], v[92:95]\n"
    "v_mfma_f32_16x16x32_bf16 v[92:95], a[148:151], a[60:63], v[92:95]\n"
    "ds_read_b128 a[84:87], v241 offset:36864\n"
    "s_waitcnt lgkmcnt(8)\n"
    "v_mfma_f32_16x16x32_bf16 v[96:99], a[152:155], a[0:3], v[96:99]\n"
    "v_mfma_f32_16x16x32_bf16 v[96:99], a[156:159], a[4:7], v[96:99]\n"
    "ds_read_b64_tr_b16 a[136:137], v243 offset:512\n"
    "v_mfma_f32_16x16x32_bf16 v[100:103], a[152:155], a[8:11], v[100:103]\n"
    "v_mfma_f32_16x16x32_bf16 v[100:103], a[156:159], a[12:15], v[100:103]\n"
    "ds_read_b64_tr_b16 a[138:139], v243 offset:1536\n"
    "v_mfma_f32_16x16x32_bf16 v[104:107], a[152:155], a[16:19], v[104:107]\n"
    "v_mfma_f32_16x16x32_bf16 v[104:107], a[156:159], a[20:23], v[104:107]\n"
    "ds_read_b64_tr_b16 a[140:141], v243 offset:8704\n"
    "v_mfma_f32_16x16x32_bf16 v[108:111], a[152:155], a[24:27], v[108:111]\n"
    "v_mfma_f32_16x16x32_bf16 v[108:111], a[156:159], a[28:31], v[108:111]\n"
    "ds_read_b64_tr_b16 a[142:143], v243 offset:9728\n"
    "v_mfma_f32_16x16x32_bf16 v[112:115], a[152:155], a[32:35], v[112:115]\n"
    "s_add_u32 m0, s58, 12288\n"
    "v_mfma_f32_16x16x32_bf16 v[112:115], a[156:159], a[36:39], v[112:115]\n"
    "buffer_load_dwordx4 v238, s[40:43], s54 offen nt lds\n"
    "v_mfma_f32_16x16x32_bf16 v[116:119], a[152:155], a[40:43], v[116:119]\n"
    "v_mfma_f32_16x16x32_bf16 v[116:119], a[156:159], a[44:47], v[116:119]\n"
    "ds_read_b128 a[88:91], v240 offset:38912\n"
    "v_mfma_f32_16x16x32_bf16 v[120:123], a[152:155], a[48:51], v[120:123]\n"
    "s_add_u32 m0, s58, 13312\n"
    "v_mfma_f32_16x16x32_bf16 v[120:123], a[156:159], a[52:55], v[120:123]\n"
    "buffer_load_dwordx4 v239, s[40:43], s54 offen nt lds\n"
    "v_mfma_f32_16x16x32_bf16 v[124:127], a[152:155], a[56:59], v[124:127]\n"
    "v_mfma_f32_16x16x32_bf16 v[124:127], a[156:159], a[60:63], v[124:127]\n"
    "ds_read_b128 a[92:95], v241 offset:38912\n"
    "s_waitcnt lgkmcnt(8)\n"
    "v_mfma_f32_16x16x32_bf16 v[128:131], a[128:131], a[0:3], v[128:131]\n"
    "v_mfma_f32_16x16x32_bf16 v[128:131], a[132:135], a[4:7], v[128:131]\n"
    "ds_read_b64_tr_b16 a[144:145], v242 offset:768\n"
    "v_mfma_f32_16x16x32_bf16 v[132:135], a[128:131], a[8:11], v[132:135]\n"
    "s_add_u32 s58, s58, 32768\n"
    "v_mfma_f32_16x16x32_bf16 v[132:135], a[132:135], a[12:15], v[132:135]\n"
    "ds_read_b64_tr_b16 a[146:147], v242 offset:1792\n"
    "v_mfma_f32_16x16x32_bf16 v[136:139], a[128:131], a[16:19], v[136:139]\n"
    "s_sub_u32 s59, s58, 98304\n"
    "v_mfma_f32_16x16x32_bf16 v[136:139], a[132:135], a[20:23], v[136:139]\n"
    "ds_read_b64_tr_b16 a[148:149], v242 offset:8960\n"
    "v_mfma_f32_16x16x32_bf16 v[140:143], a[128:131], a[24:27], v[140:143]\n"
    "s_cmp_ge_u32 s58, s67\n"
    "v_mfma_f32_16x16x32_bf16 v[140:143], a[132:135], a[28:31], v[140:143]\n"
    "ds_read_b64_tr_b16 a[150:151], v242 offset:9984\n"
    "v_mfma_f32_16x16x32_bf16 v[144:147], a[128:131], a[32:35], v[144:147]\n"
    "s_cselect_b32 s58, s59, s58\n"
    "v_mfma_f32_16x16x32_bf16 v[144:147], a[132:135], a[36:39], v[144:147]\n"
    "ds_read_b128 a[96:99], v240 offset:40960\n"
    "v_mfma_f32_16x16x32_bf16 v[148:151], a[128:131], a[40:43], v[148:151]\n"
    "v_mfma_f32_16x16x32_bf16 v[148:151], a[132:135], a[44:47], v[148:151]\n"
    "ds_read_b128 a[100:103], v241 offset:40960\n"
    "v_mfma_f32_16x16x32_bf16 v[152:155], a[128:131], a[48:51], v[152:155]\n"
    "v_mfma_f32_16x16x32_bf16 v[152:155], a[132:135], a[52:55], v[152:155]\n"
    "ds_read_b128 a[104:107], v240 offset:43008\n"
    "v_mfma_f32_16x16x32_bf16 v[156:159], a[128:131], a[56:59], v[156:159]\n"
    "v_mfma_f32_16x16x32_bf16 v[156:159], a[132:135], a[60:63], v[156:159]\n"
    "ds_read_b128 a[108:111], v241 offset:43008\n"
    "s_waitcnt lgkmcnt(10)\n"
    "v_mfma_f32_16x16x32_bf16 v[160:163], a[136:139], a[0:3], v[160:163]\n"
    "v_mfma_f32_16x16x32_bf16 v[160:163], a[140:143], a[4:7], v[160:163]\n"
    "ds_read_b64_tr_b16 a[152:153], v243 offset:768\n"
    "ds_read_b128 a[112:115], v240 offset:45056\n"
    "v_mfma_f32_16x16x32_bf16 v[164:167], a[136:139], a[8:11], v[164:167]\n"
    "s_add_u32 s59, s52, 3\n"
    "v_mfma_f32_16x16x32_bf16 v[164:167], a[140:143], a[12:15], v[164:167]\n"
    "ds_read_b64_tr_b16 a[154:155], v243 offset:1792\n"
    "ds_read_b128 a[116:119], v241 offset:45056\n"
    "v_readlane_b32 s53, v244, s59\n"
    "v_mfma_f32_16x16x32_bf16 v[168:171], a[136:139], a[16:19], v[168:171]\n"
    "v_mfma_f32_16x16x32_bf16 v[168:171], a[140:143], a[20:23], v[168:171]\n"
    "ds_read_b64_tr_b16 a[156:157], v243 offset:8960\n"
    "ds_read_b128 a[120:123], v240 offset:47104\n"
    "v_mfma_f32_16x16x32_bf16 v[172:175], a[136:139], a[24:27], v[172:175]\n"
    "s_memrealtime s[34:35]\n"
    "v_mfma_f32_16x16x32_bf16 v[172:175], a[140:143], a[28:31], v[172:175]\n"
    "ds_read_b64_tr_b16 a[158:159], v243 offset:9984\n"
    "ds_read_b128 a[124:127], v241 offset:47104\n"
    "v_mfma_f32_16x16x32_bf16 v[176:179], a[136:139], a[32:35], v[176:179]\n"
    "s_add_u32 s57, s57, 1\n"
    "v_mfma_f32_16x16x32_bf16 v[176:179], a[140:143], a[36:39], v[176:179]\n"
    "s_cmp_eq_u32 s57, 3\n"
    "v_mfma_f32_16x16x32_bf16 v[180:183], a[136:139], a[40:43], v[180:183]\n"
    "s_cselect_b32 s57, 0, s57\n"
    "s_cselect_b32 s59, 4294901760, s68\n"
    "v_mfma_f32_16x16x32_bf16 v[180:183], a[140:143], a[44:47], v[180:183]\n"
    "v_mfma_f32_16x16x32_bf16 v[184:187], a[136:139], a[48:51], v[184:187]\n"
    "v_add_u32 v242, s59, v242\n"
    "v_mfma_f32_16x16x32_bf16 v[184:187], a[140:143], a[52:55], v[184:187]\n"
    "v_add_u32 v243, s59, v243\n"
    "v_mfma_f32_16x16x32_bf16 v[188:191], a[136:139], a[56:59], v[188:191]\n"
    "v_mfma_f32_16x16x32_bf16 v[188:191], a[140:143], a[60:63], v[188:191]\n"
    "s_waitcnt vmcnt(8) lgkmcnt(0)\n"
    "s_sub_u32 s34, s34, s36\n"
    "s_cmp_lt_i32 s34, -8\n"
    "s_cbranch_scc1 .Lg19_%=\n"
    ".Lg20_%=:\n"
    "s_barrier\n"
    "v_mfma_f32_16x16x32_bf16 v[192:195], a[144:147], a[0:3], v[192:195]\n"
    "v_mfma_f32_16x16x32_bf16 v[192:195], a[148:151], a[4:7], v[192:195]\n"
    "ds_read_b64_tr_b16 a[128:129], v242 offset:0\n"
    "v_mfma_f32_16x16x32_bf16 v[196:199], a[144:147], a[8:11], v[196:199]\n"
    "v_mfma_f32_16x16x32_bf16 v[196:199], a[148:151], a[12:15], v[196:199]\n"
    "ds_read_b64_tr_b16 a[130:131], v242 offset:1024\n"
    "v_mfma_f32_16x16x32_bf16 v[200:203], a[144:147], a[16:19], v[200:203]\n"
    "v_mfma_f32_16x16x32_bf16 v[200:203], a[148:151], a[20:23], v[200:203]\n"
    "ds_read_b64_tr_b16 a[132:133], v242 offset:8192\n"
    "v_mfma_f32_16x16x32_bf16 v[204:207], a[144:147], a[24:27], v[204:207]\n"
    "v_mfma_f32_16x16x32_bf16 v[204:207], a[148:151], a[28:31], v[204:207]\n"
    "ds_read_b64_tr_b16 a[134:135], v242 offset:9216\n"
    "v_mfma_f32_16x16x32_bf16 v[208:211], a[144:147], a[32:35], v[208:211]\n"
    "s_add_u32 m0, s55, 32768\n"
    "v_mfma_f32_16x16x32_bf16 v[208:211], a[148:151], a[36:39], v[208:211]\n"
    "buffer_load_dwordx4 v224, s[44:47], s53 offen sc1 lds\n"
    "v_mfma_f32_16x16x32_bf16 v[212:215], a[144:147], a[40:43], v[212:215]\n"
    "s_add_u32 m0, s55, 33792\n"
    "v_mfma_f32_16x16x32_bf16 v[212:215], a[148:151], a[44:47], v[212:215]\n"
    "buffer_load_dwordx4 v225, s[44:47], s53 offen sc1 lds\n"
    "v_mfma_f32_16x16x32_bf16 v[216:219], a[144:147], a[48:51], v[216:219]\n"
    "s_add_u32 m0, s55, 34816\n"
    "v_mfma_f32_16x16x32_bf16 v[216:219], a[148:151], a[52:55], v[216:219]\n"
    "buffer_load_dwordx4 v226, s[44:47], s53 offen sc1 lds\n"
    "v_mfma_f32_16x16x32_bf16 v[220:223], a[144:147], a[56:59], v[220:223]\n"
    "s_add_u32 m0, s55, 35840\n"
    "v_mfma_f32_16x16x32_bf16 v[220:223], a[148:151], a[60:63], v[220:223]\n"
    "buffer_load_dwordx4 v227, s[44:47], s53 offen sc1 lds\n"
    "s_waitcnt lgkmcnt(5)\n"
    "v_mfma_f32_16x16x32_bf16 a[160:163], a[152:155], a[0:3], a[160:163]\n"
    "v_mfma_f32_16x16x32_bf16 a[160:163], a[156:159], a[4:7], a[160:163]\n"
    "ds_read_b64_tr_b16 a[136:137], v243 offset:0\n"
    "v_mfma_f32_16x16x32_bf16 a[164:167], a[152:155], a[8:11], a[164:167]\n"
    "v_mfma_f32_16x16x32_bf16 a[164:167], a[156:159], a[12:15], a[164:167]\n"
    "ds_read_b64_tr_b16 a[138:139], v243 offset:1024\n"
    "v_mfma_f32_16x16x32_bf16 a[168:171], a[152:155], a[16:19], a[168:171]\n"
    "v_mfma_f32_16x16x32_bf16 a[168:171], a[156:159], a[20:23], a[168:171]\n"
    "ds_read_b64_tr_b16 a[140:141], v243 offset:8192\n"
    "v_mfma_f32_16x16x32_bf16 a[172:175], a[152:155], a[24:27], a[172:175]\n"
    "v_mfma_f32_16x16x32_bf16 a[172:175], a[156:159], a[28:31], a[172:175]\n"
    "ds_read_b64_tr_b16 a[142:143], v243 offset:9216\n"
    "v_mfma_f32_16x16x32_bf16 a[176:179], a[152:155], a[32:35], a[176:179]\n"
    "s_add_u32 m0, s55, 36864\n"
    "v_mfma_f32_16x16x32_bf16 a[176:179], a[156:159], a[36:39], a[176:179]\n"
    "buffer_load_dwordx4 v228, s[44:47], s53 offen sc1 lds\n"
    "v_mfma_f32_16x16x32_bf16 a[180:183], a[152:155], a[40:43], a[180:183]\n"
    "s_add_u32 m0, s55, 37888\n"
    "v_mfma_f32_16x16x32_bf16 a[180:183], a[156:159], a[44:47], a[180:183]\n"
    "buffer_load_dwordx4 v229, s[44:47], s53 offen sc1 lds\n"
    "v_mfma_f32_16x16x32_bf16 a[184:187], a[152:155], a[48:51], a[184:187]\n"
    "s_add_u32 m0, s55, 38912\n"
    "v_mfma_f32_16x16x32_bf16 a[184:187], a[156:159], a[52:55], a[184:187]\n"
    "buffer_load_dwordx4 v230, s[44:47], s53 offen sc1 lds\n"
    "v_mfma_f32_16x16x32_bf16 a[188:191], a[152:155], a[56:59], a[188:191]\n"
    "s_add_u32 m0, s55, 39936\n"
    "v_mfma_f32_16x16x32_bf16 a[188:191], a[156:159], a[60:63], a[188:191]\n"
    "buffer_load_dwordx4 v231, s[44:47], s53 offen sc1 lds\n"
    "s_add_u32 s52, s52, 1\n"
    ".Lg16_%=:\n"
    "s_waitcnt lgkmcnt(4)\n"
    "v_mfma_f32_16x16x32_bf16 v[0:3], a[128:131], a[64:67], v[0:3]\n"
    "s_add_u32 s59, s52, 2\n"
    "v_mfma_f32_16x16x32_bf16 v[0:3], a[132:135], a[68:71], v[0:3]\n"
    "ds_read_b64_tr_b16 a[144:145], v242 offset:256\n"
    "v_readlane_b32 s54, v245, s59\n"
    "v_mfma_f32_16x16x32_bf16 v[4:7], a[128:131], a[72:75], v[4:7]\n"
    "v_mfma_f32_16x16x32_bf16 v[4:7], a[132:135], a[76:79], v[4:7]\n"
    "ds_read_b64_tr_b16 a[146:147], v242 offset:1280\n"
    "v_mfma_f32_16x16x32_bf16 v[8:11], a[128:131], a[80:83], v[8:11]\n"
    "buffer_load_dword v252, v246, s[88:91], 0 offen sc1\n"
    "v_mfma_f32_16x16x32_bf16 v[8:11], a[132:135], a[84:87], v[8:11]\n"
    "ds_read_b64_tr_b16 a[148:149], v242 offset:8448\n"
    "v_mfma_f32_16x16x32_bf16 v[12:15], a[128:131], a[88:91], v[12:15]\n"
    "v_mfma_f32_16x16x32_bf16 v[12:15], a[132:135], a[92:95], v[12:15]\n"
    "ds_read_b64_tr_b16 a[150:151], v242 offset:9472\n"
    "v_mfma_f32_16x16x32_bf16 v[16:19], a[128:131], a[96:99], v[16:19]\n"
    "s_add_u32 m0, s58, 0\n"
    "v_mfma_f32_16x16x32_bf16 v[16:19], a[132:135], a[100:103], v[16:19]\n"
    "buffer_load_dwordx4 v232, s[40:43], s54 offen nt lds\n"
    "v_mfma_f32_16x16x32_bf16 v[20:23], a[128:131], a[104:107], v[20:23]\n"
    "v_readlane_b32 s62, v247, s52\n"
    "v_mfma_f32_16x16x32_bf16 v[20:23], a[132:135], a[108:111], v[20:23]\n"
    "ds_read_b128 a[0:3], v240 offset:0\n"
    "v_mfma_f32_16x16x32_bf16 v[24:27], a[128:131], a[112:115], v[24:27]\n"
    "s_lshl_b64 s[78:79], -1, s62\n"
    "s_add_u32 m0, s58, 1024\n"
    "v_mfma_f32_16x16x32_bf16 v[24:27], a[132:135], a[116:119], v[24:27]\n"
    "buffer_load_dwordx4 v233, s[40:43], s54 offen nt lds\n"
    "v_mfma_f32_16x16x32_bf16 v[28:31], a[128:131], a[120:123], v[28:31]\n"
    "v_mfma_f32_16x16x32_bf16 v[28:31], a[132:135], a[124:127], v[28:31]\n"
    "ds_read_b128 a[4:7], v241 offset:0\n"
    "s_waitcnt lgkmcnt(6)\n"
    "v_mfma_f32_16x16x32_bf16 v[32:35], a[136:139], a[64:67], v[32:35]\n"
    "s_sub_u32 s61, s52, 1\n"
    "v_mfma_f32_16x16x32_bf16 v[32:35], a[140:143], a[68:71], v[32:35]\n"
    "ds_read_b64_tr_b16 a[152:153], v243 offset:256\n"
    "v_mfma_f32_16x16x32_bf16 v[36:39], a[136:139], a[72:75], v[36:39]\n"
    "s_cmp_eq_u32 s61, s56\n"
    "s_cselect_b32 s61, s66, 2147418112\n"
    "v_mfma_f32_16x16x32_bf16 v[36:39], a[140:143], a[76:79], v[36:39]\n"
    "ds_read_b64_tr_b16 a[154:155], v243 offset:1280\n"
    "s_cmp_lt_u32 s52, s33\n"
    "s_cselect_b32 s59, s32, 0\n"
    "v_mfma_f32_16x16x32_bf16 v[40:43], a[136:139], a[80:83], v[40:43]\n"
    "s_cmp_lt_u32 s52, s38\n"
    "s_cselect_b32 s59, s37, s59\n"
    "v_mfma_f32_16x16x32_bf16 v[40:43], a[140:143], a[84:87], v[40:43]\n"
    "ds_read_b64_tr_b16 a[156:157], v243 offset:8448\n"
    "v_mfma_f32_16x16x32_bf16 v[44:47], a[136:139], a[88:91], v[44:47]\n"
    "s_add_u32 s36, s36, s59\n"
    "v_mfma_f32_16x16x32_bf16 v[44:47], a[140:143], a[92:95], v[44:47]\n"
    "ds_read_b64_tr_b16 a[158:159], v243 offset:9472\n"
    "v_mfma_f32_16x16x32_bf16 v[48:51], a[136:139], a[96:99], v[48:51]\n"
    "s_add_u32 m0, s58, 4096\n"
    "v_mfma_f32_16x16x32_bf16 v[48:51], a[140:143], a[100:103], v[48:51]\n"
    "buffer_load_dwordx4 v234, s[40:43], s54 offen nt lds\n"
    "v_mfma_f32_16x16x32_bf16 v[52:55], a[136:139], a[104:107], v[52:55]\n"
    "v_mfma_f32_16x16x32_bf16 v[52:55], a[140:143], a[108:111], v[52:55]\n"
    "ds_read_b128 a[8:11], v240 offset:2048\n"
    "v_mfma_f32_16x16x32_bf16 v[56:59], a[136:139], a[112:115], v[56:59]\n"
    "s_add_u32 m0, s58, 5120\n"
    "v_mfma_f32_16x16x32_bf16 v[56:59], a[140:143], a[116:119], v[56:59]\n"
    "buffer_load_dwordx4 v235, s[40:43], s54 offen nt lds\n"
    "v_mfma_f32_16x16x32_bf16 v[60:63], a[136:139], a[120:123], v[60:63]\n"
    "v_mfma_f32_16x16x32_bf16 v[60:63], a[140:143], a[124:127], v[60:63]\n"
    "ds_read_b128 a[12:15], v241 offset:2048\n"
    "s_waitcnt lgkmcnt(8)\n"
    "v_mfma_f32_16x16x32_bf16 v[64:67], a[144:147], a[64:67], v[64:67]\n"
    "v_mfma_f32_16x16x32_bf16 v[64:67], a[148:151], a[68:71], v[64:67]\n"
    "ds_read_b64_tr_b16 a[128:129], v242 offset:512\n"
    "v_mfma_f32_16x16x32_bf16 v[68:71], a[144:147], a[72:75], v[68:71]\n"
    "v_mfma_f32_16x16x32_bf16 v[68:71], a[148:151], a[76:79], v[68:71]\n"
    "ds_read_b64_tr_b16 a[130:131], v242 offset:1536\n"
    "v_mfma_f32_16x16x32_bf16 v[72:75], a[144:147], a[80:83], v[72:75]\n"
    "v_mfma_f32_16x16x32_bf16 v[72:75], a[148:151], a[84:87], v[72:75]\n"
    "ds_read_b64_tr_b16 a[132:133], v242 offset:8704\n"
    "v_mfma_f32_16x16x32_bf16 v[76:79], a[144:147], a[88:91], v[76:79]\n"
    "v_mfma_f32_16x16x32_bf16 v[76:79], a[148:151], a[92:95], v[76:79]\n"
    "ds_read_b64_tr_b16 a[134:135], v242 offset:9728\n"
    "v_mfma_f32_16x16x32_bf16 v[80:83], a[144:147], a[96:99], v[80:83]\n"
    "s_add_u32 m0, s58, 8192\n"
    "v_mfma_f32_16x16x32_bf16 v[80:83], a[148:151], a[100:103], v[80:83]\n"
    "buffer_load_dwordx4 v236, s[40:43], s54 offen nt lds\n"
    "v_mfma_f32_16x16x32_bf16 v[84:87], a[144:147], a[104:107], v[84:87]\n"
    "v_mfma_f32_16x16x32_bf16 v[84:87], a[148:151], a[108:111], v[84:87]\n"
    "ds_read_b128 a[16:19], v240 offset:4096\n"
    "v_mfma_f32_16x16x32_bf16 v[88:91], a[144:147], a[112:115], v[88:91]\n"
    "s_add_u32 m0, s58, 9216\n"
    "v_mfma_f32_16x16x32_bf16 v[88:91], a[148:151], a[116:119], v[88:91]\n"
    "buffer_load_dwordx4 v237, s[40:43], s54 offen nt lds\n"
    "v_mfma_f32_16x16x32_bf16 v[92:95], a[144:147], a[120:123], v[92:95]\n"
    "v_mfma_f32_16x16x32_bf16 v[92:95], a[148:151], a[124:127], v[92:95]\n"
    "ds_read_b128 a[20:23], v241 offset:4096\n"
    "s_waitcnt lgkmcnt(8)\n"
    "v_mfma_f32_16x16x32_bf16 v[96:99], a[152:155], a[64:67], v[96:99]\n"
    "v_mfma_f32_16x16x32_bf16 v[96:99], a[156:159], a[68:71], v[96:99]\n"
    "ds_read_b64_tr_b16 a[136:137], v243 offset:512\n"
    "v_mfma_f32_16x16x32_bf16 v[100:103], a[152:155], a[72:75], v[100:103]\n"
    "v_mfma_f32_16x16x32_bf16 v[100:103], a[156:159], a[76:79], v[100:103]\n"
    "ds_read_b64_tr_b16 a[138:139], v243 offset:1536\n"
    "v_mfma_f32_16x16x32_bf16 v[104:107], a[152:155], a[80:83], v[104:107]\n"
    "v_mfma_f32_16x16x32_bf16 v[104:107], a[156:159], a[84:87], v[104:107]\n"
    "ds_read_b64_tr_b16 a[140:141], v243 offset:8704\n"
    "v_mfma_f32_16x16x32_bf16 v[108:111], a[152:155], a[88:91], v[108:111]\n"
    "v_mfma_f32_16x16x32_bf16 v[108:111], a[156:159], a[92:95], v[108:111]\n"
    "ds_read_b64_tr_b16 a[142:143], v243 offset:9728\n"
    "v_mfma_f32_16x16x32_bf16 v[112:115], a[152:155], a[96:99], v[112:115]\n"
    "s_add_u32 m0, s58, 12288\n"
    "v_mfma_f32_16x16x32_bf16 v[112:115], a[156:159], a[100:103], v[112:115]\n"
    "buffer_load_dwordx4 v238, s[40:43], s54 offen nt lds\n"
    "v_mfma_f32_16x16x32_bf16 v[116:119], a[152:155], a[104:107], v[116:119]\n"
    "v_mfma_f32_16x16x32_bf16 v[116:119], a[156:159], a[108:111], v[116:119]\n"
    "ds_read_b128 a[24:27], v240 offset:6144\n"
    "v_mfma_f32_16x16x32_bf16 v[120:123], a[152:155], a[112:115], v[120:123]\n"
    "s_add_u32 m0, s58, 13312\n"
    "v_mfma_f32_16x16x32_bf16 v[120:123], a[156:159], a[116:119], v[120:123]\n"
    "buffer_load_dwordx4 v239, s[40:43], s54 offen nt lds\n"
    "v_mfma_f32_16x16x32_bf16 v[124:127], a[152:155], a[120:123], v[124:127]\n"
    "v_mfma_f32_16x16x32_bf16 v[124:127], a[156:159], a[124:127], v[124:127]\n"
    "ds_read_b128 a[28:31], v241 offset:6144\n"
    "s_waitcnt lgkmcnt(8)\n"
    "v_mfma_f32_16x16x32_bf16 v[128:131], a[128:131], a[64:67], v[128:131]\n"
    "v_mfma_f32_16x16x32_bf16 v[128:131], a[132:135], a[68:71], v[128:131]\n"
    "ds_read_b64_tr_b16 a[144:145], v242 offset:768\n"
    "v_mfma_f32_16x16x32_bf16 v[132:135], a[128:131], a[72:75], v[132:135]\n"
    "s_add_u32 s58, s58, 32768\n"
    "v_mfma_f32_16x16x32_bf16 v[132:135], a[132:135], a[76:79], v[132:135]\n"
    "ds_read_b64_tr_b16 a[146:147], v242 offset:1792\n"
    "v_mfma_f32_16x16x32_bf16 v[136:139], a[128:131], a[80:83], v[136:139]\n"
    "s_sub_u32 s59, s58, 98304\n"
    "v_mfma_f32_16x16x32_bf16 v[136:139], a[132:135], a[84:87], v[136:139]\n"
    "ds_read_b64_tr_b16 a[148:149], v242 offset:8960\n"
    "v_mfma_f32_16x16x32_bf16 v[140:143], a[128:131], a[88:91], v[140:143]\n"
    "s_cmp_ge_u32 s58, s67\n"
    "v_mfma_f32_16x16x32_bf16 v[140:143], a[132:135], a[92:95], v[140:143]\n"
    "ds_read_b64_tr_b16 a[150:151], v242 offset:9984\n"
    "v_mfma_f32_16x16x32_bf16 v[144:147], a[128:131], a[96:99], v[144:147]\n"
    "s_cselect_b32 s58, s59, s58\n"
    "v_mfma_f32_16x16x32_bf16 v[144:147], a[132:135], a[100:103], v[144:147]\n"
    "ds_read_b128 a[32:35], v240 offset:8192\n"
    "v_mfma_f32_16x16x32_bf16 v[148:151], a[128:131], a[104:107], v[148:151]\n"
    "v_mfma_f32_16x16x32_bf16 v[148:151], a[132:135], a[108:111], v[148:151]\n"
    "ds_read_b128 a[36:39], v241 offset:8192\n"
    "v_mfma_f32_16x16x32_bf16 v[152:155], a[128:131], a[112:115], v[152:155]\n"
    "v_mfma_f32_16x16x32_bf16 v[152:155], a[132:135], a[116:119], v[152:155]\n"
    "ds_read_b128 a[40:43], v240 offset:10240\n"
    "v_mfma_f32_16x16x32_bf16 v[156:159], a[128:131], a[120:123], v[156:159]\n"
    "v_mfma_f32_16x16x32_bf16 v[156:159], a[132:135], a[124:127], v[156:159]\n"
    "ds_read_b128 a[44:47], v241 offset:10240\n"
    "s_waitcnt lgkmcnt(10)\n"
    "v_mfma_f32_16x16x32_bf16 v[160:163], a[136:139], a[64:67], v[160:163]\n"
    "v_mfma_f32_16x16x32_bf16 v[160:163], a[140:143], a[68:71], v[160:163]\n"
    "ds_read_b64_tr_b16 a[152:153], v243 offset:768\n"
    "ds_read_b128 a[48:51], v240 offset:12288\n"
    "v_mfma_f32_16x16x32_bf16 v[164:167], a[136:139], a[72:75], v[164:167]\n"
    "s_add_u32 s59, s52, 3\n"
    "v_mfma_f32_16x16x32_bf16 v[164:167], a[140:143], a[76:79], v[164:167]\n"
    "ds_read_b64_tr_b16 a[154:155], v243 offset:1792\n"
    "ds_read_b128 a[52:55], v241 offset:12288\n"
    "v_readlane_b32 s53, v244, s59\n"
    "v_mfma_f32_16x16x32_bf16 v[168:171], a[136:139], a[80:83], v[168:171]\n"
    "s_cmp_eq_u32 s59, 8\n"
    "s_cselect_b32 s44, s48, s44\n"
    "v_mfma_f32_16x16x32_bf16 v[168:171], a[140:143], a[84:87], v[168:171]\n"
    "ds_read_b64_tr_b16 a[156:157], v243 offset:8960\n"
    "ds_read_b128 a[56:59], v240 offset:14336\n"
    "s_cselect_b32 s45, s49, s45\n"
    "v_mfma_f32_16x16x32_bf16 v[172:175], a[136:139], a[88:91], v[172:175]\n"
    "s_memrealtime s[34:35]\n"
    "v_mfma_f32_16x16x32_bf16 v[172:175], a[140:143], a[92:95], v[172:175]\n"
    "ds_read_b64_tr_b16 a[158:159], v243 offset:9984\n"
    "ds_read_b128 a[60:63], v241 offset:14336\n"
    "v_mfma_f32_16x16x32_bf16 v[176:179], a[136:139], a[96:99], v[176:179]\n"
    "s_add_u32 s57, s57, 1\n"
    "v_mfma_f32_16x16x32_bf16 v[176:179], a[140:143], a[100:103], v[176:179]\n"
    "s_cmp_eq_u32 s57, 3\n"
    "v_mfma_f32_16x16x32_bf16 v[180:183], a[136:139], a[104:107], v[180:183]\n"
    "s_cselect_b32 s57, 0, s57\n"
    "s_cselect_b32 s59, 4294901760, s68\n"
    "v_mfma_f32_16x16x32_bf16 v[180:183], a[140:143], a[108:111], v[180:183]\n"
    "v_mfma_f32_16x16x32_bf16 v[184:187], a[136:139], a[112:115], v[184:187]\n"
    "v_add_u32 v242, s59, v242\n"
    "v_mfma_f32_16x16x32_bf16 v[184:187], a[140:143], a[116:119], v[184:187]\n"
    "v_add_u32 v243, s59, v243\n"
    "v_mfma_f32_16x16x32_bf16 v[188:191], a[136:139], a[120:123], v[188:191]\n"
    "v_mfma_f32_16x16x32_bf16 v[188:191], a[140:143], a[124:127], v[188:191]\n"
    "s_waitcnt vmcnt(8) lgkmcnt(0)\n"
    "s_sub_u32 s34, s34, s36\n"
    "s_cmp_lt_i32 s34, -8\n"
    "s_cbranch_scc1 .Lg24_%=\n"
    ".Lg25_%=:\n"
    "s_barrier\n"
    "v_mfma_f32_16x16x32_bf16 v[192:195], a[144:147], a[64:67], v[192:195]\n"
    "v_cmp_le_u32 vcc, s63, v252\n"
    "v_mfma_f32_16x16x32_bf16 v[192:195], a[148:151], a[68:71], v[192:195]\n"
    "buffer_store_dword v254, v251, s[84:87], s61 offen sc0 sc1\n"
    "ds_read_b64_tr_b16 a[128:129], v242 offset:0\n"
    "v_mfma_f32_16x16x32_bf16 v[196:199], a[144:147], a[72:75], v[196:199]\n"
    "s_or_b64 vcc, vcc, s[78:79]\n"
    "s_cmp_eq_u64 vcc, -1\n"
    "v_mfma_f32_16x16x32_bf16 v[196:199], a[148:151], a[76:79], v[196:199]\n"
    "s_cbranch_scc0 .Lg21_%=\n"
    ".Lg22_%=:\n"
    "ds_read_b64_tr_b16 a[130:131], v242 offset:1024\n"
    "v_mfma_f32_16x16x32_bf16 v[200:203], a[144:147], a[80:83], v[200:203]\n"
    "v_mfma_f32_16x16x32_bf16 v[200:203], a[148:151], a[84:87], v[200:203]\n"
    "ds_read_b64_tr_b16 a[132:133], v242 offset:8192\n"
    "v_mfma_f32_16x16x32_bf16 v[204:207], a[144:147], a[88:91], v[204:207]\n"
    "v_mfma_f32_16x16x32_bf16 v[204:207], a[148:151], a[92:95], v[204:207]\n"
    "ds_read_b64_tr_b16 a[134:135], v242 offset:9216\n"
    "v_mfma_f32_16x16x32_bf16 v[208:211], a[144:147], a[96:99], v[208:211]\n"
    "s_add_u32 m0, s55, 0\n"
    "v_mfma_f32_16x16x32_bf16 v[208:211], a[148:151], a[100:103], v[208:211]\n"
    "buffer_load_dwordx4 v224, s[44:47], s53 offen sc1 lds\n"
    "v_mfma_f32_16x16x32_bf16 v[212:215], a[144:147], a[104:107], v[212:215]\n"
    "s_add_u32 m0, s55, 1024\n"
    "v_mfma_f32_16x16x32_bf16 v[212:215], a[148:151], a[108:111], v[212:215]\n"
    "buffer_load_dwordx4 v225, s[44:47], s53 offen sc1 lds\n"
    "v_mfma_f32_16x16x32_bf16 v[216:219], a[144:147], a[112:115], v[216:219]\n"
    "s_add_u32 m0, s55, 2048\n"
    "v_mfma_f32_16x16x32_bf16 v[216:219], a[148:151], a[116:119], v[216:219]\n"
    "buffer_load_dwordx4 v226, s[44:47], s53 offen sc1 lds\n"
    "v_mfma_f32_16x16x32_bf16 v[220:223], a[144:147], a[120:123], v[220:223]\n"
    "s_add_u32 m0, s55, 3072\n"
    "v_mfma_f32_16x16x32_bf16 v[220:223], a[148:151], a[124:127], v[220:223]\n"
    "buffer_load_dwordx4 v227, s[44:47], s53 offen sc1 lds\n"
    "s_waitcnt lgkmcnt(5)\n"
    "v_mfma_f32_16x16x32_bf16 a[160:163], a[152:155], a[64:67], a[160:163]\n"
    "v_mfma_f32_16x16x32_bf16 a[160:163], a[156:159], a[68:71], a[160:163]\n"
    "ds_read_b64_tr_b16 a[136:137], v243 offset:0\n"
    "v_mfma_f32_16x16x32_bf16 a[164:167], a[152:155], a[72:75], a[164:167]\n"
    "v_mfma_f32_16x16x32_bf16 a[164:167], a[156:159], a[76:79], a[164:167]\n"
    "ds_read_b64_tr_b16 a[138:139], v243 offset:1024\n"
    "v_mfma_f32_16x16x32_bf16 a[168:171], a[152:155], a[80:83], a[168:171]\n"
    "v_mfma_f32_16x16x32_bf16 a[168:171], a[156:159], a[84:87], a[168:171]\n"
    "ds_read_b64_tr_b16 a[140:141], v243 offset:8192\n"
    "v_mfma_f32_16x16x32_bf16 a[172:175], a[152:155], a[88:91], a[172:175]\n"
    "v_mfma_f32_16x16x32_bf16 a[172:175], a[156:159], a[92:95], a[172:175]\n"
    "ds_read_b64_tr_b16 a[142:143], v243 offset:9216\n"
    "v_mfma_f32_16x16x32_bf16 a[176:179], a[152:155], a[96:99], a[176:179]\n"
    "s_add_u32 m0, s55, 4096\n"
    "v_mfma_f32_16x16x32_bf16 a[176:179], a[156:159], a[100:103], a[176:179]\n"
    "buffer_load_dwordx4 v228, s[44:47], s53 offen sc1 lds\n"
    "v_mfma_f32_16x16x32_bf16 a[180:183], a[152:155], a[104:107], a[180:183]\n"
    "s_add_u32 m0, s55, 5120\n"
    "v_mfma_f32_16x16x32_bf16 a[180:183], a[156:159], a[108:111], a[180:183]\n"
    "buffer_load_dwordx4 v229, s[44:47], s53 offen sc1 lds\n"
    "v_mfma_f32_16x16x32_bf16 a[184:187], a[152:155], a[112:115], a[184:187]\n"
    "s_add_u32 m0, s55, 6144\n"
    "v_mfma_f32_16x16x32_bf16 a[184:187], a[156:159], a[116:119], a[184:187]\n"
    "buffer_load_dwordx4 v230, s[44:47], s53 offen sc1 lds\n"
    "v_mfma_f32_16x16x32_bf16 a[188:191], a[152:155], a[120:123], a[188:191]\n"
    "s_add_u32 m0, s55, 7168\n"
    "v_mfma_f32_16x16x32_bf16 a[188:191], a[156:159], a[124:127], a[188:191]\n"
    "buffer_load_dwordx4 v231, s[44:47], s53 offen sc1 lds\n"
    "s_add_u32 s52, s52, 1\n"
    "s_sub_u32 s60, s60, 1\n"
    "s_cmp_lg_u32 s60, 0\n"
    "s_cbranch_scc1 .Lg14_%=\n"
    "s_getpc_b64 s[80:81]\n"
    ".Lg27_%=:\n"
    "s_add_u32 s80, s80, .Lg26_%=-.Lg27_%=\n"
    "s_addc_u32 s81, s81, 0\n"
    "s_and_b32 s81, s81, 0xffff\n"
    "s_mov_b32 s82, 24576\n"
    "v_mbcnt_lo_u32_b32 v253, -1, 0\n"
    "v_mbcnt_hi_u32_b32 v253, -1, v253\n"
    "v_lshlrev_b32 v253, 7, v253\n"
    "buffer_load_dword a224, v253, s[80:83], s55 offen\n"
    "s_mov_b32 s60, 21\n"
    ".Lg28_%=:\n"
    "s_waitcnt lgkmcnt(4)\n"
    "v_mfma_f32_16x16x32_bf16 v[0:3], a[128:131], a[0:3], v[0:3]\n"
    "s_add_u32 s59, s52, 2\n"
    "v_mfma_f32_16x16x32_bf16 v[0:3], a[132:135], a[4:7], v[0:3]\n"
    "ds_read_b64_tr_b16 a[144:145], v242 offset:256\n"
    "v_readlane_b32 s54, v245, s59\n"
    "v_mfma_f32_16x16x32_bf16 v[4:7], a[128:131], a[8:11], v[4:7]\n"
    "v_mfma_f32_16x16x32_bf16 v[4:7], a[132:135], a[12:15], v[4:7]\n"
    "ds_read_b64_tr_b16 a[146:147], v242 offset:1280\n"
    "v_mfma_f32_16x16x32_bf16 v[8:11], a[128:131], a[16:19], v[8:11]\n"
    "v_mfma_f32_16x16x32_bf16 v[8:11], a[132:135], a[20:23], v[8:11]\n"
    "ds_read_b64_tr_b16 a[148:149], v242 offset:8448\n"
    "v_mfma_f32_16x16x32_bf16 v[12:15], a[128:131], a[24:27], v[12:15]\n"
    "v_mfma_f32_16x16x32_bf16 v[12:15], a[132:135], a[28:31], v[12:15]\n"
    "ds_read_b64_tr_b16 a[150:151], v242 offset:9472\n"
    "v_mfma_f32_16x16x32_bf16 v[16:19], a[128:131], a[32:35], v[16:19]\n"
    "s_add_u32 m0, s58, 0\n"
    "v_mfma_f32_16x16x32_bf16 v[16:19], a[132:135], a[36:39], v[16:19]\n"
    "buffer_load_dwordx4 v232, s[40:43], s54 offen nt lds\n"
    "v_mfma_f32_16x16x32_bf16 v[20:23], a[128:131], a[40:43], v[20:23]\n"
    "v_mfma_f32_16x16x32_bf16 v[20:23], a[132:135], a[44:47], v[20:23]\n"
    "ds_read_b128 a[64:67], v240 offset:32768\n"
    "v_mfma_f32_16x16x32_bf16 v[24:27], a[128:131], a[48:51], v[24:27]\n"
    "s_add_u32 m0, s58, 1024\n"
    "v_mfma_f32_16x16x32_bf16 v[24:27], a[132:135], a[52:55], v[24:27]\n"
    "buffer_load_dwordx4 v233, s[40:43], s54 offen nt lds\n"
    "v_mfma_f32_16x16x32_bf16 v[28:31], a[128:131], a[56:59], v[28:31]\n"
    "v_mfma_f32_16x16x32_bf16 v[28:31], a[132:135], a[60:63], v[28:31]\n"
    "ds_read_b128 a[68:71], v241 offset:32768\n"
    "s_waitcnt lgkmcnt(6)\n"
    "v_mfma_f32_16x16x32_bf16 v[32:35], a[136:139], a[0:3], v[32:35]\n"
    "v_mfma_f32_16x16x32_bf16 v[32:35], a[140:143], a[4:7], v[32:35]\n"
    "ds_read_b64_tr_b16 a[152:153], v243 offset:256\n"
    "v_mfma_f32_16x16x32_bf16 v[36:39], a[136:139], a[8:11], v[36:39]\n"
    "v_mfma_f32_16x16x32_bf16 v[36:39], a[140:143], a[12:15], v[36:39]\n"
    "ds_read_b64_tr_b16 a[154:155], v243 offset:1280\n"
    "v_mfma_f32_16x16x32_bf16 v[40:43], a[136:139], a[16:19], v[40:43]\n"
    "v_mfma_f32_16x16x32_bf16 v[40:43], a[140:143], a[20:23], v[40:43]\n"
    "ds_read_b64_tr_b16 a[156:157], v243 offset:8448\n"
    "v_mfma_f32_16x16x32_bf16 v[44:47], a[136:139], a[24:27], v[44:47]\n"
    "v_mfma_f32_16x16x32_bf16 v[44:47], a[140:143], a[28:31], v[44:47]\n"
    "ds_read_b64_tr_b16 a[158:159], v243 offset:9472\n"
    "v_mfma_f32_16x16x32_bf16 v[48:51], a[136:139], a[32:35], v[48:51]\n"
    "s_add_u32 m0, s58, 4096\n"
    "v_mfma_f32_16x16x32_bf16 v[48:51], a[140:143], a[36:39], v[48:51]\n"
    "buffer_load_dwordx4 v234, s[40:43], s54 offen nt lds\n"
    "v_mfma_f32_16x16x32_bf16 v[52:55], a[136:139], a[40:43], v[52:55]\n"
    "v_mfma_f32_16x16x32_bf16 v[52:55], a[140:143], a[44:47], v[52:55]\n"
    "ds_read_b128 a[72:75], v240 offset:34816\n"
    "v_mfma_f32_16x16x32_bf16 v[56:59], a[136:139], a[48:51], v[56:59]\n"
    "s_add_u32 m0, s58, 5120\n"
    "v_mfma_f32_16x16x32_bf16 v[56:59], a[140:143], a[52:55], v[56:59]\n"
    "buffer_load_dwordx4 v235, s[40:43], s54 offen nt lds\n"
    "v_mfma_f32_16x16x32_bf16 v[60:63], a[136:139], a[56:59], v[60:63]\n"
    "v_mfma_f32_16x16x32_bf16 v[60:63], a[140:143], a[60:63], v[60:63]\n"
    "ds_read_b128 a[76:79], v241 offset:34816\n"
    "s_waitcnt lgkmcnt(8)\n"
    "v_mfma_f32_16x16x32_bf16 v[64:67], a[144:147], a[0:3], v[64:67]\n"
    "v_mfma_f32_16x16x32_bf16 v[64:67], a[148:151], a[4:7], v[64:67]\n"
    "ds_read_b64_tr_b16 a[128:129], v242 offset:512\n"
    "v_mfma_f32_16x16x32_bf16 v[68:71], a[144:147], a[8:11], v[68:71]\n"
    "v_mfma_f32_16x16x32_bf16 v[68:71], a[148:151], a[12:15], v[68:71]\n"
    "ds_read_b64_tr_b16 a[130:131], v242 offset:1536\n"
    "v_mfma_f32_16x16x32_bf16 v[72:75], a[144:147], a[16:19], v[72:75]\n"
    "v_mfma_f32_16x16x32_bf16 v[72:75], a[148:151], a[20:23], v[72:75]\n"
    "ds_read_b64_tr_b16 a[132:133], v242 offset:8704\n"
    "v_mfma_f32_16x16x32_bf16 v[76:79], a[144:147], a[24:27], v[76:79]\n"
    "v_mfma_f32_16x16x32_bf16 v[76:79], a[148:151], a[28:31], v[76:79]\n"
    "ds_read_b64_tr_b16 a[134:135], v242 offset:9728\n"
    "v_mfma_f32_16x16x32_bf16 v[80:83], a[144:147], a[32:35], v[80:83]\n"
    "s_add_u32 m0, s58, 8192\n"
    "v_mfma_f32_16x16x32_bf16 v[80:83], a[148:151], a[36:39], v[80:83]\n"
    "buffer_load_dwordx4 v236, s[40:43], s54 offen nt lds\n"
    "v_mfma_f32_16x16x32_bf16 v[84:87], a[144:147], a[40:43], v[84:87]\n"
    "v_mfma_f32_16x16x32_bf16 v[84:87], a[148:151], a[44:47], v[84:87]\n"
    "ds_read_b128 a[80:83], v240 offset:36864\n"
    "v_mfma_f32_16x16x32_bf16 v[88:91], a[144:147], a[48:51], v[88:91]\n"
    "s_add_u32 m0, s58, 9216\n"
    "v_mfma_f32_16x16x32_bf16 v[88:91], a[148:151], a[52:55], v[88:91]\n"
    "buffer_load_dwordx4 v237, s[40:43], s54 offen nt lds\n"
    "v_mfma_f32_16x16x32_bf16 v[92:95], a[144:147], a[56:59], v[92:95]\n"
    "v_mfma_f32_16x16x32_bf16 v[92:95], a[148:151], a[60:63], v[92:95]\n"
    "ds_read_b128 a[84:87], v241 offset:36864\n"
    "s_waitcnt lgkmcnt(8)\n"
    "v_mfma_f32_16x16x32_bf16 v[96:99], a[152:155], a[0:3], v[96:99]\n"
    "v_mfma_f32_16x16x32_bf16 v[96:99], a[156:159], a[4:7], v[96:99]\n"
    "ds_read_b64_tr_b16 a[136:137], v243 offset:512\n"
    "v_mfma_f32_16x16x32_bf16 v[100:103], a[152:155], a[8:11], v[100:103]\n"
    "v_mfma_f32_16x16x32_bf16 v[100:103], a[156:159], a[12:15], v[100:103]\n"
    "ds_read_b64_tr_b16 a[138:139], v243 offset:1536\n"
    "v_mfma_f32_16x16x32_bf16 v[104:107], a[152:155], a[16:19], v[104:107]\n"
    "v_mfma_f32_16x16x32_bf16 v[104:107], a[156:159], a[20:23], v[104:107]\n"
    "ds_read_b64_tr_b16 a[140:141], v243 offset:8704\n"
    "v_mfma_f32_16x16x32_bf16 v[108:111], a[152:155], a[24:27], v[108:111]\n"
    "v_mfma_f32_16x16x32_bf16 v[108:111], a[156:159], a[28:31], v[108:111]\n"
    "ds_read_b64_tr_b16 a[142:143], v243 offset:9728\n"
    "v_mfma_f32_16x16x32_bf16 v[112:115], a[152:155], a[32:35], v[112:115]\n"
    "s_add_u32 m0, s58, 12288\n"
    "v_mfma_f32_16x16x32_bf16 v[112:115], a[156:159], a[36:39], v[112:115]\n"
    "buffer_load_dwordx4 v238, s[40:43], s54 offen nt lds\n"
    "v_mfma_f32_16x16x32_bf16 v[116:119], a[152:155], a[40:43], v[116:119]\n"
    "v_mfma_f32_16x16x32_bf16 v[116:119], a[156:159], a[44:47], v[116:119]\n"
    "ds_read_b128 a[88:91], v240 offset:38912\n"
    "v_mfma_f32_16x16x32_bf16 v[120:123], a[152:155], a[48:51], v[120:123]\n"
    "s_add_u32 m0, s58, 13312\n"
    "v_mfma_f32_16x16x32_bf16 v[120:123], a[156:159], a[52:55], v[120:123]\n"
    "buffer_load_dwordx4 v239, s[40:43], s54 offen nt lds\n"
    "v_mfma_f32_16x16x32_bf16 v[124:127], a[152:155], a[56:59], v[124:127]\n"
    "v_mfma_f32_16x16x32_bf16 v[124:127], a[156:159], a[60:63], v[124:127]\n"
    "ds_read_b128 a[92:95], v241 offset:38912\n"
    "s_waitcnt lgkmcnt(8)\n"
    "v_mfma_f32_16x16x32_bf16 v[128:131], a[128:131], a[0:3], v[128:131]\n"
    "v_mfma_f32_16x16x32_bf16 v[128:131], a[132:135], a[4:7], v[128:131]\n"
    "ds_read_b64_tr_b16 a[144:145], v242 offset:768\n"
    "v_mfma_f32_16x16x32_bf16 v[132:135], a[128:131], a[8:11], v[132:135]\n"
    "s_add_u32 s58, s58, 32768\n"
    "v_mfma_f32_16x16x32_bf16 v[132:135], a[132:135], a[12:15], v[132:135]\n"
    "ds_read_b64_tr_b16 a[146:147], v242 offset:1792\n"
    "v_mfma_f32_16x16x32_bf16 v[136:139], a[128:131], a[16:19], v[136:139]\n"
    "s_sub_u32 s59, s58, 98304\n"
    "v_mfma_f32_16x16x32_bf16 v[136:139], a[132:135], a[20:23], v[136:139]\n"
    "ds_read_b64_tr_b16 a[148:149], v242 offset:8960\n"
    "v_mfma_f32_16x16x32_bf16 v[140:143], a[128:131], a[24:27], v[140:143]\n"
    "s_cmp_ge_u32 s58, s67\n"
    "v_mfma_f32_16x16x32_bf16 v[140:143], a[132:135], a[28:31], v[140:143]\n"
    "ds_read_b64_tr_b16 a[150:151], v242 offset:9984\n"
    "v_mfma_f32_16x16x32_bf16 v[144:147], a[128:131], a[32:35], v[144:147]\n"
    "s_cselect_b32 s58, s59, s58\n"
    "v_mfma_f32_16x16x32_bf16 v[144:147], a[132:135], a[36:39], v[144:147]\n"
    "ds_read_b128 a[96:99], v240 offset:40960\n"
    "v_mfma_f32_16x16x32_bf16 v[148:151], a[128:131], a[40:43], v[148:151]\n"
    "v_mfma_f32_16x16x32_bf16 v[148:151], a[132:135], a[44:47], v[148:151]\n"
    "ds_read_b128 a[100:103], v241 offset:40960\n"
    "v_mfma_f32_16x16x32_bf16 v[152:155], a[128:131], a[48:51], v[152:155]\n"
    "v_mfma_f32_16x16x32_bf16 v[152:155], a[132:135], a[52:55], v[152:155]\n"
    "ds_read_b128 a[104:107], v240 offset:43008\n"
    "v_mfma_f32_16x16x32_bf16 v[156:159], a[128:131], a[56:59], v[156:159]\n"
    "v_mfma_f32_16x16x32_bf16 v[156:159], a[132:135], a[60:63], v[156:159]\n"
    "ds_read_b128 a[108:111], v241 offset:43008\n"
    "s_waitcnt lgkmcnt(10)\n"
    "v_mfma_f32_16x16x32_bf16 v[160:163], a[136:139], a[0:3], v[160:163]\n"
    "v_mfma_f32_16x16x32_bf16 v[160:163], a[140:143], a[4:7], v[160:163]\n"
    "ds_read_b64_tr_b16 a[152:153], v243 offset:768\n"
    "ds_read_b128 a[112:115], v240 offset:45056\n"
    "v_mfma_f32_16x16x32_bf16 v[164:167], a[136:139], a[8:11], v[164:167]\n"
    "s_add_u32 s59, s52, 3\n"
    "v_mfma_f32_16x16x32_bf16 v[164:167], a[140:143], a[12:15], v[164:167]\n"
    "ds_read_b64_tr_b16 a[154:155], v243 offset:1792\n"
    "ds_read_b128 a[116:119], v241 offset:45056\n"
    "v_readlane_b32 s53, v244, s59\n"
    "v_mfma_f32_16x16x32_bf16 v[168:171], a[136:139], a[16:19], v[168:171]\n"
    "v_mfma_f32_16x16x32_bf16 v[168:171], a[140:143], a[20:23], v[168:171]\n"
    "ds_read_b64_tr_b16 a[156:157], v243 offset:8960\n"
    "ds_read_b128 a[120:123], v240 offset:47104\n"
    "v_mfma_f32_16x16x32_bf16 v[172:175], a[136:139], a[24:27], v[172:175]\n"
    "v_mfma_f32_16x16x32_bf16 v[172:175], a[140:143], a[28:31], v[172:175]\n"
    "ds_read_b64_tr_b16 a[158:159], v243 offset:9984\n"
    "ds_read_b128 a[124:127], v241 offset:47104\n"
    "v_mfma_f32_16x16x32_bf16 v[176:179], a[136:139], a[32:35], v[176:179]\n"
    "s_add_u32 s57, s57, 1\n"
    "v_mfma_f32_16x16x32_bf16 v[176:179], a[140:143], a[36:39], v[176:179]\n"
    "s_cmp_eq_u32 s57, 3\n"
    "v_mfma_f32_16x16x32_bf16 v[180:183], a[136:139], a[40:43], v[180:183]\n"
    "s_cselect_b32 s57, 0, s57\n"
    "s_cselect_b32 s59, 4294901760, s68\n"
    "v_mfma_f32_16x16x32_bf16 v[180:183], a[140:143], a[44:47], v[180:183]\n"
    "v_mfma_f32_16x16x32_bf16 v[184:187], a[136:139], a[48:51], v[184:187]\n"
    "v_add_u32 v242, s59, v242\n"
    "v_mfma_f32_16x16x32_bf16 v[184:187], a[140:143], a[52:55], v[184:187]\n"
    "v_add_u32 v243, s59, v243\n"
    "v_mfma_f32_16x16x32_bf16 v[188:191], a[136:139], a[56:59], v[188:191]\n"
    "v_mfma_f32_16x16x32_bf16 v[188:191], a[140:143], a[60:63], v[188:191]\n"
    "s_waitcnt vmcnt(8) lgkmcnt(0)\n"
    "s_barrier\n"
    "v_mfma_f32_16x16x32_bf16 v[192:195], a[144:147], a[0:3], v[192:195]\n"
    "v_mfma_f32_16x16x32_bf16 v[192:195], a[148:151], a[4:7], v[192:195]\n"
    "ds_read_b64_tr_b16 a[128:129], v242 offset:0\n"
    "v_mfma_f32_16x16x32_bf16 v[196:199], a[144:147], a[8:11], v[196:199]\n"
    "v_mfma_f32_16x16x32_bf16 v[196:199], a[148:151], a[12:15], v[196:199]\n"
    "ds_read_b64_tr_b16 a[130:131], v242 offset:1024\n"
    "v_mfma_f32_16x16x32_bf16 v[200:203], a[144:147], a[16:19], v[200:203]\n"
    "v_mfma_f32_16x16x32_bf16 v[200:203], a[148:151], a[20:23], v[200:203]\n"
    "ds_read_b64_tr_b16 a[132:133], v242 offset:8192\n"
    "v_mfma_f32_16x16x32_bf16 v[204:207], a[144:147], a[24:27], v[204:207]\n"
    "v_mfma_f32_16x16x32_bf16 v[204:207], a[148:151], a[28:31], v[204:207]\n"
    "ds_read_b64_tr_b16 a[134:135], v242 offset:9216\n"
    "v_mfma_f32_16x16x32_bf16 v[208:211], a[144:147], a[32:35], v[208:211]\n"
    "s_add_u32 m0, s55, 32768\n"
    "v_mfma_f32_16x16x32_bf16 v[208:211], a[148:151], a[36:39], v[208:211]\n"
    "buffer_load_dwordx4 v224, s[44:47], s53 offen sc1 lds\n"
    "v_mfma_f32_16x16x32_bf16 v[212:215], a[144:147], a[40:43], v[212:215]\n"
    "s_add_u32 m0, s55, 33792\n"
    "v_mfma_f32_16x16x32_bf16 v[212:215], a[148:151], a[44:47], v[212:215]\n"
    "buffer_load_dwordx4 v225, s[44:47], s53 offen sc1 lds\n"
    "v_mfma_f32_16x16x32_bf16 v[216:219], a[144:147], a[48:51], v[216:219]\n"
    "s_add_u32 m0, s55, 34816\n"
    "v_mfma_f32_16x16x32_bf16 v[216:219], a[148:151], a[52:55], v[216:219]\n"
    "buffer_load_dwordx4 v226, s[44:47], s53 offen sc1 lds\n"
    "v_mfma_f32_16x16x32_bf16 v[220:223], a[144:147], a[56:59], v[220:223]\n"
    "s_add_u32 m0, s55, 35840\n"
    "v_mfma_f32_16x16x32_bf16 v[220:223], a[148:151], a[60:63], v[220:223]\n"
    "buffer_load_dwordx4 v227, s[44:47], s53 offen sc1 lds\n"
    "s_waitcnt lgkmcnt(5)\n"
    "v_mfma_f32_16x16x32_bf16 a[160:163], a[152:155], a[0:3], a[160:163]\n"
    "v_mfma_f32_16x16x32_bf16 a[160:163], a[156:159], a[4:7], a[160:163]\n"
    "ds_read_b64_tr_b16 a[136:137], v243 offset:0\n"
    "v_mfma_f32_16x16x32_bf16 a[164:167], a[152:155], a[8:11], a[164:167]\n"
    "v_mfma_f32_16x16x32_bf16 a[164:167], a[156:159], a[12:15], a[164:167]\n"
    "ds_read_b64_tr_b16 a[138:139], v243 offset:1024\n"
    "v_mfma_f32_16x16x32_bf16 a[168:171], a[152:155], a[16:19], a[168:171]\n"
    "v_mfma_f32_16x16x32_bf16 a[168:171], a[156:159], a[20:23], a[168:171]\n"
    "ds_read_b64_tr_b16 a[140:141], v243 offset:8192\n"
    "v_mfma_f32_16x16x32_bf16 a[172:175], a[152:155], a[24:27], a[172:175]\n"
    "v_mfma_f32_16x16x32_bf16 a[172:175], a[156:159], a[28:31], a[172:175]\n"
    "ds_read_b64_tr_b16 a[142:143], v243 offset:9216\n"
    "v_mfma_f32_16x16x32_bf16 a[176:179], a[152:155], a[32:35], a[176:179]\n"
    "s_add_u32 m0, s55, 36864\n"
    "v_mfma_f32_16x16x32_bf16 a[176:179], a[156:159], a[36:39], a[176:179]\n"
    "buffer_load_dwordx4 v228, s[44:47], s53 offen sc1 lds\n"
    "v_mfma_f32_16x16x32_bf16 a[180:183], a[152:155], a[40:43], a[180:183]\n"
    "s_add_u32 m0, s55, 37888\n"
    "v_mfma_f32_16x16x32_bf16 a[180:183], a[156:159], a[44:47], a[180:183]\n"
    "buffer_load_dwordx4 v229, s[44:47], s53 offen sc1 lds\n"
    "v_mfma_f32_16x16x32_bf16 a[184:187], a[152:155], a[48:51], a[184:187]\n"
    "s_add_u32 m0, s55, 38912\n"
    "v_mfma_f32_16x16x32_bf16 a[184:187], a[156:159], a[52:55], a[184:187]\n"
    "buffer_load_dwordx4 v230, s[44:47], s53 offen sc1 lds\n"
    "v_mfma_f32_16x16x32_bf16 a[188:191], a[152:155], a[56:59], a[188:191]\n"
    "s_add_u32 m0, s55, 39936\n"
    "v_mfma_f32_16x16x32_bf16 a[188:191], a[156:159], a[60:63], a[188:191]\n"
    "buffer_load_dwordx4 v231, s[44:47], s53 offen sc1 lds\n"
    "s_add_u32 s52, s52, 1\n"
    "s_waitcnt lgkmcnt(4)\n"
    "v_mfma_f32_16x16x32_bf16 v[0:3], a[128:131], a[64:67], v[0:3]\n"
    "s_add_u32 s59, s52, 2\n"
    "v_mfma_f32_16x16x32_bf16 v[0:3], a[132:135], a[68:71], v[0:3]\n"
    "ds_read_b64_tr_b16 a[144:145], v242 offset:256\n"
    "v_readlane_b32 s54, v245, s59\n"
    "v_mfma_f32_16x16x32_bf16 v[4:7], a[128:131], a[72:75], v[4:7]\n"
    "v_mfma_f32_16x16x32_bf16 v[4:7], a[132:135], a[76:79], v[4:7]\n"
    "ds_read_b64_tr_b16 a[146:147], v242 offset:1280\n"
    "v_mfma_f32_16x16x32_bf16 v[8:11], a[128:131], a[80:83], v[8:11]\n"
    "buffer_load_dword v252, v246, s[88:91], 0 offen sc1\n"
    "v_mfma_f32_16x16x32_bf16 v[8:11], a[132:135], a[84:87], v[8:11]\n"
    "ds_read_b64_tr_b16 a[148:149], v242 offset:8448\n"
    "v_mfma_f32_16x16x32_bf16 v[12:15], a[128:131], a[88:91], v[12:15]\n"
    "v_mfma_f32_16x16x32_bf16 v[12:15], a[132:135], a[92:95], v[12:15]\n"
    "ds_read_b64_tr_b16 a[150:151], v242 offset:9472\n"
    "v_mfma_f32_16x16x32_bf16 v[16:19], a[128:131], a[96:99], v[16:19]\n"
    "s_add_u32 m0, s58, 0\n"
    "v_mfma_f32_16x16x32_bf16 v[16:19], a[132:135], a[100:103], v[16:19]\n"
    "buffer_load_dwordx4 v232, s[40:43], s54 offen nt lds\n"
    "v_mfma_f32_16x16x32_bf16 v[20:23], a[128:131], a[104:107], v[20:23]\n"
    "v_readlane_b32 s62, v247, s52\n"
    "v_mfma_f32_16x16x32_bf16 v[20:23], a[132:135], a[108:111], v[20:23]\n"
    "ds_read_b128 a[0:3], v240 offset:0\n"
    "v_mfma_f32_16x16x32_bf16 v[24:27], a[128:131], a[112:115], v[24:27]\n"
    "s_lshl_b64 s[78:79], -1, s62\n"
    "s_add_u32 m0, s58, 1024\n"
    "v_mfma_f32_16x16x32_bf16 v[24:27], a[132:135], a[116:119], v[24:27]\n"
    "buffer_load_dwordx4 v233, s[40:43], s54 offen nt lds\n"
    "v_mfma_f32_16x16x32_bf16 v[28:31], a[128:131], a[120:123], v[28:31]\n"
    "v_mfma_f32_16x16x32_bf16 v[28:31], a[132:135], a[124:127], v[28:31]\n"
    "ds_read_b128 a[4:7], v241 offset:0\n"
    "s_waitcnt lgkmcnt(6)\n"
    "v_mfma_f32_16x16x32_bf16 v[32:35], a[136:139], a[64:67], v[32:35]\n"
    "v_mfma_f32_16x16x32_bf16 v[32:35], a[140:143], a[68:71], v[32:35]\n"
    "ds_read_b64_tr_b16 a[152:153], v243 offset:256\n"
    "v_mfma_f32_16x16x32_bf16 v[36:39], a[136:139], a[72:75], v[36:39]\n"
    "v_mfma_f32_16x16x32_bf16 v[36:39], a[140:143], a[76:79], v[36:39]\n"
    "ds_read_b64_tr_b16 a[154:155], v243 offset:1280\n"
    "v_mfma_f32_16x16x32_bf16 v[40:43], a[136:139], a[80:83], v[40:43]\n"
    "v_mfma_f32_16x16x32_bf16 v[40:43], a[140:143], a[84:87], v[40:43]\n"
    "ds_read_b64_tr_b16 a[156:157], v243 offset:8448\n"
    "v_mfma_f32_16x16x32_bf16 v[44:47], a[136:139], a[88:91], v[44:47]\n"
    "v_mfma_f32_16x16x32_bf16 v[44:47], a[140:143], a[92:95], v[44:47]\n"
    "ds_read_b64_tr_b16 a[158:159], v243 offset:9472\n"
    "v_mfma_f32_16x16x32_bf16 v[48:51], a[136:139], a[96:99], v[48:51]\n"
    "s_add_u32 m0, s58, 4096\n"
    "v_mfma_f32_16x16x32_bf16 v[48:51], a[140:143], a[100:103], v[48:51]\n"
    "buffer_load_dwordx4 v234, s[40:43], s54 offen nt lds\n"
    "v_mfma_f32_16x16x32_bf16 v[52:55], a[136:139], a[104:107], v[52:55]\n"
    "v_mfma_f32_16x16x32_bf16 v[52:55], a[140:143], a[108:111], v[52:55]\n"
    "ds_read_b128 a[8:11], v240 offset:2048\n"
    "v_mfma_f32_16x16x32_bf16 v[56:59], a[136:139], a[112:115], v[56:59]\n"
    "s_add_u32 m0, s58, 5120\n"
    "v_mfma_f32_16x16x32_bf16 v[56:59], a[140:143], a[116:119], v[56:59]\n"
    "buffer_load_dwordx4 v235, s[40:43], s54 offen nt lds\n"
    "v_mfma_f32_16x16x32_bf16 v[60:63], a[136:139], a[120:123], v[60:63]\n"
    "v_mfma_f32_16x16x32_bf16 v[60:63], a[140:143], a[124:127], v[60:63]\n"
    "ds_read_b128 a[12:15], v241 offset:2048\n"
    "s_waitcnt lgkmcnt(8)\n"
    "v_mfma_f32_16x16x32_bf16 v[64:67], a[144:147], a[64:67], v[64:67]\n"
    "v_mfma_f32_16x16x32_bf16 v[64:67], a[148:151], a[68:71], v[64:67]\n"
    "ds_read_b64_tr_b16 a[128:129], v242 offset:512\n"
    "v_mfma_f32_16x16x32_bf16 v[68:71], a[144:147], a[72:75], v[68:71]\n"
    "v_mfma_f32_16x16x32_bf16 v[68:71], a[148:151], a[76:79], v[68:71]\n"
    "ds_read_b64_tr_b16 a[130:131], v242 offset:1536\n"
    "v_mfma_f32_16x16x32_bf16 v[72:75], a[144:147], a[80:83], v[72:75]\n"
    "v_mfma_f32_16x16x32_bf16 v[72:75], a[148:151], a[84:87], v[72:75]\n"
    "ds_read_b64_tr_b16 a[132:133], v242 offset:8704\n"
    "v_mfma_f32_16x16x32_bf16 v[76:79], a[144:147], a[88:91], v[76:79]\n"
    "v_mfma_f32_16x16x32_bf16 v[76:79], a[148:151], a[92:95], v[76:79]\n"
    "ds_read_b64_tr_b16 a[134:135], v242 offset:9728\n"
    "v_mfma_f32_16x16x32_bf16 v[80:83], a[144:147], a[96:99], v[80:83]\n"
    "s_add_u32 m0, s58, 8192\n"
    "v_mfma_f32_16x16x32_bf16 v[80:83], a[148:151], a[100:103], v[80:83]\n"
    "buffer_load_dwordx4 v236, s[40:43], s54 offen nt lds\n"
    "v_mfma_f32_16x16x32_bf16 v[84:87], a[144:147], a[104:107], v[84:87]\n"
    "v_mfma_f32_16x16x32_bf16 v[84:87], a[148:151], a[108:111], v[84:87]\n"
    "ds_read_b128 a[16:19], v240 offset:4096\n"
    "v_mfma_f32_16x16x32_bf16 v[88:91], a[144:147], a[112:115], v[88:91]\n"
    "s_add_u32 m0, s58, 9216\n"
    "v_mfma_f32_16x16x32_bf16 v[88:91], a[148:151], a[116:119], v[88:91]\n"
    "buffer_load_dwordx4 v237, s[40:43], s54 offen nt lds\n"
    "v_mfma_f32_16x16x32_bf16 v[92:95], a[144:147], a[120:123], v[92:95]\n"
    "v_mfma_f32_16x16x32_bf16 v[92:95], a[148:151], a[124:127], v[92:95]\n"
    "ds_read_b128 a[20:23], v241 offset:4096\n"
    "s_waitcnt lgkmcnt(8)\n"
    "v_mfma_f32_16x16x32_bf16 v[96:99], a[152:155], a[64:67], v[96:99]\n"
    "v_mfma_f32_16x16x32_bf16 v[96:99], a[156:159], a[68:71], v[96:99]\n"
    "ds_read_b64_tr_b16 a[136:137], v243 offset:512\n"
    "v_mfma_f32_16x16x32_bf16 v[100:103], a[152:155], a[72:75], v[100:103]\n"
    "v_mfma_f32_16x16x32_bf16 v[100:103], a[156:159], a[76:79], v[100:103]\n"
    "ds_read_b64_tr_b16 a[138:139], v243 offset:1536\n"
    "v_mfma_f32_16x16x32_bf16 v[104:107], a[152:155], a[80:83], v[104:107]\n"
    "v_mfma_f32_16x16x32_bf16 v[104:107], a[156:159], a[84:87], v[104:107]\n"
    "ds_read_b64_tr_b16 a[140:141], v243 offset:8704\n"
    "v_mfma_f32_16x16x32_bf16 v[108:111], a[152:155], a[88:91], v[108:111]\n"
    "v_mfma_f32_16x16x32_bf16 v[108:111], a[156:159], a[92:95], v[108:111]\n"
    "ds_read_b64_tr_b16 a[142:143], v243 offset:9728\n"
    "v_mfma_f32_16x16x32_bf16 v[112:115], a[152:155], a[96:99], v[112:115]\n"
    "s_add_u32 m0, s58, 12288\n"
    "v_mfma_f32_16x16x32_bf16 v[112:115], a[156:159], a[100:103], v[112:115]\n"
    "buffer_load_dwordx4 v238, s[40:43], s54 offen nt lds\n"
    "v_mfma_f32_16x16x32_bf16 v[116:119], a[152:155], a[104:107], v[116:119]\n"
    "v_mfma_f32_16x16x32_bf16 v[116:119], a[156:159], a[108:111], v[116:119]\n"
    "ds_read_b128 a[24:27], v240 offset:6144\n"
    "v_mfma_f32_16x16x32_bf16 v[120:123], a[152:155], a[112:115], v[120:123]\n"
    "s_add_u32 m0, s58, 13312\n"
    "v_mfma_f32_16x16x32_bf16 v[120:123], a[156:159], a[116:119], v[120:123]\n"
    "buffer_load_dwordx4 v239, s[40:43], s54 offen nt lds\n"
    "v_mfma_f32_16x16x32_bf16 v[124:127], a[152:155], a[120:123], v[124:127]\n"
    "v_mfma_f32_16x16x32_bf16 v[124:127], a[156:159], a[124:127], v[124:127]\n"
    "ds_read_b128 a[28:31], v241 offset:6144\n"
    "s_waitcnt lgkmcnt(8)\n"
    "v_mfma_f32_16x16x32_bf16 v[128:131], a[128:131], a[64:67], v[128:131]\n"
    "v_mfma_f32_16x16x32_bf16 v[128:131], a[132:135], a[68:71], v[128:131]\n"
    "ds_read_b64_tr_b16 a[144:145], v242 offset:768\n"
    "v_mfma_f32_16x16x32_bf16 v[132:135], a[128:131], a[72:75], v[132:135]\n"
    "s_add_u32 s58, s58, 32768\n"
    "v_mfma_f32_16x16x32_bf16 v[132:135], a[132:135], a[76:79], v[132:135]\n"
    "ds_read_b64_tr_b16 a[146:147], v242 offset:1792\n"
    "v_mfma_f32_16x16x32_bf16 v[136:139], a[128:131], a[80:83], v[136:139]\n"
    "s_sub_u32 s59, s58, 98304\n"
    "v_mfma_f32_16x16x32_bf16 v[136:139], a[132:135], a[84:87], v[136:139]\n"
    "ds_read_b64_tr_b16 a[148:149], v242 offset:8960\n"
    "v_mfma_f32_16x16x32_bf16 v[140:143], a[128:131], a[88:91], v[140:143]\n"
    "s_cmp_ge_u32 s58, s67\n"
    "v_mfma_f32_16x16x32_bf16 v[140:143], a[132:135], a[92:95], v[140:143]\n"
    "ds_read_b64_tr_b16 a[150:151], v242 offset:9984\n"
    "v_mfma_f32_16x16x32_bf16 v[144:147], a[128:131], a[96:99], v[144:147]\n"
    "s_cselect_b32 s58, s59, s58\n"
    "v_mfma_f32_16x16x32_bf16 v[144:147], a[132:135], a[100:103], v[144:147]\n"
    "ds_read_b128 a[32:35], v240 offset:8192\n"
    "v_mfma_f32_16x16x32_bf16 v[148:151], a[128:131], a[104:107], v[148:151]\n"
    "v_mfma_f32_16x16x32_bf16 v[148:151], a[132:135], a[108:111], v[148:151]\n"
    "ds_read_b128 a[36:39], v241 offset:8192\n"
    "v_mfma_f32_16x16x32_bf16 v[152:155], a[128:131], a[112:115], v[152:155]\n"
    "v_mfma_f32_16x16x32_bf16 v[152:155], a[132:135], a[116:119], v[152:155]\n"
    "ds_read_b128 a[40:43], v240 offset:10240\n"
    "v_mfma_f32_16x16x32_bf16 v[156:159], a[128:131], a[120:123], v[156:159]\n"
    "v_mfma_f32_16x16x32_bf16 v[156:159], a[132:135], a[124:127], v[156:159]\n"
    "ds_read_b128 a[44:47], v241 offset:10240\n"
    "s_waitcnt lgkmcnt(10)\n"
    "v_mfma_f32_16x16x32_bf16 v[160:163], a[136:139], a[64:67], v[160:163]\n"
    "v_mfma_f32_16x16x32_bf16 v[160:163], a[140:143], a[68:71], v[160:163]\n"
    "ds_read_b64_tr_b16 a[152:153], v243 offset:768\n"
    "ds_read_b128 a[48:51], v240 offset:12288\n"
    "v_mfma_f32_16x16x32_bf16 v[164:167], a[136:139], a[72:75], v[164:167]\n"
    "s_add_u32 s59, s52, 3\n"
    "v_mfma_f32_16x16x32_bf16 v[164:167], a[140:143], a[76:79], v[164:167]\n"
    "ds_read_b64_tr_b16 a[154:155], v243 offset:1792\n"
    "ds_read_b128 a[52:55], v241 offset:12288\n"
    "v_readlane_b32 s53, v244, s59\n"
    "v_mfma_f32_16x16x32_bf16 v[168:171], a[136:139], a[80:83], v[168:171]\n"
    "v_mfma_f32_16x16x32_bf16 v[168:171], a[140:143], a[84:87], v[168:171]\n"
    "ds_read_b64_tr_b16 a[156:157], v243 offset:8960\n"
    "ds_read_b128 a[56:59], v240 offset:14336\n"
    "v_mfma_f32_16x16x32_bf16 v[172:175], a[136:139], a[88:91], v[172:175]\n"
    "v_mfma_f32_16x16x32_bf16 v[172:175], a[140:143], a[92:95], v[172:175]\n"
    "ds_read_b64_tr_b16 a[158:159], v243 offset:9984\n"
    "ds_read_b128 a[60:63], v241 offset:14336\n"
    "v_mfma_f32_16x16x32_bf16 v[176:179], a[136:139], a[96:99], v[176:179]\n"
    "s_add_u32 s57, s57, 1\n"
    "v_mfma_f32_16x16x32_bf16 v[176:179], a[140:143], a[100:103], v[176:179]\n"
    "s_cmp_eq_u32 s57, 3\n"
    "v_mfma_f32_16x16x32_bf16 v[180:183], a[136:139], a[104:107], v[180:183]\n"
    "s_cselect_b32 s57, 0, s57\n"
    "s_cselect_b32 s59, 4294901760, s68\n"
    "v_mfma_f32_16x16x32_bf16 v[180:183], a[140:143], a[108:111], v[180:183]\n"
    "v_mfma_f32_16x16x32_bf16 v[184:187], a[136:139], a[112:115], v[184:187]\n"
    "v_add_u32 v242, s59, v242\n"
    "v_mfma_f32_16x16x32_bf16 v[184:187], a[140:143], a[116:119], v[184:187]\n"
    "v_add_u32 v243, s59, v243\n"
    "v_mfma_f32_16x16x32_bf16 v[188:191], a[136:139], a[120:123], v[188:191]\n"
    "v_mfma_f32_16x16x32_bf16 v[188:191], a[140:143], a[124:127], v[188:191]\n"
    "s_waitcnt vmcnt(8) lgkmcnt(0)\n"
    "s_barrier\n"
    "v_mfma_f32_16x16x32_bf16 v[192:195], a[144:147], a[64:67], v[192:195]\n"
    "v_cmp_le_u32 vcc, s63, v252\n"
    "v_mfma_f32_16x16x32_bf16 v[192:195], a[148:151], a[68:71], v[192:195]\n"
    "ds_read_b64_tr_b16 a[128:129], v242 offset:0\n"
    "v_mfma_f32_16x16x32_bf16 v[196:199], a[144:147], a[72:75], v[196:199]\n"
    "s_or_b64 vcc, vcc, s[78:79]\n"
    "s_cmp_eq_u64 vcc, -1\n"
    "v_mfma_f32_16x16x32_bf16 v[196:199], a[148:151], a[76:79], v[196:199]\n"
    "s_cbranch_scc0 .Lg29_%=\n"
    ".Lg30_%=:\n"
    "ds_read_b64_tr_b16 a[130:131], v242 offset:1024\n"
    "v_mfma_f32_16x16x32_bf16 v[200:203], a[144:147], a[80:83], v[200:203]\n"
    "v_mfma_f32_16x16x32_bf16 v[200:203], a[148:151], a[84:87], v[200:203]\n"
    "ds_read_b64_tr_b16 a[132:133], v242 offset:8192\n"
    "v_mfma_f32_16x16x32_bf16 v[204:207], a[144:147], a[88:91], v[204:207]\n"
    "v_mfma_f32_16x16x32_bf16 v[204:207], a[148:151], a[92:95], v[204:207]\n"
    "ds_read_b64_tr_b16 a[134:135], v242 offset:9216\n"
    "v_mfma_f32_16x16x32_bf16 v[208:211], a[144:147], a[96:99], v[208:211]\n"
    "s_add_u32 m0, s55, 0\n"
    "v_mfma_f32_16x16x32_bf16 v[208:211], a[148:151], a[100:103], v[208:211]\n"
    "buffer_load_dwordx4 v224, s[44:47], s53 offen sc1 lds\n"
    "v_mfma_f32_16x16x32_bf16 v[212:215], a[144:147], a[104:107], v[212:215]\n"
    "s_add_u32 m0, s55, 1024\n"
    "v_mfma_f32_16x16x32_bf16 v[212:215], a[148:151], a[108:111], v[212:215]\n"
    "buffer_load_dwordx4 v225, s[44:47], s53 offen sc1 lds\n"
    "v_mfma_f32_16x16x32_bf16 v[216:219], a[144:147], a[112:115], v[216:219]\n"
    "s_add_u32 m0, s55, 2048\n"
    "v_mfma_f32_16x16x32_bf16 v[216:219], a[148:151], a[116:119], v[216:219]\n"
    "buffer_load_dwordx4 v226, s[44:47], s53 offen sc1 lds\n"
    "v_mfma_f32_16x16x32_bf16 v[220:223], a[144:147], a[120:123], v[220:223]\n"
    "s_add_u32 m0, s55, 3072\n"
    "v_mfma_f32_16x16x32_bf16 v[220:223], a[148:151], a[124:127], v[220:223]\n"
    "buffer_load_dwordx4 v227, s[44:47], s53 offen sc1 lds\n"
    "s_waitcnt lgkmcnt(5)\n"
    "v_mfma_f32_16x16x32_bf16 a[160:163], a[152:155], a[64:67], a[160:163]\n"
    "v_mfma_f32_16x16x32_bf16 a[160:163], a[156:159], a[68:71], a[160:163]\n"
    "ds_read_b64_tr_b16 a[136:137], v243 offset:0\n"
    "v_mfma_f32_16x16x32_bf16 a[164:167], a[152:155], a[72:75], a[164:167]\n"
    "v_mfma_f32_16x16x32_bf16 a[164:167], a[156:159], a[76:79], a[164:167]\n"
    "ds_read_b64_tr_b16 a[138:139], v243 offset:1024\n"
    "v_mfma_f32_16x16x32_bf16 a[168:171], a[152:155], a[80:83], a[168:171]\n"
    "v_mfma_f32_16x16x32_bf16 a[168:171], a[156:159], a[84:87], a[168:171]\n"
    "ds_read_b64_tr_b16 a[140:141], v243 offset:8192\n"
    "v_mfma_f32_16x16x32_bf16 a[172:175], a[152:155], a[88:91], a[172:175]\n"
    "v_mfma_f32_16x16x32_bf16 a[172:175], a[156:159], a[92:95], a[172:175]\n"
    "ds_read_b64_tr_b16 a[142:143], v243 offset:9216\n"
    "v_mfma_f32_16x16x32_bf16 a[176:179], a[152:155], a[96:99], a[176:179]\n"
    "s_add_u32 m0, s55, 4096\n"
    "v_mfma_f32_16x16x32_bf16 a[176:179], a[156:159], a[100:103], a[176:179]\n"
    "buffer_load_dwordx4 v228, s[44:47], s53 offen sc1 lds\n"
    "v_mfma_f32_16x16x32_bf16 a[180:183], a[152:155], a[104:107], a[180:183]\n"
    "s_add_u32 m0, s55, 5120\n"
    "v_mfma_f32_16x16x32_bf16 a[180:183], a[156:159], a[108:111], a[180:183]\n"
    "buffer_load_dwordx4 v229, s[44:47], s53 offen sc1 lds\n"
    "v_mfma_f32_16x16x32_bf16 a[184:187], a[152:155], a[112:115], a[184:187]\n"
    "s_add_u32 m0, s55, 6144\n"
    "v_mfma_f32_16x16x32_bf16 a[184:187], a[156:159], a[116:119], a[184:187]\n"
    "buffer_load_dwordx4 v230, s[44:47], s53 offen sc1 lds\n"
    "v_mfma_f32_16x16x32_bf16 a[188:191], a[152:155], a[120:123], a[188:191]\n"
    "s_add_u32 m0, s55, 7168\n"
    "v_mfma_f32_16x16x32_bf16 a[188:191], a[156:159], a[124:127], a[188:191]\n"
    "buffer_load_dwordx4 v231, s[44:47], s53 offen sc1 lds\n"
    "s_add_u32 s52, s52, 1\n"
    "s_sub_u32 s60, s60, 1\n"
    "s_cmp_lg_u32 s60, 0\n"
    "s_cbranch_scc1 .Lg28_%=\n"
    ".Lg26_%=:\n"
    "s_waitcnt lgkmcnt(4)\n"
    "v_mfma_f32_16x16x32_bf16 v[0:3], a[128:131], a[0:3], v[0:3]\n"
    "s_add_u32 s59, s52, 2\n"
    "v_mfma_f32_16x16x32_bf16 v[0:3], a[132:135], a[4:7], v[0:3]\n"
    "ds_read_b64_tr_b16 a[144:145], v242 offset:256\n"
    "v_readlane_b32 s54, v245, s59\n"
    "v_mfma_f32_16x16x32_bf16 v[4:7], a[128:131], a[8:11], v[4:7]\n"
    "v_mfma_f32_16x16x32_bf16 v[4:7], a[132:135], a[12:15], v[4:7]\n"
    "ds_read_b64_tr_b16 a[146:147], v242 offset:1280\n"
    "v_mfma_f32_16x16x32_bf16 v[8:11], a[128:131], a[16:19], v[8:11]\n"
    "v_mfma_f32_16x16x32_bf16 v[8:11], a[132:135], a[20:23], v[8:11]\n"
    "ds_read_b64_tr_b16 a[148:149], v242 offset:8448\n"
    "v_mfma_f32_16x16x32_bf16 v[12:15], a[128:131], a[24:27], v[12:15]\n"
    "v_mfma_f32_16x16x32_bf16 v[12:15], a[132:135], a[28:31], v[12:15]\n"
    "ds_read_b64_tr_b16 a[150:151], v242 offset:9472\n"
    "v_mfma_f32_16x16x32_bf16 v[16:19], a[128:131], a[32:35], v[16:19]\n"
    "s_add_u32 m0, s58, 0\n"
    "v_mfma_f32_16x16x32_bf16 v[16:19], a[132:135], a[36:39], v[16:19]\n"
    "buffer_load_dwordx4 v232, s[40:43], s54 offen nt lds\n"
    "v_mfma_f32_16x16x32_bf16 v[20:23], a[128:131], a[40:43], v[20:23]\n"
    "v_mfma_f32_16x16x32_bf16 v[20:23], a[132:135], a[44:47], v[20:23]\n"
    "ds_read_b128 a[64:67], v240 offset:32768\n"
    "v_mfma_f32_16x16x32_bf16 v[24:27], a[128:131], a[48:51], v[24:27]\n"
    "s_add_u32 m0, s58, 1024\n"
    "v_mfma_f32_16x16x32_bf16 v[24:27], a[132:135], a[52:55], v[24:27]\n"
    "buffer_load_dwordx4 v233, s[40:43], s54 offen nt lds\n"
    "v_mfma_f32_16x16x32_bf16 v[28:31], a[128:131], a[56:59], v[28:31]\n"
    "v_mfma_f32_16x16x32_bf16 v[28:31], a[132:135], a[60:63], v[28:31]\n"
    "ds_read_b128 a[68:71], v241 offset:32768\n"
    "s_waitcnt lgkmcnt(6)\n"
    "v_mfma_f32_16x16x32_bf16 v[32:35], a[136:139], a[0:3], v[32:35]\n"
    "v_mfma_f32_16x16x32_bf16 v[32:35], a[140:143], a[4:7], v[32:35]\n"
    "ds_read_b64_tr_b16 a[152:153], v243 offset:256\n"
    "v_mfma_f32_16x16x32_bf16 v[36:39], a[136:139], a[8:11], v[36:39]\n"
    "v_mfma_f32_16x16x32_bf16 v[36:39], a[140:143], a[12:15], v[36:39]\n"
    "ds_read_b64_tr_b16 a[154:155], v243 offset:1280\n"
    "v_mfma_f32_16x16x32_bf16 v[40:43], a[136:139], a[16:19], v[40:43]\n"
    "s_mov_b32 s28, s36\n"
    "v_mfma_f32_16x16x32_bf16 v[40:43], a[140:143], a[20:23], v[40:43]\n"
    "ds_read_b64_tr_b16 a[156:157], v243 offset:8448\n"
    "v_mfma_f32_16x16x32_bf16 v[44:47], a[136:139], a[24:27], v[44:47]\n"
    "s_add_u32 s36, s36, s30\n"
    "v_mfma_f32_16x16x32_bf16 v[44:47], a[140:143], a[28:31], v[44:47]\n"
    "ds_read_b64_tr_b16 a[158:159], v243 offset:9472\n"
    "v_mfma_f32_16x16x32_bf16 v[48:51], a[136:139], a[32:35], v[48:51]\n"
    "s_add_u32 m0, s58, 4096\n"
    "v_mfma_f32_16x16x32_bf16 v[48:51], a[140:143], a[36:39], v[48:51]\n"
    "buffer_load_dwordx4 v234, s[40:43], s54 offen nt lds\n"
    "v_mfma_f32_16x16x32_bf16 v[52:55], a[136:139], a[40:43], v[52:55]\n"
    "v_mfma_f32_16x16x32_bf16 v[52:55], a[140:143], a[44:47], v[52:55]\n"
    "ds_read_b128 a[72:75], v240 offset:34816\n"
    "v_mfma_f32_16x16x32_bf16 v[56:59], a[136:139], a[48:51], v[56:59]\n"
    "s_add_u32 m0, s58, 5120\n"
    "v_mfma_f32_16x16x32_bf16 v[56:59], a[140:143], a[52:55], v[56:59]\n"
    "buffer_load_dwordx4 v235, s[40:43], s54 offen nt lds\n"
    "v_mfma_f32_16x16x32_bf16 v[60:63], a[136:139], a[56:59], v[60:63]\n"
    "v_mfma_f32_16x16x32_bf16 v[60:63], a[140:143], a[60:63], v[60:63]\n"
    "ds_read_b128 a[76:79], v241 offset:34816\n"
    "s_waitcnt lgkmcnt(8)\n"
    "v_mfma_f32_16x16x32_bf16 v[64:67], a[144:147], a[0:3], v[64:67]\n"
    "v_mfma_f32_16x16x32_bf16 v[64:67], a[148:151], a[4:7], v[64:67]\n"
    "ds_read_b64_tr_b16 a[128:129], v242 offset:512\n"
    "v_mfma_f32_16x16x32_bf16 v[68:71], a[144:147], a[8:11], v[68:71]\n"
    "v_mfma_f32_16x16x32_bf16 v[68:71], a[148:151], a[12:15], v[68:71]\n"
    "ds_read_b64_tr_b16 a[130:131], v242 offset:1536\n"
    "v_mfma_f32_16x16x32_bf16 v[72:75], a[144:147], a[16:19], v[72:75]\n"
    "v_mfma_f32_16x16x32_bf16 v[72:75], a[148:151], a[20:23], v[72:75]\n"
    "ds_read_b64_tr_b16 a[132:133], v242 offset:8704\n"
    "v_mfma_f32_16x16x32_bf16 v[76:79], a[144:147], a[24:27], v[76:79]\n"
    "v_mfma_f32_16x16x32_bf16 v[76:79], a[148:151], a[28:31], v[76:79]\n"
    "ds_read_b64_tr_b16 a[134:135], v242 offset:9728\n"
    "v_mfma_f32_16x16x32_bf16 v[80:83], a[144:147], a[32:35], v[80:83]\n"
    "s_add_u32 m0, s58, 8192\n"
    "v_mfma_f32_16x16x32_bf16 v[80:83], a[148:151], a[36:39], v[80:83]\n"
    "buffer_load_dwordx4 v236, s[40:43], s54 offen nt lds\n"
    "v_mfma_f32_16x16x32_bf16 v[84:87], a[144:147], a[40:43], v[84:87]\n"
    "v_mfma_f32_16x16x32_bf16 v[84:87], a[148:151], a[44:47], v[84:87]\n"
    "ds_read_b128 a[80:83], v240 offset:36864\n"
    "v_mfma_f32_16x16x32_bf16 v[88:91], a[144:147], a[48:51], v[88:91]\n"
    "s_add_u32 m0, s58, 9216\n"
    "v_mfma_f32_16x16x32_bf16 v[88:91], a[148:151], a[52:55], v[88:91]\n"
    "buffer_load_dwordx4 v237, s[40:43], s54 offen nt lds\n"
    "v_mfma_f32_16x16x32_bf16 v[92:95], a[144:147], a[56:59], v[92:95]\n"
    "v_mfma_f32_16x16x32_bf16 v[92:95], a[148:151], a[60:63], v[92:95]\n"
    "ds_read_b128 a[84:87], v241 offset:36864\n"
    "s_waitcnt lgkmcnt(8)\n"
    "v_mfma_f32_16x16x32_bf16 v[96:99], a[152:155], a[0:3], v[96:99]\n"
    "v_mfma_f32_16x16x32_bf16 v[96:99], a[156:159], a[4:7], v[96:99]\n"
    "ds_read_b64_tr_b16 a[136:137], v243 offset:512\n"
    "v_mfma_f32_16x16x32_bf16 v[100:103], a[152:155], a[8:11], v[100:103]\n"
    "v_mfma_f32_16x16x32_bf16 v[100:103], a[156:159], a[12:15], v[100:103]\n"
    "ds_read_b64_tr_b16 a[138:139], v243 offset:1536\n"
    "v_mfma_f32_16x16x32_bf16 v[104:107], a[152:155], a[16:19], v[104:107]\n"
    "v_mfma_f32_16x16x32_bf16 v[104:107], a[156:159], a[20:23], v[104:107]\n"
    "ds_read_b64_tr_b16 a[140:141], v243 offset:8704\n"
    "v_mfma_f32_16x16x32_bf16 v[108:111], a[152:155], a[24:27], v[108:111]\n"
    "v_mfma_f32_16x16x32_bf16 v[108:111], a[156:159], a[28:31], v[108:111]\n"
    "ds_read_b64_tr_b16 a[142:143], v243 offset:9728\n"
    "v_mfma_f32_16x16x32_bf16 v[112:115], a[152:155], a[32:35], v[112:115]\n"
    "s_add_u32 m0, s58, 12288\n"
    "v_mfma_f32_16x16x32_bf16 v[112:115], a[156:159], a[36:39], v[112:115]\n"
    "buffer_load_dwordx4 v238, s[40:43], s54 offen nt lds\n"
    "v_mfma_f32_16x16x32_bf16 v[116:119], a[152:155], a[40:43], v[116:119]\n"
    "v_mfma_f32_16x16x32_bf16 v[116:119], a[156:159], a[44:47], v[116:119]\n"
    "ds_read_b128 a[88:91], v240 offset:38912\n"
    "v_mfma_f32_16x16x32_bf16 v[120:123], a[152:155], a[48:51], v[120:123]\n"
    "s_add_u32 m0, s58, 13312\n"
    "v_mfma_f32_16x16x32_bf16 v[120:123], a[156:159], a[52:55], v[120:123]\n"
    "buffer_load_dwordx4 v239, s[40:43], s54 offen nt lds\n"
    "v_mfma_f32_16x16x32_bf16 v[124:127], a[152:155], a[56:59], v[124:127]\n"
    "v_mfma_f32_16x16x32_bf16 v[124:127], a[156:159], a[60:63], v[124:127]\n"
    "ds_read_b128 a[92:95], v241 offset:38912\n"
    "s_waitcnt lgkmcnt(8)\n"
    "v_mfma_f32_16x16x32_bf16 v[128:131], a[128:131], a[0:3], v[128:131]\n"
    "v_mfma_f32_16x16x32_bf16 v[128:131], a[132:135], a[4:7], v[128:131]\n"
    "ds_read_b64_tr_b16 a[144:145], v242 offset:768\n"
    "v_mfma_f32_16x16x32_bf16 v[132:135], a[128:131], a[8:11], v[132:135]\n"
    "s_add_u32 s58, s58, 32768\n"
    "v_mfma_f32_16x16x32_bf16 v[132:135], a[132:135], a[12:15], v[132:135]\n"
    "ds_read_b64_tr_b16 a[146:147], v242 offset:1792\n"
    "v_mfma_f32_16x16x32_bf16 v[136:139], a[128:131], a[16:19], v[136:139]\n"
    "s_sub_u32 s59, s58, 98304\n"
    "v_mfma_f32_16x16x32_bf16 v[136:139], a[132:135], a[20:23], v[136:139]\n"
    "ds_read_b64_tr_b16 a[148:149], v242 offset:8960\n"
    "v_mfma_f32_16x16x32_bf16 v[140:143], a[128:131], a[24:27], v[140:143]\n"
    "v_readlane_b32 s54, v245, 63\n"
    "s_cmp_ge_u32 s58, s67\n"
    "v_mfma_f32_16x16x32_bf16 v[140:143], a[132:135], a[28:31], v[140:143]\n"
    "ds_read_b64_tr_b16 a[150:151], v242 offset:9984\n"
    "v_mfma_f32_16x16x32_bf16 v[144:147], a[128:131], a[32:35], v[144:147]\n"
    "s_cselect_b32 s58, s59, s58\n"
    "v_mfma_f32_16x16x32_bf16 v[144:147], a[132:135], a[36:39], v[144:147]\n"
    "ds_read_b128 a[96:99], v240 offset:40960\n"
    "v_mfma_f32_16x16x32_bf16 v[148:151], a[128:131], a[40:43], v[148:151]\n"
    "v_mfma_f32_16x16x32_bf16 v[148:151], a[132:135], a[44:47], v[148:151]\n"
    "ds_read_b128 a[100:103], v241 offset:40960\n"
    "v_mfma_f32_16x16x32_bf16 v[152:155], a[128:131], a[48:51], v[152:155]\n"
    "v_mfma_f32_16x16x32_bf16 v[152:155], a[132:135], a[52:55], v[152:155]\n"
    "ds_read_b128 a[104:107], v240 offset:43008\n"
    "v_mfma_f32_16x16x32_bf16 v[156:159], a[128:131], a[56:59], v[156:159]\n"
    "v_mfma_f32_16x16x32_bf16 v[156:159], a[132:135], a[60:63], v[156:159]\n"
    "ds_read_b128 a[108:111], v241 offset:43008\n"
    "s_waitcnt lgkmcnt(10)\n"
    "v_mfma_f32_16x16x32_bf16 v[160:163], a[136:139], a[0:3], v[160:163]\n"
    "v_mfma_f32_16x16x32_bf16 v[160:163], a[140:143], a[4:7], v[160:163]\n"
    "ds_read_b64_tr_b16 a[152:153], v243 offset:768\n"
    "ds_read_b128 a[112:115], v240 offset:45056\n"
    "v_mfma_f32_16x16x32_bf16 v[164:167], a[136:139], a[8:11], v[164:167]\n"
    "s_add_u32 s59, s52, 3\n"
    "v_mfma_f32_16x16x32_bf16 v[164:167], a[140:143], a[12:15], v[164:167]\n"
    "ds_read_b64_tr_b16 a[154:155], v243 offset:1792\n"
    "ds_read_b128 a[116:119], v241 offset:45056\n"
    "v_readlane_b32 s53, v244, s59\n"
    "v_mfma_f32_16x16x32_bf16 v[168:171], a[136:139], a[16:19], v[168:171]\n"
    "v_mfma_f32_16x16x32_bf16 v[168:171], a[140:143], a[20:23], v[168:171]\n"
    "ds_read_b64_tr_b16 a[156:157], v243 offset:8960\n"
    "ds_read_b128 a[120:123], v240 offset:47104\n"
    "v_mfma_f32_16x16x32_bf16 v[172:175], a[136:139], a[24:27], v[172:175]\n"
    "s_memrealtime s[34:35]\n"
    "v_mfma_f32_16x16x32_bf16 v[172:175], a[140:143], a[28:31], v[172:175]\n"
    "ds_read_b64_tr_b16 a[158:159], v243 offset:9984\n"
    "ds_read_b128 a[124:127], v241 offset:47104\n"
    "v_mfma_f32_16x16x32_bf16 v[176:179], a[136:139], a[32:35], v[176:179]\n"
    "s_add_u32 s57, s57, 1\n"
    "v_mfma_f32_16x16x32_bf16 v[176:179], a[140:143], a[36:39], v[176:179]\n"
    "s_cmp_eq_u32 s57, 3\n"
    "v_mfma_f32_16x16x32_bf16 v[180:183], a[136:139], a[40:43], v[180:183]\n"
    "s_cselect_b32 s57, 0, s57\n"
    "s_cselect_b32 s59, 4294901760, s68\n"
    "v_mfma_f32_16x16x32_bf16 v[180:183], a[140:143], a[44:47], v[180:183]\n"
    "v_mfma_f32_16x16x32_bf16 v[184:187], a[136:139], a[48:51], v[184:187]\n"
    "v_add_u32 v242, s59, v242\n"
    "v_mfma_f32_16x16x32_bf16 v[184:187], a[140:143], a[52:55], v[184:187]\n"
    "v_add_u32 v243, s59, v243\n"
    "v_mfma_f32_16x16x32_bf16 v[188:191], a[136:139], a[56:59], v[188:191]\n"
    "v_mfma_f32_16x16x32_bf16 v[188:191], a[140:143], a[60:63], v[188:191]\n"
    "s_waitcnt vmcnt(8) lgkmcnt(0)\n"
    "s_sub_u32 s28, s34, s28\n"
    "s_min_u32 s28, s28, s31\n"
    "s_sub_u32 s34, s34, s36\n"
    "s_cmp_lt_i32 s34, -8\n"
    "s_cbranch_scc1 .Lg32_%=\n"
    ".Lg33_%=:\n"
    "s_barrier\n"
    "v_mfma_f32_16x16x32_bf16 v[192:195], a[144:147], a[0:3], v[192:195]\n"
    "v_mfma_f32_16x16x32_bf16 v[192:195], a[148:151], a[4:7], v[192:195]\n"
    "ds_read_b64_tr_b16 a[128:129], v242 offset:0\n"
    "v_mfma_f32_16x16x32_bf16 v[196:199], a[144:147], a[8:11], v[196:199]\n"
    "v_mfma_f32_16x16x32_bf16 v[196:199], a[148:151], a[12:15], v[196:199]\n"
    "ds_read_b64_tr_b16 a[130:131], v242 offset:1024\n"
    "v_mfma_f32_16x16x32_bf16 v[200:203], a[144:147], a[16:19], v[200:203]\n"
    "v_mov_b32 v253, s28\n"
    "v_mfma_f32_16x16x32_bf16 v[200:203], a[148:151], a[20:23], v[200:203]\n"
    "ds_read_b64_tr_b16 a[132:133], v242 offset:8192\n"
    "v_mfma_f32_16x16x32_bf16 v[204:207], a[144:147], a[24:27], v[204:207]\n"
    "buffer_store_dword v253, v251, s[88:91], s69 offen offset:8 sc1\n"
    "v_mfma_f32_16x16x32_bf16 v[204:207], a[148:151], a[28:31], v[204:207]\n"
    "ds_read_b64_tr_b16 a[134:135], v242 offset:9216\n"
    "v_mfma_f32_16x16x32_bf16 v[208:211], a[144:147], a[32:35], v[208:211]\n"
    "s_add_u32 m0, s55, 32768\n"
    "v_mfma_f32_16x16x32_bf16 v[208:211], a[148:151], a[36:39], v[208:211]\n"
    "buffer_load_dwordx4 v224, s[44:47], s53 offen sc1 lds\n"
    "v_mfma_f32_16x16x32_bf16 v[212:215], a[144:147], a[40:43], v[212:215]\n"
    "s_add_u32 m0, s55, 33792\n"
    "v_mfma_f32_16x16x32_bf16 v[212:215], a[148:151], a[44:47], v[212:215]\n"
    "buffer_load_dwordx4 v225, s[44:47], s53 offen sc1 lds\n"
    "v_mfma_f32_16x16x32_bf16 v[216:219], a[144:147], a[48:51], v[216:219]\n"
    "s_add_u32 m0, s55, 34816\n"
    "v_mfma_f32_16x16x32_bf16 v[216:219], a[148:151], a[52:55], v[216:219]\n"
    "buffer_load_dwordx4 v226, s[44:47], s53 offen sc1 lds\n"
    "v_mfma_f32_16x16x32_bf16 v[220:223], a[144:147], a[56:59], v[220:223]\n"
    "s_add_u32 m0, s55, 35840\n"
    "v_mfma_f32_16x16x32_bf16 v[220:223], a[148:151], a[60:63], v[220:223]\n"
    "buffer_load_dwordx4 v227, s[44:47], s53 offen sc1 lds\n"
    "s_waitcnt lgkmcnt(5)\n"
    "v_mfma_f32_16x16x32_bf16 a[160:163], a[152:155], a[0:3], a[160:163]\n"
    "v_mfma_f32_16x16x32_bf16 a[160:163], a[156:159], a[4:7], a[160:163]\n"
    "ds_read_b64_tr_b16 a[136:137], v243 offset:0\n"
    "v_mfma_f32_16x16x32_bf16 a[164:167], a[152:155], a[8:11], a[164:167]\n"
    "v_mfma_f32_16x16x32_bf16 a[164:167], a[156:159], a[12:15], a[164:167]\n"
    "ds_read_b64_tr_b16 a[138:139], v243 offset:1024\n"
    "v_mfma_f32_16x16x32_bf16 a[168:171], a[152:155], a[16:19], a[168:171]\n"
    "v_mfma_f32_16x16x32_bf16 a[168:171], a[156:159], a[20:23], a[168:171]\n"
    "ds_read_b64_tr_b16 a[140:141], v243 offset:8192\n"
    "v_mfma_f32_16x16x32_bf16 a[172:175], a[152:155], a[24:27], a[172:175]\n"
    "v_mfma_f32_16x16x32_bf16 a[172:175], a[156:159], a[28:31], a[172:175]\n"
    "ds_read_b64_tr_b16 a[142:143], v243 offset:9216\n"
    "v_mfma_f32_16x16x32_bf16 a[176:179], a[152:155], a[32:35], a[176:179]\n"
    "s_add_u32 m0, s55, 36864\n"
    "v_mfma_f32_16x16x32_bf16 a[176:179], a[156:159], a[36:39], a[176:179]\n"
    "buffer_load_dwordx4 v228, s[44:47], s53 offen sc1 lds\n"
    "v_mfma_f32_16x16x32_bf16 a[180:183], a[152:155], a[40:43], a[180:183]\n"
    "s_add_u32 m0, s55, 37888\n"
    "v_mfma_f32_16x16x32_bf16 a[180:183], a[156:159], a[44:47], a[180:183]\n"
    "buffer_load_dwordx4 v229, s[44:47], s53 offen sc1 lds\n"
    "v_mfma_f32_16x16x32_bf16 a[184:187], a[152:155], a[48:51], a[184:187]\n"
    "s_add_u32 m0, s55, 38912\n"
    "v_mfma_f32_16x16x32_bf16 a[184:187], a[156:159], a[52:55], a[184:187]\n"
    "buffer_load_dwordx4 v230, s[44:47], s53 offen sc1 lds\n"
    "v_mfma_f32_16x16x32_bf16 a[188:191], a[152:155], a[56:59], a[188:191]\n"
    "s_add_u32 m0, s55, 39936\n"
    "v_mfma_f32_16x16x32_bf16 a[188:191], a[156:159], a[60:63], a[188:191]\n"
    "buffer_load_dwordx4 v231, s[44:47], s53 offen sc1 lds\n"
    "s_add_u32 s52, s52, 1\n"
    "buffer_load_dword v253, off, s[88:91], 0 sc1\n"
    "s_waitcnt lgkmcnt(4)\n"
    "v_mfma_f32_16x16x32_bf16 v[0:3], a[128:131], a[64:67], v[0:3]\n"
    "s_add_u32 m0, s58, 0\n"
    "v_mfma_f32_16x16x32_bf16 v[0:3], a[132:135], a[68:71], v[0:3]\n"
    "ds_read_b64_tr_b16 a[144:145], v242 offset:256\n"
    "buffer_load_dwordx4 v232, s[40:43], s54 offen nt lds\n"
    "v_mfma_f32_16x16x32_bf16 v[4:7], a[128:131], a[72:75], v[4:7]\n"
    "s_add_u32 m0, s58, 1024\n"
    "v_mfma_f32_16x16x32_bf16 v[4:7], a[132:135], a[76:79], v[4:7]\n"
    "ds_read_b64_tr_b16 a[146:147], v242 offset:1280\n"
    "buffer_load_dwordx4 v233, s[40:43], s54 offen nt lds\n"
    "v_mfma_f32_16x16x32_bf16 v[8:11], a[128:131], a[80:83], v[8:11]\n"
    "s_add_u32 m0, s58, 4096\n"
    "v_mfma_f32_16x16x32_bf16 v[8:11], a[132:135], a[84:87], v[8:11]\n"
    "ds_read_b64_tr_b16 a[148:149], v242 offset:8448\n"
    "buffer_load_dwordx4 v234, s[40:43], s54 offen nt lds\n"
    "v_mfma_f32_16x16x32_bf16 v[12:15], a[128:131], a[88:91], v[12:15]\n"
    "s_add_u32 m0, s58, 5120\n"
    "v_mfma_f32_16x16x32_bf16 v[12:15], a[132:135], a[92:95], v[12:15]\n"
    "ds_read_b64_tr_b16 a[150:151], v242 offset:9472\n"
    "buffer_load_dwordx4 v235, s[40:43], s54 offen nt lds\n"
    "v_mfma_f32_16x16x32_bf16 v[16:19], a[128:131], a[96:99], v[16:19]\n"
    "s_add_u32 m0, s58, 8192\n"
    "v_mfma_f32_16x16x32_bf16 v[16:19], a[132:135], a[100:103], v[16:19]\n"
    "buffer_load_dwordx4 v236, s[40:43], s54 offen nt lds\n"
    "v_mfma_f32_16x16x32_bf16 v[20:23], a[128:131], a[104:107], v[20:23]\n"
    "s_add_u32 m0, s58, 9216\n"
    "v_mfma_f32_16x16x32_bf16 v[20:23], a[132:135], a[108:111], v[20:23]\n"
    "ds_read_b128 a[0:3], v240 offset:0\n"
    "buffer_load_dwordx4 v237, s[40:43], s54 offen nt lds\n"
    "v_mfma_f32_16x16x32_bf16 v[24:27], a[128:131], a[112:115], v[24:27]\n"
    "s_add_u32 m0, s58, 12288\n"
    "v_mfma_f32_16x16x32_bf16 v[24:27], a[132:135], a[116:119], v[24:27]\n"
    "buffer_load_dwordx4 v238, s[40:43], s54 offen nt lds\n"
    "v_mfma_f32_16x16x32_bf16 v[28:31], a[128:131], a[120:123], v[28:31]\n"
    "s_add_u32 m0, s58, 13312\n"
    "v_mfma_f32_16x16x32_bf16 v[28:31], a[132:135], a[124:127], v[28:31]\n"
    "ds_read_b128 a[4:7], v241 offset:0\n"
    "buffer_load_dwordx4 v239, s[40:43], s54 offen nt lds\n"
    "s_waitcnt lgkmcnt(6)\n"
    "v_mfma_f32_16x16x32_bf16 v[32:35], a[136:139], a[64:67], v[32:35]\n"
    "v_mfma_f32_16x16x32_bf16 v[32:35], a[140:143], a[68:71], v[32:35]\n"
    "ds_read_b64_tr_b16 a[152:153], v243 offset:256\n"
    "v_mfma_f32_16x16x32_bf16 v[36:39], a[136:139], a[72:75], v[36:39]\n"
    "v_mfma_f32_16x16x32_bf16 v[36:39], a[140:143], a[76:79], v[36:39]\n"
    "ds_read_b64_tr_b16 a[154:155], v243 offset:1280\n"
    "v_mfma_f32_16x16x32_bf16 v[40:43], a[136:139], a[80:83], v[40:43]\n"
    "v_mfma_f32_16x16x32_bf16 v[40:43], a[140:143], a[84:87], v[40:43]\n"
    "ds_read_b64_tr_b16 a[156:157], v243 offset:8448\n"
    "v_mfma_f32_16x16x32_bf16 v[44:47], a[136:139], a[88:91], v[44:47]\n"
    "v_mfma_f32_16x16x32_bf16 v[44:47], a[140:143], a[92:95], v[44:47]\n"
    "ds_read_b64_tr_b16 a[158:159], v243 offset:9472\n"
    "v_mfma_f32_16x16x32_bf16 v[48:51], a[136:139], a[96:99], v[48:51]\n"
    "v_mfma_f32_16x16x32_bf16 v[48:51], a[140:143], a[100:103], v[48:51]\n"
    "v_mfma_f32_16x16x32_bf16 v[52:55], a[136:139], a[104:107], v[52:55]\n"
    "v_mfma_f32_16x16x32_bf16 v[52:55], a[140:143], a[108:111], v[52:55]\n"
    "ds_read_b128 a[8:11], v240 offset:2048\n"
    "v_mfma_f32_16x16x32_bf16 v[56:59], a[136:139], a[112:115], v[56:59]\n"
    "v_mfma_f32_16x16x32_bf16 v[56:59], a[140:143], a[116:119], v[56:59]\n"
    "v_mfma_f32_16x16x32_bf16 v[60:63], a[136:139], a[120:123], v[60:63]\n"
    "v_mfma_f32_16x16x32_bf16 v[60:63], a[140:143], a[124:127], v[60:63]\n"
    "ds_read_b128 a[12:15], v241 offset:2048\n"
    "s_waitcnt lgkmcnt(8)\n"
    "v_mfma_f32_16x16x32_bf16 v[64:67], a[144:147], a[64:67], v[64:67]\n"
    "v_mfma_f32_16x16x32_bf16 v[64:67], a[148:151], a[68:71], v[64:67]\n"
    "ds_read_b64_tr_b16 a[128:129], v242 offset:512\n"
    "v_mfma_f32_16x16x32_bf16 v[68:71], a[144:147], a[72:75], v[68:71]\n"
    "v_mfma_f32_16x16x32_bf16 v[68:71], a[148:151], a[76:79], v[68:71]\n"
    "ds_read_b64_tr_b16 a[130:131], v242 offset:1536\n"
    "v_mfma_f32_16x16x32_bf16 v[72:75], a[144:147], a[80:83], v[72:75]\n"
    "v_mfma_f32_16x16x32_bf16 v[72:75], a[148:151], a[84:87], v[72:75]\n"
    "ds_read_b64_tr_b16 a[132:133], v242 offset:8704\n"
    "v_mfma_f32_16x16x32_bf16 v[76:79], a[144:147], a[88:91], v[76:79]\n"
    "v_mfma_f32_16x16x32_bf16 v[76:79], a[148:151], a[92:95], v[76:79]\n"
    "ds_read_b64_tr_b16 a[134:135], v242 offset:9728\n"
    "v_mfma_f32_16x16x32_bf16 v[80:83], a[144:147], a[96:99], v[80:83]\n"
    "s_mov_b32 s44, 0x100000\n"
    "v_mfma_f32_16x16x32_bf16 v[80:83], a[148:151], a[100:103], v[80:83]\n"
    "v_mfma_f32_16x16x32_bf16 v[84:87], a[144:147], a[104:107], v[84:87]\n"
    "s_mov_b32 s45, 0x200000\n"
    "v_mfma_f32_16x16x32_bf16 v[84:87], a[148:151], a[108:111], v[84:87]\n"
    "ds_read_b128 a[16:19], v240 offset:4096\n"
    "v_mfma_f32_16x16x32_bf16 v[88:91], a[144:147], a[112:115], v[88:91]\n"
    "s_mov_b32 s46, 0x300000\n"
    "v_mfma_f32_16x16x32_bf16 v[88:91], a[148:151], a[116:119], v[88:91]\n"
    "v_mfma_f32_16x16x32_bf16 v[92:95], a[144:147], a[120:123], v[92:95]\n"
    "s_mov_b32 s47, 0x400000\n"
    "v_mfma_f32_16x16x32_bf16 v[92:95], a[148:151], a[124:127], v[92:95]\n"
    "ds_read_b128 a[20:23], v241 offset:4096\n"
    "s_waitcnt lgkmcnt(8)\n"
    "v_mfma_f32_16x16x32_bf16 v[96:99], a[152:155], a[64:67], v[96:99]\n"
    "s_mov_b32 s48, 0x500000\n"
    "v_mfma_f32_16x16x32_bf16 v[96:99], a[156:159], a[68:71], v[96:99]\n"
    "ds_read_b64_tr_b16 a[136:137], v243 offset:512\n"
    "v_mfma_f32_16x16x32_bf16 v[100:103], a[152:155], a[72:75], v[100:103]\n"
    "s_mov_b32 s49, 0x600000\n"
    "v_mfma_f32_16x16x32_bf16 v[100:103], a[156:159], a[76:79], v[100:103]\n"
    "ds_read_b64_tr_b16 a[138:139], v243 offset:1536\n"
    "v_mfma_f32_16x16x32_bf16 v[104:107], a[152:155], a[80:83], v[104:107]\n"
    "s_mov_b32 s50, 0x700000\n"
    "v_mfma_f32_16x16x32_bf16 v[104:107], a[156:159], a[84:87], v[104:107]\n"
    "ds_read_b64_tr_b16 a[140:141], v243 offset:8704\n"
    "v_mfma_f32_16x16x32_bf16 v[108:111], a[152:155], a[88:91], v[108:111]\n"
    "v_add_u32 v246, 524288, v248\n"
    "v_mfma_f32_16x16x32_bf16 v[108:111], a[156:159], a[92:95], v[108:111]\n"
    "ds_read_b64_tr_b16 a[142:143], v243 offset:9728\n"
    "v_mfma_f32_16x16x32_bf16 v[112:115], a[152:155], a[96:99], v[112:115]\n"
    "v_mfma_f32_16x16x32_bf16 v[112:115], a[156:159], a[100:103], v[112:115]\n"
    "v_mfma_f32_16x16x32_bf16 v[116:119], a[152:155], a[104:107], v[116:119]\n"
    "v_mfma_f32_16x16x32_bf16 v[116:119], a[156:159], a[108:111], v[116:119]\n"
    "ds_read_b128 a[24:27], v240 offset:6144\n"
    "v_mfma_f32_16x16x32_bf16 v[120:123], a[152:155], a[112:115], v[120:123]\n"
    "v_mfma_f32_16x16x32_bf16 v[120:123], a[156:159], a[116:119], v[120:123]\n"
    "v_mfma_f32_16x16x32_bf16 v[124:127], a[152:155], a[120:123], v[124:127]\n"
    "v_mfma_f32_16x16x32_bf16 v[124:127], a[156:159], a[124:127], v[124:127]\n"
    "ds_read_b128 a[28:31], v241 offset:6144\n"
    "s_waitcnt lgkmcnt(8)\n"
    "v_mfma_f32_16x16x32_bf16 v[128:131], a[128:131], a[64:67], v[128:131]\n"
    "v_mfma_f32_16x16x32_bf16 v[128:131], a[132:135], a[68:71], v[128:131]\n"
    "ds_read_b64_tr_b16 a[144:145], v242 offset:768\n"
    "v_mfma_f32_16x16x32_bf16 v[132:135], a[128:131], a[72:75], v[132:135]\n"
    "s_add_u32 s58, s58, 32768\n"
    "v_mfma_f32_16x16x32_bf16 v[132:135], a[132:135], a[76:79], v[132:135]\n"
    "ds_read_b64_tr_b16 a[146:147], v242 offset:1792\n"
    "v_mfma_f32_16x16x32_bf16 v[136:139], a[128:131], a[80:83], v[136:139]\n"
    "s_sub_u32 s59, s58, 98304\n"
    "v_mfma_f32_16x16x32_bf16 v[136:139], a[132:135], a[84:87], v[136:139]\n"
    "ds_read_b64_tr_b16 a[148:149], v242 offset:8960\n"
    "v_mfma_f32_16x16x32_bf16 v[140:143], a[128:131], a[88:91], v[140:143]\n"
    "s_cmp_ge_u32 s58, s67\n"
    "v_mfma_f32_16x16x32_bf16 v[140:143], a[132:135], a[92:95], v[140:143]\n"
    "ds_read_b64_tr_b16 a[150:151], v242 offset:9984\n"
    "v_mfma_f32_16x16x32_bf16 v[144:147], a[128:131], a[96:99], v[144:147]\n"
    "s_cselect_b32 s58, s59, s58\n"
    "v_mfma_f32_16x16x32_bf16 v[144:147], a[132:135], a[100:103], v[144:147]\n"
    "ds_read_b128 a[32:35], v240 offset:8192\n"
    "v_mfma_f32_16x16x32_bf16 v[148:151], a[128:131], a[104:107], v[148:151]\n"
    "v_mfma_f32_16x16x32_bf16 v[148:151], a[132:135], a[108:111], v[148:151]\n"
    "ds_read_b128 a[36:39], v241 offset:8192\n"
    "v_mfma_f32_16x16x32_bf16 v[152:155], a[128:131], a[112:115], v[152:155]\n"
    "v_mfma_f32_16x16x32_bf16 v[152:155], a[132:135], a[116:119], v[152:155]\n"
    "ds_read_b128 a[40:43], v240 offset:10240\n"
    "v_mfma_f32_16x16x32_bf16 v[156:159], a[128:131], a[120:123], v[156:159]\n"
    "v_mfma_f32_16x16x32_bf16 v[156:159], a[132:135], a[124:127], v[156:159]\n"
    "ds_read_b128 a[44:47], v241 offset:10240\n"
    "s_waitcnt lgkmcnt(10)\n"
    "v_mfma_f32_16x16x32_bf16 v[160:163], a[136:139], a[64:67], v[160:163]\n"
    "v_mfma_f32_16x16x32_bf16 v[160:163], a[140:143], a[68:71], v[160:163]\n"
    "ds_read_b64_tr_b16 a[152:153], v243 offset:768\n"
    "ds_read_b128 a[48:51], v240 offset:12288\n"
    "v_mfma_f32_16x16x32_bf16 v[164:167], a[136:139], a[72:75], v[164:167]\n"
    "v_mfma_f32_16x16x32_bf16 v[164:167], a[140:143], a[76:79], v[164:167]\n"
    "ds_read_b64_tr_b16 a[154:155], v243 offset:1792\n"
    "ds_read_b128 a[52:55], v241 offset:12288\n"
    "v_mfma_f32_16x16x32_bf16 v[168:171], a[136:139], a[80:83], v[168:171]\n"
    "v_mfma_f32_16x16x32_bf16 v[168:171], a[140:143], a[84:87], v[168:171]\n"
    "ds_read_b64_tr_b16 a[156:157], v243 offset:8960\n"
    "ds_read_b128 a[56:59], v240 offset:14336\n"
    "v_mfma_f32_16x16x32_bf16 v[172:175], a[136:139], a[88:91], v[172:175]\n"
    "s_memrealtime s[34:35]\n"
    "v_mfma_f32_16x16x32_bf16 v[172:175], a[140:143], a[92:95], v[172:175]\n"
    "ds_read_b64_tr_b16 a[158:159], v243 offset:9984\n"
    "ds_read_b128 a[60:63], v241 offset:14336\n"
    "v_mfma_f32_16x16x32_bf16 v[176:179], a[136:139], a[96:99], v[176:179]\n"
    "s_add_u32 s57, s57, 1\n"
    "v_mfma_f32_16x16x32_bf16 v[176:179], a[140:143], a[100:103], v[176:179]\n"
    "s_cmp_eq_u32 s57, 3\n"
    "v_mfma_f32_16x16x32_bf16 v[180:183], a[136:139], a[104:107], v[180:183]\n"
    "s_cselect_b32 s57, 0, s57\n"
    "s_cselect_b32 s59, 4294901760, s68\n"
    "v_mfma_f32_16x16x32_bf16 v[180:183], a[140:143], a[108:111], v[180:183]\n"
    "v_mfma_f32_16x16x32_bf16 v[184:187], a[136:139], a[112:115], v[184:187]\n"
    "v_add_u32 v242, s59, v242\n"
    "v_mfma_f32_16x16x32_bf16 v[184:187], a[140:143], a[116:119], v[184:187]\n"
    "v_add_u32 v243, s59, v243\n"
    "v_mfma_f32_16x16x32_bf16 v[188:191], a[136:139], a[120:123], v[188:191]\n"
    "v_add_u32 v247, 4294901760, v242\n"
    "v_mfma_f32_16x16x32_bf16 v[188:191], a[140:143], a[124:127], v[188:191]\n"
    "v_add_u32 v252, 4294901760, v243\n"
    "s_waitcnt vmcnt(8) lgkmcnt(0)\n"
    "s_barrier\n"
    "v_mfma_f32_16x16x32_bf16 v[192:195], a[144:147], a[64:67], v[192:195]\n"
    "v_mfma_f32_16x16x32_bf16 v[192:195], a[148:151], a[68:71], v[192:195]\n"
    "ds_read_b64_tr_b16 a[128:129], v242 offset:0\n"
    "v_mfma_f32_16x16x32_bf16 v[196:199], a[144:147], a[72:75], v[196:199]\n"
    "v_mov_b32 v255, s34\n"
    "v_mfma_f32_16x16x32_bf16 v[196:199], a[148:151], a[76:79], v[196:199]\n"
    "ds_read_b64_tr_b16 a[130:131], v242 offset:1024\n"
    "v_mfma_f32_16x16x32_bf16 v[200:203], a[144:147], a[80:83], v[200:203]\n"
    "v_mfma_f32_16x16x32_bf16 v[200:203], a[148:151], a[84:87], v[200:203]\n"
    "ds_read_b64_tr_b16 a[132:133], v242 offset:8192\n"
    "v_mfma_f32_16x16x32_bf16 v[204:207], a[144:147], a[88:91], v[204:207]\n"
    "v_mfma_f32_16x16x32_bf16 v[204:207], a[148:151], a[92:95], v[204:207]\n"
    "ds_read_b64_tr_b16 a[134:135], v242 offset:9216\n"
    "v_mfma_f32_16x16x32_bf16 v[208:211], a[144:147], a[96:99], v[208:211]\n"
    "v_mfma_f32_16x16x32_bf16 v[208:211], a[148:151], a[100:103], v[208:211]\n"
    "ds_read_b128 a[192:195], v240 offset:32768\n"
    "v_mfma_f32_16x16x32_bf16 v[212:215], a[144:147], a[104:107], v[212:215]\n"
    "v_mfma_f32_16x16x32_bf16 v[212:215], a[148:151], a[108:111], v[212:215]\n"
    "ds_read_b128 a[196:199], v241 offset:32768\n"
    "v_mfma_f32_16x16x32_bf16 v[216:219], a[144:147], a[112:115], v[216:219]\n"
    "v_mfma_f32_16x16x32_bf16 v[216:219], a[148:151], a[116:119], v[216:219]\n"
    "ds_read_b128 a[200:203], v240 offset:34816\n"
    "v_mfma_f32_16x16x32_bf16 v[220:223], a[144:147], a[120:123], v[220:223]\n"
    "v_mfma_f32_16x16x32_bf16 v[220:223], a[148:151], a[124:127], v[220:223]\n"
    "ds_read_b128 a[204:207], v241 offset:34816\n"
    "s_waitcnt lgkmcnt(9)\n"
    "v_mfma_f32_16x16x32_bf16 a[160:163], a[152:155], a[64:67], a[160:163]\n"
    "v_mfma_f32_16x16x32_bf16 a[160:163], a[156:159], a[68:71], a[160:163]\n"
    "ds_read_b64_tr_b16 a[136:137], v243 offset:0\n"
    "v_mfma_f32_16x16x32_bf16 a[164:167], a[152:155], a[72:75], a[164:167]\n"
    "v_mfma_f32_16x16x32_bf16 a[164:167], a[156:159], a[76:79], a[164:167]\n"
    "ds_read_b64_tr_b16 a[138:139], v243 offset:1024\n"
    "v_mfma_f32_16x16x32_bf16 a[168:171], a[152:155], a[80:83], a[168:171]\n"
    "v_mfma_f32_16x16x32_bf16 a[168:171], a[156:159], a[84:87], a[168:171]\n"
    "ds_read_b64_tr_b16 a[140:141], v243 offset:8192\n"
    "v_mfma_f32_16x16x32_bf16 a[172:175], a[152:155], a[88:91], a[172:175]\n"
    "v_mfma_f32_16x16x32_bf16 a[172:175], a[156:159], a[92:95], a[172:175]\n"
    "ds_read_b64_tr_b16 a[142:143], v243 offset:9216\n"
    "v_mfma_f32_16x16x32_bf16 a[176:179], a[152:155], a[96:99], a[176:179]\n"
    "v_mfma_f32_16x16x32_bf16 a[176:179], a[156:159], a[100:103], a[176:179]\n"
    "ds_read_b128 a[208:211], v240 offset:36864\n"
    "v_mfma_f32_16x16x32_bf16 a[180:183], a[152:155], a[104:107], a[180:183]\n"
    "v_mfma_f32_16x16x32_bf16 a[180:183], a[156:159], a[108:111], a[180:183]\n"
    "ds_read_b128 a[212:215], v241 offset:36864\n"
    "v_mfma_f32_16x16x32_bf16 a[184:187], a[152:155], a[112:115], a[184:187]\n"
    "v_mfma_f32_16x16x32_bf16 a[184:187], a[156:159], a[116:119], a[184:187]\n"
    "ds_read_b128 a[216:219], v240 offset:38912\n"
    "v_mfma_f32_16x16x32_bf16 a[188:191], a[152:155], a[120:123], a[188:191]\n"
    "v_mfma_f32_16x16x32_bf16 a[188:191], a[156:159], a[124:127], a[188:191]\n"
    "ds_read_b128 a[220:223], v241 offset:38912\n"
    "s_add_u32 s52, s52, 1\n"
    "s_waitcnt vmcnt(0)\n"
    "s_barrier\n"
    "v_readfirstlane_b32 s59, v253\n"
    "s_cmp_eq_u32 s59, s63\n"
    "s_cbranch_scc1 .Lg34_%=\n"
    "s_waitcnt lgkmcnt(12)\n"
    "v_mfma_f32_16x16x32_bf16 v[0:3], a[128:131], a[0:3], v[0:3]\n"
    "buffer_store_dwordx2 v[254:255], v251, s[88:91], s69 offen sc1\n"
    "v_mfma_f32_16x16x32_bf16 v[0:3], a[132:135], a[4:7], v[0:3]\n"
    "ds_read_b64_tr_b16 a[144:145], v247 offset:0\n"
    "v_mfma_f32_16x16x32_bf16 v[4:7], a[128:131], a[8:11], v[4:7]\n"
    "v_mfma_f32_16x16x32_bf16 v[4:7], a[132:135], a[12:15], v[4:7]\n"
    "ds_read_b64_tr_b16 a[146:147], v247 offset:1024\n"
    "v_mfma_f32_16x16x32_bf16 v[8:11], a[128:131], a[16:19], v[8:11]\n"
    "v_mfma_f32_16x16x32_bf16 v[8:11], a[132:135], a[20:23], v[8:11]\n"
    "ds_read_b64_tr_b16 a[148:149], v247 offset:8192\n"
    "v_mfma_f32_16x16x32_bf16 v[12:15], a[128:131], a[24:27], v[12:15]\n"
    "v_mfma_f32_16x16x32_bf16 v[12:15], a[132:135], a[28:31], v[12:15]\n"
    "ds_read_b64_tr_b16 a[150:151], v247 offset:9216\n"
    "v_mfma_f32_16x16x32_bf16 v[16:19], a[128:131], a[32:35], v[16:19]\n"
    "v_mfma_f32_16x16x32_bf16 v[16:19], a[132:135], a[36:39], v[16:19]\n"
    "ds_read_b128 a[224:227], v240 offset:40960\n"
    "v_mfma_f32_16x16x32_bf16 v[20:23], a[128:131], a[40:43], v[20:23]\n"
    "v_mfma_f32_16x16x32_bf16 v[20:23], a[132:135], a[44:47], v[20:23]\n"
    "ds_read_b128 a[228:231], v241 offset:40960\n"
    "v_mfma_f32_16x16x32_bf16 v[24:27], a[128:131], a[48:51], v[24:27]\n"
    "v_mfma_f32_16x16x32_bf16 v[24:27], a[132:135], a[52:55], v[24:27]\n"
    "ds_read_b128 a[232:235], v240 offset:43008\n"
    "v_mfma_f32_16x16x32_bf16 v[28:31], a[128:131], a[56:59], v[28:31]\n"
    "v_mfma_f32_16x16x32_bf16 v[28:31], a[132:135], a[60:63], v[28:31]\n"
    "ds_read_b128 a[236:239], v241 offset:43008\n"
    "s_waitcnt lgkmcnt(12)\n"
    "v_mfma_f32_16x16x32_bf16 v[32:35], a[136:139], a[0:3], v[32:35]\n"
    "v_mfma_f32_16x16x32_bf16 v[32:35], a[140:143], a[4:7], v[32:35]\n"
    "ds_read_b64_tr_b16 a[152:153], v252 offset:0\n"
    "v_mfma_f32_16x16x32_bf16 v[36:39], a[136:139], a[8:11], v[36:39]\n"
    "v_mfma_f32_16x16x32_bf16 v[36:39], a[140:143], a[12:15], v[36:39]\n"
    "ds_read_b64_tr_b16 a[154:155], v252 offset:1024\n"
    "v_mfma_f32_16x16x32_bf16 v[40:43], a[136:139], a[16:19], v[40:43]\n"
    "v_mfma_f32_16x16x32_bf16 v[40:43], a[140:143], a[20:23], v[40:43]\n"
    "ds_read_b64_tr_b16 a[156:157], v252 offset:8192\n"
    "v_mfma_f32_16x16x32_bf16 v[44:47], a[136:139], a[24:27], v[44:47]\n"
    "v_mfma_f32_16x16x32_bf16 v[44:47], a[140:143], a[28:31], v[44:47]\n"
    "ds_read_b64_tr_b16 a[158:159], v252 offset:9216\n"
    "v_mfma_f32_16x16x32_bf16 v[48:51], a[136:139], a[32:35], v[48:51]\n"
    "v_mfma_f32_16x16x32_bf16 v[48:51], a[140:143], a[36:39], v[48:51]\n"
    "ds_read_b128 a[240:243], v240 offset:45056\n"
    "v_mfma_f32_16x16x32_bf16 v[52:55], a[136:139], a[40:43], v[52:55]\n"
    "v_mfma_f32_16x16x32_bf16 v[52:55], a[140:143], a[44:47], v[52:55]\n"
    "ds_read_b128 a[244:247], v241 offset:45056\n"
    "v_mfma_f32_16x16x32_bf16 v[56:59], a[136:139], a[48:51], v[56:59]\n"
    "v_mfma_f32_16x16x32_bf16 v[56:59], a[140:143], a[52:55], v[56:59]\n"
    "ds_read_b128 a[248:251], v240 offset:47104\n"
    "v_mfma_f32_16x16x32_bf16 v[60:63], a[136:139], a[56:59], v[60:63]\n"
    "v_mfma_f32_16x16x32_bf16 v[60:63], a[140:143], a[60:63], v[60:63]\n"
    "ds_read_b128 a[252:255], v241 offset:47104\n"
    "s_waitcnt lgkmcnt(12)\n"
    "v_mfma_f32_16x16x32_bf16 v[0:3], a[144:147], a[192:195], v[0:3]\n"
    "v_mfma_f32_16x16x32_bf16 v[0:3], a[148:151], a[196:199], v[0:3]\n"
    "ds_read_b64_tr_b16 a[128:129], v242 offset:256\n"
    "v_mfma_f32_16x16x32_bf16 v[4:7], a[144:147], a[200:203], v[4:7]\n"
    "v_mfma_f32_16x16x32_bf16 v[4:7], a[148:151], a[204:207], v[4:7]\n"
    "ds_read_b64_tr_b16 a[130:131], v242 offset:1280\n"
    "v_mfma_f32_16x16x32_bf16 v[8:11], a[144:147], a[208:211], v[8:11]\n"
    "v_mfma_f32_16x16x32_bf16 v[8:11], a[148:151], a[212:215], v[8:11]\n"
    "ds_read_b64_tr_b16 a[132:133], v242 offset:8448\n"
    "v_mfma_f32_16x16x32_bf16 v[12:15], a[144:147], a[216:219], v[12:15]\n"
    "v_mfma_f32_16x16x32_bf16 v[12:15], a[148:151], a[220:223], v[12:15]\n"
    "ds_read_b64_tr_b16 a[134:135], v242 offset:9472\n"
    "s_waitcnt lgkmcnt(12)\n"
    "v_mfma_f32_16x16x32_bf16 v[16:19], a[144:147], a[224:227], v[16:19]\n"
    "v_mfma_f32_16x16x32_bf16 v[16:19], a[148:151], a[228:231], v[16:19]\n"
    "s_mov_b32 s40, 0xbfb8aa3b\n"
    "v_mfma_f32_16x16x32_bf16 v[20:23], a[144:147], a[232:235], v[20:23]\n"
    "v_mfma_f32_16x16x32_bf16 v[20:23], a[148:151], a[236:239], v[20:23]\n"
    "s_mov_b32 s41, s40\n"
    "s_waitcnt lgkmcnt(4)\n"
    "v_mfma_f32_16x16x32_bf16 v[24:27], a[144:147], a[240:243], v[24:27]\n"
    "v_mfma_f32_16x16x32_bf16 v[24:27], a[148:151], a[244:247], v[24:27]\n"
    "s_mov_b32 s42, 0x6abcede5\n"
    "v_mfma_f32_16x16x32_bf16 v[28:31], a[144:147], a[248:251], v[28:31]\n"
    "v_mfma_f32_16x16x32_bf16 v[28:31], a[148:151], a[252:255], v[28:31]\n"
    "s_mov_b32 s43, s42\n"
    "s_waitcnt lgkmcnt(8)\n"
    "v_mfma_f32_16x16x32_bf16 v[32:35], a[152:155], a[192:195], v[32:35]\n"
    "v_mfma_f32_16x16x32_bf16 v[32:35], a[156:159], a[196:199], v[32:35]\n"
    "ds_read_b64_tr_b16 a[136:137], v243 offset:256\n"
    "v_mov_b32 v240, 0xc2ad1f97\n"
    "v_mfma_f32_16x16x32_bf16 v[36:39], a[152:155], a[200:203], v[36:39]\n"
    "v_mfma_f32_16x16x32_bf16 v[36:39], a[156:159], a[204:207], v[36:39]\n"
    "ds_read_b64_tr_b16 a[138:139], v243 offset:1280\n"
    "v_mov_b32 v241, v240\n"
    "v_mfma_f32_16x16x32_bf16 v[40:43], a[152:155], a[208:211], v[40:43]\n"
    "v_mfma_f32_16x16x32_bf16 v[40:43], a[156:159], a[212:215], v[40:43]\n"
    "ds_read_b64_tr_b16 a[140:141], v243 offset:8448\n"
    "v_mov_b32 v244, 1.0\n"
    "v_mfma_f32_16x16x32_bf16 v[44:47], a[152:155], a[216:219], v[44:47]\n"
    "v_pk_fma_f32 v[224:225], v[0:1], s[40:41], v[240:241]\n"
    "v_pk_fma_f32 v[226:227], v[2:3], s[40:41], v[240:241]\n"
    "v_mfma_f32_16x16x32_bf16 v[44:47], a[156:159], a[220:223], v[44:47]\n"
    "ds_read_b64_tr_b16 a[142:143], v243 offset:9472\n"
    "v_mov_b32 v245, 1.0\n"
    "v_mfma_f32_16x16x32_bf16 v[48:51], a[152:155], a[224:227], v[48:51]\n"
    "v_mfma_f32_16x16x32_bf16 v[48:51], a[156:159], a[228:231], v[48:51]\n"
    "v_mfma_f32_16x16x32_bf16 v[52:55], a[152:155], a[232:235], v[52:55]\n"
    "v_mfma_f32_16x16x32_bf16 v[52:55], a[156:159], a[236:239], v[52:55]\n"
    "v_mfma_f32_16x16x32_bf16 v[56:59], a[152:155], a[240:243], v[56:59]\n"
    "v_mfma_f32_16x16x32_bf16 v[56:59], a[156:159], a[244:247], v[56:59]\n"
    "v_mfma_f32_16x16x32_bf16 v[60:63], a[152:155], a[248:251], v[60:63]\n"
    "v_mfma_f32_16x16x32_bf16 v[60:63], a[156:159], a[252:255], v[60:63]\n"
    "s_waitcnt lgkmcnt(4)\n"
    "v_mfma_f32_16x16x32_bf16 v[64:67], a[128:131], a[0:3], v[64:67]\n"
    "v_mfma_f32_16x16x32_bf16 v[64:67], a[132:135], a[4:7], v[64:67]\n"
    "ds_read_b64_tr_b16 a[144:145], v247 offset:256\n"
    "v_mfma_f32_16x16x32_bf16 v[68:71], a[128:131], a[8:11], v[68:71]\n"
    "v_mfma_f32_16x16x32_bf16 v[68:71], a[132:135], a[12:15], v[68:71]\n"
    "ds_read_b64_tr_b16 a[146:147], v247 offset:1280\n"
    "v_mfma_f32_16x16x32_bf16 v[72:75], a[128:131], a[16:19], v[72:75]\n"
    "v_mfma_f32_16x16x32_bf16 v[72:75], a[132:135], a[20:23], v[72:75]\n"
    "ds_read_b64_tr_b16 a[148:149], v247 offset:8448\n"
    "v_mfma_f32_16x16x32_bf16 v[76:79], a[128:131], a[24:27], v[76:79]\n"
    "v_pk_fma_f32 v[228:229], v[32:33], s[40:41], v[240:241]\n"
    "v_pk_fma_f32 v[230:231], v[34:35], s[40:41], v[240:241]\n"
    "v_mfma_f32_16x16x32_bf16 v[76:79], a[132:135], a[28:31], v[76:79]\n"
    "ds_read_b64_tr_b16 a[150:151], v247 offset:9472\n"
    "v_mfma_f32_16x16x32_bf16 v[80:83], a[128:131], a[32:35], v[80:83]\n"
    "v_exp_f32_e64 v224, v224 clamp\n"
    "v_mfma_f32_16x16x32_bf16 v[80:83], a[132:135], a[36:39], v[80:83]\n"
    "v_exp_f32_e64 v225, v225 clamp\n"
    "v_mfma_f32_16x16x32_bf16 v[84:87], a[128:131], a[40:43], v[84:87]\n"
    "v_exp_f32_e64 v226, v226 clamp\n"
    "v_mfma_f32_16x16x32_bf16 v[84:87], a[132:135], a[44:47], v[84:87]\n"
    "v_exp_f32_e64 v227, v227 clamp\n"
    "v_mfma_f32_16x16x32_bf16 v[88:91], a[128:131], a[48:51], v[88:91]\n"
    "v_exp_f32_e64 v228, v228 clamp\n"
    "v_mfma_f32_16x16x32_bf16 v[88:91], a[132:135], a[52:55], v[88:91]\n"
    "v_exp_f32_e64 v229, v229 clamp\n"
    "v_mfma_f32_16x16x32_bf16 v[92:95], a[128:131], a[56:59], v[92:95]\n"
    "v_exp_f32_e64 v230, v230 clamp\n"
    "v_mfma_f32_16x16x32_bf16 v[92:95], a[132:135], a[60:63], v[92:95]\n"
    "v_exp_f32_e64 v231, v231 clamp\n"
    "s_waitcnt lgkmcnt(4)\n"
    "v_mfma_f32_16x16x32_bf16 v[96:99], a[136:139], a[0:3], v[96:99]\n"
    "v_pk_fma_f32 v[224:225], v[224:225], s[42:43], v[244:245]\n"
    "v_pk_fma_f32 v[226:227], v[226:227], s[42:43], v[244:245]\n"
    "v_mfma_f32_16x16x32_bf16 v[96:99], a[140:143], a[4:7], v[96:99]\n"
    "ds_read_b64_tr_b16 a[152:153], v252 offset:256\n"
    "v_pk_fma_f32 v[228:229], v[228:229], s[42:43], v[244:245]\n"
    "v_mfma_f32_16x16x32_bf16 v[100:103], a[136:139], a[8:11], v[100:103]\n"
    "v_pk_fma_f32 v[230:231], v[230:231], s[42:43], v[244:245]\n"
    "v_mfma_f32_16x16x32_bf16 v[100:103], a[140:143], a[12:15], v[100:103]\n"
    "ds_read_b64_tr_b16 a[154:155], v252 offset:1280\n"
    "v_mfma_f32_16x16x32_bf16 v[104:107], a[136:139], a[16:19], v[104:107]\n"
    "v_rcp_f32 v224, v224\n"
    "v_mfma_f32_16x16x32_bf16 v[104:107], a[140:143], a[20:23], v[104:107]\n"
    "ds_read_b64_tr_b16 a[156:157], v252 offset:8448\n"
    "v_mfma_f32_16x16x32_bf16 v[108:111], a[136:139], a[24:27], v[108:111]\n"
    "v_rcp_f32 v225, v225\n"
    "v_mfma_f32_16x16x32_bf16 v[108:111], a[140:143], a[28:31], v[108:111]\n"
    "ds_read_b64_tr_b16 a[158:159], v252 offset:9472\n"
    "v_mfma_f32_16x16x32_bf16 v[112:115], a[136:139], a[32:35], v[112:115]\n"
    "v_rcp_f32 v226, v226\n"
    "v_mfma_f32_16x16x32_bf16 v[112:115], a[140:143], a[36:39], v[112:115]\n"
    "v_rcp_f32 v227, v227\n"
    "v_mfma_f32_16x16x32_bf16 v[116:119], a[136:139], a[40:43], v[116:119]\n"
    "v_rcp_f32 v228, v228\n"
    "v_mfma_f32_16x16x32_bf16 v[116:119], a[140:143], a[44:47], v[116:119]\n"
    "v_rcp_f32 v229, v229\n"
    "v_mfma_f32_16x16x32_bf16 v[120:123], a[136:139], a[48:51], v[120:123]\n"
    "v_rcp_f32 v230, v230\n"
    "v_mfma_f32_16x16x32_bf16 v[120:123], a[140:143], a[52:55], v[120:123]\n"
    "v_rcp_f32 v231, v231\n"
    "v_mfma_f32_16x16x32_bf16 v[124:127], a[136:139], a[56:59], v[124:127]\n"
    "v_pk_mul_f32 v[0:1], v[0:1], v[224:225]\n"
    "v_pk_mul_f32 v[2:3], v[2:3], v[226:227]\n"
    "v_mfma_f32_16x16x32_bf16 v[124:127], a[140:143], a[60:63], v[124:127]\n"
    "v_pk_mul_f32 v[32:33], v[32:33], v[228:229]\n"
    "v_pk_mul_f32 v[34:35], v[34:35], v[230:231]\n"
    "s_waitcnt lgkmcnt(4)\n"
    "v_mfma_f32_16x16x32_bf16 v[64:67], a[144:147], a[192:195], v[64:67]\n"
    "v_cvt_pk_bf16_f32 v0, v0, v1\n"
    "v_cvt_pk_bf16_f32 v1, v2, v3\n"
    "v_mfma_f32_16x16x32_bf16 v[64:67], a[148:151], a[196:199], v[64:67]\n"
    "ds_read_b64_tr_b16 a[128:129], v242 offset:512\n"
    "v_cvt_pk_bf16_f32 v2, v32, v33\n"
    "v_mfma_f32_16x16x32_bf16 v[68:71], a[144:147], a[200:203], v[68:71]\n"
    "v_cvt_pk_bf16_f32 v3, v34, v35\n"
    "v_pk_fma_f32 v[232:233], v[4:5], s[40:41], v[240:241]\n"
    "v_mfma_f32_16x16x32_bf16 v[68:71], a[148:151], a[204:207], v[68:71]\n"
    "ds_read_b64_tr_b16 a[130:131], v242 offset:1536\n"
    "v_pk_fma_f32 v[234:235], v[6:7], s[40:41], v[240:241]\n"
    "v_mfma_f32_16x16x32_bf16 v[72:75], a[144:147], a[208:211], v[72:75]\n"
    "v_pk_fma_f32 v[236:237], v[36:37], s[40:41], v[240:241]\n"
    "v_pk_fma_f32 v[238:239], v[38:39], s[40:41], v[240:241]\n"
    "v_mfma_f32_16x16x32_bf16 v[72:75], a[148:151], a[212:215], v[72:75]\n"
    "ds_read_b64_tr_b16 a[132:133], v242 offset:8704\n"
    "v_permlane16_swap_b32 v0, v2\n"
    "v_mfma_f32_16x16x32_bf16 v[76:79], a[144:147], a[216:219], v[76:79]\n"
    "v_permlane16_swap_b32 v1, v3\n"
    "ds_write_b128 v249, v[0:3] offset:0\n"
    "v_mfma_f32_16x16x32_bf16 v[76:79], a[148:151], a[220:223], v[76:79]\n"
    "ds_read_b64_tr_b16 a[134:135], v242 offset:9728\n"
    "v_mfma_f32_16x16x32_bf16 v[80:83], a[144:147], a[224:227], v[80:83]\n"
    "v_exp_f32_e64 v232, v232 clamp\n"
    "v_mfma_f32_16x16x32_bf16 v[80:83], a[148:151], a[228:231], v[80:83]\n"
    "v_exp_f32_e64 v233, v233 clamp\n"
    "v_mfma_f32_16x16x32_bf16 v[84:87], a[144:147], a[232:235], v[84:87]\n"
    "v_exp_f32_e64 v234, v234 clamp\n"
    "v_mfma_f32_16x16x32_bf16 v[84:87], a[148:151], a[236:239], v[84:87]\n"
    "v_exp_f32_e64 v235, v235 clamp\n"
    "v_mfma_f32_16x16x32_bf16 v[88:91], a[144:147], a[240:243], v[88:91]\n"
    "v_exp_f32_e64 v236, v236 clamp\n"
    "v_mfma_f32_16x16x32_bf16 v[88:91], a[148:151], a[244:247], v[88:91]\n"
    "v_exp_f32_e64 v237, v237 clamp\n"
    "v_mfma_f32_16x16x32_bf16 v[92:95], a[144:147], a[248:251], v[92:95]\n"
    "v_exp_f32_e64 v238, v238 clamp\n"
    "v_mfma_f32_16x16x32_bf16 v[92:95], a[148:151], a[252:255], v[92:95]\n"
    "v_exp_f32_e64 v239, v239 clamp\n"
    "s_waitcnt lgkmcnt(5)\n"
    "v_mfma_f32_16x16x32_bf16 v[96:99], a[152:155], a[192:195], v[96:99]\n"
    "v_pk_fma_f32 v[232:233], v[232:233], s[42:43], v[244:245]\n"
    "v_pk_fma_f32 v[234:235], v[234:235], s[42:43], v[244:245]\n"
    "v_mfma_f32_16x16x32_bf16 v[96:99], a[156:159], a[196:199], v[96:99]\n"
    "ds_read_b64_tr_b16 a[136:137], v243 offset:512\n"
    "v_pk_fma_f32 v[236:237], v[236:237], s[42:43], v[244:245]\n"
    "v_mfma_f32_16x16x32_bf16 v[100:103], a[152:155], a[200:203], v[100:103]\n"
    "v_pk_fma_f32 v[238:239], v[238:239], s[42:43], v[244:245]\n"
    "v_mfma_f32_16x16x32_bf16 v[100:103], a[156:159], a[204:207], v[100:103]\n"
    "ds_read_b64_tr_b16 a[138:139], v243 offset:1536\n"
    "v_mfma_f32_16x16x32_bf16 v[104:107], a[152:155], a[208:211], v[104:107]\n"
    "v_rcp_f32 v232, v232\n"
    "v_mfma_f32_16x16x32_bf16 v[104:107], a[156:159], a[212:215], v[104:107]\n"
    "ds_read_b64_tr_b16 a[140:141], v243 offset:8704\n"
    "v_mfma_f32_16x16x32_bf16 v[108:111], a[152:155], a[216:219], v[108:111]\n"
    "v_rcp_f32 v233, v233\n"
    "v_mfma_f32_16x16x32_bf16 v[108:111], a[156:159], a[220:223], v[108:111]\n"
    "ds_read_b64_tr_b16 a[142:143], v243 offset:9728\n"
    "v_mfma_f32_16x16x32_bf16 v[112:115], a[152:155], a[224:227], v[112:115]\n"
    "v_rcp_f32 v234, v234\n"
    "v_mfma_f32_16x16x32_bf16 v[112:115], a[156:159], a[228:231], v[112:115]\n"
    "v_rcp_f32 v235, v235\n"
    "v_mfma_f32_16x16x32_bf16 v[116:119], a[152:155], a[232:235], v[116:119]\n"
    "v_rcp_f32 v236, v236\n"
    "v_mfma_f32_16x16x32_bf16 v[116:119], a[156:159], a[236:239], v[116:119]\n"
    "v_rcp_f32 v237, v237\n"
    "v_mfma_f32_16x16x32_bf16 v[120:123], a[152:155], a[240:243], v[120:123]\n"
    "v_rcp_f32 v238, v238\n"
    "v_mfma_f32_16x16x32_bf16 v[120:123], a[156:159], a[244:247], v[120:123]\n"
    "v_rcp_f32 v239, v239\n"
    "v_mfma_f32_16x16x32_bf16 v[124:127], a[152:155], a[248:251], v[124:127]\n"
    "v_pk_mul_f32 v[4:5], v[4:5], v[232:233]\n"
    "v_pk_mul_f32 v[6:7], v[6:7], v[234:235]\n"
    "v_mfma_f32_16x16x32_bf16 v[124:127], a[156:159], a[252:255], v[124:127]\n"
    "v_pk_mul_f32 v[36:37], v[36:37], v[236:237]\n"
    "v_pk_mul_f32 v[38:39], v[38:39], v[238:239]\n"
    "s_waitcnt lgkmcnt(4)\n"
    "v_mfma_f32_16x16x32_bf16 v[128:131], a[128:131], a[0:3], v[128:131]\n"
    "v_cvt_pk_bf16_f32 v4, v4, v5\n"
    "v_cvt_pk_bf16_f32 v5, v6, v7\n"
    "v_mfma_f32_16x16x32_bf16 v[128:131], a[132:135], a[4:7], v[128:131]\n"
    "ds_read_b64_tr_b16 a[144:145], v247 offset:512\n"
    "v_cvt_pk_bf16_f32 v6, v36, v37\n"
    "v_mfma_f32_16x16x32_bf16 v[132:135], a[128:131], a[8:11], v[132:135]\n"
    "v_cvt_pk_bf16_f32 v7, v38, v39\n"
    "v_pk_fma_f32 v[224:225], v[64:65], s[40:41], v[240:241]\n"
    "v_mfma_f32_16x16x32_bf16 v[132:135], a[132:135], a[12:15], v[132:135]\n"
    "ds_read_b64_tr_b16 a[146:147], v247 offset:1536\n"
    "v_pk_fma_f32 v[226:227], v[66:67], s[40:41], v[240:241]\n"
    "v_mfma_f32_16x16x32_bf16 v[136:139], a[128:131], a[16:19], v[136:139]\n"
    "v_mfma_f32_16x16x32_bf16 v[136:139], a[132:135], a[20:23], v[136:139]\n"
    "ds_read_b64_tr_b16 a[148:149], v247 offset:8704\n"
    "v_mfma_f32_16x16x32_bf16 v[140:143], a[128:131], a[24:27], v[140:143]\n"
    "v_pk_fma_f32 v[228:229], v[96:97], s[40:41], v[240:241]\n"
    "v_pk_fma_f32 v[230:231], v[98:99], s[40:41], v[240:241]\n"
    "v_mfma_f32_16x16x32_bf16 v[140:143], a[132:135], a[28:31], v[140:143]\n"
    "ds_read_b64_tr_b16 a[150:151], v247 offset:9728\n"
    "v_permlane16_swap_b32 v4, v6\n"
    "v_mfma_f32_16x16x32_bf16 v[144:147], a[128:131], a[32:35], v[144:147]\n"
    "v_permlane16_swap_b32 v5, v7\n"
    "ds_write_b128 v249, v[4:7] offset:2304\n"
    "v_mfma_f32_16x16x32_bf16 v[144:147], a[132:135], a[36:39], v[144:147]\n"
    "v_exp_f32_e64 v224, v224 clamp\n"
    "v_mfma_f32_16x16x32_bf16 v[148:151], a[128:131], a[40:43], v[148:151]\n"
    "v_exp_f32_e64 v225, v225 clamp\n"
    "v_mfma_f32_16x16x32_bf16 v[148:151], a[132:135], a[44:47], v[148:151]\n"
    "v_exp_f32_e64 v226, v226 clamp\n"
    "v_mfma_f32_16x16x32_bf16 v[152:155], a[128:131], a[48:51], v[152:155]\n"
    "v_exp_f32_e64 v227, v227 clamp\n"
    "v_mfma_f32_16x16x32_bf16 v[152:155], a[132:135], a[52:55], v[152:155]\n"
    "v_exp_f32_e64 v228, v228 clamp\n"
    "v_mfma_f32_16x16x32_bf16 v[156:159], a[128:131], a[56:59], v[156:159]\n"
    "v_exp_f32_e64 v229, v229 clamp\n"
    "v_mfma_f32_16x16x32_bf16 v[156:159], a[132:135], a[60:63], v[156:159]\n"
    "v_exp_f32_e64 v230, v230 clamp\n"
    "s_waitcnt lgkmcnt(5)\n"
    "v_mfma_f32_16x16x32_bf16 v[160:163], a[136:139], a[0:3], v[160:163]\n"
    "v_exp_f32_e64 v231, v231 clamp\n"
    "v_mfma_f32_16x16x32_bf16 v[160:163], a[140:143], a[4:7], v[160:163]\n"
    "ds_read_b64_tr_b16 a[152:153], v252 offset:512\n"
    "v_pk_fma_f32 v[224:225], v[224:225], s[42:43], v[244:245]\n"
    "v_mfma_f32_16x16x32_bf16 v[164:167], a[136:139], a[8:11], v[164:167]\n"
    "v_pk_fma_f32 v[226:227], v[226:227], s[42:43], v[244:245]\n"
    "v_pk_fma_f32 v[228:229], v[228:229], s[42:43], v[244:245]\n"
    "v_mfma_f32_16x16x32_bf16 v[164:167], a[140:143], a[12:15], v[164:167]\n"
    "ds_read_b64_tr_b16 a[154:155], v252 offset:1536\n"
    "v_pk_fma_f32 v[230:231], v[230:231], s[42:43], v[244:245]\n"
    "v_mfma_f32_16x16x32_bf16 v[168:171], a[136:139], a[16:19], v[168:171]\n"
    "v_rcp_f32 v224, v224\n"
    "v_mfma_f32_16x16x32_bf16 v[168:171], a[140:143], a[20:23], v[168:171]\n"
    "ds_read_b64_tr_b16 a[156:157], v252 offset:8704\n"
    "v_mfma_f32_16x16x32_bf16 v[172:175], a[136:139], a[24:27], v[172:175]\n"
    "v_rcp_f32 v225, v225\n"
    "v_mfma_f32_16x16x32_bf16 v[172:175], a[140:143], a[28:31], v[172:175]\n"
    "ds_read_b64_tr_b16 a[158:159], v252 offset:9728\n"
    "v_mfma_f32_16x16x32_bf16 v[176:179], a[136:139], a[32:35], v[176:179]\n"
    "v_rcp_f32 v226, v226\n"
    "v_mfma_f32_16x16x32_bf16 v[176:179], a[140:143], a[36:39], v[176:179]\n"
    "v_rcp_f32 v227, v227\n"
    "v_mfma_f32_16x16x32_bf16 v[180:183], a[136:139], a[40:43], v[180:183]\n"
    "v_rcp_f32 v228, v228\n"
    "v_mfma_f32_16x16x32_bf16 v[180:183], a[140:143], a[44:47], v[180:183]\n"
    "v_rcp_f32 v229, v229\n"
    "v_mfma_f32_16x16x32_bf16 v[184:187], a[136:139], a[48:51], v[184:187]\n"
    "v_rcp_f32 v230, v230\n"
    "v_mfma_f32_16x16x32_bf16 v[184:187], a[140:143], a[52:55], v[184:187]\n"
    "v_rcp_f32 v231, v231\n"
    "v_mfma_f32_16x16x32_bf16 v[188:191], a[136:139], a[56:59], v[188:191]\n"
    "v_pk_mul_f32 v[64:65], v[64:65], v[224:225]\n"
    "v_pk_mul_f32 v[66:67], v[66:67], v[226:227]\n"
    "v_mfma_f32_16x16x32_bf16 v[188:191], a[140:143], a[60:63], v[188:191]\n"
    "v_pk_mul_f32 v[96:97], v[96:97], v[228:229]\n"
    "v_pk_mul_f32 v[98:99], v[98:99], v[230:231]\n"
    "s_waitcnt lgkmcnt(5)\n"
    "v_mfma_f32_16x16x32_bf16 v[128:131], a[144:147], a[192:195], v[128:131]\n"
    "v_cvt_pk_bf16_f32 v64, v64, v65\n"
    "v_cvt_pk_bf16_f32 v65, v66, v67\n"
    "v_mfma_f32_16x16x32_bf16 v[128:131], a[148:151], a[196:199], v[128:131]\n"
    "ds_read_b64_tr_b16 a[128:129], v242 offset:768\n"
    "v_cvt_pk_bf16_f32 v66, v96, v97\n"
    "v_mfma_f32_16x16x32_bf16 v[132:135], a[144:147], a[200:203], v[132:135]\n"
    "v_cvt_pk_bf16_f32 v67, v98, v99\n"
    "v_pk_fma_f32 v[232:233], v[68:69], s[40:41], v[240:241]\n"
    "v_mfma_f32_16x16x32_bf16 v[132:135], a[148:151], a[204:207], v[132:135]\n"
    "ds_read_b64_tr_b16 a[130:131], v242 offset:1792\n"
    "v_pk_fma_f32 v[234:235], v[70:71], s[40:41], v[240:241]\n"
    "v_mfma_f32_16x16x32_bf16 v[136:139], a[144:147], a[208:211], v[136:139]\n"
    "v_pk_fma_f32 v[236:237], v[100:101], s[40:41], v[240:241]\n"
    "v_pk_fma_f32 v[238:239], v[102:103], s[40:41], v[240:241]\n"
    "v_mfma_f32_16x16x32_bf16 v[136:139], a[148:151], a[212:215], v[136:139]\n"
    "ds_read_b64_tr_b16 a[132:133], v242 offset:8960\n"
    "v_permlane16_swap_b32 v64, v66\n"
    "v_mfma_f32_16x16x32_bf16 v[140:143], a[144:147], a[216:219], v[140:143]\n"
    "v_permlane16_swap_b32 v65, v67\n"
    "ds_write_b128 v249, v[64:67] offset:64\n"
    "v_mfma_f32_16x16x32_bf16 v[140:143], a[148:151], a[220:223], v[140:143]\n"
    "ds_read_b64_tr_b16 a[134:135], v242 offset:9984\n"
    "ds_read_b128 v[0:3], v250 offset:0\n"
    "v_mfma_f32_16x16x32_bf16 v[144:147], a[144:147], a[224:227], v[144:147]\n"
    "ds_read_b128 v[64:67], v250 offset:1152\n"
    "v_mfma_f32_16x16x32_bf16 v[144:147], a[148:151], a[228:231], v[144:147]\n"
    "v_exp_f32_e64 v232, v232 clamp\n"
    "v_mfma_f32_16x16x32_bf16 v[148:151], a[144:147], a[232:235], v[148:151]\n"
    "v_exp_f32_e64 v233, v233 clamp\n"
    "v_mfma_f32_16x16x32_bf16 v[148:151], a[148:151], a[236:239], v[148:151]\n"
    "v_exp_f32_e64 v234, v234 clamp\n"
    "v_mfma_f32_16x16x32_bf16 v[152:155], a[144:147], a[240:243], v[152:155]\n"
    "v_exp_f32_e64 v235, v235 clamp\n"
    "v_mfma_f32_16x16x32_bf16 v[152:155], a[148:151], a[244:247], v[152:155]\n"
    "v_exp_f32_e64 v236, v236 clamp\n"
    "v_mfma_f32_16x16x32_bf16 v[156:159], a[144:147], a[248:251], v[156:159]\n"
    "v_exp_f32_e64 v237, v237 clamp\n"
    "v_mfma_f32_16x16x32_bf16 v[156:159], a[148:151], a[252:255], v[156:159]\n"
    "v_exp_f32_e64 v238, v238 clamp\n"
    "s_waitcnt lgkmcnt(7)\n"
    "v_mfma_f32_16x16x32_bf16 v[160:163], a[152:155], a[192:195], v[160:163]\n"
    "v_exp_f32_e64 v239, v239 clamp\n"
    "v_mfma_f32_16x16x32_bf16 v[160:163], a[156:159], a[196:199], v[160:163]\n"
    "ds_read_b64_tr_b16 a[136:137], v243 offset:768\n"
    "v_pk_fma_f32 v[232:233], v[232:233], s[42:43], v[244:245]\n"
    "v_mfma_f32_16x16x32_bf16 v[164:167], a[152:155], a[200:203], v[164:167]\n"
    "v_pk_fma_f32 v[234:235], v[234:235], s[42:43], v[244:245]\n"
    "v_pk_fma_f32 v[236:237], v[236:237], s[42:43], v[244:245]\n"
    "v_mfma_f32_16x16x32_bf16 v[164:167], a[156:159], a[204:207], v[164:167]\n"
    "ds_read_b64_tr_b16 a[138:139], v243 offset:1792\n"
    "v_pk_fma_f32 v[238:239], v[238:239], s[42:43], v[244:245]\n"
    "v_mfma_f32_16x16x32_bf16 v[168:171], a[152:155], a[208:211], v[168:171]\n"
    "v_rcp_f32 v232, v232\n"
    "v_mfma_f32_16x16x32_bf16 v[168:171], a[156:159], a[212:215], v[168:171]\n"
    "ds_read_b64_tr_b16 a[140:141], v243 offset:8960\n"
    "v_mfma_f32_16x16x32_bf16 v[172:175], a[152:155], a[216:219], v[172:175]\n"
    "v_rcp_f32 v233, v233\n"
    "v_mfma_f32_16x16x32_bf16 v[172:175], a[156:159], a[220:223], v[172:175]\n"
    "ds_read_b64_tr_b16 a[142:143], v243 offset:9984\n"
    "v_mfma_f32_16x16x32_bf16 v[176:179], a[152:155], a[224:227], v[176:179]\n"
    "v_rcp_f32 v234, v234\n"
    "v_mfma_f32_16x16x32_bf16 v[176:179], a[156:159], a[228:231], v[176:179]\n"
    "v_rcp_f32 v235, v235\n"
    "v_mfma_f32_16x16x32_bf16 v[180:183], a[152:155], a[232:235], v[180:183]\n"
    "v_rcp_f32 v236, v236\n"
    "v_mfma_f32_16x16x32_bf16 v[180:183], a[156:159], a[236:239], v[180:183]\n"
    "v_rcp_f32 v237, v237\n"
    "v_mfma_f32_16x16x32_bf16 v[184:187], a[152:155], a[240:243], v[184:187]\n"
    "v_rcp_f32 v238, v238\n"
    "v_mfma_f32_16x16x32_bf16 v[184:187], a[156:159], a[244:247], v[184:187]\n"
    "v_rcp_f32 v239, v239\n"
    "v_mfma_f32_16x16x32_bf16 v[188:191], a[152:155], a[248:251], v[188:191]\n"
    "s_waitcnt lgkmcnt(4)\n"
    "buffer_store_dwordx4 v[0:3], v248, s[96:99], 0 offen offset:0 sc0 sc1\n"
    "v_mfma_f32_16x16x32_bf16 v[188:191], a[156:159], a[252:255], v[188:191]\n"
    "buffer_store_dwordx4 v[64:67], v246, s[96:99], 0 offen offset:0 sc0 sc1\n"
    "v_pk_mul_f32 v[68:69], v[68:69], v[232:233]\n"
    "s_waitcnt lgkmcnt(6)\n"
    "v_mfma_f32_16x16x32_bf16 v[192:195], a[128:131], a[0:3], v[192:195]\n"
    "v_pk_mul_f32 v[70:71], v[70:71], v[234:235]\n"
    "v_pk_mul_f32 v[100:101], v[100:101], v[236:237]\n"
    "v_mfma_f32_16x16x32_bf16 v[192:195], a[132:135], a[4:7], v[192:195]\n"
    "ds_read_b64_tr_b16 a[144:145], v247 offset:768\n"
    "v_pk_mul_f32 v[102:103], v[102:103], v[238:239]\n"
    "v_mfma_f32_16x16x32_bf16 v[196:199], a[128:131], a[8:11], v[196:199]\n"
    "v_cvt_pk_bf16_f32 v68, v68, v69\n"
    "v_cvt_pk_bf16_f32 v69, v70, v71\n"
    "v_mfma_f32_16x16x32_bf16 v[196:199], a[132:135], a[12:15], v[196:199]\n"
    "ds_read_b64_tr_b16 a[146:147], v247 offset:1792\n"
    "v_cvt_pk_bf16_f32 v70, v100, v101\n"
    "v_mfma_f32_16x16x32_bf16 v[200:203], a[128:131], a[16:19], v[200:203]\n"
    "v_cvt_pk_bf16_f32 v71, v102, v103\n"
    "v_pk_fma_f32 v[224:225], v[8:9], s[40:41], v[240:241]\n"
    "v_mfma_f32_16x16x32_bf16 v[200:203], a[132:135], a[20:23], v[200:203]\n"
    "ds_read_b64_tr_b16 a[148:149], v247 offset:8960\n"
    "v_pk_fma_f32 v[226:227], v[10:11], s[40:41], v[240:241]\n"
    "v_mfma_f32_16x16x32_bf16 v[204:207], a[128:131], a[24:27], v[204:207]\n"
    "v_pk_fma_f32 v[228:229], v[40:41], s[40:41], v[240:241]\n"
    "v_pk_fma_f32 v[230:231], v[42:43], s[40:41], v[240:241]\n"
    "v_mfma_f32_16x16x32_bf16 v[204:207], a[132:135], a[28:31], v[204:207]\n"
    "ds_read_b64_tr_b16 a[150:151], v247 offset:9984\n"
    "v_permlane16_swap_b32 v68, v70\n"
    "v_mfma_f32_16x16x32_bf16 v[208:211], a[128:131], a[32:35], v[208:211]\n"
    "v_permlane16_swap_b32 v69, v71\n"
    "ds_write_b128 v249, v[68:71] offset:2368\n"
    "v_mfma_f32_16x16x32_bf16 v[208:211], a[132:135], a[36:39], v[208:211]\n"
    "ds_read_b128 v[4:7], v250 offset:2304\n"
    "ds_read_b128 v[68:71], v250 offset:3456\n"
    "v_mfma_f32_16x16x32_bf16 v[212:215], a[128:131], a[40:43], v[212:215]\n"
    "v_exp_f32_e64 v224, v224 clamp\n"
    "v_mfma_f32_16x16x32_bf16 v[212:215], a[132:135], a[44:47], v[212:215]\n"
    "v_exp_f32_e64 v225, v225 clamp\n"
    "v_mfma_f32_16x16x32_bf16 v[216:219], a[128:131], a[48:51], v[216:219]\n"
    "v_exp_f32_e64 v226, v226 clamp\n"
    "v_mfma_f32_16x16x32_bf16 v[216:219], a[132:135], a[52:55], v[216:219]\n"
    "v_exp_f32_e64 v227, v227 clamp\n"
    "v_mfma_f32_16x16x32_bf16 v[220:223], a[128:131], a[56:59], v[220:223]\n"
    "v_exp_f32_e64 v228, v228 clamp\n"
    "v_mfma_f32_16x16x32_bf16 v[220:223], a[132:135], a[60:63], v[220:223]\n"
    "v_exp_f32_e64 v229, v229 clamp\n"
    "s_waitcnt lgkmcnt(7)\n"
    "v_mfma_f32_16x16x32_bf16 a[160:163], a[136:139], a[0:3], a[160:163]\n"
    "v_exp_f32_e64 v230, v230 clamp\n"
    "v_mfma_f32_16x16x32_bf16 a[160:163], a[140:143], a[4:7], a[160:163]\n"
    "ds_read_b64_tr_b16 a[152:153], v252 offset:768\n"
    "v_mfma_f32_16x16x32_bf16 a[164:167], a[136:139], a[8:11], a[164:167]\n"
    "v_exp_f32_e64 v231, v231 clamp\n"
    "v_mfma_f32_16x16x32_bf16 a[164:167], a[140:143], a[12:15], a[164:167]\n"
    "ds_read_b64_tr_b16 a[154:155], v252 offset:1792\n"
    "v_pk_fma_f32 v[224:225], v[224:225], s[42:43], v[244:245]\n"
    "v_mfma_f32_16x16x32_bf16 a[168:171], a[136:139], a[16:19], a[168:171]\n"
    "v_pk_fma_f32 v[226:227], v[226:227], s[42:43], v[244:245]\n"
    "v_pk_fma_f32 v[228:229], v[228:229], s[42:43], v[244:245]\n"
    "v_mfma_f32_16x16x32_bf16 a[168:171], a[140:143], a[20:23], a[168:171]\n"
    "ds_read_b64_tr_b16 a[156:157], v252 offset:8960\n"
    "v_pk_fma_f32 v[230:231], v[230:231], s[42:43], v[244:245]\n"
    "v_mfma_f32_16x16x32_bf16 a[172:175], a[136:139], a[24:27], a[172:175]\n"
    "v_rcp_f32 v224, v224\n"
    "v_mfma_f32_16x16x32_bf16 a[172:175], a[140:143], a[28:31], a[172:175]\n"
    "ds_read_b64_tr_b16 a[158:159], v252 offset:9984\n"
    "v_mfma_f32_16x16x32_bf16 a[176:179], a[136:139], a[32:35], a[176:179]\n"
    "v_rcp_f32 v225, v225\n"
    "v_mfma_f32_16x16x32_bf16 a[176:179], a[140:143], a[36:39], a[176:179]\n"
    "v_rcp_f32 v226, v226\n"
    "v_mfma_f32_16x16x32_bf16 a[180:183], a[136:139], a[40:43], a[180:183]\n"
    "v_rcp_f32 v227, v227\n"
    "v_mfma_f32_16x16x32_bf16 a[180:183], a[140:143], a[44:47], a[180:183]\n"
    "v_rcp_f32 v228, v228\n"
    "v_mfma_f32_16x16x32_bf16 a[184:187], a[136:139], a[48:51], a[184:187]\n"
    "v_rcp_f32 v229, v229\n"
    "v_mfma_f32_16x16x32_bf16 a[184:187], a[140:143], a[52:55], a[184:187]\n"
    "v_rcp_f32 v230, v230\n"
    "v_mfma_f32_16x16x32_bf16 a[188:191], a[136:139], a[56:59], a[188:191]\n"
    "v_rcp_f32 v231, v231\n"
    "v_mfma_f32_16x16x32_bf16 a[188:191], a[140:143], a[60:63], a[188:191]\n"
    "s_waitcnt lgkmcnt(4)\n"
    "buffer_store_dwordx4 v[4:7], v248, s[96:99], s44 offen offset:0 sc0 sc1\n"
    "s_waitcnt lgkmcnt(7)\n"
    "v_mfma_f32_16x16x32_bf16 v[192:195], a[144:147], a[192:195], v[192:195]\n"
    "buffer_store_dwordx4 v[68:71], v246, s[96:99], s44 offen offset:0 sc0 sc1\n"
    "v_pk_mul_f32 v[8:9], v[8:9], v[224:225]\n"
    "v_mfma_f32_16x16x32_bf16 v[192:195], a[148:151], a[196:199], v[192:195]\n"
    "v_pk_mul_f32 v[10:11], v[10:11], v[226:227]\n"
    "v_pk_mul_f32 v[40:41], v[40:41], v[228:229]\n"
    "v_mfma_f32_16x16x32_bf16 v[196:199], a[144:147], a[200:203], v[196:199]\n"
    "v_pk_mul_f32 v[42:43], v[42:43], v[230:231]\n"
    "v_cvt_pk_bf16_f32 v8, v8, v9\n"
    "v_mfma_f32_16x16x32_bf16 v[196:199], a[148:151], a[204:207], v[196:199]\n"
    "v_cvt_pk_bf16_f32 v9, v10, v11\n"
    "v_cvt_pk_bf16_f32 v10, v40, v41\n"
    "v_mfma_f32_16x16x32_bf16 v[200:203], a[144:147], a[208:211], v[200:203]\n"
    "v_cvt_pk_bf16_f32 v11, v42, v43\n"
    "v_pk_fma_f32 v[232:233], v[72:73], s[40:41], v[240:241]\n"
    "v_mfma_f32_16x16x32_bf16 v[200:203], a[148:151], a[212:215], v[200:203]\n"
    "v_pk_fma_f32 v[234:235], v[74:75], s[40:41], v[240:241]\n"
    "v_pk_fma_f32 v[236:237], v[104:105], s[40:41], v[240:241]\n"
    "v_mfma_f32_16x16x32_bf16 v[204:207], a[144:147], a[216:219], v[204:207]\n"
    "v_pk_fma_f32 v[238:239], v[106:107], s[40:41], v[240:241]\n"
    "v_permlane16_swap_b32 v8, v10\n"
    "v_mfma_f32_16x16x32_bf16 v[204:207], a[148:151], a[220:223], v[204:207]\n"
    "v_permlane16_swap_b32 v9, v11\n"
    "ds_write_b128 v249, v[8:11] offset:4608\n"
    "v_mfma_f32_16x16x32_bf16 v[208:211], a[144:147], a[224:227], v[208:211]\n"
    "v_exp_f32_e64 v232, v232 clamp\n"
    "v_mfma_f32_16x16x32_bf16 v[208:211], a[148:151], a[228:231], v[208:211]\n"
    "v_exp_f32_e64 v233, v233 clamp\n"
    "v_mfma_f32_16x16x32_bf16 v[212:215], a[144:147], a[232:235], v[212:215]\n"
    "v_exp_f32_e64 v234, v234 clamp\n"
    "v_mfma_f32_16x16x32_bf16 v[212:215], a[148:151], a[236:239], v[212:215]\n"
    "v_exp_f32_e64 v235, v235 clamp\n"
    "v_mfma_f32_16x16x32_bf16 v[216:219], a[144:147], a[240:243], v[216:219]\n"
    "v_exp_f32_e64 v236, v236 clamp\n"
    "v_mfma_f32_16x16x32_bf16 v[216:219], a[148:151], a[244:247], v[216:219]\n"
    "v_exp_f32_e64 v237, v237 clamp\n"
    "v_mfma_f32_16x16x32_bf16 v[220:223], a[144:147], a[248:251], v[220:223]\n"
    "v_exp_f32_e64 v238, v238 clamp\n"
    "v_mfma_f32_16x16x32_bf16 v[220:223], a[148:151], a[252:255], v[220:223]\n"
    "v_exp_f32_e64 v239, v239 clamp\n"
    "s_waitcnt lgkmcnt(1)\n"
    "v_mfma_f32_16x16x32_bf16 a[160:163], a[152:155], a[192:195], a[160:163]\n"
    "v_pk_fma_f32 v[232:233], v[232:233], s[42:43], v[244:245]\n"
    "v_pk_fma_f32 v[234:235], v[234:235], s[42:43], v[244:245]\n"
    "v_mfma_f32_16x16x32_bf16 a[160:163], a[156:159], a[196:199], a[160:163]\n"
    "v_pk_fma_f32 v[236:237], v[236:237], s[42:43], v[244:245]\n"
    "v_pk_fma_f32 v[238:239], v[238:239], s[42:43], v[244:245]\n"
    "v_mfma_f32_16x16x32_bf16 a[164:167], a[152:155], a[200:203], a[164:167]\n"
    "v_rcp_f32 v232, v232\n"
    "v_mfma_f32_16x16x32_bf16 a[164:167], a[156:159], a[204:207], a[164:167]\n"
    "v_rcp_f32 v233, v233\n"
    "v_mfma_f32_16x16x32_bf16 a[168:171], a[152:155], a[208:211], a[168:171]\n"
    "v_rcp_f32 v234, v234\n"
    "v_mfma_f32_16x16x32_bf16 a[168:171], a[156:159], a[212:215], a[168:171]\n"
    "v_rcp_f32 v235, v235\n"
    "v_mfma_f32_16x16x32_bf16 a[172:175], a[152:155], a[216:219], a[172:175]\n"
    "v_rcp_f32 v236, v236\n"
    "v_mfma_f32_16x16x32_bf16 a[172:175], a[156:159], a[220:223], a[172:175]\n"
    "v_rcp_f32 v237, v237\n"
    "v_mfma_f32_16x16x32_bf16 a[176:179], a[152:155], a[224:227], a[176:179]\n"
    "v_rcp_f32 v238, v238\n"
    "v_mfma_f32_16x16x32_bf16 a[176:179], a[156:159], a[228:231], a[176:179]\n"
    "v_rcp_f32 v239, v239\n"
    "v_mfma_f32_16x16x32_bf16 a[180:183], a[152:155], a[232:235], a[180:183]\n"
    "v_pk_mul_f32 v[72:73], v[72:73], v[232:233]\n"
    "v_pk_mul_f32 v[74:75], v[74:75], v[234:235]\n"
    "v_mfma_f32_16x16x32_bf16 a[180:183], a[156:159], a[236:239], a[180:183]\n"
    "v_pk_mul_f32 v[104:105], v[104:105], v[236:237]\n"
    "v_pk_mul_f32 v[106:107], v[106:107], v[238:239]\n"
    "v_mfma_f32_16x16x32_bf16 a[184:187], a[152:155], a[240:243], a[184:187]\n"
    "v_cvt_pk_bf16_f32 v72, v72, v73\n"
    "v_cvt_pk_bf16_f32 v73, v74, v75\n"
    "v_mfma_f32_16x16x32_bf16 a[184:187], a[156:159], a[244:247], a[184:187]\n"
    "v_cvt_pk_bf16_f32 v74, v104, v105\n"
    "v_cvt_pk_bf16_f32 v75, v106, v107\n"
    "v_mfma_f32_16x16x32_bf16 a[188:191], a[152:155], a[248:251], a[188:191]\n"
    "v_pk_fma_f32 v[224:225], v[12:13], s[40:41], v[240:241]\n"
    "v_pk_fma_f32 v[226:227], v[14:15], s[40:41], v[240:241]\n"
    "v_mfma_f32_16x16x32_bf16 a[188:191], a[156:159], a[252:255], a[188:191]\n"
    "v_pk_fma_f32 v[228:229], v[44:45], s[40:41], v[240:241]\n"
    "v_pk_fma_f32 v[230:231], v[46:47], s[40:41], v[240:241]\n"
    "s_nop 7\n"
    "v_permlane16_swap_b32 v72, v74\n"
    "v_permlane16_swap_b32 v73, v75\n"
    "ds_write_b128 v249, v[72:75] offset:4672\n"
    "ds_read_b128 v[8:11], v250 offset:4608\n"
    "ds_read_b128 v[72:75], v250 offset:5760\n"
    "v_exp_f32_e64 v224, v224 clamp\n"
    "v_exp_f32_e64 v225, v225 clamp\n"
    "v_exp_f32_e64 v226, v226 clamp\n"
    "v_exp_f32_e64 v227, v227 clamp\n"
    "v_exp_f32_e64 v228, v228 clamp\n"
    "v_exp_f32_e64 v229, v229 clamp\n"
    "v_exp_f32_e64 v230, v230 clamp\n"
    "v_exp_f32_e64 v231, v231 clamp\n"
    "v_pk_fma_f32 v[224:225], v[224:225], s[42:43], v[244:245]\n"
    "v_pk_fma_f32 v[226:227], v[226:227], s[42:43], v[244:245]\n"
    "v_pk_fma_f32 v[228:229], v[228:229], s[42:43], v[244:245]\n"
    "v_pk_fma_f32 v[230:231], v[230:231], s[42:43], v[244:245]\n"
    "v_rcp_f32 v224, v224\n"
    "v_rcp_f32 v225, v225\n"
    "v_rcp_f32 v226, v226\n"
    "v_rcp_f32 v227, v227\n"
    "v_rcp_f32 v228, v228\n"
    "v_rcp_f32 v229, v229\n"
    "v_rcp_f32 v230, v230\n"
    "v_rcp_f32 v231, v231\n"
    "s_waitcnt lgkmcnt(0)\n"
    "buffer_store_dwordx4 v[8:11], v248, s[96:99], s45 offen offset:0 sc0 sc1\n"
    "buffer_store_dwordx4 v[72:75], v246, s[96:99], s45 offen offset:0 sc0 sc1\n"
    "v_pk_mul_f32 v[12:13], v[12:13], v[224:225]\n"
    "v_pk_mul_f32 v[14:15], v[14:15], v[226:227]\n"
    "v_pk_mul_f32 v[44:45], v[44:45], v[228:229]\n"
    "v_pk_mul_f32 v[46:47], v[46:47], v[230:231]\n"
    "v_cvt_pk_bf16_f32 v12, v12, v13\n"
    "v_cvt_pk_bf16_f32 v13, v14, v15\n"
    "v_cvt_pk_bf16_f32 v14, v44, v45\n"
    "v_cvt_pk_bf16_f32 v15, v46, v47\n"
    "v_pk_fma_f32 v[232:233], v[76:77], s[40:41], v[240:241]\n"
    "v_pk_fma_f32 v[234:235], v[78:79], s[40:41], v[240:241]\n"
    "v_pk_fma_f32 v[236:237], v[108:109], s[40:41], v[240:241]\n"
    "v_pk_fma_f32 v[238:239], v[110:111], s[40:41], v[240:241]\n"
    "v_permlane16_swap_b32 v12, v14\n"
    "v_permlane16_swap_b32 v13, v15\n"
    "ds_write_b128 v249, v[12:15] offset:6912\n"
    "v_exp_f32_e64 v232, v232 clamp\n"
    "v_exp_f32_e64 v233, v233 clamp\n"
    "v_exp_f32_e64 v234, v234 clamp\n"
    "v_exp_f32_e64 v235, v235 clamp\n"
    "v_exp_f32_e64 v236, v236 clamp\n"
    "v_exp_f32_e64 v237, v237 clamp\n"
    "v_exp_f32_e64 v238, v238 clamp\n"
    "v_exp_f32_e64 v239, v239 clamp\n"
    "v_pk_fma_f32 v[232:233], v[232:233], s[42:43], v[244:245]\n"
    "v_pk_fma_f32 v[234:235], v[234:235], s[42:43], v[244:245]\n"
    "v_pk_fma_f32 v[236:237], v[236:237], s[42:43], v[244:245]\n"
    "v_pk_fma_f32 v[238:239], v[238:239], s[42:43], v[244:245]\n"
    "v_rcp_f32 v232, v232\n"
    "v_rcp_f32 v233, v233\n"
    "v_rcp_f32 v234, v234\n"
    "v_rcp_f32 v235, v235\n"
    "v_rcp_f32 v236, v236\n"
    "v_rcp_f32 v237, v237\n"
    "v_rcp_f32 v238, v238\n"
    "v_rcp_f32 v239, v239\n"
    "v_pk_mul_f32 v[76:77], v[76:77], v[232:233]\n"
    "v_pk_mul_f32 v[78:79], v[78:79], v[234:235]\n"
    "v_pk_mul_f32 v[108:109], v[108:109], v[236:237]\n"
    "v_pk_mul_f32 v[110:111], v[110:111], v[238:239]\n"
    "v_cvt_pk_bf16_f32 v76, v76, v77\n"
    "v_cvt_pk_bf16_f32 v77, v78, v79\n"
    "v_cvt_pk_bf16_f32 v78, v108, v109\n"
    "v_cvt_pk_bf16_f32 v79, v110, v111\n"
    "v_pk_fma_f32 v[224:225], v[16:17], s[40:41], v[240:241]\n"
    "v_pk_fma_f32 v[226:227], v[18:19], s[40:41], v[240:241]\n"
    "v_pk_fma_f32 v[228:229], v[48:49], s[40:41], v[240:241]\n"
    "v_pk_fma_f32 v[230:231], v[50:51], s[40:41], v[240:241]\n"
    "v_permlane16_swap_b32 v76, v78\n"
    "v_permlane16_swap_b32 v77, v79\n"
    "ds_write_b128 v249, v[76:79] offset:6976\n"
    "ds_read_b128 v[12:15], v250 offset:6912\n"
    "ds_read_b128 v[76:79], v250 offset:8064\n"
    "v_exp_f32_e64 v224, v224 clamp\n"
    "v_exp_f32_e64 v225, v225 clamp\n"
    "v_exp_f32_e64 v226, v226 clamp\n"
    "v_exp_f32_e64 v227, v227 clamp\n"
    "v_exp_f32_e64 v228, v228 clamp\n"
    "v_exp_f32_e64 v229, v229 clamp\n"
    "v_exp_f32_e64 v230, v230 clamp\n"
    "v_exp_f32_e64 v231, v231 clamp\n"
    "v_pk_fma_f32 v[224:225], v[224:225], s[42:43], v[244:245]\n"
    "v_pk_fma_f32 v[226:227], v[226:227], s[42:43], v[244:245]\n"
    "v_pk_fma_f32 v[228:229], v[228:229], s[42:43], v[244:245]\n"
    "v_pk_fma_f32 v[230:231], v[230:231], s[42:43], v[244:245]\n"
    "v_rcp_f32 v224, v224\n"
    "v_rcp_f32 v225, v225\n"
    "v_rcp_f32 v226, v226\n"
    "v_rcp_f32 v227, v227\n"
    "v_rcp_f32 v228, v228\n"
    "v_rcp_f32 v229, v229\n"
    "v_rcp_f32 v230, v230\n"
    "v_rcp_f32 v231, v231\n"
    "s_waitcnt lgkmcnt(0)\n"
    "buffer_store_dwordx4 v[12:15], v248, s[96:99], s46 offen offset:0 sc0 sc1\n"
    "buffer_store_dwordx4 v[76:79], v246, s[96:99], s46 offen offset:0 sc0 sc1\n"
    "v_pk_mul_f32 v[16:17], v[16:17], v[224:225]\n"
    "v_pk_mul_f32 v[18:19], v[18:19], v[226:227]\n"
    "v_pk_mul_f32 v[48:49], v[48:49], v[228:229]\n"
    "v_pk_mul_f32 v[50:51], v[50:51], v[230:231]\n"
    "v_cvt_pk_bf16_f32 v16, v16, v17\n"
    "v_cvt_pk_bf16_f32 v17, v18, v19\n"
    "v_cvt_pk_bf16_f32 v18, v48, v49\n"
    "v_cvt_pk_bf16_f32 v19, v50, v51\n"
    "v_pk_fma_f32 v[232:233], v[80:81], s[40:41], v[240:241]\n"
    "v_pk_fma_f32 v[234:235], v[82:83], s[40:41], v[240:241]\n"
    "v_pk_fma_f32 v[236:237], v[112:113], s[40:41], v[240:241]\n"
    "v_pk_fma_f32 v[238:239], v[114:115], s[40:41], v[240:241]\n"
    "v_permlane16_swap_b32 v16, v18\n"
    "v_permlane16_swap_b32 v17, v19\n"
    "ds_write_b128 v249, v[16:19] offset:0\n"
    "v_exp_f32_e64 v232, v232 clamp\n"
    "v_exp_f32_e64 v233, v233 clamp\n"
    "v_exp_f32_e64 v234, v234 clamp\n"
    "v_exp_f32_e64 v235, v235 clamp\n"
    "v_exp_f32_e64 v236, v236 clamp\n"
    "v_exp_f32_e64 v237, v237 clamp\n"
    "v_exp_f32_e64 v238, v238 clamp\n"
    "v_exp_f32_e64 v239, v239 clamp\n"
    "v_pk_fma_f32 v[232:233], v[232:233], s[42:43], v[244:245]\n"
    "v_pk_fma_f32 v[234:235], v[234:235], s[42:43], v[244:245]\n"
    "v_pk_fma_f32 v[236:237], v[236:237], s[42:43], v[244:245]\n"
    "v_pk_fma_f32 v[238:239], v[238:239], s[42:43], v[244:245]\n"
    "v_rcp_f32 v232, v232\n"
    "v_rcp_f32 v233, v233\n"
    "v_rcp_f32 v234, v234\n"
    "v_rcp_f32 v235, v235\n"
    "v_rcp_f32 v236, v236\n"
    "v_rcp_f32 v237, v237\n"
    "v_rcp_f32 v238, v238\n"
    "v_rcp_f32 v239, v239\n"
    "v_pk_mul_f32 v[80:81], v[80:81], v[232:233]\n"
    "v_pk_mul_f32 v[82:83], v[82:83], v[234:235]\n"
    "v_pk_mul_f32 v[112:113], v[112:113], v[236:237]\n"
    "v_pk_mul_f32 v[114:115], v[114:115], v[238:239]\n"
    "v_cvt_pk_bf16_f32 v80, v80, v81\n"
    "v_cvt_pk_bf16_f32 v81, v82, v83\n"
    "v_cvt_pk_bf16_f32 v82, v112, v113\n"
    "v_cvt_pk_bf16_f32 v83, v114, v115\n"
    "v_pk_fma_f32 v[224:225], v[20:21], s[40:41], v[240:241]\n"
    "v_pk_fma_f32 v[226:227], v[22:23], s[40:41], v[240:241]\n"
    "v_pk_fma_f32 v[228:229], v[52:53], s[40:41], v[240:241]\n"
    "v_pk_fma_f32 v[230:231], v[54:55], s[40:41], v[240:241]\n"
    "v_permlane16_swap_b32 v80, v82\n"
    "v_permlane16_swap_b32 v81, v83\n"
    "ds_write_b128 v249, v[80:83] offset:64\n"
    "ds_read_b128 v[16:19], v250 offset:0\n"
    "ds_read_b128 v[80:83], v250 offset:1152\n"
    "v_exp_f32_e64 v224, v224 clamp\n"
    "v_exp_f32_e64 v225, v225 clamp\n"
    "v_exp_f32_e64 v226, v226 clamp\n"
    "v_exp_f32_e64 v227, v227 clamp\n"
    "v_exp_f32_e64 v228, v228 clamp\n"
    "v_exp_f32_e64 v229, v229 clamp\n"
    "v_exp_f32_e64 v230, v230 clamp\n"
    "v_exp_f32_e64 v231, v231 clamp\n"
    "v_pk_fma_f32 v[224:225], v[224:225], s[42:43], v[244:245]\n"
    "v_pk_fma_f32 v[226:227], v[226:227], s[42:43], v[244:245]\n"
    "v_pk_fma_f32 v[228:229], v[228:229], s[42:43], v[244:245]\n"
    "v_pk_fma_f32 v[230:231], v[230:231], s[42:43], v[244:245]\n"
    "v_rcp_f32 v224, v224\n"
    "v_rcp_f32 v225, v225\n"
    "v_rcp_f32 v226, v226\n"
    "v_rcp_f32 v227, v227\n"
    "v_rcp_f32 v228, v228\n"
    "v_rcp_f32 v229, v229\n"
    "v_rcp_f32 v230, v230\n"
    "v_rcp_f32 v231, v231\n"
    "s_waitcnt lgkmcnt(0)\n"
    "buffer_store_dwordx4 v[16:19], v248, s[96:99], s47 offen offset:0 sc0 sc1\n"
    "buffer_store_dwordx4 v[80:83], v246, s[96:99], s47 offen offset:0 sc0 sc1\n"
    "v_pk_mul_f32 v[20:21], v[20:21], v[224:225]\n"
    "v_pk_mul_f32 v[22:23], v[22:23], v[226:227]\n"
    "v_pk_mul_f32 v[52:53], v[52:53], v[228:229]\n"
    "v_pk_mul_f32 v[54:55], v[54:55], v[230:231]\n"
    "v_cvt_pk_bf16_f32 v20, v20, v21\n"
    "v_cvt_pk_bf16_f32 v21, v22, v23\n"
    "v_cvt_pk_bf16_f32 v22, v52, v53\n"
    "v_cvt_pk_bf16_f32 v23, v54, v55\n"
    "v_pk_fma_f32 v[232:233], v[84:85], s[40:41], v[240:241]\n"
    "v_pk_fma_f32 v[234:235], v[86:87], s[40:41], v[240:241]\n"
    "v_pk_fma_f32 v[236:237], v[116:117], s[40:41], v[240:241]\n"
    "v_pk_fma_f32 v[238:239], v[118:119], s[40:41], v[240:241]\n"
    "v_permlane16_swap_b32 v20, v22\n"
    "v_permlane16_swap_b32 v21, v23\n"
    "ds_write_b128 v249, v[20:23] offset:2304\n"
    "v_exp_f32_e64 v232, v232 clamp\n"
    "v_exp_f32_e64 v233, v233 clamp\n"
    "v_exp_f32_e64 v234, v234 clamp\n"
    "v_exp_f32_e64 v235, v235 clamp\n"
    "v_exp_f32_e64 v236, v236 clamp\n"
    "v_exp_f32_e64 v237, v237 clamp\n"
    "v_exp_f32_e64 v238, v238 clamp\n"
    "v_exp_f32_e64 v239, v239 clamp\n"
    "v_pk_fma_f32 v[232:233], v[232:233], s[42:43], v[244:245]\n"
    "v_pk_fma_f32 v[234:235], v[234:235], s[42:43], v[244:245]\n"
    "v_pk_fma_f32 v[236:237], v[236:237], s[42:43], v[244:245]\n"
    "v_pk_fma_f32 v[238:239], v[238:239], s[42:43], v[244:245]\n"
    "v_rcp_f32 v232, v232\n"
    "v_rcp_f32 v233, v233\n"
    "v_rcp_f32 v234, v234\n"
    "v_rcp_f32 v235, v235\n"
    "v_rcp_f32 v236, v236\n"
    "v_rcp_f32 v237, v237\n"
    "v_rcp_f32 v238, v238\n"
    "v_rcp_f32 v239, v239\n"
    "v_pk_mul_f32 v[84:85], v[84:85], v[232:233]\n"
    "v_pk_mul_f32 v[86:87], v[86:87], v[234:235]\n"
    "v_pk_mul_f32 v[116:117], v[116:117], v[236:237]\n"
    "v_pk_mul_f32 v[118:119], v[118:119], v[238:239]\n"
    "v_cvt_pk_bf16_f32 v84, v84, v85\n"
    "v_cvt_pk_bf16_f32 v85, v86, v87\n"
    "v_cvt_pk_bf16_f32 v86, v116, v117\n"
    "v_cvt_pk_bf16_f32 v87, v118, v119\n"
    "v_pk_fma_f32 v[224:225], v[24:25], s[40:41], v[240:241]\n"
    "v_pk_fma_f32 v[226:227], v[26:27], s[40:41], v[240:241]\n"
    "v_pk_fma_f32 v[228:229], v[56:57], s[40:41], v[240:241]\n"
    "v_pk_fma_f32 v[230:231], v[58:59], s[40:41], v[240:241]\n"
    "v_permlane16_swap_b32 v84, v86\n"
    "v_permlane16_swap_b32 v85, v87\n"
    "ds_write_b128 v249, v[84:87] offset:2368\n"
    "ds_read_b128 v[20:23], v250 offset:2304\n"
    "ds_read_b128 v[84:87], v250 offset:3456\n"
    "v_exp_f32_e64 v224, v224 clamp\n"
    "v_exp_f32_e64 v225, v225 clamp\n"
    "v_exp_f32_e64 v226, v226 clamp\n"
    "v_exp_f32_e64 v227, v227 clamp\n"
    "v_exp_f32_e64 v228, v228 clamp\n"
    "v_exp_f32_e64 v229, v229 clamp\n"
    "v_exp_f32_e64 v230, v230 clamp\n"
    "v_exp_f32_e64 v231, v231 clamp\n"
    "v_pk_fma_f32 v[224:225], v[224:225], s[42:43], v[244:245]\n"
    "v_pk_fma_f32 v[226:227], v[226:227], s[42:43], v[244:245]\n"
    "v_pk_fma_f32 v[228:229], v[228:229], s[42:43], v[244:245]\n"
    "v_pk_fma_f32 v[230:231], v[230:231], s[42:43], v[244:245]\n"
    "v_rcp_f32 v224, v224\n"
    "v_rcp_f32 v225, v225\n"
    "v_rcp_f32 v226, v226\n"
    "v_rcp_f32 v227, v227\n"
    "v_rcp_f32 v228, v228\n"
    "v_rcp_f32 v229, v229\n"
    "v_rcp_f32 v230, v230\n"
    "v_rcp_f32 v231, v231\n"
    "s_waitcnt lgkmcnt(0)\n"
    "buffer_store_dwordx4 v[20:23], v248, s[96:99], s48 offen offset:0 sc0 sc1\n"
    "buffer_store_dwordx4 v[84:87], v246, s[96:99], s48 offen offset:0 sc0 sc1\n"
    "v_pk_mul_f32 v[24:25], v[24:25], v[224:225]\n"
    "v_pk_mul_f32 v[26:27], v[26:27], v[226:227]\n"
    "v_pk_mul_f32 v[56:57], v[56:57], v[228:229]\n"
    "v_pk_mul_f32 v[58:59], v[58:59], v[230:231]\n"
    "v_cvt_pk_bf16_f32 v24, v24, v25\n"
    "v_cvt_pk_bf16_f32 v25, v26, v27\n"
    "v_cvt_pk_bf16_f32 v26, v56, v57\n"
    "v_cvt_pk_bf16_f32 v27, v58, v59\n"
    "v_pk_fma_f32 v[232:233], v[88:89], s[40:41], v[240:241]\n"
    "v_pk_fma_f32 v[234:235], v[90:91], s[40:41], v[240:241]\n"
    "v_pk_fma_f32 v[236:237], v[120:121], s[40:41], v[240:241]\n"
    "v_pk_fma_f32 v[238:239], v[122:123], s[40:41], v[240:241]\n"
    "v_permlane16_swap_b32 v24, v26\n"
    "v_permlane16_swap_b32 v25, v27\n"
    "ds_write_b128 v249, v[24:27] offset:4608\n"
    "v_exp_f32_e64 v232, v232 clamp\n"
    "v_exp_f32_e64 v233, v233 clamp\n"
    "v_exp_f32_e64 v234, v234 clamp\n"
    "v_exp_f32_e64 v235, v235 clamp\n"
    "v_exp_f32_e64 v236, v236 clamp\n"
    "v_exp_f32_e64 v237, v237 clamp\n"
    "v_exp_f32_e64 v238, v238 clamp\n"
    "v_exp_f32_e64 v239, v239 clamp\n"
    "v_pk_fma_f32 v[232:233], v[232:233], s[42:43], v[244:245]\n"
    "v_pk_fma_f32 v[234:235], v[234:235], s[42:43], v[244:245]\n"
    "v_pk_fma_f32 v[236:237], v[236:237], s[42:43], v[244:245]\n"
    "v_pk_fma_f32 v[238:239], v[238:239], s[42:43], v[244:245]\n"
    "v_rcp_f32 v232, v232\n"
    "v_rcp_f32 v233, v233\n"
    "v_rcp_f32 v234, v234\n"
    "v_rcp_f32 v235, v235\n"
    "v_rcp_f32 v236, v236\n"
    "v_rcp_f32 v237, v237\n"
    "v_rcp_f32 v238, v238\n"
    "v_rcp_f32 v239, v239\n"
    "v_pk_mul_f32 v[88:89], v[88:89], v[232:233]\n"
    "v_pk_mul_f32 v[90:91], v[90:91], v[234:235]\n"
    "v_pk_mul_f32 v[120:121], v[120:121], v[236:237]\n"
    "v_pk_mul_f32 v[122:123], v[122:123], v[238:239]\n"
    "v_cvt_pk_bf16_f32 v88, v88, v89\n"
    "v_cvt_pk_bf16_f32 v89, v90, v91\n"
    "v_cvt_pk_bf16_f32 v90, v120, v121\n"
    "v_cvt_pk_bf16_f32 v91, v122, v123\n"
    "v_pk_fma_f32 v[224:225], v[28:29], s[40:41], v[240:241]\n"
    "v_pk_fma_f32 v[226:227], v[30:31], s[40:41], v[240:241]\n"
    "v_pk_fma_f32 v[228:229], v[60:61], s[40:41], v[240:241]\n"
    "v_pk_fma_f32 v[230:231], v[62:63], s[40:41], v[240:241]\n"
    "v_permlane16_swap_b32 v88, v90\n"
    "v_permlane16_swap_b32 v89, v91\n"
    "ds_write_b128 v249, v[88:91] offset:4672\n"
    "ds_read_b128 v[24:27], v250 offset:4608\n"
    "ds_read_b128 v[88:91], v250 offset:5760\n"
    "v_exp_f32_e64 v224, v224 clamp\n"
    "v_exp_f32_e64 v225, v225 clamp\n"
    "v_exp_f32_e64 v226, v226 clamp\n"
    "v_exp_f32_e64 v227, v227 clamp\n"
    "v_exp_f32_e64 v228, v228 clamp\n"
    "v_exp_f32_e64 v229, v229 clamp\n"
    "v_exp_f32_e64 v230, v230 clamp\n"
    "v_exp_f32_e64 v231, v231 clamp\n"
    "v_pk_fma_f32 v[224:225], v[224:225], s[42:43], v[244:245]\n"
    "v_pk_fma_f32 v[226:227], v[226:227], s[42:43], v[244:245]\n"
    "v_pk_fma_f32 v[228:229], v[228:229], s[42:43], v[244:245]\n"
    "v_pk_fma_f32 v[230:231], v[230:231], s[42:43], v[244:245]\n"
    "v_rcp_f32 v224, v224\n"
    "v_rcp_f32 v225, v225\n"
    "v_rcp_f32 v226, v226\n"
    "v_rcp_f32 v227, v227\n"
    "v_rcp_f32 v228, v228\n"
    "v_rcp_f32 v229, v229\n"
    "v_rcp_f32 v230, v230\n"
    "v_rcp_f32 v231, v231\n"
    "s_waitcnt lgkmcnt(0)\n"
    "buffer_store_dwordx4 v[24:27], v248, s[96:99], s49 offen offset:0 sc0 sc1\n"
    "buffer_store_dwordx4 v[88:91], v246, s[96:99], s49 offen offset:0 sc0 sc1\n"
    "v_pk_mul_f32 v[28:29], v[28:29], v[224:225]\n"
    "v_pk_mul_f32 v[30:31], v[30:31], v[226:227]\n"
    "v_pk_mul_f32 v[60:61], v[60:61], v[228:229]\n"
    "v_pk_mul_f32 v[62:63], v[62:63], v[230:231]\n"
    "v_cvt_pk_bf16_f32 v28, v28, v29\n"
    "v_cvt_pk_bf16_f32 v29, v30, v31\n"
    "v_cvt_pk_bf16_f32 v30, v60, v61\n"
    "v_cvt_pk_bf16_f32 v31, v62, v63\n"
    "v_pk_fma_f32 v[232:233], v[92:93], s[40:41], v[240:241]\n"
    "v_pk_fma_f32 v[234:235], v[94:95], s[40:41], v[240:241]\n"
    "v_pk_fma_f32 v[236:237], v[124:125], s[40:41], v[240:241]\n"
    "v_pk_fma_f32 v[238:239], v[126:127], s[40:41], v[240:241]\n"
    "v_permlane16_swap_b32 v28, v30\n"
    "v_permlane16_swap_b32 v29, v31\n"
    "ds_write_b128 v249, v[28:31] offset:6912\n"
    "v_exp_f32_e64 v232, v232 clamp\n"
    "v_exp_f32_e64 v233, v233 clamp\n"
    "v_exp_f32_e64 v234, v234 clamp\n"
    "v_exp_f32_e64 v235, v235 clamp\n"
    "v_exp_f32_e64 v236, v236 clamp\n"
    "v_exp_f32_e64 v237, v237 clamp\n"
    "v_exp_f32_e64 v238, v238 clamp\n"
    "v_exp_f32_e64 v239, v239 clamp\n"
    "v_pk_fma_f32 v[232:233], v[232:233], s[42:43], v[244:245]\n"
    "v_pk_fma_f32 v[234:235], v[234:235], s[42:43], v[244:245]\n"
    "v_pk_fma_f32 v[236:237], v[236:237], s[42:43], v[244:245]\n"
    "v_pk_fma_f32 v[238:239], v[238:239], s[42:43], v[244:245]\n"
    "v_rcp_f32 v232, v232\n"
    "v_rcp_f32 v233, v233\n"
    "v_rcp_f32 v234, v234\n"
    "v_rcp_f32 v235, v235\n"
    "v_rcp_f32 v236, v236\n"
    "v_rcp_f32 v237, v237\n"
    "v_rcp_f32 v238, v238\n"
    "v_rcp_f32 v239, v239\n"
    "v_pk_mul_f32 v[92:93], v[92:93], v[232:233]\n"
    "v_pk_mul_f32 v[94:95], v[94:95], v[234:235]\n"
    "v_pk_mul_f32 v[124:125], v[124:125], v[236:237]\n"
    "v_pk_mul_f32 v[126:127], v[126:127], v[238:239]\n"
    "v_cvt_pk_bf16_f32 v92, v92, v93\n"
    "v_cvt_pk_bf16_f32 v93, v94, v95\n"
    "v_cvt_pk_bf16_f32 v94, v124, v125\n"
    "v_cvt_pk_bf16_f32 v95, v126, v127\n"
    "v_pk_fma_f32 v[224:225], v[128:129], s[40:41], v[240:241]\n"
    "v_pk_fma_f32 v[226:227], v[130:131], s[40:41], v[240:241]\n"
    "v_pk_fma_f32 v[228:229], v[160:161], s[40:41], v[240:241]\n"
    "v_pk_fma_f32 v[230:231], v[162:163], s[40:41], v[240:241]\n"
    "v_permlane16_swap_b32 v92, v94\n"
    "v_permlane16_swap_b32 v93, v95\n"
    "ds_write_b128 v249, v[92:95] offset:6976\n"
    "ds_read_b128 v[28:31], v250 offset:6912\n"
    "ds_read_b128 v[92:95], v250 offset:8064\n"
    "v_exp_f32_e64 v224, v224 clamp\n"
    "v_exp_f32_e64 v225, v225 clamp\n"
    "v_exp_f32_e64 v226, v226 clamp\n"
    "v_exp_f32_e64 v227, v227 clamp\n"
    "v_exp_f32_e64 v228, v228 clamp\n"
    "v_exp_f32_e64 v229, v229 clamp\n"
    "v_exp_f32_e64 v230, v230 clamp\n"
    "v_exp_f32_e64 v231, v231 clamp\n"
    "v_pk_fma_f32 v[224:225], v[224:225], s[42:43], v[244:245]\n"
    "v_pk_fma_f32 v[226:227], v[226:227], s[42:43], v[244:245]\n"
    "v_pk_fma_f32 v[228:229], v[228:229], s[42:43], v[244:245]\n"
    "v_pk_fma_f32 v[230:231], v[230:231], s[42:43], v[244:245]\n"
    "v_rcp_f32 v224, v224\n"
    "v_rcp_f32 v225, v225\n"
    "v_rcp_f32 v226, v226\n"
    "v_rcp_f32 v227, v227\n"
    "v_rcp_f32 v228, v228\n"
    "v_rcp_f32 v229, v229\n"
    "v_rcp_f32 v230, v230\n"
    "v_rcp_f32 v231, v231\n"
    "s_waitcnt lgkmcnt(0)\n"
    "buffer_store_dwordx4 v[28:31], v248, s[96:99], s50 offen offset:0 sc0 sc1\n"
    "buffer_store_dwordx4 v[92:95], v246, s[96:99], s50 offen offset:0 sc0 sc1\n"
    "v_pk_mul_f32 v[128:129], v[128:129], v[224:225]\n"
    "v_pk_mul_f32 v[130:131], v[130:131], v[226:227]\n"
    "v_pk_mul_f32 v[160:161], v[160:161], v[228:229]\n"
    "v_pk_mul_f32 v[162:163], v[162:163], v[230:231]\n"
    "v_cvt_pk_bf16_f32 v128, v128, v129\n"
    "v_cvt_pk_bf16_f32 v129, v130, v131\n"
    "v_cvt_pk_bf16_f32 v130, v160, v161\n"
    "v_cvt_pk_bf16_f32 v131, v162, v163\n"
    "v_accvgpr_read_b32 v32, a160\n"
    "v_accvgpr_read_b32 v33, a161\n"
    "v_accvgpr_read_b32 v34, a162\n"
    "v_accvgpr_read_b32 v35, a163\n"
    "v_pk_fma_f32 v[232:233], v[192:193], s[40:41], v[240:241]\n"
    "v_pk_fma_f32 v[234:235], v[194:195], s[40:41], v[240:241]\n"
    "v_pk_fma_f32 v[236:237], v[32:33], s[40:41], v[240:241]\n"
    "v_pk_fma_f32 v[238:239], v[34:35], s[40:41], v[240:241]\n"
    "v_permlane16_swap_b32 v128, v130\n"
    "v_permlane16_swap_b32 v129, v131\n"
    "ds_write_b128 v249, v[128:131] offset:0\n"
    "v_exp_f32_e64 v232, v232 clamp\n"
    "v_exp_f32_e64 v233, v233 clamp\n"
    "v_exp_f32_e64 v234, v234 clamp\n"
    "v_exp_f32_e64 v235, v235 clamp\n"
    "v_exp_f32_e64 v236, v236 clamp\n"
    "v_exp_f32_e64 v237, v237 clamp\n"
    "v_exp_f32_e64 v238, v238 clamp\n"
    "v_exp_f32_e64 v239, v239 clamp\n"
    "v_pk_fma_f32 v[232:233], v[232:233], s[42:43], v[244:245]\n"
    "v_pk_fma_f32 v[234:235], v[234:235], s[42:43], v[244:245]\n"
    "v_pk_fma_f32 v[236:237], v[236:237], s[42:43], v[244:245]\n"
    "v_pk_fma_f32 v[238:239], v[238:239], s[42:43], v[244:245]\n"
    "v_rcp_f32 v232, v232\n"
    "v_rcp_f32 v233, v233\n"
    "v_rcp_f32 v234, v234\n"
    "v_rcp_f32 v235, v235\n"
    "v_rcp_f32 v236, v236\n"
    "v_rcp_f32 v237, v237\n"
    "v_rcp_f32 v238, v238\n"
    "v_rcp_f32 v239, v239\n"
    "v_pk_mul_f32 v[192:193], v[192:193], v[232:233]\n"
    "v_pk_mul_f32 v[194:195], v[194:195], v[234:235]\n"
    "v_pk_mul_f32 v[32:33], v[32:33], v[236:237]\n"
    "v_pk_mul_f32 v[34:35], v[34:35], v[238:239]\n"
    "v_cvt_pk_bf16_f32 v192, v192, v193\n"
    "v_cvt_pk_bf16_f32 v193, v194, v195\n"
    "v_cvt_pk_bf16_f32 v194, v32, v33\n"
    "v_cvt_pk_bf16_f32 v195, v34, v35\n"
    "v_pk_fma_f32 v[224:225], v[132:133], s[40:41], v[240:241]\n"
    "v_pk_fma_f32 v[226:227], v[134:135], s[40:41], v[240:241]\n"
    "v_pk_fma_f32 v[228:229], v[164:165], s[40:41], v[240:241]\n"
    "v_pk_fma_f32 v[230:231], v[166:167], s[40:41], v[240:241]\n"
    "v_permlane16_swap_b32 v192, v194\n"
    "v_permlane16_swap_b32 v193, v195\n"
    "ds_write_b128 v249, v[192:195] offset:64\n"
    "ds_read_b128 v[128:131], v250 offset:0\n"
    "ds_read_b128 v[192:195], v250 offset:1152\n"
    "v_exp_f32_e64 v224, v224 clamp\n"
    "v_exp_f32_e64 v225, v225 clamp\n"
    "v_exp_f32_e64 v226, v226 clamp\n"
    "v_exp_f32_e64 v227, v227 clamp\n"
    "v_exp_f32_e64 v228, v228 clamp\n"
    "v_exp_f32_e64 v229, v229 clamp\n"
    "v_exp_f32_e64 v230, v230 clamp\n"
    "v_exp_f32_e64 v231, v231 clamp\n"
    "v_pk_fma_f32 v[224:225], v[224:225], s[42:43], v[244:245]\n"
    "v_pk_fma_f32 v[226:227], v[226:227], s[42:43], v[244:245]\n"
    "v_pk_fma_f32 v[228:229], v[228:229], s[42:43], v[244:245]\n"
    "v_pk_fma_f32 v[230:231], v[230:231], s[42:43], v[244:245]\n"
    "v_rcp_f32 v224, v224\n"
    "v_rcp_f32 v225, v225\n"
    "v_rcp_f32 v226, v226\n"
    "v_rcp_f32 v227, v227\n"
    "v_rcp_f32 v228, v228\n"
    "v_rcp_f32 v229, v229\n"
    "v_rcp_f32 v230, v230\n"
    "v_rcp_f32 v231, v231\n"
    "s_waitcnt lgkmcnt(0)\n"
    "buffer_store_dwordx4 v[128:131], v248, s[96:99], 0 offen offset:128 sc0 sc1\n"
    "buffer_store_dwordx4 v[192:195], v246, s[96:99], 0 offen offset:128 sc0 sc1\n"
    "v_pk_mul_f32 v[132:133], v[132:133], v[224:225]\n"
    "v_pk_mul_f32 v[134:135], v[134:135], v[226:227]\n"
    "v_pk_mul_f32 v[164:165], v[164:165], v[228:229]\n"
    "v_pk_mul_f32 v[166:167], v[166:167], v[230:231]\n"
    "v_cvt_pk_bf16_f32 v132, v132, v133\n"
    "v_cvt_pk_bf16_f32 v133, v134, v135\n"
    "v_cvt_pk_bf16_f32 v134, v164, v165\n"
    "v_cvt_pk_bf16_f32 v135, v166, v167\n"
    "v_accvgpr_read_b32 v36, a164\n"
    "v_accvgpr_read_b32 v37, a165\n"
    "v_accvgpr_read_b32 v38, a166\n"
    "v_accvgpr_read_b32 v39, a167\n"
    "v_pk_fma_f32 v[232:233], v[196:197], s[40:41], v[240:241]\n"
    "v_pk_fma_f32 v[234:235], v[198:199], s[40:41], v[240:241]\n"
    "v_pk_fma_f32 v[236:237], v[36:37], s[40:41], v[240:241]\n"
    "v_pk_fma_f32 v[238:239], v[38:39], s[40:41], v[240:241]\n"
    "v_permlane16_swap_b32 v132, v134\n"
    "v_permlane16_swap_b32 v133, v135\n"
    "ds_write_b128 v249, v[132:135] offset:2304\n"
    "v_exp_f32_e64 v232, v232 clamp\n"
    "v_exp_f32_e64 v233, v233 clamp\n"
    "v_exp_f32_e64 v234, v234 clamp\n"
    "v_exp_f32_e64 v235, v235 clamp\n"
    "v_exp_f32_e64 v236, v236 clamp\n"
    "v_exp_f32_e64 v237, v237 clamp\n"
    "v_exp_f32_e64 v238, v238 clamp\n"
    "v_exp_f32_e64 v239, v239 clamp\n"
    "v_pk_fma_f32 v[232:233], v[232:233], s[42:43], v[244:245]\n"
    "v_pk_fma_f32 v[234:235], v[234:235], s[42:43], v[244:245]\n"
    "v_pk_fma_f32 v[236:237], v[236:237], s[42:43], v[244:245]\n"
    "v_pk_fma_f32 v[238:239], v[238:239], s[42:43], v[244:245]\n"
    "v_rcp_f32 v232, v232\n"
    "v_rcp_f32 v233, v233\n"
    "v_rcp_f32 v234, v234\n"
    "v_rcp_f32 v235, v235\n"
    "v_rcp_f32 v236, v236\n"
    "v_rcp_f32 v237, v237\n"
    "v_rcp_f32 v238, v238\n"
    "v_rcp_f32 v239, v239\n"
    "v_pk_mul_f32 v[196:197], v[196:197], v[232:233]\n"
    "v_pk_mul_f32 v[198:199], v[198:199], v[234:235]\n"
    "v_pk_mul_f32 v[36:37], v[36:37], v[236:237]\n"
    "v_pk_mul_f32 v[38:39], v[38:39], v[238:239]\n"
    "v_cvt_pk_bf16_f32 v196, v196, v197\n"
    "v_cvt_pk_bf16_f32 v197, v198, v199\n"
    "v_cvt_pk_bf16_f32 v198, v36, v37\n"
    "v_cvt_pk_bf16_f32 v199, v38, v39\n"
    "v_pk_fma_f32 v[224:225], v[136:137], s[40:41], v[240:241]\n"
    "v_pk_fma_f32 v[226:227], v[138:139], s[40:41], v[240:241]\n"
    "v_pk_fma_f32 v[228:229], v[168:169], s[40:41], v[240:241]\n"
    "v_pk_fma_f32 v[230:231], v[170:171], s[40:41], v[240:241]\n"
    "v_permlane16_swap_b32 v196, v198\n"
    "v_permlane16_swap_b32 v197, v199\n"
    "ds_write_b128 v249, v[196:199] offset:2368\n"
    "ds_read_b128 v[132:135], v250 offset:2304\n"
    "ds_read_b128 v[196:199], v250 offset:3456\n"
    "v_exp_f32_e64 v224, v224 clamp\n"
    "v_exp_f32_e64 v225, v225 clamp\n"
    "v_exp_f32_e64 v226, v226 clamp\n"
    "v_exp_f32_e64 v227, v227 clamp\n"
    "v_exp_f32_e64 v228, v228 clamp\n"
    "v_exp_f32_e64 v229, v229 clamp\n"
    "v_exp_f32_e64 v230, v230 clamp\n"
    "v_exp_f32_e64 v231, v231 clamp\n"
    "v_pk_fma_f32 v[224:225], v[224:225], s[42:43], v[244:245]\n"
    "v_pk_fma_f32 v[226:227], v[226:227], s[42:43], v[244:245]\n"
    "v_pk_fma_f32 v[228:229], v[228:229], s[42:43], v[244:245]\n"
    "v_pk_fma_f32 v[230:231], v[230:231], s[42:43], v[244:245]\n"
    "v_rcp_f32 v224, v224\n"
    "v_rcp_f32 v225, v225\n"
    "v_rcp_f32 v226, v226\n"
    "v_rcp_f32 v227, v227\n"
    "v_rcp_f32 v228, v228\n"
    "v_rcp_f32 v229, v229\n"
    "v_rcp_f32 v230, v230\n"
    "v_rcp_f32 v231, v231\n"
    "s_waitcnt lgkmcnt(0)\n"
    "buffer_store_dwordx4 v[132:135], v248, s[96:99], s44 offen offset:128 sc0 sc1\n"
    "buffer_store_dwordx4 v[196:199], v246, s[96:99], s44 offen offset:128 sc0 sc1\n"
    "v_pk_mul_f32 v[136:137], v[136:137], v[224:225]\n"
    "v_pk_mul_f32 v[138:139], v[138:139], v[226:227]\n"
    "v_pk_mul_f32 v[168:169], v[168:169], v[228:229]\n"
    "v_pk_mul_f32 v[170:171], v[170:171], v[230:231]\n"
    "v_cvt_pk_bf16_f32 v136, v136, v137\n"
    "v_cvt_pk_bf16_f32 v137, v138, v139\n"
    "v_cvt_pk_bf16_f32 v138, v168, v169\n"
    "v_cvt_pk_bf16_f32 v139, v170, v171\n"
    "v_accvgpr_read_b32 v40, a168\n"
    "v_accvgpr_read_b32 v41, a169\n"
    "v_accvgpr_read_b32 v42, a170\n"
    "v_accvgpr_read_b32 v43, a171\n"
    "v_pk_fma_f32 v[232:233], v[200:201], s[40:41], v[240:241]\n"
    "v_pk_fma_f32 v[234:235], v[202:203], s[40:41], v[240:241]\n"
    "v_pk_fma_f32 v[236:237], v[40:41], s[40:41], v[240:241]\n"
    "v_pk_fma_f32 v[238:239], v[42:43], s[40:41], v[240:241]\n"
    "v_permlane16_swap_b32 v136, v138\n"
    "v_permlane16_swap_b32 v137, v139\n"
    "ds_write_b128 v249, v[136:139] offset:4608\n"
    "v_exp_f32_e64 v232, v232 clamp\n"
    "v_exp_f32_e64 v233, v233 clamp\n"
    "v_exp_f32_e64 v234, v234 clamp\n"
    "v_exp_f32_e64 v235, v235 clamp\n"
    "v_exp_f32_e64 v236, v236 clamp\n"
    "v_exp_f32_e64 v237, v237 clamp\n"
    "v_exp_f32_e64 v238, v238 clamp\n"
    "v_exp_f32_e64 v239, v239 clamp\n"
    "v_pk_fma_f32 v[232:233], v[232:233], s[42:43], v[244:245]\n"
    "v_pk_fma_f32 v[234:235], v[234:235], s[42:43], v[244:245]\n"
    "v_pk_fma_f32 v[236:237], v[236:237], s[42:43], v[244:245]\n"
    "v_pk_fma_f32 v[238:239], v[238:239], s[42:43], v[244:245]\n"
    "v_rcp_f32 v232, v232\n"
    "v_rcp_f32 v233, v233\n"
    "v_rcp_f32 v234, v234\n"
    "v_rcp_f32 v235, v235\n"
    "v_rcp_f32 v236, v236\n"
    "v_rcp_f32 v237, v237\n"
    "v_rcp_f32 v238, v238\n"
    "v_rcp_f32 v239, v239\n"
    "v_pk_mul_f32 v[200:201], v[200:201], v[232:233]\n"
    "v_pk_mul_f32 v[202:203], v[202:203], v[234:235]\n"
    "v_pk_mul_f32 v[40:41], v[40:41], v[236:237]\n"
    "v_pk_mul_f32 v[42:43], v[42:43], v[238:239]\n"
    "v_cvt_pk_bf16_f32 v200, v200, v201\n"
    "v_cvt_pk_bf16_f32 v201, v202, v203\n"
    "v_cvt_pk_bf16_f32 v202, v40, v41\n"
    "v_cvt_pk_bf16_f32 v203, v42, v43\n"
    "v_pk_fma_f32 v[224:225], v[140:141], s[40:41], v[240:241]\n"
    "v_pk_fma_f32 v[226:227], v[142:143], s[40:41], v[240:241]\n"
    "v_pk_fma_f32 v[228:229], v[172:173], s[40:41], v[240:241]\n"
    "v_pk_fma_f32 v[230:231], v[174:175], s[40:41], v[240:241]\n"
    "v_permlane16_swap_b32 v200, v202\n"
    "v_permlane16_swap_b32 v201, v203\n"
    "ds_write_b128 v249, v[200:203] offset:4672\n"
    "ds_read_b128 v[136:139], v250 offset:4608\n"
    "ds_read_b128 v[200:203], v250 offset:5760\n"
    "v_exp_f32_e64 v224, v224 clamp\n"
    "v_exp_f32_e64 v225, v225 clamp\n"
    "v_exp_f32_e64 v226, v226 clamp\n"
    "v_exp_f32_e64 v227, v227 clamp\n"
    "v_exp_f32_e64 v228, v228 clamp\n"
    "v_exp_f32_e64 v229, v229 clamp\n"
    "v_exp_f32_e64 v230, v230 clamp\n"
    "v_exp_f32_e64 v231, v231 clamp\n"
    "v_pk_fma_f32 v[224:225], v[224:225], s[42:43], v[244:245]\n"
    "v_pk_fma_f32 v[226:227], v[226:227], s[42:43], v[244:245]\n"
    "v_pk_fma_f32 v[228:229], v[228:229], s[42:43], v[244:245]\n"
    "v_pk_fma_f32 v[230:231], v[230:231], s[42:43], v[244:245]\n"
    "v_rcp_f32 v224, v224\n"
    "v_rcp_f32 v225, v225\n"
    "v_rcp_f32 v226, v226\n"
    "v_rcp_f32 v227, v227\n"
    "v_rcp_f32 v228, v228\n"
    "v_rcp_f32 v229, v229\n"
    "v_rcp_f32 v230, v230\n"
    "v_rcp_f32 v231, v231\n"
    "s_waitcnt lgkmcnt(0)\n"
    "buffer_store_dwordx4 v[136:139], v248, s[96:99], s45 offen offset:128 sc0 sc1\n"
    "buffer_store_dwordx4 v[200:203], v246, s[96:99], s45 offen offset:128 sc0 sc1\n"
    "v_pk_mul_f32 v[140:141], v[140:141], v[224:225]\n"
    "v_pk_mul_f32 v[142:143], v[142:143], v[226:227]\n"
    "v_pk_mul_f32 v[172:173], v[172:173], v[228:229]\n"
    "v_pk_mul_f32 v[174:175], v[174:175], v[230:231]\n"
    "v_cvt_pk_bf16_f32 v140, v140, v141\n"
    "v_cvt_pk_bf16_f32 v141, v142, v143\n"
    "v_cvt_pk_bf16_f32 v142, v172, v173\n"
    "v_cvt_pk_bf16_f32 v143, v174, v175\n"
    "v_accvgpr_read_b32 v44, a172\n"
    "v_accvgpr_read_b32 v45, a173\n"
    "v_accvgpr_read_b32 v46, a174\n"
    "v_accvgpr_read_b32 v47, a175\n"
    "v_pk_fma_f32 v[232:233], v[204:205], s[40:41], v[240:241]\n"
    "v_pk_fma_f32 v[234:235], v[206:207], s[40:41], v[240:241]\n"
    "v_pk_fma_f32 v[236:237], v[44:45], s[40:41], v[240:241]\n"
    "v_pk_fma_f32 v[238:239], v[46:47], s[40:41], v[240:241]\n"
    "v_permlane16_swap_b32 v140, v142\n"
    "v_permlane16_swap_b32 v141, v143\n"
    "ds_write_b128 v249, v[140:143] offset:6912\n"
    "v_exp_f32_e64 v232, v232 clamp\n"
    "v_exp_f32_e64 v233, v233 clamp\n"
    "v_exp_f32_e64 v234, v234 clamp\n"
    "v_exp_f32_e64 v235, v235 clamp\n"
    "v_exp_f32_e64 v236, v236 clamp\n"
    "v_exp_f32_e64 v237, v237 clamp\n"
    "v_exp_f32_e64 v238, v238 clamp\n"
    "v_exp_f32_e64 v239, v239 clamp\n"
    "v_pk_fma_f32 v[232:233], v[232:233], s[42:43], v[244:245]\n"
    "v_pk_fma_f32 v[234:235], v[234:235], s[42:43], v[244:245]\n"
    "v_pk_fma_f32 v[236:237], v[236:237], s[42:43], v[244:245]\n"
    "v_pk_fma_f32 v[238:239], v[238:239], s[42:43], v[244:245]\n"
    "v_rcp_f32 v232, v232\n"
    "v_rcp_f32 v233, v233\n"
    "v_rcp_f32 v234, v234\n"
    "v_rcp_f32 v235, v235\n"
    "v_rcp_f32 v236, v236\n"
    "v_rcp_f32 v237, v237\n"
    "v_rcp_f32 v238, v238\n"
    "v_rcp_f32 v239, v239\n"
    "v_pk_mul_f32 v[204:205], v[204:205], v[232:233]\n"
    "v_pk_mul_f32 v[206:207], v[206:207], v[234:235]\n"
    "v_pk_mul_f32 v[44:45], v[44:45], v[236:237]\n"
    "v_pk_mul_f32 v[46:47], v[46:47], v[238:239]\n"
    "v_cvt_pk_bf16_f32 v204, v204, v205\n"
    "v_cvt_pk_bf16_f32 v205, v206, v207\n"
    "v_cvt_pk_bf16_f32 v206, v44, v45\n"
    "v_cvt_pk_bf16_f32 v207, v46, v47\n"
    "v_pk_fma_f32 v[224:225], v[144:145], s[40:41], v[240:241]\n"
    "v_pk_fma_f32 v[226:227], v[146:147], s[40:41], v[240:241]\n"
    "v_pk_fma_f32 v[228:229], v[176:177], s[40:41], v[240:241]\n"
    "v_pk_fma_f32 v[230:231], v[178:179], s[40:41], v[240:241]\n"
    "v_permlane16_swap_b32 v204, v206\n"
    "v_permlane16_swap_b32 v205, v207\n"
    "ds_write_b128 v249, v[204:207] offset:6976\n"
    "ds_read_b128 v[140:143], v250 offset:6912\n"
    "ds_read_b128 v[204:207], v250 offset:8064\n"
    "v_exp_f32_e64 v224, v224 clamp\n"
    "v_exp_f32_e64 v225, v225 clamp\n"
    "v_exp_f32_e64 v226, v226 clamp\n"
    "v_exp_f32_e64 v227, v227 clamp\n"
    "v_exp_f32_e64 v228, v228 clamp\n"
    "v_exp_f32_e64 v229, v229 clamp\n"
    "v_exp_f32_e64 v230, v230 clamp\n"
    "v_exp_f32_e64 v231, v231 clamp\n"
    "v_pk_fma_f32 v[224:225], v[224:225], s[42:43], v[244:245]\n"
    "v_pk_fma_f32 v[226:227], v[226:227], s[42:43], v[244:245]\n"
    "v_pk_fma_f32 v[228:229], v[228:229], s[42:43], v[244:245]\n"
    "v_pk_fma_f32 v[230:231], v[230:231], s[42:43], v[244:245]\n"
    "v_rcp_f32 v224, v224\n"
    "v_rcp_f32 v225, v225\n"
    "v_rcp_f32 v226, v226\n"
    "v_rcp_f32 v227, v227\n"
    "v_rcp_f32 v228, v228\n"
    "v_rcp_f32 v229, v229\n"
    "v_rcp_f32 v230, v230\n"
    "v_rcp_f32 v231, v231\n"
    "s_waitcnt lgkmcnt(0)\n"
    "buffer_store_dwordx4 v[140:143], v248, s[96:99], s46 offen offset:128 sc0 sc1\n"
    "buffer_store_dwordx4 v[204:207], v246, s[96:99], s46 offen offset:128 sc0 sc1\n"
    "v_pk_mul_f32 v[144:145], v[144:145], v[224:225]\n"
    "v_pk_mul_f32 v[146:147], v[146:147], v[226:227]\n"
    "v_pk_mul_f32 v[176:177], v[176:177], v[228:229]\n"
    "v_pk_mul_f32 v[178:179], v[178:179], v[230:231]\n"
    "v_cvt_pk_bf16_f32 v144, v144, v145\n"
    "v_cvt_pk_bf16_f32 v145, v146, v147\n"
    "v_cvt_pk_bf16_f32 v146, v176, v177\n"
    "v_cvt_pk_bf16_f32 v147, v178, v179\n"
    "v_accvgpr_read_b32 v48, a176\n"
    "v_accvgpr_read_b32 v49, a177\n"
    "v_accvgpr_read_b32 v50, a178\n"
    "v_accvgpr_read_b32 v51, a179\n"
    "v_pk_fma_f32 v[232:233], v[208:209], s[40:41], v[240:241]\n"
    "v_pk_fma_f32 v[234:235], v[210:211], s[40:41], v[240:241]\n"
    "v_pk_fma_f32 v[236:237], v[48:49], s[40:41], v[240:241]\n"
    "v_pk_fma_f32 v[238:239], v[50:51], s[40:41], v[240:241]\n"
    "v_permlane16_swap_b32 v144, v146\n"
    "v_permlane16_swap_b32 v145, v147\n"
    "ds_write_b128 v249, v[144:147] offset:0\n"
    "v_exp_f32_e64 v232, v232 clamp\n"
    "v_exp_f32_e64 v233, v233 clamp\n"
    "v_exp_f32_e64 v234, v234 clamp\n"
    "v_exp_f32_e64 v235, v235 clamp\n"
    "v_exp_f32_e64 v236, v236 clamp\n"
    "v_exp_f32_e64 v237, v237 clamp\n"
    "v_exp_f32_e64 v238, v238 clamp\n"
    "v_exp_f32_e64 v239, v239 clamp\n"
    "v_pk_fma_f32 v[232:233], v[232:233], s[42:43], v[244:245]\n"
    "v_pk_fma_f32 v[234:235], v[234:235], s[42:43], v[244:245]\n"
    "v_pk_fma_f32 v[236:237], v[236:237], s[42:43], v[244:245]\n"
    "v_pk_fma_f32 v[238:239], v[238:239], s[42:43], v[244:245]\n"
    "v_rcp_f32 v232, v232\n"
    "v_rcp_f32 v233, v233\n"
    "v_rcp_f32 v234, v234\n"
    "v_rcp_f32 v235, v235\n"
    "v_rcp_f32 v236, v236\n"
    "v_rcp_f32 v237, v237\n"
    "v_rcp_f32 v238, v238\n"
    "v_rcp_f32 v239, v239\n"
    "v_pk_mul_f32 v[208:209], v[208:209], v[232:233]\n"
    "v_pk_mul_f32 v[210:211], v[210:211], v[234:235]\n"
    "v_pk_mul_f32 v[48:49], v[48:49], v[236:237]\n"
    "v_pk_mul_f32 v[50:51], v[50:51], v[238:239]\n"
    "v_cvt_pk_bf16_f32 v208, v208, v209\n"
    "v_cvt_pk_bf16_f32 v209, v210, v211\n"
    "v_cvt_pk_bf16_f32 v210, v48, v49\n"
    "v_cvt_pk_bf16_f32 v211, v50, v51\n"
    "v_pk_fma_f32 v[224:225], v[148:149], s[40:41], v[240:241]\n"
    "v_pk_fma_f32 v[226:227], v[150:151], s[40:41], v[240:241]\n"
    "v_pk_fma_f32 v[228:229], v[180:181], s[40:41], v[240:241]\n"
    "v_pk_fma_f32 v[230:231], v[182:183], s[40:41], v[240:241]\n"
    "v_permlane16_swap_b32 v208, v210\n"
    "v_permlane16_swap_b32 v209, v211\n"
    "ds_write_b128 v249, v[208:211] offset:64\n"
    "ds_read_b128 v[144:147], v250 offset:0\n"
    "ds_read_b128 v[208:211], v250 offset:1152\n"
    "v_exp_f32_e64 v224, v224 clamp\n"
    "v_exp_f32_e64 v225, v225 clamp\n"
    "v_exp_f32_e64 v226, v226 clamp\n"
    "v_exp_f32_e64 v227, v227 clamp\n"
    "v_exp_f32_e64 v228, v228 clamp\n"
    "v_exp_f32_e64 v229, v229 clamp\n"
    "v_exp_f32_e64 v230, v230 clamp\n"
    "v_exp_f32_e64 v231, v231 clamp\n"
    "v_pk_fma_f32 v[224:225], v[224:225], s[42:43], v[244:245]\n"
    "v_pk_fma_f32 v[226:227], v[226:227], s[42:43], v[244:245]\n"
    "v_pk_fma_f32 v[228:229], v[228:229], s[42:43], v[244:245]\n"
    "v_pk_fma_f32 v[230:231], v[230:231], s[42:43], v[244:245]\n"
    "v_rcp_f32 v224, v224\n"
    "v_rcp_f32 v225, v225\n"
    "v_rcp_f32 v226, v226\n"
    "v_rcp_f32 v227, v227\n"
    "v_rcp_f32 v228, v228\n"
    "v_rcp_f32 v229, v229\n"
    "v_rcp_f32 v230, v230\n"
    "v_rcp_f32 v231, v231\n"
    "s_waitcnt lgkmcnt(0)\n"
    "buffer_store_dwordx4 v[144:147], v248, s[96:99], s47 offen offset:128 sc0 sc1\n"
    "buffer_store_dwordx4 v[208:211], v246, s[96:99], s47 offen offset:128 sc0 sc1\n"
    "v_pk_mul_f32 v[148:149], v[148:149], v[224:225]\n"
    "v_pk_mul_f32 v[150:151], v[150:151], v[226:227]\n"
    "v_pk_mul_f32 v[180:181], v[180:181], v[228:229]\n"
    "v_pk_mul_f32 v[182:183], v[182:183], v[230:231]\n"
    "v_cvt_pk_bf16_f32 v148, v148, v149\n"
    "v_cvt_pk_bf16_f32 v149, v150, v151\n"
    "v_cvt_pk_bf16_f32 v150, v180, v181\n"
    "v_cvt_pk_bf16_f32 v151, v182, v183\n"
    "v_accvgpr_read_b32 v52, a180\n"
    "v_accvgpr_read_b32 v53, a181\n"
    "v_accvgpr_read_b32 v54, a182\n"
    "v_accvgpr_read_b32 v55, a183\n"
    "v_pk_fma_f32 v[232:233], v[212:213], s[40:41], v[240:241]\n"
    "v_pk_fma_f32 v[234:235], v[214:215], s[40:41], v[240:241]\n"
    "v_pk_fma_f32 v[236:237], v[52:53], s[40:41], v[240:241]\n"
    "v_pk_fma_f32 v[238:239], v[54:55], s[40:41], v[240:241]\n"
    "v_permlane16_swap_b32 v148, v150\n"
    "v_permlane16_swap_b32 v149, v151\n"
    "ds_write_b128 v249, v[148:151] offset:2304\n"
    "v_exp_f32_e64 v232, v232 clamp\n"
    "v_exp_f32_e64 v233, v233 clamp\n"
    "v_exp_f32_e64 v234, v234 clamp\n"
    "v_exp_f32_e64 v235, v235 clamp\n"
    "v_exp_f32_e64 v236, v236 clamp\n"
    "v_exp_f32_e64 v237, v237 clamp\n"
    "v_exp_f32_e64 v238, v238 clamp\n"
    "v_exp_f32_e64 v239, v239 clamp\n"
    "v_pk_fma_f32 v[232:233], v[232:233], s[42:43], v[244:245]\n"
    "v_pk_fma_f32 v[234:235], v[234:235], s[42:43], v[244:245]\n"
    "v_pk_fma_f32 v[236:237], v[236:237], s[42:43], v[244:245]\n"
    "v_pk_fma_f32 v[238:239], v[238:239], s[42:43], v[244:245]\n"
    "v_rcp_f32 v232, v232\n"
    "v_rcp_f32 v233, v233\n"
    "v_rcp_f32 v234, v234\n"
    "v_rcp_f32 v235, v235\n"
    "v_rcp_f32 v236, v236\n"
    "v_rcp_f32 v237, v237\n"
    "v_rcp_f32 v238, v238\n"
    "v_rcp_f32 v239, v239\n"
    "v_pk_mul_f32 v[212:213], v[212:213], v[232:233]\n"
    "v_pk_mul_f32 v[214:215], v[214:215], v[234:235]\n"
    "v_pk_mul_f32 v[52:53], v[52:53], v[236:237]\n"
    "v_pk_mul_f32 v[54:55], v[54:55], v[238:239]\n"
    "v_cvt_pk_bf16_f32 v212, v212, v213\n"
    "v_cvt_pk_bf16_f32 v213, v214, v215\n"
    "v_cvt_pk_bf16_f32 v214, v52, v53\n"
    "v_cvt_pk_bf16_f32 v215, v54, v55\n"
    "v_pk_fma_f32 v[224:225], v[152:153], s[40:41], v[240:241]\n"
    "v_pk_fma_f32 v[226:227], v[154:155], s[40:41], v[240:241]\n"
    "v_pk_fma_f32 v[228:229], v[184:185], s[40:41], v[240:241]\n"
    "v_pk_fma_f32 v[230:231], v[186:187], s[40:41], v[240:241]\n"
    "v_permlane16_swap_b32 v212, v214\n"
    "v_permlane16_swap_b32 v213, v215\n"
    "ds_write_b128 v249, v[212:215] offset:2368\n"
    "ds_read_b128 v[148:151], v250 offset:2304\n"
    "ds_read_b128 v[212:215], v250 offset:3456\n"
    "v_exp_f32_e64 v224, v224 clamp\n"
    "v_exp_f32_e64 v225, v225 clamp\n"
    "v_exp_f32_e64 v226, v226 clamp\n"
    "v_exp_f32_e64 v227, v227 clamp\n"
    "v_exp_f32_e64 v228, v228 clamp\n"
    "v_exp_f32_e64 v229, v229 clamp\n"
    "v_exp_f32_e64 v230, v230 clamp\n"
    "v_exp_f32_e64 v231, v231 clamp\n"
    "v_pk_fma_f32 v[224:225], v[224:225], s[42:43], v[244:245]\n"
    "v_pk_fma_f32 v[226:227], v[226:227], s[42:43], v[244:245]\n"
    "v_pk_fma_f32 v[228:229], v[228:229], s[42:43], v[244:245]\n"
    "v_pk_fma_f32 v[230:231], v[230:231], s[42:43], v[244:245]\n"
    "v_rcp_f32 v224, v224\n"
    "v_rcp_f32 v225, v225\n"
    "v_rcp_f32 v226, v226\n"
    "v_rcp_f32 v227, v227\n"
    "v_rcp_f32 v228, v228\n"
    "v_rcp_f32 v229, v229\n"
    "v_rcp_f32 v230, v230\n"
    "v_rcp_f32 v231, v231\n"
    "s_waitcnt lgkmcnt(0)\n"
    "buffer_store_dwordx4 v[148:151], v248, s[96:99], s48 offen offset:128 sc0 sc1\n"
    "buffer_store_dwordx4 v[212:215], v246, s[96:99], s48 offen offset:128 sc0 sc1\n"
    "v_pk_mul_f32 v[152:153], v[152:153], v[224:225]\n"
    "v_pk_mul_f32 v[154:155], v[154:155], v[226:227]\n"
    "v_pk_mul_f32 v[184:185], v[184:185], v[228:229]\n"
    "v_pk_mul_f32 v[186:187], v[186:187], v[230:231]\n"
    "v_cvt_pk_bf16_f32 v152, v152, v153\n"
    "v_cvt_pk_bf16_f32 v153, v154, v155\n"
    "v_cvt_pk_bf16_f32 v154, v184, v185\n"
    "v_cvt_pk_bf16_f32 v155, v186, v187\n"
    "v_accvgpr_read_b32 v56, a184\n"
    "v_accvgpr_read_b32 v57, a185\n"
    "v_accvgpr_read_b32 v58, a186\n"
    "v_accvgpr_read_b32 v59, a187\n"
    "v_pk_fma_f32 v[232:233], v[216:217], s[40:41], v[240:241]\n"
    "v_pk_fma_f32 v[234:235], v[218:219], s[40:41], v[240:241]\n"
    "v_pk_fma_f32 v[236:237], v[56:57], s[40:41], v[240:241]\n"
    "v_pk_fma_f32 v[238:239], v[58:59], s[40:41], v[240:241]\n"
    "v_permlane16_swap_b32 v152, v154\n"
    "v_permlane16_swap_b32 v153, v155\n"
    "ds_write_b128 v249, v[152:155] offset:4608\n"
    "v_exp_f32_e64 v232, v232 clamp\n"
    "v_exp_f32_e64 v233, v233 clamp\n"
    "v_exp_f32_e64 v234, v234 clamp\n"
    "v_exp_f32_e64 v235, v235 clamp\n"
    "v_exp_f32_e64 v236, v236 clamp\n"
    "v_exp_f32_e64 v237, v237 clamp\n"
    "v_exp_f32_e64 v238, v238 clamp\n"
    "v_exp_f32_e64 v239, v239 clamp\n"
    "v_pk_fma_f32 v[232:233], v[232:233], s[42:43], v[244:245]\n"
    "v_pk_fma_f32 v[234:235], v[234:235], s[42:43], v[244:245]\n"
    "v_pk_fma_f32 v[236:237], v[236:237], s[42:43], v[244:245]\n"
    "v_pk_fma_f32 v[238:239], v[238:239], s[42:43], v[244:245]\n"
    "v_rcp_f32 v232, v232\n"
    "v_rcp_f32 v233, v233\n"
    "v_rcp_f32 v234, v234\n"
    "v_rcp_f32 v235, v235\n"
    "v_rcp_f32 v236, v236\n"
    "v_rcp_f32 v237, v237\n"
    "v_rcp_f32 v238, v238\n"
    "v_rcp_f32 v239, v239\n"
    "v_pk_mul_f32 v[216:217], v[216:217], v[232:233]\n"
    "v_pk_mul_f32 v[218:219], v[218:219], v[234:235]\n"
    "v_pk_mul_f32 v[56:57], v[56:57], v[236:237]\n"
    "v_pk_mul_f32 v[58:59], v[58:59], v[238:239]\n"
    "v_cvt_pk_bf16_f32 v216, v216, v217\n"
    "v_cvt_pk_bf16_f32 v217, v218, v219\n"
    "v_cvt_pk_bf16_f32 v218, v56, v57\n"
    "v_cvt_pk_bf16_f32 v219, v58, v59\n"
    "v_pk_fma_f32 v[224:225], v[156:157], s[40:41], v[240:241]\n"
    "v_pk_fma_f32 v[226:227], v[158:159], s[40:41], v[240:241]\n"
    "v_pk_fma_f32 v[228:229], v[188:189], s[40:41], v[240:241]\n"
    "v_pk_fma_f32 v[230:231], v[190:191], s[40:41], v[240:241]\n"
    "v_permlane16_swap_b32 v216, v218\n"
    "v_permlane16_swap_b32 v217, v219\n"
    "ds_write_b128 v249, v[216:219] offset:4672\n"
    "ds_read_b128 v[152:155], v250 offset:4608\n"
    "ds_read_b128 v[216:219], v250 offset:5760\n"
    "v_exp_f32_e64 v224, v224 clamp\n"
    "v_exp_f32_e64 v225, v225 clamp\n"
    "v_exp_f32_e64 v226, v226 clamp\n"
    "v_exp_f32_e64 v227, v227 clamp\n"
    "v_exp_f32_e64 v228, v228 clamp\n"
    "v_exp_f32_e64 v229, v229 clamp\n"
    "v_exp_f32_e64 v230, v230 clamp\n"
    "v_exp_f32_e64 v231, v231 clamp\n"
    "v_pk_fma_f32 v[224:225], v[224:225], s[42:43], v[244:245]\n"
    "v_pk_fma_f32 v[226:227], v[226:227], s[42:43], v[244:245]\n"
    "v_pk_fma_f32 v[228:229], v[228:229], s[42:43], v[244:245]\n"
    "v_pk_fma_f32 v[230:231], v[230:231], s[42:43], v[244:245]\n"
    "v_rcp_f32 v224, v224\n"
    "v_rcp_f32 v225, v225\n"
    "v_rcp_f32 v226, v226\n"
    "v_rcp_f32 v227, v227\n"
    "v_rcp_f32 v228, v228\n"
    "v_rcp_f32 v229, v229\n"
    "v_rcp_f32 v230, v230\n"
    "v_rcp_f32 v231, v231\n"
    "s_waitcnt lgkmcnt(0)\n"
    "buffer_store_dwordx4 v[152:155], v248, s[96:99], s49 offen offset:128 sc0 sc1\n"
    "buffer_store_dwordx4 v[216:219], v246, s[96:99], s49 offen offset:128 sc0 sc1\n"
    "v_pk_mul_f32 v[156:157], v[156:157], v[224:225]\n"
    "v_pk_mul_f32 v[158:159], v[158:159], v[226:227]\n"
    "v_pk_mul_f32 v[188:189], v[188:189], v[228:229]\n"
    "v_pk_mul_f32 v[190:191], v[190:191], v[230:231]\n"
    "v_cvt_pk_bf16_f32 v156, v156, v157\n"
    "v_cvt_pk_bf16_f32 v157, v158, v159\n"
    "v_cvt_pk_bf16_f32 v158, v188, v189\n"
    "v_cvt_pk_bf16_f32 v159, v190, v191\n"
    "v_accvgpr_read_b32 v60, a188\n"
    "v_accvgpr_read_b32 v61, a189\n"
    "v_accvgpr_read_b32 v62, a190\n"
    "v_accvgpr_read_b32 v63, a191\n"
    "v_pk_fma_f32 v[232:233], v[220:221], s[40:41], v[240:241]\n"
    "v_pk_fma_f32 v[234:235], v[222:223], s[40:41], v[240:241]\n"
    "v_pk_fma_f32 v[236:237], v[60:61], s[40:41], v[240:241]\n"
    "v_pk_fma_f32 v[238:239], v[62:63], s[40:41], v[240:241]\n"
    "v_permlane16_swap_b32 v156, v158\n"
    "v_permlane16_swap_b32 v157, v159\n"
    "ds_write_b128 v249, v[156:159] offset:6912\n"
    "v_exp_f32_e64 v232, v232 clamp\n"
    "v_exp_f32_e64 v233, v233 clamp\n"
    "v_exp_f32_e64 v234, v234 clamp\n"
    "v_exp_f32_e64 v235, v235 clamp\n"
    "v_exp_f32_e64 v236, v236 clamp\n"
    "v_exp_f32_e64 v237, v237 clamp\n"
    "v_exp_f32_e64 v238, v238 clamp\n"
    "v_exp_f32_e64 v239, v239 clamp\n"
    "v_pk_fma_f32 v[232:233], v[232:233], s[42:43], v[244:245]\n"
    "v_pk_fma_f32 v[234:235], v[234:235], s[42:43], v[244:245]\n"
    "v_pk_fma_f32 v[236:237], v[236:237], s[42:43], v[244:245]\n"
    "v_pk_fma_f32 v[238:239], v[238:239], s[42:43], v[244:245]\n"
    "v_rcp_f32 v232, v232\n"
    "v_rcp_f32 v233, v233\n"
    "v_rcp_f32 v234, v234\n"
    "v_rcp_f32 v235, v235\n"
    "v_rcp_f32 v236, v236\n"
    "v_rcp_f32 v237, v237\n"
    "v_rcp_f32 v238, v238\n"
    "v_rcp_f32 v239, v239\n"
    "v_pk_mul_f32 v[220:221], v[220:221], v[232:233]\n"
    "v_pk_mul_f32 v[222:223], v[222:223], v[234:235]\n"
    "v_pk_mul_f32 v[60:61], v[60:61], v[236:237]\n"
    "v_pk_mul_f32 v[62:63], v[62:63], v[238:239]\n"
    "v_cvt_pk_bf16_f32 v220, v220, v221\n"
    "v_cvt_pk_bf16_f32 v221, v222, v223\n"
    "v_cvt_pk_bf16_f32 v222, v60, v61\n"
    "v_cvt_pk_bf16_f32 v223, v62, v63\n"
    "s_nop 1\n"
    "v_permlane16_swap_b32 v220, v222\n"
    "v_permlane16_swap_b32 v221, v223\n"
    "ds_write_b128 v249, v[220:223] offset:6976\n"
    "ds_read_b128 v[156:159], v250 offset:6912\n"
    "ds_read_b128 v[220:223], v250 offset:8064\n"
    "s_waitcnt lgkmcnt(0)\n"
    "buffer_store_dwordx4 v[156:159], v248, s[96:99], s50 offen offset:128 sc0 sc1\n"
    "buffer_store_dwordx4 v[220:223], v246, s[96:99], s50 offen offset:128 sc0 sc1\n"
    "s_endpgm\n"
    ".Lg3_%=:\n"
    "s_waitcnt lgkmcnt(4)\n"
    "v_mfma_f32_16x16x32_bf16 v[0:3], a[128:131], a[0:3], 0\n"
    "buffer_load_dwordx4 a[192:195], v224, s[80:83], s64 offen offset:0\n"
    "s_add_u32 s59, s52, 2\n"
    "v_mfma_f32_16x16x32_bf16 v[0:3], a[132:135], a[4:7], v[0:3]\n"
    "ds_read_b64_tr_b16 a[144:145], v242 offset:256\n"
    "buffer_load_dwordx4 a[196:199], v225, s[80:83], s64 offen offset:0\n"
    "v_readlane_b32 s54, v245, s59\n"
    "v_mfma_f32_16x16x32_bf16 v[4:7], a[128:131], a[8:11], 0\n"
    "buffer_load_dwordx4 a[200:203], v226, s[80:83], s64 offen offset:0\n"
    "v_mfma_f32_16x16x32_bf16 v[4:7], a[132:135], a[12:15], v[4:7]\n"
    "ds_read_b64_tr_b16 a[146:147], v242 offset:1280\n"
    "buffer_load_dwordx4 a[204:207], v227, s[80:83], s64 offen offset:0\n"
    "v_mfma_f32_16x16x32_bf16 v[8:11], a[128:131], a[16:19], 0\n"
    "buffer_load_dwordx4 a[208:211], v228, s[80:83], s64 offen offset:0\n"
    "v_mfma_f32_16x16x32_bf16 v[8:11], a[132:135], a[20:23], v[8:11]\n"
    "ds_read_b64_tr_b16 a[148:149], v242 offset:8448\n"
    "buffer_load_dwordx4 a[212:215], v229, s[80:83], s64 offen offset:0\n"
    "v_mfma_f32_16x16x32_bf16 v[12:15], a[128:131], a[24:27], 0\n"
    "buffer_load_dwordx4 a[216:219], v230, s[80:83], s64 offen offset:0\n"
    "v_mfma_f32_16x16x32_bf16 v[12:15], a[132:135], a[28:31], v[12:15]\n"
    "ds_read_b64_tr_b16 a[150:151], v242 offset:9472\n"
    "buffer_load_dwordx4 a[220:223], v231, s[80:83], s64 offen offset:0\n"
    "v_mfma_f32_16x16x32_bf16 v[16:19], a[128:131], a[32:35], 0\n"
    "s_add_u32 m0, s58, 0\n"
    "v_mfma_f32_16x16x32_bf16 v[16:19], a[132:135], a[36:39], v[16:19]\n"
    "buffer_load_dwordx4 v232, s[40:43], s54 offen nt lds\n"
    "v_mfma_f32_16x16x32_bf16 v[20:23], a[128:131], a[40:43], 0\n"
    "v_mfma_f32_16x16x32_bf16 v[20:23], a[132:135], a[44:47], v[20:23]\n"
    "ds_read_b128 a[64:67], v240 offset:32768\n"
    "v_mfma_f32_16x16x32_bf16 v[24:27], a[128:131], a[48:51], 0\n"
    "s_add_u32 m0, s58, 1024\n"
    "v_mfma_f32_16x16x32_bf16 v[24:27], a[132:135], a[52:55], v[24:27]\n"
    "buffer_load_dwordx4 v233, s[40:43], s54 offen nt lds\n"
    "v_mfma_f32_16x16x32_bf16 v[28:31], a[128:131], a[56:59], 0\n"
    "v_mfma_f32_16x16x32_bf16 v[28:31], a[132:135], a[60:63], v[28:31]\n"
    "ds_read_b128 a[68:71], v241 offset:32768\n"
    "s_waitcnt lgkmcnt(6)\n"
    "v_mfma_f32_16x16x32_bf16 v[32:35], a[136:139], a[0:3], 0\n"
    "v_mfma_f32_16x16x32_bf16 v[32:35], a[140:143], a[4:7], v[32:35]\n"
    "ds_read_b64_tr_b16 a[152:153], v243 offset:256\n"
    "v_mfma_f32_16x16x32_bf16 v[36:39], a[136:139], a[8:11], 0\n"
    "v_mfma_f32_16x16x32_bf16 v[36:39], a[140:143], a[12:15], v[36:39]\n"
    "ds_read_b64_tr_b16 a[154:155], v243 offset:1280\n"
    "s_cmp_lt_u32 s52, s33\n"
    "s_cselect_b32 s59, s32, 0\n"
    "v_mfma_f32_16x16x32_bf16 v[40:43], a[136:139], a[16:19], 0\n"
    "s_cmp_lt_u32 s52, s38\n"
    "s_cselect_b32 s59, s37, s59\n"
    "v_mfma_f32_16x16x32_bf16 v[40:43], a[140:143], a[20:23], v[40:43]\n"
    "ds_read_b64_tr_b16 a[156:157], v243 offset:8448\n"
    "v_mfma_f32_16x16x32_bf16 v[44:47], a[136:139], a[24:27], 0\n"
    "s_add_u32 s36, s36, s59\n"
    "v_mfma_f32_16x16x32_bf16 v[44:47], a[140:143], a[28:31], v[44:47]\n"
    "ds_read_b64_tr_b16 a[158:159], v243 offset:9472\n"
    "v_mfma_f32_16x16x32_bf16 v[48:51], a[136:139], a[32:35], 0\n"
    "s_add_u32 m0, s58, 4096\n"
    "v_mfma_f32_16x16x32_bf16 v[48:51], a[140:143], a[36:39], v[48:51]\n"
    "buffer_load_dwordx4 v234, s[40:43], s54 offen nt lds\n"
    "v_mfma_f32_16x16x32_bf16 v[52:55], a[136:139], a[40:43], 0\n"
    "v_mfma_f32_16x16x32_bf16 v[52:55], a[140:143], a[44:47], v[52:55]\n"
    "ds_read_b128 a[72:75], v240 offset:34816\n"
    "v_mfma_f32_16x16x32_bf16 v[56:59], a[136:139], a[48:51], 0\n"
    "s_add_u32 m0, s58, 5120\n"
    "v_mfma_f32_16x16x32_bf16 v[56:59], a[140:143], a[52:55], v[56:59]\n"
    "buffer_load_dwordx4 v235, s[40:43], s54 offen nt lds\n"
    "v_mfma_f32_16x16x32_bf16 v[60:63], a[136:139], a[56:59], 0\n"
    "v_mfma_f32_16x16x32_bf16 v[60:63], a[140:143], a[60:63], v[60:63]\n"
    "ds_read_b128 a[76:79], v241 offset:34816\n"
    "s_waitcnt lgkmcnt(8)\n"
    "v_mfma_f32_16x16x32_bf16 v[64:67], a[144:147], a[0:3], 0\n"
    "v_mfma_f32_16x16x32_bf16 v[64:67], a[148:151], a[4:7], v[64:67]\n"
    "ds_read_b64_tr_b16 a[128:129], v242 offset:512\n"
    "v_mfma_f32_16x16x32_bf16 v[68:71], a[144:147], a[8:11], 0\n"
    "v_mfma_f32_16x16x32_bf16 v[68:71], a[148:151], a[12:15], v[68:71]\n"
    "ds_read_b64_tr_b16 a[130:131], v242 offset:1536\n"
    "v_mfma_f32_16x16x32_bf16 v[72:75], a[144:147], a[16:19], 0\n"
    "v_mfma_f32_16x16x32_bf16 v[72:75], a[148:151], a[20:23], v[72:75]\n"
    "ds_read_b64_tr_b16 a[132:133], v242 offset:8704\n"
    "v_mfma_f32_16x16x32_bf16 v[76:79], a[144:147], a[24:27], 0\n"
    "v_mfma_f32_16x16x32_bf16 v[76:79], a[148:151], a[28:31], v[76:79]\n"
    "ds_read_b64_tr_b16 a[134:135], v242 offset:9728\n"
    "v_mfma_f32_16x16x32_bf16 v[80:83], a[144:147], a[32:35], 0\n"
    "s_add_u32 m0, s58, 8192\n"
    "v_mfma_f32_16x16x32_bf16 v[80:83], a[148:151], a[36:39], v[80:83]\n"
    "buffer_load_dwordx4 v236, s[40:43], s54 offen nt lds\n"
    "v_mfma_f32_16x16x32_bf16 v[84:87], a[144:147], a[40:43], 0\n"
    "v_mfma_f32_16x16x32_bf16 v[84:87], a[148:151], a[44:47], v[84:87]\n"
    "ds_read_b128 a[80:83], v240 offset:36864\n"
    "v_mfma_f32_16x16x32_bf16 v[88:91], a[144:147], a[48:51], 0\n"
    "s_add_u32 m0, s58, 9216\n"
    "v_mfma_f32_16x16x32_bf16 v[88:91], a[148:151], a[52:55], v[88:91]\n"
    "buffer_load_dwordx4 v237, s[40:43], s54 offen nt lds\n"
    "v_mfma_f32_16x16x32_bf16 v[92:95], a[144:147], a[56:59], 0\n"
    "v_mfma_f32_16x16x32_bf16 v[92:95], a[148:151], a[60:63], v[92:95]\n"
    "ds_read_b128 a[84:87], v241 offset:36864\n"
    "s_waitcnt lgkmcnt(8)\n"
    "v_mfma_f32_16x16x32_bf16 v[96:99], a[152:155], a[0:3], 0\n"
    "v_mfma_f32_16x16x32_bf16 v[96:99], a[156:159], a[4:7], v[96:99]\n"
    "ds_read_b64_tr_b16 a[136:137], v243 offset:512\n"
    "v_mfma_f32_16x16x32_bf16 v[100:103], a[152:155], a[8:11], 0\n"
    "v_mfma_f32_16x16x32_bf16 v[100:103], a[156:159], a[12:15], v[100:103]\n"
    "ds_read_b64_tr_b16 a[138:139], v243 offset:1536\n"
    "s_waitcnt vmcnt(6)\n"
    "v_mfma_f32_16x16x32_bf16 v[104:107], a[152:155], a[16:19], 0\n"
    "v_mfma_f32_16x16x32_bf16 v[104:107], a[156:159], a[20:23], v[104:107]\n"
    "ds_read_b64_tr_b16 a[140:141], v243 offset:8704\n"
    "buffer_store_dwordx4 a[192:195], v224, s[84:87], s65 offen offset:0 sc0 sc1\n"
    "v_mfma_f32_16x16x32_bf16 v[108:111], a[152:155], a[24:27], 0\n"
    "v_mfma_f32_16x16x32_bf16 v[108:111], a[156:159], a[28:31], v[108:111]\n"
    "ds_read_b64_tr_b16 a[142:143], v243 offset:9728\n"
    "v_mfma_f32_16x16x32_bf16 v[112:115], a[152:155], a[32:35], 0\n"
    "s_add_u32 m0, s58, 12288\n"
    "v_mfma_f32_16x16x32_bf16 v[112:115], a[156:159], a[36:39], v[112:115]\n"
    "buffer_load_dwordx4 v238, s[40:43], s54 offen nt lds\n"
    "v_mfma_f32_16x16x32_bf16 v[116:119], a[152:155], a[40:43], 0\n"
    "v_mfma_f32_16x16x32_bf16 v[116:119], a[156:159], a[44:47], v[116:119]\n"
    "ds_read_b128 a[88:91], v240 offset:38912\n"
    "v_mfma_f32_16x16x32_bf16 v[120:123], a[152:155], a[48:51], 0\n"
    "s_add_u32 m0, s58, 13312\n"
    "v_mfma_f32_16x16x32_bf16 v[120:123], a[156:159], a[52:55], v[120:123]\n"
    "buffer_store_dwordx4 a[196:199], v225, s[84:87], s65 offen offset:0 sc0 sc1\n"
    "buffer_load_dwordx4 v239, s[40:43], s54 offen nt lds\n"
    "v_mfma_f32_16x16x32_bf16 v[124:127], a[152:155], a[56:59], 0\n"
    "v_mfma_f32_16x16x32_bf16 v[124:127], a[156:159], a[60:63], v[124:127]\n"
    "ds_read_b128 a[92:95], v241 offset:38912\n"
    "s_waitcnt lgkmcnt(8)\n"
    "v_mfma_f32_16x16x32_bf16 v[128:131], a[128:131], a[0:3], 0\n"
    "v_mfma_f32_16x16x32_bf16 v[128:131], a[132:135], a[4:7], v[128:131]\n"
    "ds_read_b64_tr_b16 a[144:145], v242 offset:768\n"
    "v_mfma_f32_16x16x32_bf16 v[132:135], a[128:131], a[8:11], 0\n"
    "s_add_u32 s58, s58, 32768\n"
    "v_mfma_f32_16x16x32_bf16 v[132:135], a[132:135], a[12:15], v[132:135]\n"
    "ds_read_b64_tr_b16 a[146:147], v242 offset:1792\n"
    "v_mfma_f32_16x16x32_bf16 v[136:139], a[128:131], a[16:19], 0\n"
    "s_sub_u32 s59, s58, 98304\n"
    "v_mfma_f32_16x16x32_bf16 v[136:139], a[132:135], a[20:23], v[136:139]\n"
    "ds_read_b64_tr_b16 a[148:149], v242 offset:8960\n"
    "buffer_store_dwordx4 a[200:203], v226, s[84:87], s65 offen offset:0 sc0 sc1\n"
    "v_mfma_f32_16x16x32_bf16 v[140:143], a[128:131], a[24:27], 0\n"
    "s_cmp_ge_u32 s58, s67\n"
    "v_mfma_f32_16x16x32_bf16 v[140:143], a[132:135], a[28:31], v[140:143]\n"
    "ds_read_b64_tr_b16 a[150:151], v242 offset:9984\n"
    "v_mfma_f32_16x16x32_bf16 v[144:147], a[128:131], a[32:35], 0\n"
    "s_cselect_b32 s58, s59, s58\n"
    "v_mfma_f32_16x16x32_bf16 v[144:147], a[132:135], a[36:39], v[144:147]\n"
    "ds_read_b128 a[96:99], v240 offset:40960\n"
    "v_mfma_f32_16x16x32_bf16 v[148:151], a[128:131], a[40:43], 0\n"
    "v_mfma_f32_16x16x32_bf16 v[148:151], a[132:135], a[44:47], v[148:151]\n"
    "ds_read_b128 a[100:103], v241 offset:40960\n"
    "v_mfma_f32_16x16x32_bf16 v[152:155], a[128:131], a[48:51], 0\n"
    "v_mfma_f32_16x16x32_bf16 v[152:155], a[132:135], a[52:55], v[152:155]\n"
    "ds_read_b128 a[104:107], v240 offset:43008\n"
    "buffer_store_dwordx4 a[204:207], v227, s[84:87], s65 offen offset:0 sc0 sc1\n"
    "v_mfma_f32_16x16x32_bf16 v[156:159], a[128:131], a[56:59], 0\n"
    "v_mfma_f32_16x16x32_bf16 v[156:159], a[132:135], a[60:63], v[156:159]\n"
    "ds_read_b128 a[108:111], v241 offset:43008\n"
    "s_waitcnt lgkmcnt(10)\n"
    "v_mfma_f32_16x16x32_bf16 v[160:163], a[136:139], a[0:3], 0\n"
    "v_mfma_f32_16x16x32_bf16 v[160:163], a[140:143], a[4:7], v[160:163]\n"
    "ds_read_b64_tr_b16 a[152:153], v243 offset:768\n"
    "ds_read_b128 a[112:115], v240 offset:45056\n"
    "v_mfma_f32_16x16x32_bf16 v[164:167], a[136:139], a[8:11], 0\n"
    "s_add_u32 s59, s52, 3\n"
    "v_mfma_f32_16x16x32_bf16 v[164:167], a[140:143], a[12:15], v[164:167]\n"
    "ds_read_b64_tr_b16 a[154:155], v243 offset:1792\n"
    "ds_read_b128 a[116:119], v241 offset:45056\n"
    "v_readlane_b32 s53, v244, s59\n"
    "v_mfma_f32_16x16x32_bf16 v[168:171], a[136:139], a[16:19], 0\n"
    "v_mfma_f32_16x16x32_bf16 v[168:171], a[140:143], a[20:23], v[168:171]\n"
    "ds_read_b64_tr_b16 a[156:157], v243 offset:8960\n"
    "ds_read_b128 a[120:123], v240 offset:47104\n"
    "buffer_store_dwordx4 a[208:211], v228, s[84:87], s65 offen offset:0 sc0 sc1\n"
    "v_mfma_f32_16x16x32_bf16 v[172:175], a[136:139], a[24:27], 0\n"
    "s_memrealtime s[34:35]\n"
    "v_mfma_f32_16x16x32_bf16 v[172:175], a[140:143], a[28:31], v[172:175]\n"
    "ds_read_b64_tr_b16 a[158:159], v243 offset:9984\n"
    "ds_read_b128 a[124:127], v241 offset:47104\n"
    "v_mfma_f32_16x16x32_bf16 v[176:179], a[136:139], a[32:35], 0\n"
    "s_add_u32 s57, s57, 1\n"
    "v_mfma_f32_16x16x32_bf16 v[176:179], a[140:143], a[36:39], v[176:179]\n"
    "s_cmp_eq_u32 s57, 3\n"
    "v_mfma_f32_16x16x32_bf16 v[180:183], a[136:139], a[40:43], 0\n"
    "s_cselect_b32 s57, 0, s57\n"
    "s_cselect_b32 s59, 4294901760, s68\n"
    "v_mfma_f32_16x16x32_bf16 v[180:183], a[140:143], a[44:47], v[180:183]\n"
    "v_mfma_f32_16x16x32_bf16 v[184:187], a[136:139], a[48:51], 0\n"
    "v_add_u32 v242, s59, v242\n"
    "v_mfma_f32_16x16x32_bf16 v[184:187], a[140:143], a[52:55], v[184:187]\n"
    "buffer_store_dwordx4 a[212:215], v229, s[84:87], s65 offen offset:0 sc0 sc1\n"
    "v_add_u32 v243, s59, v243\n"
    "v_mfma_f32_16x16x32_bf16 v[188:191], a[136:139], a[56:59], 0\n"
    "v_mfma_f32_16x16x32_bf16 v[188:191], a[140:143], a[60:63], v[188:191]\n"
    "s_waitcnt vmcnt(14) lgkmcnt(0)\n"
    "s_sub_u32 s34, s34, s36\n"
    "s_cmp_lt_i32 s34, -8\n"
    "s_cbranch_scc1 .Lg5_%=\n"
    ".Lg6_%=:\n"
    "s_barrier\n"
    "v_mfma_f32_16x16x32_bf16 v[192:195], a[144:147], a[0:3], 0\n"
    "v_mfma_f32_16x16x32_bf16 v[192:195], a[148:151], a[4:7], v[192:195]\n"
    "ds_read_b64_tr_b16 a[128:129], v242 offset:0\n"
    "v_mfma_f32_16x16x32_bf16 v[196:199], a[144:147], a[8:11], 0\n"
    "v_mfma_f32_16x16x32_bf16 v[196:199], a[148:151], a[12:15], v[196:199]\n"
    "ds_read_b64_tr_b16 a[130:131], v242 offset:1024\n"
    "v_mfma_f32_16x16x32_bf16 v[200:203], a[144:147], a[16:19], 0\n"
    "v_mfma_f32_16x16x32_bf16 v[200:203], a[148:151], a[20:23], v[200:203]\n"
    "ds_read_b64_tr_b16 a[132:133], v242 offset:8192\n"
    "v_mfma_f32_16x16x32_bf16 v[204:207], a[144:147], a[24:27], 0\n"
    "v_mfma_f32_16x16x32_bf16 v[204:207], a[148:151], a[28:31], v[204:207]\n"
    "buffer_store_dwordx4 a[216:219], v230, s[84:87], s65 offen offset:0 sc0 sc1\n"
    "ds_read_b64_tr_b16 a[134:135], v242 offset:9216\n"
    "v_mfma_f32_16x16x32_bf16 v[208:211], a[144:147], a[32:35], 0\n"
    "s_add_u32 m0, s55, 32768\n"
    "v_mfma_f32_16x16x32_bf16 v[208:211], a[148:151], a[36:39], v[208:211]\n"
    "buffer_load_dwordx4 v224, s[44:47], s53 offen sc1 lds\n"
    "v_mfma_f32_16x16x32_bf16 v[212:215], a[144:147], a[40:43], 0\n"
    "s_add_u32 m0, s55, 33792\n"
    "v_mfma_f32_16x16x32_bf16 v[212:215], a[148:151], a[44:47], v[212:215]\n"
    "buffer_load_dwordx4 v225, s[44:47], s53 offen sc1 lds\n"
    "v_mfma_f32_16x16x32_bf16 v[216:219], a[144:147], a[48:51], 0\n"
    "s_add_u32 m0, s55, 34816\n"
    "v_mfma_f32_16x16x32_bf16 v[216:219], a[148:151], a[52:55], v[216:219]\n"
    "buffer_load_dwordx4 v226, s[44:47], s53 offen sc1 lds\n"
    "v_mfma_f32_16x16x32_bf16 v[220:223], a[144:147], a[56:59], 0\n"
    "s_add_u32 m0, s55, 35840\n"
    "v_mfma_f32_16x16x32_bf16 v[220:223], a[148:151], a[60:63], v[220:223]\n"
    "buffer_load_dwordx4 v227, s[44:47], s53 offen sc1 lds\n"
    "s_waitcnt lgkmcnt(5)\n"
    "v_mfma_f32_16x16x32_bf16 a[160:163], a[152:155], a[0:3], 0\n"
    "v_mfma_f32_16x16x32_bf16 a[160:163], a[156:159], a[4:7], a[160:163]\n"
    "ds_read_b64_tr_b16 a[136:137], v243 offset:0\n"
    "v_mfma_f32_16x16x32_bf16 a[164:167], a[152:155], a[8:11], 0\n"
    "v_mfma_f32_16x16x32_bf16 a[164:167], a[156:159], a[12:15], a[164:167]\n"
    "ds_read_b64_tr_b16 a[138:139], v243 offset:1024\n"
    "v_mfma_f32_16x16x32_bf16 a[168:171], a[152:155], a[16:19], 0\n"
    "v_mfma_f32_16x16x32_bf16 a[168:171], a[156:159], a[20:23], a[168:171]\n"
    "buffer_store_dwordx4 a[220:223], v231, s[84:87], s65 offen offset:0 sc0 sc1\n"
    "ds_read_b64_tr_b16 a[140:141], v243 offset:8192\n"
    "v_mfma_f32_16x16x32_bf16 a[172:175], a[152:155], a[24:27], 0\n"
    "v_mfma_f32_16x16x32_bf16 a[172:175], a[156:159], a[28:31], a[172:175]\n"
    "ds_read_b64_tr_b16 a[142:143], v243 offset:9216\n"
    "v_mfma_f32_16x16x32_bf16 a[176:179], a[152:155], a[32:35], 0\n"
    "s_add_u32 m0, s55, 36864\n"
    "v_mfma_f32_16x16x32_bf16 a[176:179], a[156:159], a[36:39], a[176:179]\n"
    "buffer_load_dwordx4 v228, s[44:47], s53 offen sc1 lds\n"
    "v_mfma_f32_16x16x32_bf16 a[180:183], a[152:155], a[40:43], 0\n"
    "s_add_u32 m0, s55, 37888\n"
    "v_mfma_f32_16x16x32_bf16 a[180:183], a[156:159], a[44:47], a[180:183]\n"
    "buffer_load_dwordx4 v229, s[44:47], s53 offen sc1 lds\n"
    "v_mfma_f32_16x16x32_bf16 a[184:187], a[152:155], a[48:51], 0\n"
    "s_add_u32 m0, s55, 38912\n"
    "v_mfma_f32_16x16x32_bf16 a[184:187], a[156:159], a[52:55], a[184:187]\n"
    "buffer_load_dwordx4 v230, s[44:47], s53 offen sc1 lds\n"
    "v_mfma_f32_16x16x32_bf16 a[188:191], a[152:155], a[56:59], 0\n"
    "s_add_u32 m0, s55, 39936\n"
    "v_mfma_f32_16x16x32_bf16 a[188:191], a[156:159], a[60:63], a[188:191]\n"
    "buffer_load_dwordx4 v231, s[44:47], s53 offen sc1 lds\n"
    "s_add_u32 s52, s52, 1\n"
    "s_branch .Lg4_%=\n"
    ".Lg5_%=:\n"
    "s_sleep 1\n"
    "s_memrealtime s[34:35]\n"
    "s_waitcnt lgkmcnt(0)\n"
    "s_sub_u32 s34, s34, s36\n"
    "s_cmp_lt_i32 s34, 0\n"
    "s_cbranch_scc1 .Lg5_%=\n"
    "s_mov_b32 s34, 0\n"
    "s_branch .Lg6_%=\n"
    ".Lg7_%=:\n"
    "s_sleep 1\n"
    "s_memrealtime s[34:35]\n"
    "s_waitcnt lgkmcnt(0)\n"
    "s_sub_u32 s34, s34, s36\n"
    "s_cmp_lt_i32 s34, 0\n"
    "s_cbranch_scc1 .Lg7_%=\n"
    "s_mov_b32 s34, 0\n"
    "s_branch .Lg8_%=\n"
    ".Lg9_%=:\n"
    "s_memrealtime s[92:93]\n"
    "s_waitcnt lgkmcnt(0)\n"
    ".Lg11_%=:\n"
    "buffer_load_dword v252, v246, s[88:91], 0 offen sc0 sc1\n"
    "buffer_load_dword v253, off, s[88:91], 0 sc1\n"
    "s_memrealtime s[94:95]\n"
    "s_waitcnt vmcnt(0) lgkmcnt(0)\n"
    "v_cmp_le_u32 vcc, s63, v252\n"
    "s_or_b64 vcc, vcc, s[78:79]\n"
    "s_cmp_eq_u64 vcc, -1\n"
    "s_cbranch_scc1 .Lg10_%=\n"
    "v_readfirstlane_b32 s59, v253\n"
    "s_cmp_eq_u32 s59, s63\n"
    "s_cbranch_scc1 .Lg10_%=\n"
    "s_sleep 4\n"
    "s_sub_u32 s94, s94, s92\n"
    "s_cmp_lt_u32 s94, 50000000\n"
    "s_cbranch_scc1 .Lg11_%=\n"
    "buffer_store_dword v254, v251, s[88:91], 0 offen sc1\n"
    "s_waitcnt vmcnt(0)\n"
    "s_branch .Lg10_%=\n"
    ".Lg12_%=:\n"
    "s_sleep 1\n"
    "s_memrealtime s[34:35]\n"
    "s_waitcnt lgkmcnt(0)\n"
    "s_sub_u32 s34, s34, s36\n"
    "s_cmp_lt_i32 s34, 0\n"
    "s_cbranch_scc1 .Lg12_%=\n"
    "s_mov_b32 s34, 0\n"
    "s_branch .Lg13_%=\n"
    ".Lg15_%=:\n"
    "s_waitcnt lgkmcnt(4)\n"
    "v_mfma_f32_16x16x32_bf16 v[0:3], a[128:131], a[0:3], v[0:3]\n"
    "buffer_load_dwordx4 a[192:195], v224, s[80:83], s64 offen offset:0\n"
    "s_add_u32 s59, s52, 2\n"
    "v_mfma_f32_16x16x32_bf16 v[0:3], a[132:135], a[4:7], v[0:3]\n"
    "ds_read_b64_tr_b16 a[144:145], v242 offset:256\n"
    "buffer_load_dwordx4 a[196:199], v225, s[80:83], s64 offen offset:0\n"
    "v_readlane_b32 s54, v245, s59\n"
    "v_mfma_f32_16x16x32_bf16 v[4:7], a[128:131], a[8:11], v[4:7]\n"
    "buffer_load_dwordx4 a[200:203], v226, s[80:83], s64 offen offset:0\n"
    "v_mfma_f32_16x16x32_bf16 v[4:7], a[132:135], a[12:15], v[4:7]\n"
    "ds_read_b64_tr_b16 a[146:147], v242 offset:1280\n"
    "buffer_load_dwordx4 a[204:207], v227, s[80:83], s64 offen offset:0\n"
    "v_mfma_f32_16x16x32_bf16 v[8:11], a[128:131], a[16:19], v[8:11]\n"
    "buffer_load_dwordx4 a[208:211], v228, s[80:83], s64 offen offset:0\n"
    "v_mfma_f32_16x16x32_bf16 v[8:11], a[132:135], a[20:23], v[8:11]\n"
    "ds_read_b64_tr_b16 a[148:149], v242 offset:8448\n"
    "buffer_load_dwordx4 a[212:215], v229, s[80:83], s64 offen offset:0\n"
    "v_mfma_f32_16x16x32_bf16 v[12:15], a[128:131], a[24:27], v[12:15]\n"
    "buffer_load_dwordx4 a[216:219], v230, s[80:83], s64 offen offset:0\n"
    "v_mfma_f32_16x16x32_bf16 v[12:15], a[132:135], a[28:31], v[12:15]\n"
    "ds_read_b64_tr_b16 a[150:151], v242 offset:9472\n"
    "buffer_load_dwordx4 a[220:223], v231, s[80:83], s64 offen offset:0\n"
    "v_mfma_f32_16x16x32_bf16 v[16:19], a[128:131], a[32:35], v[16:19]\n"
    "s_add_u32 m0, s58, 0\n"
    "v_mfma_f32_16x16x32_bf16 v[16:19], a[132:135], a[36:39], v[16:19]\n"
    "buffer_load_dwordx4 v232, s[40:43], s54 offen nt lds\n"
    "v_mfma_f32_16x16x32_bf16 v[20:23], a[128:131], a[40:43], v[20:23]\n"
    "v_mfma_f32_16x16x32_bf16 v[20:23], a[132:135], a[44:47], v[20:23]\n"
    "ds_read_b128 a[64:67], v240 offset:32768\n"
    "v_mfma_f32_16x16x32_bf16 v[24:27], a[128:131], a[48:51], v[24:27]\n"
    "s_add_u32 m0, s58, 1024\n"
    "v_mfma_f32_16x16x32_bf16 v[24:27], a[132:135], a[52:55], v[24:27]\n"
    "buffer_load_dwordx4 v233, s[40:43], s54 offen nt lds\n"
    "v_mfma_f32_16x16x32_bf16 v[28:31], a[128:131], a[56:59], v[28:31]\n"
    "v_mfma_f32_16x16x32_bf16 v[28:31], a[132:135], a[60:63], v[28:31]\n"
    "ds_read_b128 a[68:71], v241 offset:32768\n"
    "s_waitcnt lgkmcnt(6)\n"
    "v_mfma_f32_16x16x32_bf16 v[32:35], a[136:139], a[0:3], v[32:35]\n"
    "v_mfma_f32_16x16x32_bf16 v[32:35], a[140:143], a[4:7], v[32:35]\n"
    "ds_read_b64_tr_b16 a[152:153], v243 offset:256\n"
    "v_mfma_f32_16x16x32_bf16 v[36:39], a[136:139], a[8:11], v[36:39]\n"
    "v_mfma_f32_16x16x32_bf16 v[36:39], a[140:143], a[12:15], v[36:39]\n"
    "ds_read_b64_tr_b16 a[154:155], v243 offset:1280\n"
    "s_cmp_lt_u32 s52, s33\n"
    "s_cselect_b32 s59, s32, 0\n"
    "v_mfma_f32_16x16x32_bf16 v[40:43], a[136:139], a[16:19], v[40:43]\n"
    "s_cmp_lt_u32 s52, s38\n"
    "s_cselect_b32 s59, s37, s59\n"
    "v_mfma_f32_16x16x32_bf16 v[40:43], a[140:143], a[20:23], v[40:43]\n"
    "ds_read_b64_tr_b16 a[156:157], v243 offset:8448\n"
    "v_mfma_f32_16x16x32_bf16 v[44:47], a[136:139], a[24:27], v[44:47]\n"
    "s_add_u32 s36, s36, s59\n"
    "v_mfma_f32_16x16x32_bf16 v[44:47], a[140:143], a[28:31], v[44:47]\n"
    "ds_read_b64_tr_b16 a[158:159], v243 offset:9472\n"
    "v_mfma_f32_16x16x32_bf16 v[48:51], a[136:139], a[32:35], v[48:51]\n"
    "s_add_u32 m0, s58, 4096\n"
    "v_mfma_f32_16x16x32_bf16 v[48:51], a[140:143], a[36:39], v[48:51]\n"
    "buffer_load_dwordx4 v234, s[40:43], s54 offen nt lds\n"
    "v_mfma_f32_16x16x32_bf16 v[52:55], a[136:139], a[40:43], v[52:55]\n"
    "v_mfma_f32_16x16x32_bf16 v[52:55], a[140:143], a[44:47], v[52:55]\n"
    "ds_read_b128 a[72:75], v240 offset:34816\n"
    "v_mfma_f32_16x16x32_bf16 v[56:59], a[136:139], a[48:51], v[56:59]\n"
    "s_add_u32 m0, s58, 5120\n"
    "v_mfma_f32_16x16x32_bf16 v[56:59], a[140:143], a[52:55], v[56:59]\n"
    "buffer_load_dwordx4 v235, s[40:43], s54 offen nt lds\n"
    "v_mfma_f32_16x16x32_bf16 v[60:63], a[136:139], a[56:59], v[60:63]\n"
    "v_mfma_f32_16x16x32_bf16 v[60:63], a[140:143], a[60:63], v[60:63]\n"
    "ds_read_b128 a[76:79], v241 offset:34816\n"
    "s_waitcnt lgkmcnt(8)\n"
    "v_mfma_f32_16x16x32_bf16 v[64:67], a[144:147], a[0:3], v[64:67]\n"
    "v_mfma_f32_16x16x32_bf16 v[64:67], a[148:151], a[4:7], v[64:67]\n"
    "ds_read_b64_tr_b16 a[128:129], v242 offset:512\n"
    "v_mfma_f32_16x16x32_bf16 v[68:71], a[144:147], a[8:11], v[68:71]\n"
    "v_mfma_f32_16x16x32_bf16 v[68:71], a[148:151], a[12:15], v[68:71]\n"
    "ds_read_b64_tr_b16 a[130:131], v242 offset:1536\n"
    "v_mfma_f32_16x16x32_bf16 v[72:75], a[144:147], a[16:19], v[72:75]\n"
    "v_mfma_f32_16x16x32_bf16 v[72:75], a[148:151], a[20:23], v[72:75]\n"
    "ds_read_b64_tr_b16 a[132:133], v242 offset:8704\n"
    "v_mfma_f32_16x16x32_bf16 v[76:79], a[144:147], a[24:27], v[76:79]\n"
    "v_mfma_f32_16x16x32_bf16 v[76:79], a[148:151], a[28:31], v[76:79]\n"
    "ds_read_b64_tr_b16 a[134:135], v242 offset:9728\n"
    "v_mfma_f32_16x16x32_bf16 v[80:83], a[144:147], a[32:35], v[80:83]\n"
    "s_add_u32 m0, s58, 8192\n"
    "v_mfma_f32_16x16x32_bf16 v[80:83], a[148:151], a[36:39], v[80:83]\n"
    "buffer_load_dwordx4 v236, s[40:43], s54 offen nt lds\n"
    "v_mfma_f32_16x16x32_bf16 v[84:87], a[144:147], a[40:43], v[84:87]\n"
    "v_mfma_f32_16x16x32_bf16 v[84:87], a[148:151], a[44:47], v[84:87]\n"
    "ds_read_b128 a[80:83], v240 offset:36864\n"
    "v_mfma_f32_16x16x32_bf16 v[88:91], a[144:147], a[48:51], v[88:91]\n"
    "s_add_u32 m0, s58, 9216\n"
    "v_mfma_f32_16x16x32_bf16 v[88:91], a[148:151], a[52:55], v[88:91]\n"
    "buffer_load_dwordx4 v237, s[40:43], s54 offen nt lds\n"
    "v_mfma_f32_16x16x32_bf16 v[92:95], a[144:147], a[56:59], v[92:95]\n"
    "v_mfma_f32_16x16x32_bf16 v[92:95], a[148:151], a[60:63], v[92:95]\n"
    "ds_read_b128 a[84:87], v241 offset:36864\n"
    "s_waitcnt lgkmcnt(8)\n"
    "v_mfma_f32_16x16x32_bf16 v[96:99], a[152:155], a[0:3], v[96:99]\n"
    "v_mfma_f32_16x16x32_bf16 v[96:99], a[156:159], a[4:7], v[96:99]\n"
    "ds_read_b64_tr_b16 a[136:137], v243 offset:512\n"
    "v_mfma_f32_16x16x32_bf16 v[100:103], a[152:155], a[8:11], v[100:103]\n"
    "v_mfma_f32_16x16x32_bf16 v[100:103], a[156:159], a[12:15], v[100:103]\n"
    "ds_read_b64_tr_b16 a[138:139], v243 offset:1536\n"
    "s_waitcnt vmcnt(6)\n"
    "v_mfma_f32_16x16x32_bf16 v[104:107], a[152:155], a[16:19], v[104:107]\n"
    "v_mfma_f32_16x16x32_bf16 v[104:107], a[156:159], a[20:23], v[104:107]\n"
    "ds_read_b64_tr_b16 a[140:141], v243 offset:8704\n"
    "buffer_store_dwordx4 a[192:195], v224, s[84:87], s65 offen offset:0 sc0 sc1\n"
    "v_mfma_f32_16x16x32_bf16 v[108:111], a[152:155], a[24:27], v[108:111]\n"
    "v_mfma_f32_16x16x32_bf16 v[108:111], a[156:159], a[28:31], v[108:111]\n"
    "ds_read_b64_tr_b16 a[142:143], v243 offset:9728\n"
    "v_mfma_f32_16x16x32_bf16 v[112:115], a[152:155], a[32:35], v[112:115]\n"
    "s_add_u32 m0, s58, 12288\n"
    "v_mfma_f32_16x16x32_bf16 v[112:115], a[156:159], a[36:39], v[112:115]\n"
    "buffer_load_dwordx4 v238, s[40:43], s54 offen nt lds\n"
    "v_mfma_f32_16x16x32_bf16 v[116:119], a[152:155], a[40:43], v[116:119]\n"
    "v_mfma_f32_16x16x32_bf16 v[116:119], a[156:159], a[44:47], v[116:119]\n"
    "ds_read_b128 a[88:91], v240 offset:38912\n"
    "v_mfma_f32_16x16x32_bf16 v[120:123], a[152:155], a[48:51], v[120:123]\n"
    "s_add_u32 m0, s58, 13312\n"
    "v_mfma_f32_16x16x32_bf16 v[120:123], a[156:159], a[52:55], v[120:123]\n"
    "buffer_store_dwordx4 a[196:199], v225, s[84:87], s65 offen offset:0 sc0 sc1\n"
    "buffer_load_dwordx4 v239, s[40:43], s54 offen nt lds\n"
    "v_mfma_f32_16x16x32_bf16 v[124:127], a[152:155], a[56:59], v[124:127]\n"
    "v_mfma_f32_16x16x32_bf16 v[124:127], a[156:159], a[60:63], v[124:127]\n"
    "ds_read_b128 a[92:95], v241 offset:38912\n"
    "s_waitcnt lgkmcnt(8)\n"
    "v_mfma_f32_16x16x32_bf16 v[128:131], a[128:131], a[0:3], v[128:131]\n"
    "v_mfma_f32_16x16x32_bf16 v[128:131], a[132:135], a[4:7], v[128:131]\n"
    "ds_read_b64_tr_b16 a[144:145], v242 offset:768\n"
    "v_mfma_f32_16x16x32_bf16 v[132:135], a[128:131], a[8:11], v[132:135]\n"
    "s_add_u32 s58, s58, 32768\n"
    "v_mfma_f32_16x16x32_bf16 v[132:135], a[132:135], a[12:15], v[132:135]\n"
    "ds_read_b64_tr_b16 a[146:147], v242 offset:1792\n"
    "v_mfma_f32_16x16x32_bf16 v[136:139], a[128:131], a[16:19], v[136:139]\n"
    "s_sub_u32 s59, s58, 98304\n"
    "v_mfma_f32_16x16x32_bf16 v[136:139], a[132:135], a[20:23], v[136:139]\n"
    "ds_read_b64_tr_b16 a[148:149], v242 offset:8960\n"
    "buffer_store_dwordx4 a[200:203], v226, s[84:87], s65 offen offset:0 sc0 sc1\n"
    "v_mfma_f32_16x16x32_bf16 v[140:143], a[128:131], a[24:27], v[140:143]\n"
    "s_cmp_ge_u32 s58, s67\n"
    "v_mfma_f32_16x16x32_bf16 v[140:143], a[132:135], a[28:31], v[140:143]\n"
    "ds_read_b64_tr_b16 a[150:151], v242 offset:9984\n"
    "v_mfma_f32_16x16x32_bf16 v[144:147], a[128:131], a[32:35], v[144:147]\n"
    "s_cselect_b32 s58, s59, s58\n"
    "v_mfma_f32_16x16x32_bf16 v[144:147], a[132:135], a[36:39], v[144:147]\n"
    "ds_read_b128 a[96:99], v240 offset:40960\n"
    "v_mfma_f32_16x16x32_bf16 v[148:151], a[128:131], a[40:43], v[148:151]\n"
    "v_mfma_f32_16x16x32_bf16 v[148:151], a[132:135], a[44:47], v[148:151]\n"
    "ds_read_b128 a[100:103], v241 offset:40960\n"
    "v_mfma_f32_16x16x32_bf16 v[152:155], a[128:131], a[48:51], v[152:155]\n"
    "v_mfma_f32_16x16x32_bf16 v[152:155], a[132:135], a[52:55], v[152:155]\n"
    "ds_read_b128 a[104:107], v240 offset:43008\n"
    "buffer_store_dwordx4 a[204:207], v227, s[84:87], s65 offen offset:0 sc0 sc1\n"
    "v_mfma_f32_16x16x32_bf16 v[156:159], a[128:131], a[56:59], v[156:159]\n"
    "v_mfma_f32_16x16x32_bf16 v[156:159], a[132:135], a[60:63], v[156:159]\n"
    "ds_read_b128 a[108:111], v241 offset:43008\n"
    "s_waitcnt lgkmcnt(10)\n"
    "v_mfma_f32_16x16x32_bf16 v[160:163], a[136:139], a[0:3], v[160:163]\n"
    "v_mfma_f32_16x16x32_bf16 v[160:163], a[140:143], a[4:7], v[160:163]\n"
    "ds_read_b64_tr_b16 a[152:153], v243 offset:768\n"
    "ds_read_b128 a[112:115], v240 offset:45056\n"
    "v_mfma_f32_16x16x32_bf16 v[164:167], a[136:139], a[8:11], v[164:167]\n"
    "s_add_u32 s59, s52, 3\n"
    "v_mfma_f32_16x16x32_bf16 v[164:167], a[140:143], a[12:15], v[164:167]\n"
    "ds_read_b64_tr_b16 a[154:155], v243 offset:1792\n"
    "ds_read_b128 a[116:119], v241 offset:45056\n"
    "v_readlane_b32 s53, v244, s59\n"
    "v_mfma_f32_16x16x32_bf16 v[168:171], a[136:139], a[16:19], v[168:171]\n"
    "v_mfma_f32_16x16x32_bf16 v[168:171], a[140:143], a[20:23], v[168:171]\n"
    "ds_read_b64_tr_b16 a[156:157], v243 offset:8960\n"
    "ds_read_b128 a[120:123], v240 offset:47104\n"
    "buffer_store_dwordx4 a[208:211], v228, s[84:87], s65 offen offset:0 sc0 sc1\n"
    "v_mfma_f32_16x16x32_bf16 v[172:175], a[136:139], a[24:27], v[172:175]\n"
    "s_memrealtime s[34:35]\n"
    "v_mfma_f32_16x16x32_bf16 v[172:175], a[140:143], a[28:31], v[172:175]\n"
    "ds_read_b64_tr_b16 a[158:159], v243 offset:9984\n"
    "ds_read_b128 a[124:127], v241 offset:47104\n"
    "v_mfma_f32_16x16x32_bf16 v[176:179], a[136:139], a[32:35], v[176:179]\n"
    "s_add_u32 s57, s57, 1\n"
    "v_mfma_f32_16x16x32_bf16 v[176:179], a[140:143], a[36:39], v[176:179]\n"
    "s_cmp_eq_u32 s57, 3\n"
    "v_mfma_f32_16x16x32_bf16 v[180:183], a[136:139], a[40:43], v[180:183]\n"
    "s_cselect_b32 s57, 0, s57\n"
    "s_cselect_b32 s59, 4294901760, s68\n"
    "v_mfma_f32_16x16x32_bf16 v[180:183], a[140:143], a[44:47], v[180:183]\n"
    "v_mfma_f32_16x16x32_bf16 v[184:187], a[136:139], a[48:51], v[184:187]\n"
    "v_add_u32 v242, s59, v242\n"
    "v_mfma_f32_16x16x32_bf16 v[184:187], a[140:143], a[52:55], v[184:187]\n"
    "buffer_store_dwordx4 a[212:215], v229, s[84:87], s65 offen offset:0 sc0 sc1\n"
    "v_add_u32 v243, s59, v243\n"
    "v_mfma_f32_16x16x32_bf16 v[188:191], a[136:139], a[56:59], v[188:191]\n"
    "v_mfma_f32_16x16x32_bf16 v[188:191], a[140:143], a[60:63], v[188:191]\n"
    "s_waitcnt vmcnt(14) lgkmcnt(0)\n"
    "s_sub_u32 s34, s34, s36\n"
    "s_cmp_lt_i32 s34, -8\n"
    "s_cbranch_scc1 .Lg17_%=\n"
    ".Lg18_%=:\n"
    "s_barrier\n"
    "v_mfma_f32_16x16x32_bf16 v[192:195], a[144:147], a[0:3], v[192:195]\n"
    "v_mfma_f32_16x16x32_bf16 v[192:195], a[148:151], a[4:7], v[192:195]\n"
    "ds_read_b64_tr_b16 a[128:129], v242 offset:0\n"
    "v_mfma_f32_16x16x32_bf16 v[196:199], a[144:147], a[8:11], v[196:199]\n"
    "v_mfma_f32_16x16x32_bf16 v[196:199], a[148:151], a[12:15], v[196:199]\n"
    "ds_read_b64_tr_b16 a[130:131], v242 offset:1024\n"
    "v_mfma_f32_16x16x32_bf16 v[200:203], a[144:147], a[16:19], v[200:203]\n"
    "v_mfma_f32_16x16x32_bf16 v[200:203], a[148:151], a[20:23], v[200:203]\n"
    "ds_read_b64_tr_b16 a[132:133], v242 offset:8192\n"
    "v_mfma_f32_16x16x32_bf16 v[204:207], a[144:147], a[24:27], v[204:207]\n"
    "v_mfma_f32_16x16x32_bf16 v[204:207], a[148:151], a[28:31], v[204:207]\n"
    "buffer_store_dwordx4 a[216:219], v230, s[84:87], s65 offen offset:0 sc0 sc1\n"
    "ds_read_b64_tr_b16 a[134:135], v242 offset:9216\n"
    "v_mfma_f32_16x16x32_bf16 v[208:211], a[144:147], a[32:35], v[208:211]\n"
    "s_add_u32 m0, s55, 32768\n"
    "v_mfma_f32_16x16x32_bf16 v[208:211], a[148:151], a[36:39], v[208:211]\n"
    "buffer_load_dwordx4 v224, s[44:47], s53 offen sc1 lds\n"
    "v_mfma_f32_16x16x32_bf16 v[212:215], a[144:147], a[40:43], v[212:215]\n"
    "s_add_u32 m0, s55, 33792\n"
    "v_mfma_f32_16x16x32_bf16 v[212:215], a[148:151], a[44:47], v[212:215]\n"
    "buffer_load_dwordx4 v225, s[44:47], s53 offen sc1 lds\n"
    "v_mfma_f32_16x16x32_bf16 v[216:219], a[144:147], a[48:51], v[216:219]\n"
    "s_add_u32 m0, s55, 34816\n"
    "v_mfma_f32_16x16x32_bf16 v[216:219], a[148:151], a[52:55], v[216:219]\n"
    "buffer_load_dwordx4 v226, s[44:47], s53 offen sc1 lds\n"
    "v_mfma_f32_16x16x32_bf16 v[220:223], a[144:147], a[56:59], v[220:223]\n"
    "s_add_u32 m0, s55, 35840\n"
    "v_mfma_f32_16x16x32_bf16 v[220:223], a[148:151], a[60:63], v[220:223]\n"
    "buffer_load_dwordx4 v227, s[44:47], s53 offen sc1 lds\n"
    "s_waitcnt lgkmcnt(5)\n"
    "v_mfma_f32_16x16x32_bf16 a[160:163], a[152:155], a[0:3], a[160:163]\n"
    "v_mfma_f32_16x16x32_bf16 a[160:163], a[156:159], a[4:7], a[160:163]\n"
    "ds_read_b64_tr_b16 a[136:137], v243 offset:0\n"
    "v_mfma_f32_16x16x32_bf16 a[164:167], a[152:155], a[8:11], a[164:167]\n"
    "v_mfma_f32_16x16x32_bf16 a[164:167], a[156:159], a[12:15], a[164:167]\n"
    "ds_read_b64_tr_b16 a[138:139], v243 offset:1024\n"
    "v_mfma_f32_16x16x32_bf16 a[168:171], a[152:155], a[16:19], a[168:171]\n"
    "v_mfma_f32_16x16x32_bf16 a[168:171], a[156:159], a[20:23], a[168:171]\n"
    "buffer_store_dwordx4 a[220:223], v231, s[84:87], s65 offen offset:0 sc0 sc1\n"
    "ds_read_b64_tr_b16 a[140:141], v243 offset:8192\n"
    "v_mfma_f32_16x16x32_bf16 a[172:175], a[152:155], a[24:27], a[172:175]\n"
    "v_mfma_f32_16x16x32_bf16 a[172:175], a[156:159], a[28:31], a[172:175]\n"
    "ds_read_b64_tr_b16 a[142:143], v243 offset:9216\n"
    "v_mfma_f32_16x16x32_bf16 a[176:179], a[152:155], a[32:35], a[176:179]\n"
    "s_add_u32 m0, s55, 36864\n"
    "v_mfma_f32_16x16x32_bf16 a[176:179], a[156:159], a[36:39], a[176:179]\n"
    "buffer_load_dwordx4 v228, s[44:47], s53 offen sc1 lds\n"
    "v_mfma_f32_16x16x32_bf16 a[180:183], a[152:155], a[40:43], a[180:183]\n"
    "s_add_u32 m0, s55, 37888\n"
    "v_mfma_f32_16x16x32_bf16 a[180:183], a[156:159], a[44:47], a[180:183]\n"
    "buffer_load_dwordx4 v229, s[44:47], s53 offen sc1 lds\n"
    "v_mfma_f32_16x16x32_bf16 a[184:187], a[152:155], a[48:51], a[184:187]\n"
    "s_add_u32 m0, s55, 38912\n"
    "v_mfma_f32_16x16x32_bf16 a[184:187], a[156:159], a[52:55], a[184:187]\n"
    "buffer_load_dwordx4 v230, s[44:47], s53 offen sc1 lds\n"
    "v_mfma_f32_16x16x32_bf16 a[188:191], a[152:155], a[56:59], a[188:191]\n"
    "s_add_u32 m0, s55, 39936\n"
    "v_mfma_f32_16x16x32_bf16 a[188:191], a[156:159], a[60:63], a[188:191]\n"
    "buffer_load_dwordx4 v231, s[44:47], s53 offen sc1 lds\n"
    "s_add_u32 s52, s52, 1\n"
    "s_branch .Lg16_%=\n"
    ".Lg17_%=:\n"
    "s_sleep 1\n"
    "s_memrealtime s[34:35]\n"
    "s_waitcnt lgkmcnt(0)\n"
    "s_sub_u32 s34, s34, s36\n"
    "s_cmp_lt_i32 s34, 0\n"
    "s_cbranch_scc1 .Lg17_%=\n"
    "s_mov_b32 s34, 0\n"
    "s_branch .Lg18_%=\n"
    ".Lg19_%=:\n"
    "s_sleep 1\n"
    "s_memrealtime s[34:35]\n"
    "s_waitcnt lgkmcnt(0)\n"
    "s_sub_u32 s34, s34, s36\n"
    "s_cmp_lt_i32 s34, 0\n"
    "s_cbranch_scc1 .Lg19_%=\n"
    "s_mov_b32 s34, 0\n"
    "s_branch .Lg20_%=\n"
    ".Lg21_%=:\n"
    "s_memrealtime s[92:93]\n"
    "s_waitcnt lgkmcnt(0)\n"
    ".Lg23_%=:\n"
    "buffer_load_dword v252, v246, s[88:91], 0 offen sc0 sc1\n"
    "buffer_load_dword v253, off, s[88:91], 0 sc1\n"
    "s_memrealtime s[94:95]\n"
    "s_waitcnt vmcnt(0) lgkmcnt(0)\n"
    "v_cmp_le_u32 vcc, s63, v252\n"
    "s_or_b64 vcc, vcc, s[78:79]\n"
    "s_cmp_eq_u64 vcc, -1\n"
    "s_cbranch_scc1 .Lg22_%=\n"
    "v_readfirstlane_b32 s59, v253\n"
    "s_cmp_eq_u32 s59, s63\n"
    "s_cbranch_scc1 .Lg22_%=\n"
    "s_sleep 4\n"
    "s_sub_u32 s94, s94, s92\n"
    "s_cmp_lt_u32 s94, 50000000\n"
    "s_cbranch_scc1 .Lg23_%=\n"
    "buffer_store_dword v254, v251, s[88:91], 0 offen sc1\n"
    "s_waitcnt vmcnt(0)\n"
    "s_branch .Lg22_%=\n"
    ".Lg24_%=:\n"
    "s_sleep 1\n"
    "s_memrealtime s[34:35]\n"
    "s_waitcnt lgkmcnt(0)\n"
    "s_sub_u32 s34, s34, s36\n"
    "s_cmp_lt_i32 s34, 0\n"
    "s_cbranch_scc1 .Lg24_%=\n"
    "s_mov_b32 s34, 0\n"
    "s_branch .Lg25_%=\n"
    ".Lg29_%=:\n"
    "s_memrealtime s[92:93]\n"
    "s_waitcnt lgkmcnt(0)\n"
    ".Lg31_%=:\n"
    "buffer_load_dword v252, v246, s[88:91], 0 offen sc0 sc1\n"
    "buffer_load_dword v253, off, s[88:91], 0 sc1\n"
    "s_memrealtime s[94:95]\n"
    "s_waitcnt vmcnt(0) lgkmcnt(0)\n"
    "v_cmp_le_u32 vcc, s63, v252\n"
    "s_or_b64 vcc, vcc, s[78:79]\n"
    "s_cmp_eq_u64 vcc, -1\n"
    "s_cbranch_scc1 .Lg30_%=\n"
    "v_readfirstlane_b32 s59, v253\n"
    "s_cmp_eq_u32 s59, s63\n"
    "s_cbranch_scc1 .Lg30_%=\n"
    "s_sleep 4\n"
    "s_sub_u32 s94, s94, s92\n"
    "s_cmp_lt_u32 s94, 50000000\n"
    "s_cbranch_scc1 .Lg31_%=\n"
    "buffer_store_dword v254, v251, s[88:91], 0 offen sc1\n"
    "s_waitcnt vmcnt(0)\n"
    "s_branch .Lg30_%=\n"
    ".Lg32_%=:\n"
    "s_sleep 1\n"
    "s_memrealtime s[34:35]\n"
    "s_waitcnt lgkmcnt(0)\n"
    "s_sub_u32 s34, s34, s36\n"
    "s_cmp_lt_i32 s34, 0\n"
    "s_cbranch_scc1 .Lg32_%=\n"
    "s_mov_b32 s34, 0\n"
    "s_branch .Lg33_%=\n"
    ".Lg34_%=:\n"
    "s_nop 7\n"
    "s_nop 7\n"
    "buffer_store_dwordx2 v[254:255], v251, s[88:91], s69 offen sc1\n"
    "v_mov_b32 v0, 0x7fc07fc0\n"
    "v_mov_b32 v1, 0x7fc07fc0\n"
    "v_mov_b32 v2, 0x7fc07fc0\n"
    "v_mov_b32 v3, 0x7fc07fc0\n"
    "buffer_store_dwordx4 v[0:3], v248, s[96:99], 0 offen offset:0\n"
    "buffer_store_dwordx4 v[0:3], v246, s[96:99], 0 offen offset:0\n"
    "buffer_store_dwordx4 v[0:3], v248, s[96:99], 0 offen offset:128\n"
    "buffer_store_dwordx4 v[0:3], v246, s[96:99], 0 offen offset:128\n"
    "buffer_store_dwordx4 v[0:3], v248, s[96:99], s44 offen offset:0\n"
    "buffer_store_dwordx4 v[0:3], v246, s[96:99], s44 offen offset:0\n"
    "buffer_store_dwordx4 v[0:3], v248, s[96:99], s44 offen offset:128\n"
    "buffer_store_dwordx4 v[0:3], v246, s[96:99], s44 offen offset:128\n"
    "buffer_store_dwordx4 v[0:3], v248, s[96:99], s45 offen offset:0\n"
    "buffer_store_dwordx4 v[0:3], v246, s[96:99], s45 offen offset:0\n"
    "buffer_store_dwordx4 v[0:3], v248, s[96:99], s45 offen offset:128\n"
    "buffer_store_dwordx4 v[0:3], v246, s[96:99], s45 offen offset:128\n"
    "buffer_store_dwordx4 v[0:3], v248, s[96:99], s46 offen offset:0\n"
    "buffer_store_dwordx4 v[0:3], v246, s[96:99], s46 offen offset:0\n"
    "buffer_store_dwordx4 v[0:3], v248, s[96:99], s46 offen offset:128\n"
    "buffer_store_dwordx4 v[0:3], v246, s[96:99], s46 offen offset:128\n"
    "buffer_store_dwordx4 v[0:3], v248, s[96:99], s47 offen offset:0\n"
    "buffer_store_dwordx4 v[0:3], v246, s[96:99], s47 offen offset:0\n"
    "buffer_store_dwordx4 v[0:3], v248, s[96:99], s47 offen offset:128\n"
    "buffer_store_dwordx4 v[0:3], v246, s[96:99], s47 offen offset:128\n"
    "buffer_store_dwordx4 v[0:3], v248, s[96:99], s48 offen offset:0\n"
    "buffer_store_dwordx4 v[0:3], v246, s[96:99], s48 offen offset:0\n"
    "buffer_store_dwordx4 v[0:3], v248, s[96:99], s48 offen offset:128\n"
    "buffer_store_dwordx4 v[0:3], v246, s[96:99], s48 offen offset:128\n"
    "buffer_store_dwordx4 v[0:3], v248, s[96:99], s49 offen offset:0\n"
    "buffer_store_dwordx4 v[0:3], v246, s[96:99], s49 offen offset:0\n"
    "buffer_store_dwordx4 v[0:3], v248, s[96:99], s49 offen offset:128\n"
    "buffer_store_dwordx4 v[0:3], v246, s[96:99], s49 offen offset:128\n"
    "buffer_store_dwordx4 v[0:3], v248, s[96:99], s50 offen offset:0\n"
    "buffer_store_dwordx4 v[0:3], v246, s[96:99], s50 offen offset:0\n"
    "buffer_store_dwordx4 v[0:3], v248, s[96:99], s50 offen offset:128\n"
    "buffer_store_dwordx4 v[0:3], v246, s[96:99], s50 offen offset:128\n"
    "s_waitcnt lgkmcnt(0)\n"
    "s_branch .Lg1_%=\n"
    ".Lg2_%=:\n"
    "s_mov_b32 s52, 0\n"
    "s_cmp_eq_u32 s52, s56\n"
    "s_cbranch_scc1 .Lg35_%=\n"
    "s_waitcnt lgkmcnt(4)\n"
    "v_mfma_f32_16x16x32_bf16 v[0:3], a[128:131], a[0:3], 0\n"
    "s_add_u32 s59, s52, 2\n"
    "v_mfma_f32_16x16x32_bf16 v[0:3], a[132:135], a[4:7], v[0:3]\n"
    "ds_read_b64_tr_b16 a[144:145], v242 offset:256\n"
    "v_readlane_b32 s54, v245, s59\n"
    "v_mfma_f32_16x16x32_bf16 v[4:7], a[128:131], a[8:11], 0\n"
    "v_mfma_f32_16x16x32_bf16 v[4:7], a[132:135], a[12:15], v[4:7]\n"
    "ds_read_b64_tr_b16 a[146:147], v242 offset:1280\n"
    "v_mfma_f32_16x16x32_bf16 v[8:11], a[128:131], a[16:19], 0\n"
    "v_mfma_f32_16x16x32_bf16 v[8:11], a[132:135], a[20:23], v[8:11]\n"
    "ds_read_b64_tr_b16 a[148:149], v242 offset:8448\n"
    "v_mfma_f32_16x16x32_bf16 v[12:15], a[128:131], a[24:27], 0\n"
    "v_mfma_f32_16x16x32_bf16 v[12:15], a[132:135], a[28:31], v[12:15]\n"
    "ds_read_b64_tr_b16 a[150:151], v242 offset:9472\n"
    "v_mfma_f32_16x16x32_bf16 v[16:19], a[128:131], a[32:35], 0\n"
    "s_add_u32 m0, s58, 0\n"
    "v_mfma_f32_16x16x32_bf16 v[16:19], a[132:135], a[36:39], v[16:19]\n"
    "buffer_load_dwordx4 v232, s[40:43], s54 offen nt lds\n"
    "v_mfma_f32_16x16x32_bf16 v[20:23], a[128:131], a[40:43], 0\n"
    "v_mfma_f32_16x16x32_bf16 v[20:23], a[132:135], a[44:47], v[20:23]\n"
    "ds_read_b128 a[64:67], v240 offset:32768\n"
    "v_mfma_f32_16x16x32_bf16 v[24:27], a[128:131], a[48:51], 0\n"
    "s_add_u32 m0, s58, 1024\n"
    "v_mfma_f32_16x16x32_bf16 v[24:27], a[132:135], a[52:55], v[24:27]\n"
    "buffer_load_dwordx4 v233, s[40:43], s54 offen nt lds\n"
    "v_mfma_f32_16x16x32_bf16 v[28:31], a[128:131], a[56:59], 0\n"
    "v_mfma_f32_16x16x32_bf16 v[28:31], a[132:135], a[60:63], v[28:31]\n"
    "ds_read_b128 a[68:71], v241 offset:32768\n"
    "s_waitcnt lgkmcnt(6)\n"
    "v_mfma_f32_16x16x32_bf16 v[32:35], a[136:139], a[0:3], 0\n"
    "v_mfma_f32_16x16x32_bf16 v[32:35], a[140:143], a[4:7], v[32:35]\n"
    "ds_read_b64_tr_b16 a[152:153], v243 offset:256\n"
    "v_mfma_f32_16x16x32_bf16 v[36:39], a[136:139], a[8:11], 0\n"
    "v_mfma_f32_16x16x32_bf16 v[36:39], a[140:143], a[12:15], v[36:39]\n"
    "ds_read_b64_tr_b16 a[154:155], v243 offset:1280\n"
    "v_mfma_f32_16x16x32_bf16 v[40:43], a[136:139], a[16:19], 0\n"
    "v_mfma_f32_16x16x32_bf16 v[40:43], a[140:143], a[20:23], v[40:43]\n"
    "ds_read_b64_tr_b16 a[156:157], v243 offset:8448\n"
    "v_mfma_f32_16x16x32_bf16 v[44:47], a[136:139], a[24:27], 0\n"
    "v_mfma_f32_16x16x32_bf16 v[44:47], a[140:143], a[28:31], v[44:47]\n"
    "ds_read_b64_tr_b16 a[158:159], v243 offset:9472\n"
    "v_mfma_f32_16x16x32_bf16 v[48:51], a[136:139], a[32:35], 0\n"
    "s_add_u32 m0, s58, 4096\n"
    "v_mfma_f32_16x16x32_bf16 v[48:51], a[140:143], a[36:39], v[48:51]\n"
    "buffer_load_dwordx4 v234, s[40:43], s54 offen nt lds\n"
    "v_mfma_f32_16x16x32_bf16 v[52:55], a[136:139], a[40:43], 0\n"
    "v_mfma_f32_16x16x32_bf16 v[52:55], a[140:143], a[44:47], v[52:55]\n"
    "ds_read_b128 a[72:75], v240 offset:34816\n"
    "v_mfma_f32_16x16x32_bf16 v[56:59], a[136:139], a[48:51], 0\n"
    "s_add_u32 m0, s58, 5120\n"
    "v_mfma_f32_16x16x32_bf16 v[56:59], a[140:143], a[52:55], v[56:59]\n"
    "buffer_load_dwordx4 v235, s[40:43], s54 offen nt lds\n"
    "v_mfma_f32_16x16x32_bf16 v[60:63], a[136:139], a[56:59], 0\n"
    "v_mfma_f32_16x16x32_bf16 v[60:63], a[140:143], a[60:63], v[60:63]\n"
    "ds_read_b128 a[76:79], v241 offset:34816\n"
    "s_waitcnt lgkmcnt(8)\n"
    "v_mfma_f32_16x16x32_bf16 v[64:67], a[144:147], a[0:3], 0\n"
    "v_mfma_f32_16x16x32_bf16 v[64:67], a[148:151], a[4:7], v[64:67]\n"
    "ds_read_b64_tr_b16 a[128:129], v242 offset:512\n"
    "v_mfma_f32_16x16x32_bf16 v[68:71], a[144:147], a[8:11], 0\n"
    "v_mfma_f32_16x16x32_bf16 v[68:71], a[148:151], a[12:15], v[68:71]\n"
    "ds_read_b64_tr_b16 a[130:131], v242 offset:1536\n"
    "v_mfma_f32_16x16x32_bf16 v[72:75], a[144:147], a[16:19], 0\n"
    "v_mfma_f32_16x16x32_bf16 v[72:75], a[148:151], a[20:23], v[72:75]\n"
    "ds_read_b64_tr_b16 a[132:133], v242 offset:8704\n"
    "v_mfma_f32_16x16x32_bf16 v[76:79], a[144:147], a[24:27], 0\n"
    "v_mfma_f32_16x16x32_bf16 v[76:79], a[148:151], a[28:31], v[76:79]\n"
    "ds_read_b64_tr_b16 a[134:135], v242 offset:9728\n"
    "v_mfma_f32_16x16x32_bf16 v[80:83], a[144:147], a[32:35], 0\n"
    "s_add_u32 m0, s58, 8192\n"
    "v_mfma_f32_16x16x32_bf16 v[80:83], a[148:151], a[36:39], v[80:83]\n"
    "buffer_load_dwordx4 v236, s[40:43], s54 offen nt lds\n"
    "v_mfma_f32_16x16x32_bf16 v[84:87], a[144:147], a[40:43], 0\n"
    "v_mfma_f32_16x16x32_bf16 v[84:87], a[148:151], a[44:47], v[84:87]\n"
    "ds_read_b128 a[80:83], v240 offset:36864\n"
    "v_mfma_f32_16x16x32_bf16 v[88:91], a[144:147], a[48:51], 0\n"
    "s_add_u32 m0, s58, 9216\n"
    "v_mfma_f32_16x16x32_bf16 v[88:91], a[148:151], a[52:55], v[88:91]\n"
    "buffer_load_dwordx4 v237, s[40:43], s54 offen nt lds\n"
    "v_mfma_f32_16x16x32_bf16 v[92:95], a[144:147], a[56:59], 0\n"
    "v_mfma_f32_16x16x32_bf16 v[92:95], a[148:151], a[60:63], v[92:95]\n"
    "ds_read_b128 a[84:87], v241 offset:36864\n"
    "s_waitcnt lgkmcnt(8)\n"
    "v_mfma_f32_16x16x32_bf16 v[96:99], a[152:155], a[0:3], 0\n"
    "v_mfma_f32_16x16x32_bf16 v[96:99], a[156:159], a[4:7], v[96:99]\n"
    "ds_read_b64_tr_b16 a[136:137], v243 offset:512\n"
    "v_mfma_f32_16x16x32_bf16 v[100:103], a[152:155], a[8:11], 0\n"
    "v_mfma_f32_16x16x32_bf16 v[100:103], a[156:159], a[12:15], v[100:103]\n"
    "ds_read_b64_tr_b16 a[138:139], v243 offset:1536\n"
    "v_mfma_f32_16x16x32_bf16 v[104:107], a[152:155], a[16:19], 0\n"
    "v_mfma_f32_16x16x32_bf16 v[104:107], a[156:159], a[20:23], v[104:107]\n"
    "ds_read_b64_tr_b16 a[140:141], v243 offset:8704\n"
    "v_mfma_f32_16x16x32_bf16 v[108:111], a[152:155], a[24:27], 0\n"
    "v_mfma_f32_16x16x32_bf16 v[108:111], a[156:159], a[28:31], v[108:111]\n"
    "ds_read_b64_tr_b16 a[142:143], v243 offset:9728\n"
    "v_mfma_f32_16x16x32_bf16 v[112:115], a[152:155], a[32:35], 0\n"
    "s_add_u32 m0, s58, 12288\n"
    "v_mfma_f32_16x16x32_bf16 v[112:115], a[156:159], a[36:39], v[112:115]\n"
    "buffer_load_dwordx4 v238, s[40:43], s54 offen nt lds\n"
    "v_mfma_f32_16x16x32_bf16 v[116:119], a[152:155], a[40:43], 0\n"
    "v_mfma_f32_16x16x32_bf16 v[116:119], a[156:159], a[44:47], v[116:119]\n"
    "ds_read_b128 a[88:91], v240 offset:38912\n"
    "v_mfma_f32_16x16x32_bf16 v[120:123], a[152:155], a[48:51], 0\n"
    "s_add_u32 m0, s58, 13312\n"
    "v_mfma_f32_16x16x32_bf16 v[120:123], a[156:159], a[52:55], v[120:123]\n"
    "buffer_load_dwordx4 v239, s[40:43], s54 offen nt lds\n"
    "v_mfma_f32_16x16x32_bf16 v[124:127], a[152:155], a[56:59], 0\n"
    "v_mfma_f32_16x16x32_bf16 v[124:127], a[156:159], a[60:63], v[124:127]\n"
    "ds_read_b128 a[92:95], v241 offset:38912\n"
    "s_waitcnt lgkmcnt(8)\n"
    "v_mfma_f32_16x16x32_bf16 v[128:131], a[128:131], a[0:3], 0\n"
    "v_mfma_f32_16x16x32_bf16 v[128:131], a[132:135], a[4:7], v[128:131]\n"
    "ds_read_b64_tr_b16 a[144:145], v242 offset:768\n"
    "v_mfma_f32_16x16x32_bf16 v[132:135], a[128:131], a[8:11], 0\n"
    "s_add_u32 s58, s58, 32768\n"
    "v_mfma_f32_16x16x32_bf16 v[132:135], a[132:135], a[12:15], v[132:135]\n"
    "ds_read_b64_tr_b16 a[146:147], v242 offset:1792\n"
    "v_mfma_f32_16x16x32_bf16 v[136:139], a[128:131], a[16:19], 0\n"
    "s_sub_u32 s59, s58, 98304\n"
    "v_mfma_f32_16x16x32_bf16 v[136:139], a[132:135], a[20:23], v[136:139]\n"
    "ds_read_b64_tr_b16 a[148:149], v242 offset:8960\n"
    "v_mfma_f32_16x16x32_bf16 v[140:143], a[128:131], a[24:27], 0\n"
    "s_cmp_ge_u32 s58, s67\n"
    "v_mfma_f32_16x16x32_bf16 v[140:143], a[132:135], a[28:31], v[140:143]\n"
    "ds_read_b64_tr_b16 a[150:151], v242 offset:9984\n"
    "v_mfma_f32_16x16x32_bf16 v[144:147], a[128:131], a[32:35], 0\n"
    "s_cselect_b32 s58, s59, s58\n"
    "v_mfma_f32_16x16x32_bf16 v[144:147], a[132:135], a[36:39], v[144:147]\n"
    "ds_read_b128 a[96:99], v240 offset:40960\n"
    "v_mfma_f32_16x16x32_bf16 v[148:151], a[128:131], a[40:43], 0\n"
    "v_mfma_f32_16x16x32_bf16 v[148:151], a[132:135], a[44:47], v[148:151]\n"
    "ds_read_b128 a[100:103], v241 offset:40960\n"
    "v_mfma_f32_16x16x32_bf16 v[152:155], a[128:131], a[48:51], 0\n"
    "v_mfma_f32_16x16x32_bf16 v[152:155], a[132:135], a[52:55], v[152:155]\n"
    "ds_read_b128 a[104:107], v240 offset:43008\n"
    "v_mfma_f32_16x16x32_bf16 v[156:159], a[128:131], a[56:59], 0\n"
    "v_mfma_f32_16x16x32_bf16 v[156:159], a[132:135], a[60:63], v[156:159]\n"
    "ds_read_b128 a[108:111], v241 offset:43008\n"
    "s_waitcnt lgkmcnt(10)\n"
    "v_mfma_f32_16x16x32_bf16 v[160:163], a[136:139], a[0:3], 0\n"
    "v_mfma_f32_16x16x32_bf16 v[160:163], a[140:143], a[4:7], v[160:163]\n"
    "ds_read_b64_tr_b16 a[152:153], v243 offset:768\n"
    "ds_read_b128 a[112:115], v240 offset:45056\n"
    "v_mfma_f32_16x16x32_bf16 v[164:167], a[136:139], a[8:11], 0\n"
    "s_add_u32 s59, s52, 3\n"
    "v_mfma_f32_16x16x32_bf16 v[164:167], a[140:143], a[12:15], v[164:167]\n"
    "ds_read_b64_tr_b16 a[154:155], v243 offset:1792\n"
    "ds_read_b128 a[116:119], v241 offset:45056\n"
    "v_readlane_b32 s53, v244, s59\n"
    "v_mfma_f32_16x16x32_bf16 v[168:171], a[136:139], a[16:19], 0\n"
    "v_mfma_f32_16x16x32_bf16 v[168:171], a[140:143], a[20:23], v[168:171]\n"
    "ds_read_b64_tr_b16 a[156:157], v243 offset:8960\n"
    "ds_read_b128 a[120:123], v240 offset:47104\n"
    "v_mfma_f32_16x16x32_bf16 v[172:175], a[136:139], a[24:27], 0\n"
    "v_mfma_f32_16x16x32_bf16 v[172:175], a[140:143], a[28:31], v[172:175]\n"
    "ds_read_b64_tr_b16 a[158:159], v243 offset:9984\n"
    "ds_read_b128 a[124:127], v241 offset:47104\n"
    "v_mfma_f32_16x16x32_bf16 v[176:179], a[136:139], a[32:35], 0\n"
    "s_add_u32 s57, s57, 1\n"
    "v_mfma_f32_16x16x32_bf16 v[176:179], a[140:143], a[36:39], v[176:179]\n"
    "s_cmp_eq_u32 s57, 3\n"
    "v_mfma_f32_16x16x32_bf16 v[180:183], a[136:139], a[40:43], 0\n"
    "s_cselect_b32 s57, 0, s57\n"
    "s_cselect_b32 s59, 4294901760, s68\n"
    "v_mfma_f32_16x16x32_bf16 v[180:183], a[140:143], a[44:47], v[180:183]\n"
    "v_mfma_f32_16x16x32_bf16 v[184:187], a[136:139], a[48:51], 0\n"
    "v_add_u32 v242, s59, v242\n"
    "v_mfma_f32_16x16x32_bf16 v[184:187], a[140:143], a[52:55], v[184:187]\n"
    "v_add_u32 v243, s59, v243\n"
    "v_mfma_f32_16x16x32_bf16 v[188:191], a[136:139], a[56:59], 0\n"
    "v_mfma_f32_16x16x32_bf16 v[188:191], a[140:143], a[60:63], v[188:191]\n"
    "s_waitcnt vmcnt(8) lgkmcnt(0)\n"
    "s_barrier\n"
    "v_mfma_f32_16x16x32_bf16 v[192:195], a[144:147], a[0:3], 0\n"
    "v_mfma_f32_16x16x32_bf16 v[192:195], a[148:151], a[4:7], v[192:195]\n"
    "ds_read_b64_tr_b16 a[128:129], v242 offset:0\n"
    "v_mfma_f32_16x16x32_bf16 v[196:199], a[144:147], a[8:11], 0\n"
    "v_mfma_f32_16x16x32_bf16 v[196:199], a[148:151], a[12:15], v[196:199]\n"
    "ds_read_b64_tr_b16 a[130:131], v242 offset:1024\n"
    "v_mfma_f32_16x16x32_bf16 v[200:203], a[144:147], a[16:19], 0\n"
    "v_mfma_f32_16x16x32_bf16 v[200:203], a[148:151], a[20:23], v[200:203]\n"
    "ds_read_b64_tr_b16 a[132:133], v242 offset:8192\n"
    "v_mfma_f32_16x16x32_bf16 v[204:207], a[144:147], a[24:27], 0\n"
    "v_mfma_f32_16x16x32_bf16 v[204:207], a[148:151], a[28:31], v[204:207]\n"
    "ds_read_b64_tr_b16 a[134:135], v242 offset:9216\n"
    "v_mfma_f32_16x16x32_bf16 v[208:211], a[144:147], a[32:35], 0\n"
    "s_add_u32 m0, s55, 32768\n"
    "v_mfma_f32_16x16x32_bf16 v[208:211], a[148:151], a[36:39], v[208:211]\n"
    "buffer_load_dwordx4 v224, s[44:47], s53 offen sc1 lds\n"
    "v_mfma_f32_16x16x32_bf16 v[212:215], a[144:147], a[40:43], 0\n"
    "s_add_u32 m0, s55, 33792\n"
    "v_mfma_f32_16x16x32_bf16 v[212:215], a[148:151], a[44:47], v[212:215]\n"
    "buffer_load_dwordx4 v225, s[44:47], s53 offen sc1 lds\n"
    "v_mfma_f32_16x16x32_bf16 v[216:219], a[144:147], a[48:51], 0\n"
    "s_add_u32 m0, s55, 34816\n"
    "v_mfma_f32_16x16x32_bf16 v[216:219], a[148:151], a[52:55], v[216:219]\n"
    "buffer_load_dwordx4 v226, s[44:47], s53 offen sc1 lds\n"
    "v_mfma_f32_16x16x32_bf16 v[220:223], a[144:147], a[56:59], 0\n"
    "s_add_u32 m0, s55, 35840\n"
    "v_mfma_f32_16x16x32_bf16 v[220:223], a[148:151], a[60:63], v[220:223]\n"
    "buffer_load_dwordx4 v227, s[44:47], s53 offen sc1 lds\n"
    "s_waitcnt lgkmcnt(5)\n"
    "v_mfma_f32_16x16x32_bf16 a[160:163], a[152:155], a[0:3], 0\n"
    "v_mfma_f32_16x16x32_bf16 a[160:163], a[156:159], a[4:7], a[160:163]\n"
    "ds_read_b64_tr_b16 a[136:137], v243 offset:0\n"
    "v_mfma_f32_16x16x32_bf16 a[164:167], a[152:155], a[8:11], 0\n"
    "v_mfma_f32_16x16x32_bf16 a[164:167], a[156:159], a[12:15], a[164:167]\n"
    "ds_read_b64_tr_b16 a[138:139], v243 offset:1024\n"
    "v_mfma_f32_16x16x32_bf16 a[168:171], a[152:155], a[16:19], 0\n"
    "v_mfma_f32_16x16x32_bf16 a[168:171], a[156:159], a[20:23], a[168:171]\n"
    "ds_read_b64_tr_b16 a[140:141], v243 offset:8192\n"
    "v_mfma_f32_16x16x32_bf16 a[172:175], a[152:155], a[24:27], 0\n"
    "v_mfma_f32_16x16x32_bf16 a[172:175], a[156:159], a[28:31], a[172:175]\n"
    "ds_read_b64_tr_b16 a[142:143], v243 offset:9216\n"
    "v_mfma_f32_16x16x32_bf16 a[176:179], a[152:155], a[32:35], 0\n"
    "s_add_u32 m0, s55, 36864\n"
    "v_mfma_f32_16x16x32_bf16 a[176:179], a[156:159], a[36:39], a[176:179]\n"
    "buffer_load_dwordx4 v228, s[44:47], s53 offen sc1 lds\n"
    "v_mfma_f32_16x16x32_bf16 a[180:183], a[152:155], a[40:43], 0\n"
    "s_add_u32 m0, s55, 37888\n"
    "v_mfma_f32_16x16x32_bf16 a[180:183], a[156:159], a[44:47], a[180:183]\n"
    "buffer_load_dwordx4 v229, s[44:47], s53 offen sc1 lds\n"
    "v_mfma_f32_16x16x32_bf16 a[184:187], a[152:155], a[48:51], 0\n"
    "s_add_u32 m0, s55, 38912\n"
    "v_mfma_f32_16x16x32_bf16 a[184:187], a[156:159], a[52:55], a[184:187]\n"
    "buffer_load_dwordx4 v230, s[44:47], s53 offen sc1 lds\n"
    "v_mfma_f32_16x16x32_bf16 a[188:191], a[152:155], a[56:59], 0\n"
    "s_add_u32 m0, s55, 39936\n"
    "v_mfma_f32_16x16x32_bf16 a[188:191], a[156:159], a[60:63], a[188:191]\n"
    "buffer_load_dwordx4 v231, s[44:47], s53 offen sc1 lds\n"
    "s_add_u32 s52, s52, 1\n"
    ".Lg36_%=:\n"
    "s_waitcnt lgkmcnt(4)\n"
    "v_mfma_f32_16x16x32_bf16 v[0:3], a[128:131], a[64:67], v[0:3]\n"
    "s_add_u32 s59, s52, 2\n"
    "v_mfma_f32_16x16x32_bf16 v[0:3], a[132:135], a[68:71], v[0:3]\n"
    "ds_read_b64_tr_b16 a[144:145], v242 offset:256\n"
    "v_readlane_b32 s54, v245, s59\n"
    "v_mfma_f32_16x16x32_bf16 v[4:7], a[128:131], a[72:75], v[4:7]\n"
    "v_mfma_f32_16x16x32_bf16 v[4:7], a[132:135], a[76:79], v[4:7]\n"
    "ds_read_b64_tr_b16 a[146:147], v242 offset:1280\n"
    "v_mfma_f32_16x16x32_bf16 v[8:11], a[128:131], a[80:83], v[8:11]\n"
    "buffer_load_dword v252, v246, s[88:91], 0 offen sc1\n"
    "v_mfma_f32_16x16x32_bf16 v[8:11], a[132:135], a[84:87], v[8:11]\n"
    "ds_read_b64_tr_b16 a[148:149], v242 offset:8448\n"
    "v_mfma_f32_16x16x32_bf16 v[12:15], a[128:131], a[88:91], v[12:15]\n"
    "v_mfma_f32_16x16x32_bf16 v[12:15], a[132:135], a[92:95], v[12:15]\n"
    "ds_read_b64_tr_b16 a[150:151], v242 offset:9472\n"
    "v_mfma_f32_16x16x32_bf16 v[16:19], a[128:131], a[96:99], v[16:19]\n"
    "s_add_u32 m0, s58, 0\n"
    "v_mfma_f32_16x16x32_bf16 v[16:19], a[132:135], a[100:103], v[16:19]\n"
    "buffer_load_dwordx4 v232, s[40:43], s54 offen nt lds\n"
    "v_mfma_f32_16x16x32_bf16 v[20:23], a[128:131], a[104:107], v[20:23]\n"
    "v_readlane_b32 s62, v247, s52\n"
    "v_mfma_f32_16x16x32_bf16 v[20:23], a[132:135], a[108:111], v[20:23]\n"
    "ds_read_b128 a[0:3], v240 offset:0\n"
    "v_mfma_f32_16x16x32_bf16 v[24:27], a[128:131], a[112:115], v[24:27]\n"
    "s_lshl_b64 s[78:79], -1, s62\n"
    "s_add_u32 m0, s58, 1024\n"
    "v_mfma_f32_16x16x32_bf16 v[24:27], a[132:135], a[116:119], v[24:27]\n"
    "buffer_load_dwordx4 v233, s[40:43], s54 offen nt lds\n"
    "v_mfma_f32_16x16x32_bf16 v[28:31], a[128:131], a[120:123], v[28:31]\n"
    "v_mfma_f32_16x16x32_bf16 v[28:31], a[132:135], a[124:127], v[28:31]\n"
    "ds_read_b128 a[4:7], v241 offset:0\n"
    "s_waitcnt lgkmcnt(6)\n"
    "v_mfma_f32_16x16x32_bf16 v[32:35], a[136:139], a[64:67], v[32:35]\n"
    "s_sub_u32 s61, s52, 1\n"
    "v_mfma_f32_16x16x32_bf16 v[32:35], a[140:143], a[68:71], v[32:35]\n"
    "ds_read_b64_tr_b16 a[152:153], v243 offset:256\n"
    "v_mfma_f32_16x16x32_bf16 v[36:39], a[136:139], a[72:75], v[36:39]\n"
    "s_cmp_eq_u32 s61, s56\n"
    "s_cselect_b32 s61, s66, 2147418112\n"
    "v_mfma_f32_16x16x32_bf16 v[36:39], a[140:143], a[76:79], v[36:39]\n"
    "ds_read_b64_tr_b16 a[154:155], v243 offset:1280\n"
    "v_mfma_f32_16x16x32_bf16 v[40:43], a[136:139], a[80:83], v[40:43]\n"
    "v_mfma_f32_16x16x32_bf16 v[40:43], a[140:143], a[84:87], v[40:43]\n"
    "ds_read_b64_tr_b16 a[156:157], v243 offset:8448\n"
    "v_mfma_f32_16x16x32_bf16 v[44:47], a[136:139], a[88:91], v[44:47]\n"
    "v_mfma_f32_16x16x32_bf16 v[44:47], a[140:143], a[92:95], v[44:47]\n"
    "ds_read_b64_tr_b16 a[158:159], v243 offset:9472\n"
    "v_mfma_f32_16x16x32_bf16 v[48:51], a[136:139], a[96:99], v[48:51]\n"
    "s_add_u32 m0, s58, 4096\n"
    "v_mfma_f32_16x16x32_bf16 v[48:51], a[140:143], a[100:103], v[48:51]\n"
    "buffer_load_dwordx4 v234, s[40:43], s54 offen nt lds\n"
    "v_mfma_f32_16x16x32_bf16 v[52:55], a[136:139], a[104:107], v[52:55]\n"
    "v_mfma_f32_16x16x32_bf16 v[52:55], a[140:143], a[108:111], v[52:55]\n"
    "ds_read_b128 a[8:11], v240 offset:2048\n"
    "v_mfma_f32_16x16x32_bf16 v[56:59], a[136:139], a[112:115], v[56:59]\n"
    "s_add_u32 m0, s58, 5120\n"
    "v_mfma_f32_16x16x32_bf16 v[56:59], a[140:143], a[116:119], v[56:59]\n"
    "buffer_load_dwordx4 v235, s[40:43], s54 offen nt lds\n"
    "v_mfma_f32_16x16x32_bf16 v[60:63], a[136:139], a[120:123], v[60:63]\n"
    "v_mfma_f32_16x16x32_bf16 v[60:63], a[140:143], a[124:127], v[60:63]\n"
    "ds_read_b128 a[12:15], v241 offset:2048\n"
    "s_waitcnt lgkmcnt(8)\n"
    "v_mfma_f32_16x16x32_bf16 v[64:67], a[144:147], a[64:67], v[64:67]\n"
    "v_mfma_f32_16x16x32_bf16 v[64:67], a[148:151], a[68:71], v[64:67]\n"
    "ds_read_b64_tr_b16 a[128:129], v242 offset:512\n"
    "v_mfma_f32_16x16x32_bf16 v[68:71], a[144:147], a[72:75], v[68:71]\n"
    "v_mfma_f32_16x16x32_bf16 v[68:71], a[148:151], a[76:79], v[68:71]\n"
    "ds_read_b64_tr_b16 a[130:131], v242 offset:1536\n"
    "v_mfma_f32_16x16x32_bf16 v[72:75], a[144:147], a[80:83], v[72:75]\n"
    "v_mfma_f32_16x16x32_bf16 v[72:75], a[148:151], a[84:87], v[72:75]\n"
    "ds_read_b64_tr_b16 a[132:133], v242 offset:8704\n"
    "v_mfma_f32_16x16x32_bf16 v[76:79], a[144:147], a[88:91], v[76:79]\n"
    "v_mfma_f32_16x16x32_bf16 v[76:79], a[148:151], a[92:95], v[76:79]\n"
    "ds_read_b64_tr_b16 a[134:135], v242 offset:9728\n"
    "v_mfma_f32_16x16x32_bf16 v[80:83], a[144:147], a[96:99], v[80:83]\n"
    "s_add_u32 m0, s58, 8192\n"
    "v_mfma_f32_16x16x32_bf16 v[80:83], a[148:151], a[100:103], v[80:83]\n"
    "buffer_load_dwordx4 v236, s[40:43], s54 offen nt lds\n"
    "v_mfma_f32_16x16x32_bf16 v[84:87], a[144:147], a[104:107], v[84:87]\n"
    "v_mfma_f32_16x16x32_bf16 v[84:87], a[148:151], a[108:111], v[84:87]\n"
    "ds_read_b128 a[16:19], v240 offset:4096\n"
    "v_mfma_f32_16x16x32_bf16 v[88:91], a[144:147], a[112:115], v[88:91]\n"
    "s_add_u32 m0, s58, 9216\n"
    "v_mfma_f32_16x16x32_bf16 v[88:91], a[148:151], a[116:119], v[88:91]\n"
    "buffer_load_dwordx4 v237, s[40:43], s54 offen nt lds\n"
    "v_mfma_f32_16x16x32_bf16 v[92:95], a[144:147], a[120:123], v[92:95]\n"
    "v_mfma_f32_16x16x32_bf16 v[92:95], a[148:151], a[124:127], v[92:95]\n"
    "ds_read_b128 a[20:23], v241 offset:4096\n"
    "s_waitcnt lgkmcnt(8)\n"
    "v_mfma_f32_16x16x32_bf16 v[96:99], a[152:155], a[64:67], v[96:99]\n"
    "v_mfma_f32_16x16x32_bf16 v[96:99], a[156:159], a[68:71], v[96:99]\n"
    "ds_read_b64_tr_b16 a[136:137], v243 offset:512\n"
    "v_mfma_f32_16x16x32_bf16 v[100:103], a[152:155], a[72:75], v[100:103]\n"
    "v_mfma_f32_16x16x32_bf16 v[100:103], a[156:159], a[76:79], v[100:103]\n"
    "ds_read_b64_tr_b16 a[138:139], v243 offset:1536\n"
    "v_mfma_f32_16x16x32_bf16 v[104:107], a[152:155], a[80:83], v[104:107]\n"
    "v_mfma_f32_16x16x32_bf16 v[104:107], a[156:159], a[84:87], v[104:107]\n"
    "ds_read_b64_tr_b16 a[140:141], v243 offset:8704\n"
    "v_mfma_f32_16x16x32_bf16 v[108:111], a[152:155], a[88:91], v[108:111]\n"
    "v_mfma_f32_16x16x32_bf16 v[108:111], a[156:159], a[92:95], v[108:111]\n"
    "ds_read_b64_tr_b16 a[142:143], v243 offset:9728\n"
    "v_mfma_f32_16x16x32_bf16 v[112:115], a[152:155], a[96:99], v[112:115]\n"
    "s_add_u32 m0, s58, 12288\n"
    "v_mfma_f32_16x16x32_bf16 v[112:115], a[156:159], a[100:103], v[112:115]\n"
    "buffer_load_dwordx4 v238, s[40:43], s54 offen nt lds\n"
    "v_mfma_f32_16x16x32_bf16 v[116:119], a[152:155], a[104:107], v[116:119]\n"
    "v_mfma_f32_16x16x32_bf16 v[116:119], a[156:159], a[108:111], v[116:119]\n"
    "ds_read_b128 a[24:27], v240 offset:6144\n"
    "v_mfma_f32_16x16x32_bf16 v[120:123], a[152:155], a[112:115], v[120:123]\n"
    "s_add_u32 m0, s58, 13312\n"
    "v_mfma_f32_16x16x32_bf16 v[120:123], a[156:159], a[116:119], v[120:123]\n"
    "buffer_load_dwordx4 v239, s[40:43], s54 offen nt lds\n"
    "v_mfma_f32_16x16x32_bf16 v[124:127], a[152:155], a[120:123], v[124:127]\n"
    "v_mfma_f32_16x16x32_bf16 v[124:127], a[156:159], a[124:127], v[124:127]\n"
    "ds_read_b128 a[28:31], v241 offset:6144\n"
    "s_waitcnt lgkmcnt(8)\n"
    "v_mfma_f32_16x16x32_bf16 v[128:131], a[128:131], a[64:67], v[128:131]\n"
    "v_mfma_f32_16x16x32_bf16 v[128:131], a[132:135], a[68:71], v[128:131]\n"
    "ds_read_b64_tr_b16 a[144:145], v242 offset:768\n"
    "v_mfma_f32_16x16x32_bf16 v[132:135], a[128:131], a[72:75], v[132:135]\n"
    "s_add_u32 s58, s58, 32768\n"
    "v_mfma_f32_16x16x32_bf16 v[132:135], a[132:135], a[76:79], v[132:135]\n"
    "ds_read_b64_tr_b16 a[146:147], v242 offset:1792\n"
    "v_mfma_f32_16x16x32_bf16 v[136:139], a[128:131], a[80:83], v[136:139]\n"
    "s_sub_u32 s59, s58, 98304\n"
    "v_mfma_f32_16x16x32_bf16 v[136:139], a[132:135], a[84:87], v[136:139]\n"
    "ds_read_b64_tr_b16 a[148:149], v242 offset:8960\n"
    "v_mfma_f32_16x16x32_bf16 v[140:143], a[128:131], a[88:91], v[140:143]\n"
    "s_cmp_ge_u32 s58, s67\n"
    "v_mfma_f32_16x16x32_bf16 v[140:143], a[132:135], a[92:95], v[140:143]\n"
    "ds_read_b64_tr_b16 a[150:151], v242 offset:9984\n"
    "v_mfma_f32_16x16x32_bf16 v[144:147], a[128:131], a[96:99], v[144:147]\n"
    "s_cselect_b32 s58, s59, s58\n"
    "v_mfma_f32_16x16x32_bf16 v[144:147], a[132:135], a[100:103], v[144:147]\n"
    "ds_read_b128 a[32:35], v240 offset:8192\n"
    "v_mfma_f32_16x16x32_bf16 v[148:151], a[128:131], a[104:107], v[148:151]\n"
    "v_mfma_f32_16x16x32_bf16 v[148:151], a[132:135], a[108:111], v[148:151]\n"
    "ds_read_b128 a[36:39], v241 offset:8192\n"
    "v_mfma_f32_16x16x32_bf16 v[152:155], a[128:131], a[112:115], v[152:155]\n"
    "v_mfma_f32_16x16x32_bf16 v[152:155], a[132:135], a[116:119], v[152:155]\n"
    "ds_read_b128 a[40:43], v240 offset:10240\n"
    "v_mfma_f32_16x16x32_bf16 v[156:159], a[128:131], a[120:123], v[156:159]\n"
    "v_mfma_f32_16x16x32_bf16 v[156:159], a[132:135], a[124:127], v[156:159]\n"
    "ds_read_b128 a[44:47], v241 offset:10240\n"
    "s_waitcnt lgkmcnt(10)\n"
    "v_mfma_f32_16x16x32_bf16 v[160:163], a[136:139], a[64:67], v[160:163]\n"
    "v_mfma_f32_16x16x32_bf16 v[160:163], a[140:143], a[68:71], v[160:163]\n"
    "ds_read_b64_tr_b16 a[152:153], v243 offset:768\n"
    "ds_read_b128 a[48:51], v240 offset:12288\n"
    "v_mfma_f32_16x16x32_bf16 v[164:167], a[136:139], a[72:75], v[164:167]\n"
    "s_add_u32 s59, s52, 3\n"
    "v_mfma_f32_16x16x32_bf16 v[164:167], a[140:143], a[76:79], v[164:167]\n"
    "ds_read_b64_tr_b16 a[154:155], v243 offset:1792\n"
    "ds_read_b128 a[52:55], v241 offset:12288\n"
    "v_readlane_b32 s53, v244, s59\n"
    "v_mfma_f32_16x16x32_bf16 v[168:171], a[136:139], a[80:83], v[168:171]\n"
    "s_cmp_eq_u32 s59, 8\n"
    "s_cselect_b32 s44, s48, s44\n"
    "v_mfma_f32_16x16x32_bf16 v[168:171], a[140:143], a[84:87], v[168:171]\n"
    "ds_read_b64_tr_b16 a[156:157], v243 offset:8960\n"
    "ds_read_b128 a[56:59], v240 offset:14336\n"
    "s_cselect_b32 s45, s49, s45\n"
    "v_mfma_f32_16x16x32_bf16 v[172:175], a[136:139], a[88:91], v[172:175]\n"
    "v_mfma_f32_16x16x32_bf16 v[172:175], a[140:143], a[92:95], v[172:175]\n"
    "ds_read_b64_tr_b16 a[158:159], v243 offset:9984\n"
    "ds_read_b128 a[60:63], v241 offset:14336\n"
    "v_mfma_f32_16x16x32_bf16 v[176:179], a[136:139], a[96:99], v[176:179]\n"
    "s_add_u32 s57, s57, 1\n"
    "v_mfma_f32_16x16x32_bf16 v[176:179], a[140:143], a[100:103], v[176:179]\n"
    "s_cmp_eq_u32 s57, 3\n"
    "v_mfma_f32_16x16x32_bf16 v[180:183], a[136:139], a[104:107], v[180:183]\n"
    "s_cselect_b32 s57, 0, s57\n"
    "s_cselect_b32 s59, 4294901760, s68\n"
    "v_mfma_f32_16x16x32_bf16 v[180:183], a[140:143], a[108:111], v[180:183]\n"
    "v_mfma_f32_16x16x32_bf16 v[184:187], a[136:139], a[112:115], v[184:187]\n"
    "v_add_u32 v242, s59, v242\n"
    "v_mfma_f32_16x16x32_bf16 v[184:187], a[140:143], a[116:119], v[184:187]\n"
    "v_add_u32 v243, s59, v243\n"
    "v_mfma_f32_16x16x32_bf16 v[188:191], a[136:139], a[120:123], v[188:191]\n"
    "v_mfma_f32_16x16x32_bf16 v[188:191], a[140:143], a[124:127], v[188:191]\n"
    "s_waitcnt vmcnt(8) lgkmcnt(0)\n"
    "s_barrier\n"
    "v_mfma_f32_16x16x32_bf16 v[192:195], a[144:147], a[64:67], v[192:195]\n"
    "v_cmp_le_u32 vcc, s63, v252\n"
    "v_mfma_f32_16x16x32_bf16 v[192:195], a[148:151], a[68:71], v[192:195]\n"
    "buffer_store_dword v254, v251, s[84:87], s61 offen sc0 sc1\n"
    "ds_read_b64_tr_b16 a[128:129], v242 offset:0\n"
    "v_mfma_f32_16x16x32_bf16 v[196:199], a[144:147], a[72:75], v[196:199]\n"
    "s_or_b64 vcc, vcc, s[78:79]\n"
    "s_cmp_eq_u64 vcc, -1\n"
    "v_mfma_f32_16x16x32_bf16 v[196:199], a[148:151], a[76:79], v[196:199]\n"
    "s_cbranch_scc0 .Lg37_%=\n"
    ".Lg38_%=:\n"
    "ds_read_b64_tr_b16 a[130:131], v242 offset:1024\n"
    "v_mfma_f32_16x16x32_bf16 v[200:203], a[144:147], a[80:83], v[200:203]\n"
    "v_mfma_f32_16x16x32_bf16 v[200:203], a[148:151], a[84:87], v[200:203]\n"
    "ds_read_b64_tr_b16 a[132:133], v242 offset:8192\n"
    "v_mfma_f32_16x16x32_bf16 v[204:207], a[144:147], a[88:91], v[204:207]\n"
    "v_mfma_f32_16x16x32_bf16 v[204:207], a[148:151], a[92:95], v[204:207]\n"
    "ds_read_b64_tr_b16 a[134:135], v242 offset:9216\n"
    "v_mfma_f32_16x16x32_bf16 v[208:211], a[144:147], a[96:99], v[208:211]\n"
    "s_add_u32 m0, s55, 0\n"
    "v_mfma_f32_16x16x32_bf16 v[208:211], a[148:151], a[100:103], v[208:211]\n"
    "buffer_load_dwordx4 v224, s[44:47], s53 offen sc1 lds\n"
    "v_mfma_f32_16x16x32_bf16 v[212:215], a[144:147], a[104:107], v[212:215]\n"
    "s_add_u32 m0, s55, 1024\n"
    "v_mfma_f32_16x16x32_bf16 v[212:215], a[148:151], a[108:111], v[212:215]\n"
    "buffer_load_dwordx4 v225, s[44:47], s53 offen sc1 lds\n"
    "v_mfma_f32_16x16x32_bf16 v[216:219], a[144:147], a[112:115], v[216:219]\n"
    "s_add_u32 m0, s55, 2048\n"
    "v_mfma_f32_16x16x32_bf16 v[216:219], a[148:151], a[116:119], v[216:219]\n"
    "buffer_load_dwordx4 v226, s[44:47], s53 offen sc1 lds\n"
    "v_mfma_f32_16x16x32_bf16 v[220:223], a[144:147], a[120:123], v[220:223]\n"
    "s_add_u32 m0, s55, 3072\n"
    "v_mfma_f32_16x16x32_bf16 v[220:223], a[148:151], a[124:127], v[220:223]\n"
    "buffer_load_dwordx4 v227, s[44:47], s53 offen sc1 lds\n"
    "s_waitcnt lgkmcnt(5)\n"
    "v_mfma_f32_16x16x32_bf16 a[160:163], a[152:155], a[64:67], a[160:163]\n"
    "v_mfma_f32_16x16x32_bf16 a[160:163], a[156:159], a[68:71], a[160:163]\n"
    "ds_read_b64_tr_b16 a[136:137], v243 offset:0\n"
    "v_mfma_f32_16x16x32_bf16 a[164:167], a[152:155], a[72:75], a[164:167]\n"
    "v_mfma_f32_16x16x32_bf16 a[164:167], a[156:159], a[76:79], a[164:167]\n"
    "ds_read_b64_tr_b16 a[138:139], v243 offset:1024\n"
    "v_mfma_f32_16x16x32_bf16 a[168:171], a[152:155], a[80:83], a[168:171]\n"
    "v_mfma_f32_16x16x32_bf16 a[168:171], a[156:159], a[84:87], a[168:171]\n"
    "ds_read_b64_tr_b16 a[140:141], v243 offset:8192\n"
    "v_mfma_f32_16x16x32_bf16 a[172:175], a[152:155], a[88:91], a[172:175]\n"
    "v_mfma_f32_16x16x32_bf16 a[172:175], a[156:159], a[92:95], a[172:175]\n"
    "ds_read_b64_tr_b16 a[142:143], v243 offset:9216\n"
    "v_mfma_f32_16x16x32_bf16 a[176:179], a[152:155], a[96:99], a[176:179]\n"
    "s_add_u32 m0, s55, 4096\n"
    "v_mfma_f32_16x16x32_bf16 a[176:179], a[156:159], a[100:103], a[176:179]\n"
    "buffer_load_dwordx4 v228, s[44:47], s53 offen sc1 lds\n"
    "v_mfma_f32_16x16x32_bf16 a[180:183], a[152:155], a[104:107], a[180:183]\n"
    "s_add_u32 m0, s55, 5120\n"
    "v_mfma_f32_16x16x32_bf16 a[180:183], a[156:159], a[108:111], a[180:183]\n"
    "buffer_load_dwordx4 v229, s[44:47], s53 offen sc1 lds\n"
    "v_mfma_f32_16x16x32_bf16 a[184:187], a[152:155], a[112:115], a[184:187]\n"
    "s_add_u32 m0, s55, 6144\n"
    "v_mfma_f32_16x16x32_bf16 a[184:187], a[156:159], a[116:119], a[184:187]\n"
    "buffer_load_dwordx4 v230, s[44:47], s53 offen sc1 lds\n"
    "v_mfma_f32_16x16x32_bf16 a[188:191], a[152:155], a[120:123], a[188:191]\n"
    "s_add_u32 m0, s55, 7168\n"
    "v_mfma_f32_16x16x32_bf16 a[188:191], a[156:159], a[124:127], a[188:191]\n"
    "buffer_load_dwordx4 v231, s[44:47], s53 offen sc1 lds\n"
    "s_add_u32 s52, s52, 1\n"
    "s_mov_b32 s60, 8\n"
    ".Lg40_%=:\n"
    "s_cmp_eq_u32 s52, s56\n"
    "s_cbranch_scc1 .Lg41_%=\n"
    "s_waitcnt lgkmcnt(4)\n"
    "v_mfma_f32_16x16x32_bf16 v[0:3], a[128:131], a[0:3], v[0:3]\n"
    "s_add_u32 s59, s52, 2\n"
    "v_mfma_f32_16x16x32_bf16 v[0:3], a[132:135], a[4:7], v[0:3]\n"
    "ds_read_b64_tr_b16 a[144:145], v242 offset:256\n"
    "v_readlane_b32 s54, v245, s59\n"
    "v_mfma_f32_16x16x32_bf16 v[4:7], a[128:131], a[8:11], v[4:7]\n"
    "v_mfma_f32_16x16x32_bf16 v[4:7], a[132:135], a[12:15], v[4:7]\n"
    "ds_read_b64_tr_b16 a[146:147], v242 offset:1280\n"
    "v_mfma_f32_16x16x32_bf16 v[8:11], a[128:131], a[16:19], v[8:11]\n"
    "v_mfma_f32_16x16x32_bf16 v[8:11], a[132:135], a[20:23], v[8:11]\n"
    "ds_read_b64_tr_b16 a[148:149], v242 offset:8448\n"
    "v_mfma_f32_16x16x32_bf16 v[12:15], a[128:131], a[24:27], v[12:15]\n"
    "v_mfma_f32_16x16x32_bf16 v[12:15], a[132:135], a[28:31], v[12:15]\n"
    "ds_read_b64_tr_b16 a[150:151], v242 offset:9472\n"
    "v_mfma_f32_16x16x32_bf16 v[16:19], a[128:131], a[32:35], v[16:19]\n"
    "s_add_u32 m0, s58, 0\n"
    "v_mfma_f32_16x16x32_bf16 v[16:19], a[132:135], a[36:39], v[16:19]\n"
    "buffer_load_dwordx4 v232, s[40:43], s54 offen lds\n"
    "v_mfma_f32_16x16x32_bf16 v[20:23], a[128:131], a[40:43], v[20:23]\n"
    "v_mfma_f32_16x16x32_bf16 v[20:23], a[132:135], a[44:47], v[20:23]\n"
    "ds_read_b128 a[64:67], v240 offset:32768\n"
    "v_mfma_f32_16x16x32_bf16 v[24:27], a[128:131], a[48:51], v[24:27]\n"
    "s_add_u32 m0, s58, 1024\n"
    "v_mfma_f32_16x16x32_bf16 v[24:27], a[132:135], a[52:55], v[24:27]\n"
    "buffer_load_dwordx4 v233, s[40:43], s54 offen lds\n"
    "v_mfma_f32_16x16x32_bf16 v[28:31], a[128:131], a[56:59], v[28:31]\n"
    "v_mfma_f32_16x16x32_bf16 v[28:31], a[132:135], a[60:63], v[28:31]\n"
    "ds_read_b128 a[68:71], v241 offset:32768\n"
    "s_waitcnt lgkmcnt(6)\n"
    "v_mfma_f32_16x16x32_bf16 v[32:35], a[136:139], a[0:3], v[32:35]\n"
    "v_mfma_f32_16x16x32_bf16 v[32:35], a[140:143], a[4:7], v[32:35]\n"
    "ds_read_b64_tr_b16 a[152:153], v243 offset:256\n"
    "v_mfma_f32_16x16x32_bf16 v[36:39], a[136:139], a[8:11], v[36:39]\n"
    "v_mfma_f32_16x16x32_bf16 v[36:39], a[140:143], a[12:15], v[36:39]\n"
    "ds_read_b64_tr_b16 a[154:155], v243 offset:1280\n"
    "v_mfma_f32_16x16x32_bf16 v[40:43], a[136:139], a[16:19], v[40:43]\n"
    "v_mfma_f32_16x16x32_bf16 v[40:43], a[140:143], a[20:23], v[40:43]\n"
    "ds_read_b64_tr_b16 a[156:157], v243 offset:8448\n"
    "v_mfma_f32_16x16x32_bf16 v[44:47], a[136:139], a[24:27], v[44:47]\n"
    "v_mfma_f32_16x16x32_bf16 v[44:47], a[140:143], a[28:31], v[44:47]\n"
    "ds_read_b64_tr_b16 a[158:159], v243 offset:9472\n"
    "v_mfma_f32_16x16x32_bf16 v[48:51], a[136:139], a[32:35], v[48:51]\n"
    "s_add_u32 m0, s58, 4096\n"
    "v_mfma_f32_16x16x32_bf16 v[48:51], a[140:143], a[36:39], v[48:51]\n"
    "buffer_load_dwordx4 v234, s[40:43], s54 offen lds\n"
    "v_mfma_f32_16x16x32_bf16 v[52:55], a[136:139], a[40:43], v[52:55]\n"
    "v_mfma_f32_16x16x32_bf16 v[52:55], a[140:143], a[44:47], v[52:55]\n"
    "ds_read_b128 a[72:75], v240 offset:34816\n"
    "v_mfma_f32_16x16x32_bf16 v[56:59], a[136:139], a[48:51], v[56:59]\n"
    "s_add_u32 m0, s58, 5120\n"
    "v_mfma_f32_16x16x32_bf16 v[56:59], a[140:143], a[52:55], v[56:59]\n"
    "buffer_load_dwordx4 v235, s[40:43], s54 offen lds\n"
    "v_mfma_f32_16x16x32_bf16 v[60:63], a[136:139], a[56:59], v[60:63]\n"
    "v_mfma_f32_16x16x32_bf16 v[60:63], a[140:143], a[60:63], v[60:63]\n"
    "ds_read_b128 a[76:79], v241 offset:34816\n"
    "s_waitcnt lgkmcnt(8)\n"
    "v_mfma_f32_16x16x32_bf16 v[64:67], a[144:147], a[0:3], v[64:67]\n"
    "v_mfma_f32_16x16x32_bf16 v[64:67], a[148:151], a[4:7], v[64:67]\n"
    "ds_read_b64_tr_b16 a[128:129], v242 offset:512\n"
    "v_mfma_f32_16x16x32_bf16 v[68:71], a[144:147], a[8:11], v[68:71]\n"
    "v_mfma_f32_16x16x32_bf16 v[68:71], a[148:151], a[12:15], v[68:71]\n"
    "ds_read_b64_tr_b16 a[130:131], v242 offset:1536\n"
    "v_mfma_f32_16x16x32_bf16 v[72:75], a[144:147], a[16:19], v[72:75]\n"
    "v_mfma_f32_16x16x32_bf16 v[72:75], a[148:151], a[20:23], v[72:75]\n"
    "ds_read_b64_tr_b16 a[132:133], v242 offset:8704\n"
    "v_mfma_f32_16x16x32_bf16 v[76:79], a[144:147], a[24:27], v[76:79]\n"
    "v_mfma_f32_16x16x32_bf16 v[76:79], a[148:151], a[28:31], v[76:79]\n"
    "ds_read_b64_tr_b16 a[134:135], v242 offset:9728\n"
    "v_mfma_f32_16x16x32_bf16 v[80:83], a[144:147], a[32:35], v[80:83]\n"
    "s_add_u32 m0, s58, 8192\n"
    "v_mfma_f32_16x16x32_bf16 v[80:83], a[148:151], a[36:39], v[80:83]\n"
    "buffer_load_dwordx4 v236, s[40:43], s54 offen lds\n"
    "v_mfma_f32_16x16x32_bf16 v[84:87], a[144:147], a[40:43], v[84:87]\n"
    "v_mfma_f32_16x16x32_bf16 v[84:87], a[148:151], a[44:47], v[84:87]\n"
    "ds_read_b128 a[80:83], v240 offset:36864\n"
    "v_mfma_f32_16x16x32_bf16 v[88:91], a[144:147], a[48:51], v[88:91]\n"
    "s_add_u32 m0, s58, 9216\n"
    "v_mfma_f32_16x16x32_bf16 v[88:91], a[148:151], a[52:55], v[88:91]\n"
    "buffer_load_dwordx4 v237, s[40:43], s54 offen lds\n"
    "v_mfma_f32_16x16x32_bf16 v[92:95], a[144:147], a[56:59], v[92:95]\n"
    "v_mfma_f32_16x16x32_bf16 v[92:95], a[148:151], a[60:63], v[92:95]\n"
    "ds_read_b128 a[84:87], v241 offset:36864\n"
    "s_waitcnt lgkmcnt(8)\n"
    "v_mfma_f32_16x16x32_bf16 v[96:99], a[152:155], a[0:3], v[96:99]\n"
    "v_mfma_f32_16x16x32_bf16 v[96:99], a[156:159], a[4:7], v[96:99]\n"
    "ds_read_b64_tr_b16 a[136:137], v243 offset:512\n"
    "v_mfma_f32_16x16x32_bf16 v[100:103], a[152:155], a[8:11], v[100:103]\n"
    "v_mfma_f32_16x16x32_bf16 v[100:103], a[156:159], a[12:15], v[100:103]\n"
    "ds_read_b64_tr_b16 a[138:139], v243 offset:1536\n"
    "v_mfma_f32_16x16x32_bf16 v[104:107], a[152:155], a[16:19], v[104:107]\n"
    "v_mfma_f32_16x16x32_bf16 v[104:107], a[156:159], a[20:23], v[104:107]\n"
    "ds_read_b64_tr_b16 a[140:141], v243 offset:8704\n"
    "v_mfma_f32_16x16x32_bf16 v[108:111], a[152:155], a[24:27], v[108:111]\n"
    "v_mfma_f32_16x16x32_bf16 v[108:111], a[156:159], a[28:31], v[108:111]\n"
    "ds_read_b64_tr_b16 a[142:143], v243 offset:9728\n"
    "v_mfma_f32_16x16x32_bf16 v[112:115], a[152:155], a[32:35], v[112:115]\n"
    "s_add_u32 m0, s58, 12288\n"
    "v_mfma_f32_16x16x32_bf16 v[112:115], a[156:159], a[36:39], v[112:115]\n"
    "buffer_load_dwordx4 v238, s[40:43], s54 offen lds\n"
    "v_mfma_f32_16x16x32_bf16 v[116:119], a[152:155], a[40:43], v[116:119]\n"
    "v_mfma_f32_16x16x32_bf16 v[116:119], a[156:159], a[44:47], v[116:119]\n"
    "ds_read_b128 a[88:91], v240 offset:38912\n"
    "v_mfma_f32_16x16x32_bf16 v[120:123], a[152:155], a[48:51], v[120:123]\n"
    "s_add_u32 m0, s58, 13312\n"
    "v_mfma_f32_16x16x32_bf16 v[120:123], a[156:159], a[52:55], v[120:123]\n"
    "buffer_load_dwordx4 v239, s[40:43], s54 offen lds\n"
    "v_mfma_f32_16x16x32_bf16 v[124:127], a[152:155], a[56:59], v[124:127]\n"
    "v_mfma_f32_16x16x32_bf16 v[124:127], a[156:159], a[60:63], v[124:127]\n"
    "ds_read_b128 a[92:95], v241 offset:38912\n"
    "s_waitcnt lgkmcnt(8)\n"
    "v_mfma_f32_16x16x32_bf16 v[128:131], a[128:131], a[0:3], v[128:131]\n"
    "v_mfma_f32_16x16x32_bf16 v[128:131], a[132:135], a[4:7], v[128:131]\n"
    "ds_read_b64_tr_b16 a[144:145], v242 offset:768\n"
    "v_mfma_f32_16x16x32_bf16 v[132:135], a[128:131], a[8:11], v[132:135]\n"
    "s_add_u32 s58, s58, 32768\n"
    "v_mfma_f32_16x16x32_bf16 v[132:135], a[132:135], a[12:15], v[132:135]\n"
    "ds_read_b64_tr_b16 a[146:147], v242 offset:1792\n"
    "v_mfma_f32_16x16x32_bf16 v[136:139], a[128:131], a[16:19], v[136:139]\n"
    "s_sub_u32 s59, s58, 98304\n"
    "v_mfma_f32_16x16x32_bf16 v[136:139], a[132:135], a[20:23], v[136:139]\n"
    "ds_read_b64_tr_b16 a[148:149], v242 offset:8960\n"
    "v_mfma_f32_16x16x32_bf16 v[140:143], a[128:131], a[24:27], v[140:143]\n"
    "s_cmp_ge_u32 s58, s67\n"
    "v_mfma_f32_16x16x32_bf16 v[140:143], a[132:135], a[28:31], v[140:143]\n"
    "ds_read_b64_tr_b16 a[150:151], v242 offset:9984\n"
    "v_mfma_f32_16x16x32_bf16 v[144:147], a[128:131], a[32:35], v[144:147]\n"
    "s_cselect_b32 s58, s59, s58\n"
    "v_mfma_f32_16x16x32_bf16 v[144:147], a[132:135], a[36:39], v[144:147]\n"
    "ds_read_b128 a[96:99], v240 offset:40960\n"
    "v_mfma_f32_16x16x32_bf16 v[148:151], a[128:131], a[40:43], v[148:151]\n"
    "v_mfma_f32_16x16x32_bf16 v[148:151], a[132:135], a[44:47], v[148:151]\n"
    "ds_read_b128 a[100:103], v241 offset:40960\n"
    "v_mfma_f32_16x16x32_bf16 v[152:155], a[128:131], a[48:51], v[152:155]\n"
    "v_mfma_f32_16x16x32_bf16 v[152:155], a[132:135], a[52:55], v[152:155]\n"
    "ds_read_b128 a[104:107], v240 offset:43008\n"
    "v_mfma_f32_16x16x32_bf16 v[156:159], a[128:131], a[56:59], v[156:159]\n"
    "v_mfma_f32_16x16x32_bf16 v[156:159], a[132:135], a[60:63], v[156:159]\n"
    "ds_read_b128 a[108:111], v241 offset:43008\n"
    "s_waitcnt lgkmcnt(10)\n"
    "v_mfma_f32_16x16x32_bf16 v[160:163], a[136:139], a[0:3], v[160:163]\n"
    "v_mfma_f32_16x16x32_bf16 v[160:163], a[140:143], a[4:7], v[160:163]\n"
    "ds_read_b64_tr_b16 a[152:153], v243 offset:768\n"
    "ds_read_b128 a[112:115], v240 offset:45056\n"
    "v_mfma_f32_16x16x32_bf16 v[164:167], a[136:139], a[8:11], v[164:167]\n"
    "s_add_u32 s59, s52, 3\n"
    "v_mfma_f32_16x16x32_bf16 v[164:167], a[140:143], a[12:15], v[164:167]\n"
    "ds_read_b64_tr_b16 a[154:155], v243 offset:1792\n"
    "ds_read_b128 a[116:119], v241 offset:45056\n"
    "v_readlane_b32 s53, v244, s59\n"
    "v_mfma_f32_16x16x32_bf16 v[168:171], a[136:139], a[16:19], v[168:171]\n"
    "v_mfma_f32_16x16x32_bf16 v[168:171], a[140:143], a[20:23], v[168:171]\n"
    "ds_read_b64_tr_b16 a[156:157], v243 offset:8960\n"
    "ds_read_b128 a[120:123], v240 offset:47104\n"
    "v_mfma_f32_16x16x32_bf16 v[172:175], a[136:139], a[24:27], v[172:175]\n"
    "v_mfma_f32_16x16x32_bf16 v[172:175], a[140:143], a[28:31], v[172:175]\n"
    "ds_read_b64_tr_b16 a[158:159], v243 offset:9984\n"
    "ds_read_b128 a[124:127], v241 offset:47104\n"
    "v_mfma_f32_16x16x32_bf16 v[176:179], a[136:139], a[32:35], v[176:179]\n"
    "s_add_u32 s57, s57, 1\n"
    "v_mfma_f32_16x16x32_bf16 v[176:179], a[140:143], a[36:39], v[176:179]\n"
    "s_cmp_eq_u32 s57, 3\n"
    "v_mfma_f32_16x16x32_bf16 v[180:183], a[136:139], a[40:43], v[180:183]\n"
    "s_cselect_b32 s57, 0, s57\n"
    "s_cselect_b32 s59, 4294901760, s68\n"
    "v_mfma_f32_16x16x32_bf16 v[180:183], a[140:143], a[44:47], v[180:183]\n"
    "v_mfma_f32_16x16x32_bf16 v[184:187], a[136:139], a[48:51], v[184:187]\n"
    "v_add_u32 v242, s59, v242\n"
    "v_mfma_f32_16x16x32_bf16 v[184:187], a[140:143], a[52:55], v[184:187]\n"
    "v_add_u32 v243, s59, v243\n"
    "v_mfma_f32_16x16x32_bf16 v[188:191], a[136:139], a[56:59], v[188:191]\n"
    "v_mfma_f32_16x16x32_bf16 v[188:191], a[140:143], a[60:63], v[188:191]\n"
    "s_waitcnt vmcnt(8) lgkmcnt(0)\n"
    "s_barrier\n"
    "v_mfma_f32_16x16x32_bf16 v[192:195], a[144:147], a[0:3], v[192:195]\n"
    "v_mfma_f32_16x16x32_bf16 v[192:195], a[148:151], a[4:7], v[192:195]\n"
    "ds_read_b64_tr_b16 a[128:129], v242 offset:0\n"
    "v_mfma_f32_16x16x32_bf16 v[196:199], a[144:147], a[8:11], v[196:199]\n"
    "v_mfma_f32_16x16x32_bf16 v[196:199], a[148:151], a[12:15], v[196:199]\n"
    "ds_read_b64_tr_b16 a[130:131], v242 offset:1024\n"
    "v_mfma_f32_16x16x32_bf16 v[200:203], a[144:147], a[16:19], v[200:203]\n"
    "v_mfma_f32_16x16x32_bf16 v[200:203], a[148:151], a[20:23], v[200:203]\n"
    "ds_read_b64_tr_b16 a[132:133], v242 offset:8192\n"
    "v_mfma_f32_16x16x32_bf16 v[204:207], a[144:147], a[24:27], v[204:207]\n"
    "v_mfma_f32_16x16x32_bf16 v[204:207], a[148:151], a[28:31], v[204:207]\n"
    "ds_read_b64_tr_b16 a[134:135], v242 offset:9216\n"
    "v_mfma_f32_16x16x32_bf16 v[208:211], a[144:147], a[32:35], v[208:211]\n"
    "s_add_u32 m0, s55, 32768\n"
    "v_mfma_f32_16x16x32_bf16 v[208:211], a[148:151], a[36:39], v[208:211]\n"
    "buffer_load_dwordx4 v224, s[44:47], s53 offen sc1 lds\n"
    "v_mfma_f32_16x16x32_bf16 v[212:215], a[144:147], a[40:43], v[212:215]\n"
    "s_add_u32 m0, s55, 33792\n"
    "v_mfma_f32_16x16x32_bf16 v[212:215], a[148:151], a[44:47], v[212:215]\n"
    "buffer_load_dwordx4 v225, s[44:47], s53 offen sc1 lds\n"
    "v_mfma_f32_16x16x32_bf16 v[216:219], a[144:147], a[48:51], v[216:219]\n"
    "s_add_u32 m0, s55, 34816\n"
    "v_mfma_f32_16x16x32_bf16 v[216:219], a[148:151], a[52:55], v[216:219]\n"
    "buffer_load_dwordx4 v226, s[44:47], s53 offen sc1 lds\n"
    "v_mfma_f32_16x16x32_bf16 v[220:223], a[144:147], a[56:59], v[220:223]\n"
    "s_add_u32 m0, s55, 35840\n"
    "v_mfma_f32_16x16x32_bf16 v[220:223], a[148:151], a[60:63], v[220:223]\n"
    "buffer_load_dwordx4 v227, s[44:47], s53 offen sc1 lds\n"
    "s_waitcnt lgkmcnt(5)\n"
    "v_mfma_f32_16x16x32_bf16 a[160:163], a[152:155], a[0:3], a[160:163]\n"
    "v_mfma_f32_16x16x32_bf16 a[160:163], a[156:159], a[4:7], a[160:163]\n"
    "ds_read_b64_tr_b16 a[136:137], v243 offset:0\n"
    "v_mfma_f32_16x16x32_bf16 a[164:167], a[152:155], a[8:11], a[164:167]\n"
    "v_mfma_f32_16x16x32_bf16 a[164:167], a[156:159], a[12:15], a[164:167]\n"
    "ds_read_b64_tr_b16 a[138:139], v243 offset:1024\n"
    "v_mfma_f32_16x16x32_bf16 a[168:171], a[152:155], a[16:19], a[168:171]\n"
    "v_mfma_f32_16x16x32_bf16 a[168:171], a[156:159], a[20:23], a[168:171]\n"
    "ds_read_b64_tr_b16 a[140:141], v243 offset:8192\n"
    "v_mfma_f32_16x16x32_bf16 a[172:175], a[152:155], a[24:27], a[172:175]\n"
    "v_mfma_f32_16x16x32_bf16 a[172:175], a[156:159], a[28:31], a[172:175]\n"
    "ds_read_b64_tr_b16 a[142:143], v243 offset:9216\n"
    "v_mfma_f32_16x16x32_bf16 a[176:179], a[152:155], a[32:35], a[176:179]\n"
    "s_add_u32 m0, s55, 36864\n"
    "v_mfma_f32_16x16x32_bf16 a[176:179], a[156:159], a[36:39], a[176:179]\n"
    "buffer_load_dwordx4 v228, s[44:47], s53 offen sc1 lds\n"
    "v_mfma_f32_16x16x32_bf16 a[180:183], a[152:155], a[40:43], a[180:183]\n"
    "s_add_u32 m0, s55, 37888\n"
    "v_mfma_f32_16x16x32_bf16 a[180:183], a[156:159], a[44:47], a[180:183]\n"
    "buffer_load_dwordx4 v229, s[44:47], s53 offen sc1 lds\n"
    "v_mfma_f32_16x16x32_bf16 a[184:187], a[152:155], a[48:51], a[184:187]\n"
    "s_add_u32 m0, s55, 38912\n"
    "v_mfma_f32_16x16x32_bf16 a[184:187], a[156:159], a[52:55], a[184:187]\n"
    "buffer_load_dwordx4 v230, s[44:47], s53 offen sc1 lds\n"
    "v_mfma_f32_16x16x32_bf16 a[188:191], a[152:155], a[56:59], a[188:191]\n"
    "s_add_u32 m0, s55, 39936\n"
    "v_mfma_f32_16x16x32_bf16 a[188:191], a[156:159], a[60:63], a[188:191]\n"
    "buffer_load_dwordx4 v231, s[44:47], s53 offen sc1 lds\n"
    "s_add_u32 s52, s52, 1\n"
    ".Lg42_%=:\n"
    "s_waitcnt lgkmcnt(4)\n"
    "v_mfma_f32_16x16x32_bf16 v[0:3], a[128:131], a[64:67], v[0:3]\n"
    "s_add_u32 s59, s52, 2\n"
    "v_mfma_f32_16x16x32_bf16 v[0:3], a[132:135], a[68:71], v[0:3]\n"
    "ds_read_b64_tr_b16 a[144:145], v242 offset:256\n"
    "v_readlane_b32 s54, v245, s59\n"
    "v_mfma_f32_16x16x32_bf16 v[4:7], a[128:131], a[72:75], v[4:7]\n"
    "v_mfma_f32_16x16x32_bf16 v[4:7], a[132:135], a[76:79], v[4:7]\n"
    "ds_read_b64_tr_b16 a[146:147], v242 offset:1280\n"
    "v_mfma_f32_16x16x32_bf16 v[8:11], a[128:131], a[80:83], v[8:11]\n"
    "buffer_load_dword v252, v246, s[88:91], 0 offen sc1\n"
    "v_mfma_f32_16x16x32_bf16 v[8:11], a[132:135], a[84:87], v[8:11]\n"
    "ds_read_b64_tr_b16 a[148:149], v242 offset:8448\n"
    "v_mfma_f32_16x16x32_bf16 v[12:15], a[128:131], a[88:91], v[12:15]\n"
    "v_mfma_f32_16x16x32_bf16 v[12:15], a[132:135], a[92:95], v[12:15]\n"
    "ds_read_b64_tr_b16 a[150:151], v242 offset:9472\n"
    "v_mfma_f32_16x16x32_bf16 v[16:19], a[128:131], a[96:99], v[16:19]\n"
    "s_add_u32 m0, s58, 0\n"
    "v_mfma_f32_16x16x32_bf16 v[16:19], a[132:135], a[100:103], v[16:19]\n"
    "buffer_load_dwordx4 v232, s[40:43], s54 offen lds\n"
    "v_mfma_f32_16x16x32_bf16 v[20:23], a[128:131], a[104:107], v[20:23]\n"
    "v_readlane_b32 s62, v247, s52\n"
    "v_mfma_f32_16x16x32_bf16 v[20:23], a[132:135], a[108:111], v[20:23]\n"
    "ds_read_b128 a[0:3], v240 offset:0\n"
    "v_mfma_f32_16x16x32_bf16 v[24:27], a[128:131], a[112:115], v[24:27]\n"
    "s_lshl_b64 s[78:79], -1, s62\n"
    "s_add_u32 m0, s58, 1024\n"
    "v_mfma_f32_16x16x32_bf16 v[24:27], a[132:135], a[116:119], v[24:27]\n"
    "buffer_load_dwordx4 v233, s[40:43], s54 offen lds\n"
    "v_mfma_f32_16x16x32_bf16 v[28:31], a[128:131], a[120:123], v[28:31]\n"
    "v_mfma_f32_16x16x32_bf16 v[28:31], a[132:135], a[124:127], v[28:31]\n"
    "ds_read_b128 a[4:7], v241 offset:0\n"
    "s_waitcnt lgkmcnt(6)\n"
    "v_mfma_f32_16x16x32_bf16 v[32:35], a[136:139], a[64:67], v[32:35]\n"
    "s_sub_u32 s61, s52, 1\n"
    "v_mfma_f32_16x16x32_bf16 v[32:35], a[140:143], a[68:71], v[32:35]\n"
    "ds_read_b64_tr_b16 a[152:153], v243 offset:256\n"
    "v_mfma_f32_16x16x32_bf16 v[36:39], a[136:139], a[72:75], v[36:39]\n"
    "s_cmp_eq_u32 s61, s56\n"
    "s_cselect_b32 s61, s66, 2147418112\n"
    "v_mfma_f32_16x16x32_bf16 v[36:39], a[140:143], a[76:79], v[36:39]\n"
    "ds_read_b64_tr_b16 a[154:155], v243 offset:1280\n"
    "v_mfma_f32_16x16x32_bf16 v[40:43], a[136:139], a[80:83], v[40:43]\n"
    "v_mfma_f32_16x16x32_bf16 v[40:43], a[140:143], a[84:87], v[40:43]\n"
    "ds_read_b64_tr_b16 a[156:157], v243 offset:8448\n"
    "v_mfma_f32_16x16x32_bf16 v[44:47], a[136:139], a[88:91], v[44:47]\n"
    "v_mfma_f32_16x16x32_bf16 v[44:47], a[140:143], a[92:95], v[44:47]\n"
    "ds_read_b64_tr_b16 a[158:159], v243 offset:9472\n"
    "v_mfma_f32_16x16x32_bf16 v[48:51], a[136:139], a[96:99], v[48:51]\n"
    "s_add_u32 m0, s58, 4096\n"
    "v_mfma_f32_16x16x32_bf16 v[48:51], a[140:143], a[100:103], v[48:51]\n"
    "buffer_load_dwordx4 v234, s[40:43], s54 offen lds\n"
    "v_mfma_f32_16x16x32_bf16 v[52:55], a[136:139], a[104:107], v[52:55]\n"
    "v_mfma_f32_16x16x32_bf16 v[52:55], a[140:143], a[108:111], v[52:55]\n"
    "ds_read_b128 a[8:11], v240 offset:2048\n"
    "v_mfma_f32_16x16x32_bf16 v[56:59], a[136:139], a[112:115], v[56:59]\n"
    "s_add_u32 m0, s58, 5120\n"
    "v_mfma_f32_16x16x32_bf16 v[56:59], a[140:143], a[116:119], v[56:59]\n"
    "buffer_load_dwordx4 v235, s[40:43], s54 offen lds\n"
    "v_mfma_f32_16x16x32_bf16 v[60:63], a[136:139], a[120:123], v[60:63]\n"
    "v_mfma_f32_16x16x32_bf16 v[60:63], a[140:143], a[124:127], v[60:63]\n"
    "ds_read_b128 a[12:15], v241 offset:2048\n"
    "s_waitcnt lgkmcnt(8)\n"
    "v_mfma_f32_16x16x32_bf16 v[64:67], a[144:147], a[64:67], v[64:67]\n"
    "v_mfma_f32_16x16x32_bf16 v[64:67], a[148:151], a[68:71], v[64:67]\n"
    "ds_read_b64_tr_b16 a[128:129], v242 offset:512\n"
    "v_mfma_f32_16x16x32_bf16 v[68:71], a[144:147], a[72:75], v[68:71]\n"
    "v_mfma_f32_16x16x32_bf16 v[68:71], a[148:151], a[76:79], v[68:71]\n"
    "ds_read_b64_tr_b16 a[130:131], v242 offset:1536\n"
    "v_mfma_f32_16x16x32_bf16 v[72:75], a[144:147], a[80:83], v[72:75]\n"
    "v_mfma_f32_16x16x32_bf16 v[72:75], a[148:151], a[84:87], v[72:75]\n"
    "ds_read_b64_tr_b16 a[132:133], v242 offset:8704\n"
    "v_mfma_f32_16x16x32_bf16 v[76:79], a[144:147], a[88:91], v[76:79]\n"
    "v_mfma_f32_16x16x32_bf16 v[76:79], a[148:151], a[92:95], v[76:79]\n"
    "ds_read_b64_tr_b16 a[134:135], v242 offset:9728\n"
    "v_mfma_f32_16x16x32_bf16 v[80:83], a[144:147], a[96:99], v[80:83]\n"
    "s_add_u32 m0, s58, 8192\n"
    "v_mfma_f32_16x16x32_bf16 v[80:83], a[148:151], a[100:103], v[80:83]\n"
    "buffer_load_dwordx4 v236, s[40:43], s54 offen lds\n"
    "v_mfma_f32_16x16x32_bf16 v[84:87], a[144:147], a[104:107], v[84:87]\n"
    "v_mfma_f32_16x16x32_bf16 v[84:87], a[148:151], a[108:111], v[84:87]\n"
    "ds_read_b128 a[16:19], v240 offset:4096\n"
    "v_mfma_f32_16x16x32_bf16 v[88:91], a[144:147], a[112:115], v[88:91]\n"
    "s_add_u32 m0, s58, 9216\n"
    "v_mfma_f32_16x16x32_bf16 v[88:91], a[148:151], a[116:119], v[88:91]\n"
    "buffer_load_dwordx4 v237, s[40:43], s54 offen lds\n"
    "v_mfma_f32_16x16x32_bf16 v[92:95], a[144:147], a[120:123], v[92:95]\n"
    "v_mfma_f32_16x16x32_bf16 v[92:95], a[148:151], a[124:127], v[92:95]\n"
    "ds_read_b128 a[20:23], v241 offset:4096\n"
    "s_waitcnt lgkmcnt(8)\n"
    "v_mfma_f32_16x16x32_bf16 v[96:99], a[152:155], a[64:67], v[96:99]\n"
    "v_mfma_f32_16x16x32_bf16 v[96:99], a[156:159], a[68:71], v[96:99]\n"
    "ds_read_b64_tr_b16 a[136:137], v243 offset:512\n"
    "v_mfma_f32_16x16x32_bf16 v[100:103], a[152:155], a[72:75], v[100:103]\n"
    "v_mfma_f32_16x16x32_bf16 v[100:103], a[156:159], a[76:79], v[100:103]\n"
    "ds_read_b64_tr_b16 a[138:139], v243 offset:1536\n"
    "v_mfma_f32_16x16x32_bf16 v[104:107], a[152:155], a[80:83], v[104:107]\n"
    "v_mfma_f32_16x16x32_bf16 v[104:107], a[156:159], a[84:87], v[104:107]\n"
    "ds_read_b64_tr_b16 a[140:141], v243 offset:8704\n"
    "v_mfma_f32_16x16x32_bf16 v[108:111], a[152:155], a[88:91], v[108:111]\n"
    "v_mfma_f32_16x16x32_bf16 v[108:111], a[156:159], a[92:95], v[108:111]\n"
    "ds_read_b64_tr_b16 a[142:143], v243 offset:9728\n"
    "v_mfma_f32_16x16x32_bf16 v[112:115], a[152:155], a[96:99], v[112:115]\n"
    "s_add_u32 m0, s58, 12288\n"
    "v_mfma_f32_16x16x32_bf16 v[112:115], a[156:159], a[100:103], v[112:115]\n"
    "buffer_load_dwordx4 v238, s[40:43], s54 offen lds\n"
    "v_mfma_f32_16x16x32_bf16 v[116:119], a[152:155], a[104:107], v[116:119]\n"
    "v_mfma_f32_16x16x32_bf16 v[116:119], a[156:159], a[108:111], v[116:119]\n"
    "ds_read_b128 a[24:27], v240 offset:6144\n"
    "v_mfma_f32_16x16x32_bf16 v[120:123], a[152:155], a[112:115], v[120:123]\n"
    "s_add_u32 m0, s58, 13312\n"
    "v_mfma_f32_16x16x32_bf16 v[120:123], a[156:159], a[116:119], v[120:123]\n"
    "buffer_load_dwordx4 v239, s[40:43], s54 offen lds\n"
    "v_mfma_f32_16x16x32_bf16 v[124:127], a[152:155], a[120:123], v[124:127]\n"
    "v_mfma_f32_16x16x32_bf16 v[124:127], a[156:159], a[124:127], v[124:127]\n"
    "ds_read_b128 a[28:31], v241 offset:6144\n"
    "s_waitcnt lgkmcnt(8)\n"
    "v_mfma_f32_16x16x32_bf16 v[128:131], a[128:131], a[64:67], v[128:131]\n"
    "v_mfma_f32_16x16x32_bf16 v[128:131], a[132:135], a[68:71], v[128:131]\n"
    "ds_read_b64_tr_b16 a[144:145], v242 offset:768\n"
    "v_mfma_f32_16x16x32_bf16 v[132:135], a[128:131], a[72:75], v[132:135]\n"
    "s_add_u32 s58, s58, 32768\n"
    "v_mfma_f32_16x16x32_bf16 v[132:135], a[132:135], a[76:79], v[132:135]\n"
    "ds_read_b64_tr_b16 a[146:147], v242 offset:1792\n"
    "v_mfma_f32_16x16x32_bf16 v[136:139], a[128:131], a[80:83], v[136:139]\n"
    "s_sub_u32 s59, s58, 98304\n"
    "v_mfma_f32_16x16x32_bf16 v[136:139], a[132:135], a[84:87], v[136:139]\n"
    "ds_read_b64_tr_b16 a[148:149], v242 offset:8960\n"
    "v_mfma_f32_16x16x32_bf16 v[140:143], a[128:131], a[88:91], v[140:143]\n"
    "s_cmp_ge_u32 s58, s67\n"
    "v_mfma_f32_16x16x32_bf16 v[140:143], a[132:135], a[92:95], v[140:143]\n"
    "ds_read_b64_tr_b16 a[150:151], v242 offset:9984\n"
    "v_mfma_f32_16x16x32_bf16 v[144:147], a[128:131], a[96:99], v[144:147]\n"
    "s_cselect_b32 s58, s59, s58\n"
    "v_mfma_f32_16x16x32_bf16 v[144:147], a[132:135], a[100:103], v[144:147]\n"
    "ds_read_b128 a[32:35], v240 offset:8192\n"
    "v_mfma_f32_16x16x32_bf16 v[148:151], a[128:131], a[104:107], v[148:151]\n"
    "v_mfma_f32_16x16x32_bf16 v[148:151], a[132:135], a[108:111], v[148:151]\n"
    "ds_read_b128 a[36:39], v241 offset:8192\n"
    "v_mfma_f32_16x16x32_bf16 v[152:155], a[128:131], a[112:115], v[152:155]\n"
    "v_mfma_f32_16x16x32_bf16 v[152:155], a[132:135], a[116:119], v[152:155]\n"
    "ds_read_b128 a[40:43], v240 offset:10240\n"
    "v_mfma_f32_16x16x32_bf16 v[156:159], a[128:131], a[120:123], v[156:159]\n"
    "v_mfma_f32_16x16x32_bf16 v[156:159], a[132:135], a[124:127], v[156:159]\n"
    "ds_read_b128 a[44:47], v241 offset:10240\n"
    "s_waitcnt lgkmcnt(10)\n"
    "v_mfma_f32_16x16x32_bf16 v[160:163], a[136:139], a[64:67], v[160:163]\n"
    "v_mfma_f32_16x16x32_bf16 v[160:163], a[140:143], a[68:71], v[160:163]\n"
    "ds_read_b64_tr_b16 a[152:153], v243 offset:768\n"
    "ds_read_b128 a[48:51], v240 offset:12288\n"
    "v_mfma_f32_16x16x32_bf16 v[164:167], a[136:139], a[72:75], v[164:167]\n"
    "s_add_u32 s59, s52, 3\n"
    "v_mfma_f32_16x16x32_bf16 v[164:167], a[140:143], a[76:79], v[164:167]\n"
    "ds_read_b64_tr_b16 a[154:155], v243 offset:1792\n"
    "ds_read_b128 a[52:55], v241 offset:12288\n"
    "v_readlane_b32 s53, v244, s59\n"
    "v_mfma_f32_16x16x32_bf16 v[168:171], a[136:139], a[80:83], v[168:171]\n"
    "s_cmp_eq_u32 s59, 8\n"
    "s_cselect_b32 s44, s48, s44\n"
    "v_mfma_f32_16x16x32_bf16 v[168:171], a[140:143], a[84:87], v[168:171]\n"
    "ds_read_b64_tr_b16 a[156:157], v243 offset:8960\n"
    "ds_read_b128 a[56:59], v240 offset:14336\n"
    "s_cselect_b32 s45, s49, s45\n"
    "v_mfma_f32_16x16x32_bf16 v[172:175], a[136:139], a[88:91], v[172:175]\n"
    "v_mfma_f32_16x16x32_bf16 v[172:175], a[140:143], a[92:95], v[172:175]\n"
    "ds_read_b64_tr_b16 a[158:159], v243 offset:9984\n"
    "ds_read_b128 a[60:63], v241 offset:14336\n"
    "v_mfma_f32_16x16x32_bf16 v[176:179], a[136:139], a[96:99], v[176:179]\n"
    "s_add_u32 s57, s57, 1\n"
    "v_mfma_f32_16x16x32_bf16 v[176:179], a[140:143], a[100:103], v[176:179]\n"
    "s_cmp_eq_u32 s57, 3\n"
    "v_mfma_f32_16x16x32_bf16 v[180:183], a[136:139], a[104:107], v[180:183]\n"
    "s_cselect_b32 s57, 0, s57\n"
    "s_cselect_b32 s59, 4294901760, s68\n"
    "v_mfma_f32_16x16x32_bf16 v[180:183], a[140:143], a[108:111], v[180:183]\n"
    "v_mfma_f32_16x16x32_bf16 v[184:187], a[136:139], a[112:115], v[184:187]\n"
    "v_add_u32 v242, s59, v242\n"
    "v_mfma_f32_16x16x32_bf16 v[184:187], a[140:143], a[116:119], v[184:187]\n"
    "v_add_u32 v243, s59, v243\n"
    "v_mfma_f32_16x16x32_bf16 v[188:191], a[136:139], a[120:123], v[188:191]\n"
    "v_mfma_f32_16x16x32_bf16 v[188:191], a[140:143], a[124:127], v[188:191]\n"
    "s_waitcnt vmcnt(8) lgkmcnt(0)\n"
    "s_barrier\n"
    "v_mfma_f32_16x16x32_bf16 v[192:195], a[144:147], a[64:67], v[192:195]\n"
    "v_cmp_le_u32 vcc, s63, v252\n"
    "v_mfma_f32_16x16x32_bf16 v[192:195], a[148:151], a[68:71], v[192:195]\n"
    "buffer_store_dword v254, v251, s[84:87], s61 offen sc0 sc1\n"
    "ds_read_b64_tr_b16 a[128:129], v242 offset:0\n"
    "v_mfma_f32_16x16x32_bf16 v[196:199], a[144:147], a[72:75], v[196:199]\n"
    "s_or_b64 vcc, vcc, s[78:79]\n"
    "s_cmp_eq_u64 vcc, -1\n"
    "v_mfma_f32_16x16x32_bf16 v[196:199], a[148:151], a[76:79], v[196:199]\n"
    "s_cbranch_scc0 .Lg43_%=\n"
    ".Lg44_%=:\n"
    "ds_read_b64_tr_b16 a[130:131], v242 offset:1024\n"
    "v_mfma_f32_16x16x32_bf16 v[200:203], a[144:147], a[80:83], v[200:203]\n"
    "v_mfma_f32_16x16x32_bf16 v[200:203], a[148:151], a[84:87], v[200:203]\n"
    "ds_read_b64_tr_b16 a[132:133], v242 offset:8192\n"
    "v_mfma_f32_16x16x32_bf16 v[204:207], a[144:147], a[88:91], v[204:207]\n"
    "v_mfma_f32_16x16x32_bf16 v[204:207], a[148:151], a[92:95], v[204:207]\n"
    "ds_read_b64_tr_b16 a[134:135], v242 offset:9216\n"
    "v_mfma_f32_16x16x32_bf16 v[208:211], a[144:147], a[96:99], v[208:211]\n"
    "s_add_u32 m0, s55, 0\n"
    "v_mfma_f32_16x16x32_bf16 v[208:211], a[148:151], a[100:103], v[208:211]\n"
    "buffer_load_dwordx4 v224, s[44:47], s53 offen sc1 lds\n"
    "v_mfma_f32_16x16x32_bf16 v[212:215], a[144:147], a[104:107], v[212:215]\n"
    "s_add_u32 m0, s55, 1024\n"
    "v_mfma_f32_16x16x32_bf16 v[212:215], a[148:151], a[108:111], v[212:215]\n"
    "buffer_load_dwordx4 v225, s[44:47], s53 offen sc1 lds\n"
    "v_mfma_f32_16x16x32_bf16 v[216:219], a[144:147], a[112:115], v[216:219]\n"
    "s_add_u32 m0, s55, 2048\n"
    "v_mfma_f32_16x16x32_bf16 v[216:219], a[148:151], a[116:119], v[216:219]\n"
    "buffer_load_dwordx4 v226, s[44:47], s53 offen sc1 lds\n"
    "v_mfma_f32_16x16x32_bf16 v[220:223], a[144:147], a[120:123], v[220:223]\n"
    "s_add_u32 m0, s55, 3072\n"
    "v_mfma_f32_16x16x32_bf16 v[220:223], a[148:151], a[124:127], v[220:223]\n"
    "buffer_load_dwordx4 v227, s[44:47], s53 offen sc1 lds\n"
    "s_waitcnt lgkmcnt(5)\n"
    "v_mfma_f32_16x16x32_bf16 a[160:163], a[152:155], a[64:67], a[160:163]\n"
    "v_mfma_f32_16x16x32_bf16 a[160:163], a[156:159], a[68:71], a[160:163]\n"
    "ds_read_b64_tr_b16 a[136:137], v243 offset:0\n"
    "v_mfma_f32_16x16x32_bf16 a[164:167], a[152:155], a[72:75], a[164:167]\n"
    "v_mfma_f32_16x16x32_bf16 a[164:167], a[156:159], a[76:79], a[164:167]\n"
    "ds_read_b64_tr_b16 a[138:139], v243 offset:1024\n"
    "v_mfma_f32_16x16x32_bf16 a[168:171], a[152:155], a[80:83], a[168:171]\n"
    "v_mfma_f32_16x16x32_bf16 a[168:171], a[156:159], a[84:87], a[168:171]\n"
    "ds_read_b64_tr_b16 a[140:141], v243 offset:8192\n"
    "v_mfma_f32_16x16x32_bf16 a[172:175], a[152:155], a[88:91], a[172:175]\n"
    "v_mfma_f32_16x16x32_bf16 a[172:175], a[156:159], a[92:95], a[172:175]\n"
    "ds_read_b64_tr_b16 a[142:143], v243 offset:9216\n"
    "v_mfma_f32_16x16x32_bf16 a[176:179], a[152:155], a[96:99], a[176:179]\n"
    "s_add_u32 m0, s55, 4096\n"
    "v_mfma_f32_16x16x32_bf16 a[176:179], a[156:159], a[100:103], a[176:179]\n"
    "buffer_load_dwordx4 v228, s[44:47], s53 offen sc1 lds\n"
    "v_mfma_f32_16x16x32_bf16 a[180:183], a[152:155], a[104:107], a[180:183]\n"
    "s_add_u32 m0, s55, 5120\n"
    "v_mfma_f32_16x16x32_bf16 a[180:183], a[156:159], a[108:111], a[180:183]\n"
    "buffer_load_dwordx4 v229, s[44:47], s53 offen sc1 lds\n"
    "v_mfma_f32_16x16x32_bf16 a[184:187], a[152:155], a[112:115], a[184:187]\n"
    "s_add_u32 m0, s55, 6144\n"
    "v_mfma_f32_16x16x32_bf16 a[184:187], a[156:159], a[116:119], a[184:187]\n"
    "buffer_load_dwordx4 v230, s[44:47], s53 offen sc1 lds\n"
    "v_mfma_f32_16x16x32_bf16 a[188:191], a[152:155], a[120:123], a[188:191]\n"
    "s_add_u32 m0, s55, 7168\n"
    "v_mfma_f32_16x16x32_bf16 a[188:191], a[156:159], a[124:127], a[188:191]\n"
    "buffer_load_dwordx4 v231, s[44:47], s53 offen sc1 lds\n"
    "s_add_u32 s52, s52, 1\n"
    "s_sub_u32 s60, s60, 1\n"
    "s_cmp_lg_u32 s60, 0\n"
    "s_cbranch_scc1 .Lg40_%=\n"
    "s_mov_b32 s60, 21\n"
    ".Lg47_%=:\n"
    "s_waitcnt lgkmcnt(4)\n"
    "v_mfma_f32_16x16x32_bf16 v[0:3], a[128:131], a[0:3], v[0:3]\n"
    "s_add_u32 s59, s52, 2\n"
    "v_mfma_f32_16x16x32_bf16 v[0:3], a[132:135], a[4:7], v[0:3]\n"
    "ds_read_b64_tr_b16 a[144:145], v242 offset:256\n"
    "v_readlane_b32 s54, v245, s59\n"
    "v_mfma_f32_16x16x32_bf16 v[4:7], a[128:131], a[8:11], v[4:7]\n"
    "v_mfma_f32_16x16x32_bf16 v[4:7], a[132:135], a[12:15], v[4:7]\n"
    "ds_read_b64_tr_b16 a[146:147], v242 offset:1280\n"
    "v_mfma_f32_16x16x32_bf16 v[8:11], a[128:131], a[16:19], v[8:11]\n"
    "v_mfma_f32_16x16x32_bf16 v[8:11], a[132:135], a[20:23], v[8:11]\n"
    "ds_read_b64_tr_b16 a[148:149], v242 offset:8448\n"
    "v_mfma_f32_16x16x32_bf16 v[12:15], a[128:131], a[24:27], v[12:15]\n"
    "v_mfma_f32_16x16x32_bf16 v[12:15], a[132:135], a[28:31], v[12:15]\n"
    "ds_read_b64_tr_b16 a[150:151], v242 offset:9472\n"
    "v_mfma_f32_16x16x32_bf16 v[16:19], a[128:131], a[32:35], v[16:19]\n"
    "s_add_u32 m0, s58, 0\n"
    "v_mfma_f32_16x16x32_bf16 v[16:19], a[132:135], a[36:39], v[16:19]\n"
    "buffer_load_dwordx4 v232, s[40:43], s54 offen lds\n"
    "v_mfma_f32_16x16x32_bf16 v[20:23], a[128:131], a[40:43], v[20:23]\n"
    "v_mfma_f32_16x16x32_bf16 v[20:23], a[132:135], a[44:47], v[20:23]\n"
    "ds_read_b128 a[64:67], v240 offset:32768\n"
    "v_mfma_f32_16x16x32_bf16 v[24:27], a[128:131], a[48:51], v[24:27]\n"
    "s_add_u32 m0, s58, 1024\n"
    "v_mfma_f32_16x16x32_bf16 v[24:27], a[132:135], a[52:55], v[24:27]\n"
    "buffer_load_dwordx4 v233, s[40:43], s54 offen lds\n"
    "v_mfma_f32_16x16x32_bf16 v[28:31], a[128:131], a[56:59], v[28:31]\n"
    "v_mfma_f32_16x16x32_bf16 v[28:31], a[132:135], a[60:63], v[28:31]\n"
    "ds_read_b128 a[68:71], v241 offset:32768\n"
    "s_waitcnt lgkmcnt(6)\n"
    "v_mfma_f32_16x16x32_bf16 v[32:35], a[136:139], a[0:3], v[32:35]\n"
    "v_mfma_f32_16x16x32_bf16 v[32:35], a[140:143], a[4:7], v[32:35]\n"
    "ds_read_b64_tr_b16 a[152:153], v243 offset:256\n"
    "v_mfma_f32_16x16x32_bf16 v[36:39], a[136:139], a[8:11], v[36:39]\n"
    "v_mfma_f32_16x16x32_bf16 v[36:39], a[140:143], a[12:15], v[36:39]\n"
    "ds_read_b64_tr_b16 a[154:155], v243 offset:1280\n"
    "v_mfma_f32_16x16x32_bf16 v[40:43], a[136:139], a[16:19], v[40:43]\n"
    "v_mfma_f32_16x16x32_bf16 v[40:43], a[140:143], a[20:23], v[40:43]\n"
    "ds_read_b64_tr_b16 a[156:157], v243 offset:8448\n"
    "v_mfma_f32_16x16x32_bf16 v[44:47], a[136:139], a[24:27], v[44:47]\n"
    "v_mfma_f32_16x16x32_bf16 v[44:47], a[140:143], a[28:31], v[44:47]\n"
    "ds_read_b64_tr_b16 a[158:159], v243 offset:9472\n"
    "v_mfma_f32_16x16x32_bf16 v[48:51], a[136:139], a[32:35], v[48:51]\n"
    "s_add_u32 m0, s58, 4096\n"
    "v_mfma_f32_16x16x32_bf16 v[48:51], a[140:143], a[36:39], v[48:51]\n"
    "buffer_load_dwordx4 v234, s[40:43], s54 offen lds\n"
    "v_mfma_f32_16x16x32_bf16 v[52:55], a[136:139], a[40:43], v[52:55]\n"
    "v_mfma_f32_16x16x32_bf16 v[52:55], a[140:143], a[44:47], v[52:55]\n"
    "ds_read_b128 a[72:75], v240 offset:34816\n"
    "v_mfma_f32_16x16x32_bf16 v[56:59], a[136:139], a[48:51], v[56:59]\n"
    "s_add_u32 m0, s58, 5120\n"
    "v_mfma_f32_16x16x32_bf16 v[56:59], a[140:143], a[52:55], v[56:59]\n"
    "buffer_load_dwordx4 v235, s[40:43], s54 offen lds\n"
    "v_mfma_f32_16x16x32_bf16 v[60:63], a[136:139], a[56:59], v[60:63]\n"
    "v_mfma_f32_16x16x32_bf16 v[60:63], a[140:143], a[60:63], v[60:63]\n"
    "ds_read_b128 a[76:79], v241 offset:34816\n"
    "s_waitcnt lgkmcnt(8)\n"
    "v_mfma_f32_16x16x32_bf16 v[64:67], a[144:147], a[0:3], v[64:67]\n"
    "v_mfma_f32_16x16x32_bf16 v[64:67], a[148:151], a[4:7], v[64:67]\n"
    "ds_read_b64_tr_b16 a[128:129], v242 offset:512\n"
    "v_mfma_f32_16x16x32_bf16 v[68:71], a[144:147], a[8:11], v[68:71]\n"
    "v_mfma_f32_16x16x32_bf16 v[68:71], a[148:151], a[12:15], v[68:71]\n"
    "ds_read_b64_tr_b16 a[130:131], v242 offset:1536\n"
    "v_mfma_f32_16x16x32_bf16 v[72:75], a[144:147], a[16:19], v[72:75]\n"
    "v_mfma_f32_16x16x32_bf16 v[72:75], a[148:151], a[20:23], v[72:75]\n"
    "ds_read_b64_tr_b16 a[132:133], v242 offset:8704\n"
    "v_mfma_f32_16x16x32_bf16 v[76:79], a[144:147], a[24:27], v[76:79]\n"
    "v_mfma_f32_16x16x32_bf16 v[76:79], a[148:151], a[28:31], v[76:79]\n"
    "ds_read_b64_tr_b16 a[134:135], v242 offset:9728\n"
    "v_mfma_f32_16x16x32_bf16 v[80:83], a[144:147], a[32:35], v[80:83]\n"
    "s_add_u32 m0, s58, 8192\n"
    "v_mfma_f32_16x16x32_bf16 v[80:83], a[148:151], a[36:39], v[80:83]\n"
    "buffer_load_dwordx4 v236, s[40:43], s54 offen lds\n"
    "v_mfma_f32_16x16x32_bf16 v[84:87], a[144:147], a[40:43], v[84:87]\n"
    "v_mfma_f32_16x16x32_bf16 v[84:87], a[148:151], a[44:47], v[84:87]\n"
    "ds_read_b128 a[80:83], v240 offset:36864\n"
    "v_mfma_f32_16x16x32_bf16 v[88:91], a[144:147], a[48:51], v[88:91]\n"
    "s_add_u32 m0, s58, 9216\n"
    "v_mfma_f32_16x16x32_bf16 v[88:91], a[148:151], a[52:55], v[88:91]\n"
    "buffer_load_dwordx4 v237, s[40:43], s54 offen lds\n"
    "v_mfma_f32_16x16x32_bf16 v[92:95], a[144:147], a[56:59], v[92:95]\n"
    "v_mfma_f32_16x16x32_bf16 v[92:95], a[148:151], a[60:63], v[92:95]\n"
    "ds_read_b128 a[84:87], v241 offset:36864\n"
    "s_waitcnt lgkmcnt(8)\n"
    "v_mfma_f32_16x16x32_bf16 v[96:99], a[152:155], a[0:3], v[96:99]\n"
    "v_mfma_f32_16x16x32_bf16 v[96:99], a[156:159], a[4:7], v[96:99]\n"
    "ds_read_b64_tr_b16 a[136:137], v243 offset:512\n"
    "v_mfma_f32_16x16x32_bf16 v[100:103], a[152:155], a[8:11], v[100:103]\n"
    "v_mfma_f32_16x16x32_bf16 v[100:103], a[156:159], a[12:15], v[100:103]\n"
    "ds_read_b64_tr_b16 a[138:139], v243 offset:1536\n"
    "v_mfma_f32_16x16x32_bf16 v[104:107], a[152:155], a[16:19], v[104:107]\n"
    "v_mfma_f32_16x16x32_bf16 v[104:107], a[156:159], a[20:23], v[104:107]\n"
    "ds_read_b64_tr_b16 a[140:141], v243 offset:8704\n"
    "v_mfma_f32_16x16x32_bf16 v[108:111], a[152:155], a[24:27], v[108:111]\n"
    "v_mfma_f32_16x16x32_bf16 v[108:111], a[156:159], a[28:31], v[108:111]\n"
    "ds_read_b64_tr_b16 a[142:143], v243 offset:9728\n"
    "v_mfma_f32_16x16x32_bf16 v[112:115], a[152:155], a[32:35], v[112:115]\n"
    "s_add_u32 m0, s58, 12288\n"
    "v_mfma_f32_16x16x32_bf16 v[112:115], a[156:159], a[36:39], v[112:115]\n"
    "buffer_load_dwordx4 v238, s[40:43], s54 offen lds\n"
    "v_mfma_f32_16x16x32_bf16 v[116:119], a[152:155], a[40:43], v[116:119]\n"
    "v_mfma_f32_16x16x32_bf16 v[116:119], a[156:159], a[44:47], v[116:119]\n"
    "ds_read_b128 a[88:91], v240 offset:38912\n"
    "v_mfma_f32_16x16x32_bf16 v[120:123], a[152:155], a[48:51], v[120:123]\n"
    "s_add_u32 m0, s58, 13312\n"
    "v_mfma_f32_16x16x32_bf16 v[120:123], a[156:159], a[52:55], v[120:123]\n"
    "buffer_load_dwordx4 v239, s[40:43], s54 offen lds\n"
    "v_mfma_f32_16x16x32_bf16 v[124:127], a[152:155], a[56:59], v[124:127]\n"
    "v_mfma_f32_16x16x32_bf16 v[124:127], a[156:159], a[60:63], v[124:127]\n"
    "ds_read_b128 a[92:95], v241 offset:38912\n"
    "s_waitcnt lgkmcnt(8)\n"
    "v_mfma_f32_16x16x32_bf16 v[128:131], a[128:131], a[0:3], v[128:131]\n"
    "v_mfma_f32_16x16x32_bf16 v[128:131], a[132:135], a[4:7], v[128:131]\n"
    "ds_read_b64_tr_b16 a[144:145], v242 offset:768\n"
    "v_mfma_f32_16x16x32_bf16 v[132:135], a[128:131], a[8:11], v[132:135]\n"
    "s_add_u32 s58, s58, 32768\n"
    "v_mfma_f32_16x16x32_bf16 v[132:135], a[132:135], a[12:15], v[132:135]\n"
    "ds_read_b64_tr_b16 a[146:147], v242 offset:1792\n"
    "v_mfma_f32_16x16x32_bf16 v[136:139], a[128:131], a[16:19], v[136:139]\n"
    "s_sub_u32 s59, s58, 98304\n"
    "v_mfma_f32_16x16x32_bf16 v[136:139], a[132:135], a[20:23], v[136:139]\n"
    "ds_read_b64_tr_b16 a[148:149], v242 offset:8960\n"
    "v_mfma_f32_16x16x32_bf16 v[140:143], a[128:131], a[24:27], v[140:143]\n"
    "s_cmp_ge_u32 s58, s67\n"
    "v_mfma_f32_16x16x32_bf16 v[140:143], a[132:135], a[28:31], v[140:143]\n"
    "ds_read_b64_tr_b16 a[150:151], v242 offset:9984\n"
    "v_mfma_f32_16x16x32_bf16 v[144:147], a[128:131], a[32:35], v[144:147]\n"
    "s_cselect_b32 s58, s59, s58\n"
    "v_mfma_f32_16x16x32_bf16 v[144:147], a[132:135], a[36:39], v[144:147]\n"
    "ds_read_b128 a[96:99], v240 offset:40960\n"
    "v_mfma_f32_16x16x32_bf16 v[148:151], a[128:131], a[40:43], v[148:151]\n"
    "v_mfma_f32_16x16x32_bf16 v[148:151], a[132:135], a[44:47], v[148:151]\n"
    "ds_read_b128 a[100:103], v241 offset:40960\n"
    "v_mfma_f32_16x16x32_bf16 v[152:155], a[128:131], a[48:51], v[152:155]\n"
    "v_mfma_f32_16x16x32_bf16 v[152:155], a[132:135], a[52:55], v[152:155]\n"
    "ds_read_b128 a[104:107], v240 offset:43008\n"
    "v_mfma_f32_16x16x32_bf16 v[156:159], a[128:131], a[56:59], v[156:159]\n"
    "v_mfma_f32_16x16x32_bf16 v[156:159], a[132:135], a[60:63], v[156:159]\n"
    "ds_read_b128 a[108:111], v241 offset:43008\n"
    "s_waitcnt lgkmcnt(10)\n"
    "v_mfma_f32_16x16x32_bf16 v[160:163], a[136:139], a[0:3], v[160:163]\n"
    "v_mfma_f32_16x16x32_bf16 v[160:163], a[140:143], a[4:7], v[160:163]\n"
    "ds_read_b64_tr_b16 a[152:153], v243 offset:768\n"
    "ds_read_b128 a[112:115], v240 offset:45056\n"
    "v_mfma_f32_16x16x32_bf16 v[164:167], a[136:139], a[8:11], v[164:167]\n"
    "s_add_u32 s59, s52, 3\n"
    "v_mfma_f32_16x16x32_bf16 v[164:167], a[140:143], a[12:15], v[164:167]\n"
    "ds_read_b64_tr_b16 a[154:155], v243 offset:1792\n"
    "ds_read_b128 a[116:119], v241 offset:45056\n"
    "v_readlane_b32 s53, v244, s59\n"
    "v_mfma_f32_16x16x32_bf16 v[168:171], a[136:139], a[16:19], v[168:171]\n"
    "v_mfma_f32_16x16x32_bf16 v[168:171], a[140:143], a[20:23], v[168:171]\n"
    "ds_read_b64_tr_b16 a[156:157], v243 offset:8960\n"
    "ds_read_b128 a[120:123], v240 offset:47104\n"
    "v_mfma_f32_16x16x32_bf16 v[172:175], a[136:139], a[24:27], v[172:175]\n"
    "v_mfma_f32_16x16x32_bf16 v[172:175], a[140:143], a[28:31], v[172:175]\n"
    "ds_read_b64_tr_b16 a[158:159], v243 offset:9984\n"
    "ds_read_b128 a[124:127], v241 offset:47104\n"
    "v_mfma_f32_16x16x32_bf16 v[176:179], a[136:139], a[32:35], v[176:179]\n"
    "s_add_u32 s57, s57, 1\n"
    "v_mfma_f32_16x16x32_bf16 v[176:179], a[140:143], a[36:39], v[176:179]\n"
    "s_cmp_eq_u32 s57, 3\n"
    "v_mfma_f32_16x16x32_bf16 v[180:183], a[136:139], a[40:43], v[180:183]\n"
    "s_cselect_b32 s57, 0, s57\n"
    "s_cselect_b32 s59, 4294901760, s68\n"
    "v_mfma_f32_16x16x32_bf16 v[180:183], a[140:143], a[44:47], v[180:183]\n"
    "v_mfma_f32_16x16x32_bf16 v[184:187], a[136:139], a[48:51], v[184:187]\n"
    "v_add_u32 v242, s59, v242\n"
    "v_mfma_f32_16x16x32_bf16 v[184:187], a[140:143], a[52:55], v[184:187]\n"
    "v_add_u32 v243, s59, v243\n"
    "v_mfma_f32_16x16x32_bf16 v[188:191], a[136:139], a[56:59], v[188:191]\n"
    "v_mfma_f32_16x16x32_bf16 v[188:191], a[140:143], a[60:63], v[188:191]\n"
    "s_waitcnt vmcnt(8) lgkmcnt(0)\n"
    "s_barrier\n"
    "v_mfma_f32_16x16x32_bf16 v[192:195], a[144:147], a[0:3], v[192:195]\n"
    "v_mfma_f32_16x16x32_bf16 v[192:195], a[148:151], a[4:7], v[192:195]\n"
    "ds_read_b64_tr_b16 a[128:129], v242 offset:0\n"
    "v_mfma_f32_16x16x32_bf16 v[196:199], a[144:147], a[8:11], v[196:199]\n"
    "v_mfma_f32_16x16x32_bf16 v[196:199], a[148:151], a[12:15], v[196:199]\n"
    "ds_read_b64_tr_b16 a[130:131], v242 offset:1024\n"
    "v_mfma_f32_16x16x32_bf16 v[200:203], a[144:147], a[16:19], v[200:203]\n"
    "v_mfma_f32_16x16x32_bf16 v[200:203], a[148:151], a[20:23], v[200:203]\n"
    "ds_read_b64_tr_b16 a[132:133], v242 offset:8192\n"
    "v_mfma_f32_16x16x32_bf16 v[204:207], a[144:147], a[24:27], v[204:207]\n"
    "v_mfma_f32_16x16x32_bf16 v[204:207], a[148:151], a[28:31], v[204:207]\n"
    "ds_read_b64_tr_b16 a[134:135], v242 offset:9216\n"
    "v_mfma_f32_16x16x32_bf16 v[208:211], a[144:147], a[32:35], v[208:211]\n"
    "s_add_u32 m0, s55, 32768\n"
    "v_mfma_f32_16x16x32_bf16 v[208:211], a[148:151], a[36:39], v[208:211]\n"
    "buffer_load_dwordx4 v224, s[44:47], s53 offen sc1 lds\n"
    "v_mfma_f32_16x16x32_bf16 v[212:215], a[144:147], a[40:43], v[212:215]\n"
    "s_add_u32 m0, s55, 33792\n"
    "v_mfma_f32_16x16x32_bf16 v[212:215], a[148:151], a[44:47], v[212:215]\n"
    "buffer_load_dwordx4 v225, s[44:47], s53 offen sc1 lds\n"
    "v_mfma_f32_16x16x32_bf16 v[216:219], a[144:147], a[48:51], v[216:219]\n"
    "s_add_u32 m0, s55, 34816\n"
    "v_mfma_f32_16x16x32_bf16 v[216:219], a[148:151], a[52:55], v[216:219]\n"
    "buffer_load_dwordx4 v226, s[44:47], s53 offen sc1 lds\n"
    "v_mfma_f32_16x16x32_bf16 v[220:223], a[144:147], a[56:59], v[220:223]\n"
    "s_add_u32 m0, s55, 35840\n"
    "v_mfma_f32_16x16x32_bf16 v[220:223], a[148:151], a[60:63], v[220:223]\n"
    "buffer_load_dwordx4 v227, s[44:47], s53 offen sc1 lds\n"
    "s_waitcnt lgkmcnt(5)\n"
    "v_mfma_f32_16x16x32_bf16 a[160:163], a[152:155], a[0:3], a[160:163]\n"
    "v_mfma_f32_16x16x32_bf16 a[160:163], a[156:159], a[4:7], a[160:163]\n"
    "ds_read_b64_tr_b16 a[136:137], v243 offset:0\n"
    "v_mfma_f32_16x16x32_bf16 a[164:167], a[152:155], a[8:11], a[164:167]\n"
    "v_mfma_f32_16x16x32_bf16 a[164:167], a[156:159], a[12:15], a[164:167]\n"
    "ds_read_b64_tr_b16 a[138:139], v243 offset:1024\n"
    "v_mfma_f32_16x16x32_bf16 a[168:171], a[152:155], a[16:19], a[168:171]\n"
    "v_mfma_f32_16x16x32_bf16 a[168:171], a[156:159], a[20:23], a[168:171]\n"
    "ds_read_b64_tr_b16 a[140:141], v243 offset:8192\n"
    "v_mfma_f32_16x16x32_bf16 a[172:175], a[152:155], a[24:27], a[172:175]\n"
    "v_mfma_f32_16x16x32_bf16 a[172:175], a[156:159], a[28:31], a[172:175]\n"
    "ds_read_b64_tr_b16 a[142:143], v243 offset:9216\n"
    "v_mfma_f32_16x16x32_bf16 a[176:179], a[152:155], a[32:35], a[176:179]\n"
    "s_add_u32 m0, s55, 36864\n"
    "v_mfma_f32_16x16x32_bf16 a[176:179], a[156:159], a[36:39], a[176:179]\n"
    "buffer_load_dwordx4 v228, s[44:47], s53 offen sc1 lds\n"
    "v_mfma_f32_16x16x32_bf16 a[180:183], a[152:155], a[40:43], a[180:183]\n"
    "s_add_u32 m0, s55, 37888\n"
    "v_mfma_f32_16x16x32_bf16 a[180:183], a[156:159], a[44:47], a[180:183]\n"
    "buffer_load_dwordx4 v229, s[44:47], s53 offen sc1 lds\n"
    "v_mfma_f32_16x16x32_bf16 a[184:187], a[152:155], a[48:51], a[184:187]\n"
    "s_add_u32 m0, s55, 38912\n"
    "v_mfma_f32_16x16x32_bf16 a[184:187], a[156:159], a[52:55], a[184:187]\n"
    "buffer_load_dwordx4 v230, s[44:47], s53 offen sc1 lds\n"
    "v_mfma_f32_16x16x32_bf16 a[188:191], a[152:155], a[56:59], a[188:191]\n"
    "s_add_u32 m0, s55, 39936\n"
    "v_mfma_f32_16x16x32_bf16 a[188:191], a[156:159], a[60:63], a[188:191]\n"
    "buffer_load_dwordx4 v231, s[44:47], s53 offen sc1 lds\n"
    "s_add_u32 s52, s52, 1\n"
    "s_waitcnt lgkmcnt(4)\n"
    "v_mfma_f32_16x16x32_bf16 v[0:3], a[128:131], a[64:67], v[0:3]\n"
    "s_add_u32 s59, s52, 2\n"
    "v_mfma_f32_16x16x32_bf16 v[0:3], a[132:135], a[68:71], v[0:3]\n"
    "ds_read_b64_tr_b16 a[144:145], v242 offset:256\n"
    "v_readlane_b32 s54, v245, s59\n"
    "v_mfma_f32_16x16x32_bf16 v[4:7], a[128:131], a[72:75], v[4:7]\n"
    "v_mfma_f32_16x16x32_bf16 v[4:7], a[132:135], a[76:79], v[4:7]\n"
    "ds_read_b64_tr_b16 a[146:147], v242 offset:1280\n"
    "v_mfma_f32_16x16x32_bf16 v[8:11], a[128:131], a[80:83], v[8:11]\n"
    "buffer_load_dword v252, v246, s[88:91], 0 offen sc1\n"
    "v_mfma_f32_16x16x32_bf16 v[8:11], a[132:135], a[84:87], v[8:11]\n"
    "ds_read_b64_tr_b16 a[148:149], v242 offset:8448\n"
    "v_mfma_f32_16x16x32_bf16 v[12:15], a[128:131], a[88:91], v[12:15]\n"
    "v_mfma_f32_16x16x32_bf16 v[12:15], a[132:135], a[92:95], v[12:15]\n"
    "ds_read_b64_tr_b16 a[150:151], v242 offset:9472\n"
    "v_mfma_f32_16x16x32_bf16 v[16:19], a[128:131], a[96:99], v[16:19]\n"
    "s_add_u32 m0, s58, 0\n"
    "v_mfma_f32_16x16x32_bf16 v[16:19], a[132:135], a[100:103], v[16:19]\n"
    "buffer_load_dwordx4 v232, s[40:43], s54 offen lds\n"
    "v_mfma_f32_16x16x32_bf16 v[20:23], a[128:131], a[104:107], v[20:23]\n"
    "v_readlane_b32 s62, v247, s52\n"
    "v_mfma_f32_16x16x32_bf16 v[20:23], a[132:135], a[108:111], v[20:23]\n"
    "ds_read_b128 a[0:3], v240 offset:0\n"
    "v_mfma_f32_16x16x32_bf16 v[24:27], a[128:131], a[112:115], v[24:27]\n"
    "s_lshl_b64 s[78:79], -1, s62\n"
    "s_add_u32 m0, s58, 1024\n"
    "v_mfma_f32_16x16x32_bf16 v[24:27], a[132:135], a[116:119], v[24:27]\n"
    "buffer_load_dwordx4 v233, s[40:43], s54 offen lds\n"
    "v_mfma_f32_16x16x32_bf16 v[28:31], a[128:131], a[120:123], v[28:31]\n"
    "v_mfma_f32_16x16x32_bf16 v[28:31], a[132:135], a[124:127], v[28:31]\n"
    "ds_read_b128 a[4:7], v241 offset:0\n"
    "s_waitcnt lgkmcnt(6)\n"
    "v_mfma_f32_16x16x32_bf16 v[32:35], a[136:139], a[64:67], v[32:35]\n"
    "v_mfma_f32_16x16x32_bf16 v[32:35], a[140:143], a[68:71], v[32:35]\n"
    "ds_read_b64_tr_b16 a[152:153], v243 offset:256\n"
    "v_mfma_f32_16x16x32_bf16 v[36:39], a[136:139], a[72:75], v[36:39]\n"
    "v_mfma_f32_16x16x32_bf16 v[36:39], a[140:143], a[76:79], v[36:39]\n"
    "ds_read_b64_tr_b16 a[154:155], v243 offset:1280\n"
    "v_mfma_f32_16x16x32_bf16 v[40:43], a[136:139], a[80:83], v[40:43]\n"
    "v_mfma_f32_16x16x32_bf16 v[40:43], a[140:143], a[84:87], v[40:43]\n"
    "ds_read_b64_tr_b16 a[156:157], v243 offset:8448\n"
    "v_mfma_f32_16x16x32_bf16 v[44:47], a[136:139], a[88:91], v[44:47]\n"
    "v_mfma_f32_16x16x32_bf16 v[44:47], a[140:143], a[92:95], v[44:47]\n"
    "ds_read_b64_tr_b16 a[158:159], v243 offset:9472\n"
    "v_mfma_f32_16x16x32_bf16 v[48:51], a[136:139], a[96:99], v[48:51]\n"
    "s_add_u32 m0, s58, 4096\n"
    "v_mfma_f32_16x16x32_bf16 v[48:51], a[140:143], a[100:103], v[48:51]\n"
    "buffer_load_dwordx4 v234, s[40:43], s54 offen lds\n"
    "v_mfma_f32_16x16x32_bf16 v[52:55], a[136:139], a[104:107], v[52:55]\n"
    "v_mfma_f32_16x16x32_bf16 v[52:55], a[140:143], a[108:111], v[52:55]\n"
    "ds_read_b128 a[8:11], v240 offset:2048\n"
    "v_mfma_f32_16x16x32_bf16 v[56:59], a[136:139], a[112:115], v[56:59]\n"
    "s_add_u32 m0, s58, 5120\n"
    "v_mfma_f32_16x16x32_bf16 v[56:59], a[140:143], a[116:119], v[56:59]\n"
    "buffer_load_dwordx4 v235, s[40:43], s54 offen lds\n"
    "v_mfma_f32_16x16x32_bf16 v[60:63], a[136:139], a[120:123], v[60:63]\n"
    "v_mfma_f32_16x16x32_bf16 v[60:63], a[140:143], a[124:127], v[60:63]\n"
    "ds_read_b128 a[12:15], v241 offset:2048\n"
    "s_waitcnt lgkmcnt(8)\n"
    "v_mfma_f32_16x16x32_bf16 v[64:67], a[144:147], a[64:67], v[64:67]\n"
    "v_mfma_f32_16x16x32_bf16 v[64:67], a[148:151], a[68:71], v[64:67]\n"
    "ds_read_b64_tr_b16 a[128:129], v242 offset:512\n"
    "v_mfma_f32_16x16x32_bf16 v[68:71], a[144:147], a[72:75], v[68:71]\n"
    "v_mfma_f32_16x16x32_bf16 v[68:71], a[148:151], a[76:79], v[68:71]\n"
    "ds_read_b64_tr_b16 a[130:131], v242 offset:1536\n"
    "v_mfma_f32_16x16x32_bf16 v[72:75], a[144:147], a[80:83], v[72:75]\n"
    "v_mfma_f32_16x16x32_bf16 v[72:75], a[148:151], a[84:87], v[72:75]\n"
    "ds_read_b64_tr_b16 a[132:133], v242 offset:8704\n"
    "v_mfma_f32_16x16x32_bf16 v[76:79], a[144:147], a[88:91], v[76:79]\n"
    "v_mfma_f32_16x16x32_bf16 v[76:79], a[148:151], a[92:95], v[76:79]\n"
    "ds_read_b64_tr_b16 a[134:135], v242 offset:9728\n"
    "v_mfma_f32_16x16x32_bf16 v[80:83], a[144:147], a[96:99], v[80:83]\n"
    "s_add_u32 m0, s58, 8192\n"
    "v_mfma_f32_16x16x32_bf16 v[80:83], a[148:151], a[100:103], v[80:83]\n"
    "buffer_load_dwordx4 v236, s[40:43], s54 offen lds\n"
    "v_mfma_f32_16x16x32_bf16 v[84:87], a[144:147], a[104:107], v[84:87]\n"
    "v_mfma_f32_16x16x32_bf16 v[84:87], a[148:151], a[108:111], v[84:87]\n"
    "ds_read_b128 a[16:19], v240 offset:4096\n"
    "v_mfma_f32_16x16x32_bf16 v[88:91], a[144:147], a[112:115], v[88:91]\n"
    "s_add_u32 m0, s58, 9216\n"
    "v_mfma_f32_16x16x32_bf16 v[88:91], a[148:151], a[116:119], v[88:91]\n"
    "buffer_load_dwordx4 v237, s[40:43], s54 offen lds\n"
    "v_mfma_f32_16x16x32_bf16 v[92:95], a[144:147], a[120:123], v[92:95]\n"
    "v_mfma_f32_16x16x32_bf16 v[92:95], a[148:151], a[124:127], v[92:95]\n"
    "ds_read_b128 a[20:23], v241 offset:4096\n"
    "s_waitcnt lgkmcnt(8)\n"
    "v_mfma_f32_16x16x32_bf16 v[96:99], a[152:155], a[64:67], v[96:99]\n"
    "v_mfma_f32_16x16x32_bf16 v[96:99], a[156:159], a[68:71], v[96:99]\n"
    "ds_read_b64_tr_b16 a[136:137], v243 offset:512\n"
    "v_mfma_f32_16x16x32_bf16 v[100:103], a[152:155], a[72:75], v[100:103]\n"
    "v_mfma_f32_16x16x32_bf16 v[100:103], a[156:159], a[76:79], v[100:103]\n"
    "ds_read_b64_tr_b16 a[138:139], v243 offset:1536\n"
    "v_mfma_f32_16x16x32_bf16 v[104:107], a[152:155], a[80:83], v[104:107]\n"
    "v_mfma_f32_16x16x32_bf16 v[104:107], a[156:159], a[84:87], v[104:107]\n"
    "ds_read_b64_tr_b16 a[140:141], v243 offset:8704\n"
    "v_mfma_f32_16x16x32_bf16 v[108:111], a[152:155], a[88:91], v[108:111]\n"
    "v_mfma_f32_16x16x32_bf16 v[108:111], a[156:159], a[92:95], v[108:111]\n"
    "ds_read_b64_tr_b16 a[142:143], v243 offset:9728\n"
    "v_mfma_f32_16x16x32_bf16 v[112:115], a[152:155], a[96:99], v[112:115]\n"
    "s_add_u32 m0, s58, 12288\n"
    "v_mfma_f32_16x16x32_bf16 v[112:115], a[156:159], a[100:103], v[112:115]\n"
    "buffer_load_dwordx4 v238, s[40:43], s54 offen lds\n"
    "v_mfma_f32_16x16x32_bf16 v[116:119], a[152:155], a[104:107], v[116:119]\n"
    "v_mfma_f32_16x16x32_bf16 v[116:119], a[156:159], a[108:111], v[116:119]\n"
    "ds_read_b128 a[24:27], v240 offset:6144\n"
    "v_mfma_f32_16x16x32_bf16 v[120:123], a[152:155], a[112:115], v[120:123]\n"
    "s_add_u32 m0, s58, 13312\n"
    "v_mfma_f32_16x16x32_bf16 v[120:123], a[156:159], a[116:119], v[120:123]\n"
    "buffer_load_dwordx4 v239, s[40:43], s54 offen lds\n"
    "v_mfma_f32_16x16x32_bf16 v[124:127], a[152:155], a[120:123], v[124:127]\n"
    "v_mfma_f32_16x16x32_bf16 v[124:127], a[156:159], a[124:127], v[124:127]\n"
    "ds_read_b128 a[28:31], v241 offset:6144\n"
    "s_waitcnt lgkmcnt(8)\n"
    "v_mfma_f32_16x16x32_bf16 v[128:131], a[128:131], a[64:67], v[128:131]\n"
    "v_mfma_f32_16x16x32_bf16 v[128:131], a[132:135], a[68:71], v[128:131]\n"
    "ds_read_b64_tr_b16 a[144:145], v242 offset:768\n"
    "v_mfma_f32_16x16x32_bf16 v[132:135], a[128:131], a[72:75], v[132:135]\n"
    "s_add_u32 s58, s58, 32768\n"
    "v_mfma_f32_16x16x32_bf16 v[132:135], a[132:135], a[76:79], v[132:135]\n"
    "ds_read_b64_tr_b16 a[146:147], v242 offset:1792\n"
    "v_mfma_f32_16x16x32_bf16 v[136:139], a[128:131], a[80:83], v[136:139]\n"
    "s_sub_u32 s59, s58, 98304\n"
    "v_mfma_f32_16x16x32_bf16 v[136:139], a[132:135], a[84:87], v[136:139]\n"
    "ds_read_b64_tr_b16 a[148:149], v242 offset:8960\n"
    "v_mfma_f32_16x16x32_bf16 v[140:143], a[128:131], a[88:91], v[140:143]\n"
    "s_cmp_ge_u32 s58, s67\n"
    "v_mfma_f32_16x16x32_bf16 v[140:143], a[132:135], a[92:95], v[140:143]\n"
    "ds_read_b64_tr_b16 a[150:151], v242 offset:9984\n"
    "v_mfma_f32_16x16x32_bf16 v[144:147], a[128:131], a[96:99], v[144:147]\n"
    "s_cselect_b32 s58, s59, s58\n"
    "v_mfma_f32_16x16x32_bf16 v[144:147], a[132:135], a[100:103], v[144:147]\n"
    "ds_read_b128 a[32:35], v240 offset:8192\n"
    "v_mfma_f32_16x16x32_bf16 v[148:151], a[128:131], a[104:107], v[148:151]\n"
    "v_mfma_f32_16x16x32_bf16 v[148:151], a[132:135], a[108:111], v[148:151]\n"
    "ds_read_b128 a[36:39], v241 offset:8192\n"
    "v_mfma_f32_16x16x32_bf16 v[152:155], a[128:131], a[112:115], v[152:155]\n"
    "v_mfma_f32_16x16x32_bf16 v[152:155], a[132:135], a[116:119], v[152:155]\n"
    "ds_read_b128 a[40:43], v240 offset:10240\n"
    "v_mfma_f32_16x16x32_bf16 v[156:159], a[128:131], a[120:123], v[156:159]\n"
    "v_mfma_f32_16x16x32_bf16 v[156:159], a[132:135], a[124:127], v[156:159]\n"
    "ds_read_b128 a[44:47], v241 offset:10240\n"
    "s_waitcnt lgkmcnt(10)\n"
    "v_mfma_f32_16x16x32_bf16 v[160:163], a[136:139], a[64:67], v[160:163]\n"
    "v_mfma_f32_16x16x32_bf16 v[160:163], a[140:143], a[68:71], v[160:163]\n"
    "ds_read_b64_tr_b16 a[152:153], v243 offset:768\n"
    "ds_read_b128 a[48:51], v240 offset:12288\n"
    "v_mfma_f32_16x16x32_bf16 v[164:167], a[136:139], a[72:75], v[164:167]\n"
    "s_add_u32 s59, s52, 3\n"
    "v_mfma_f32_16x16x32_bf16 v[164:167], a[140:143], a[76:79], v[164:167]\n"
    "ds_read_b64_tr_b16 a[154:155], v243 offset:1792\n"
    "ds_read_b128 a[52:55], v241 offset:12288\n"
    "v_readlane_b32 s53, v244, s59\n"
    "v_mfma_f32_16x16x32_bf16 v[168:171], a[136:139], a[80:83], v[168:171]\n"
    "v_mfma_f32_16x16x32_bf16 v[168:171], a[140:143], a[84:87], v[168:171]\n"
    "ds_read_b64_tr_b16 a[156:157], v243 offset:8960\n"
    "ds_read_b128 a[56:59], v240 offset:14336\n"
    "v_mfma_f32_16x16x32_bf16 v[172:175], a[136:139], a[88:91], v[172:175]\n"
    "v_mfma_f32_16x16x32_bf16 v[172:175], a[140:143], a[92:95], v[172:175]\n"
    "ds_read_b64_tr_b16 a[158:159], v243 offset:9984\n"
    "ds_read_b128 a[60:63], v241 offset:14336\n"
    "v_mfma_f32_16x16x32_bf16 v[176:179], a[136:139], a[96:99], v[176:179]\n"
    "s_add_u32 s57, s57, 1\n"
    "v_mfma_f32_16x16x32_bf16 v[176:179], a[140:143], a[100:103], v[176:179]\n"
    "s_cmp_eq_u32 s57, 3\n"
    "v_mfma_f32_16x16x32_bf16 v[180:183], a[136:139], a[104:107], v[180:183]\n"
    "s_cselect_b32 s57, 0, s57\n"
    "s_cselect_b32 s59, 4294901760, s68\n"
    "v_mfma_f32_16x16x32_bf16 v[180:183], a[140:143], a[108:111], v[180:183]\n"
    "v_mfma_f32_16x16x32_bf16 v[184:187], a[136:139], a[112:115], v[184:187]\n"
    "v_add_u32 v242, s59, v242\n"
    "v_mfma_f32_16x16x32_bf16 v[184:187], a[140:143], a[116:119], v[184:187]\n"
    "v_add_u32 v243, s59, v243\n"
    "v_mfma_f32_16x16x32_bf16 v[188:191], a[136:139], a[120:123], v[188:191]\n"
    "v_mfma_f32_16x16x32_bf16 v[188:191], a[140:143], a[124:127], v[188:191]\n"
    "s_waitcnt vmcnt(8) lgkmcnt(0)\n"
    "s_barrier\n"
    "v_mfma_f32_16x16x32_bf16 v[192:195], a[144:147], a[64:67], v[192:195]\n"
    "v_cmp_le_u32 vcc, s63, v252\n"
    "v_mfma_f32_16x16x32_bf16 v[192:195], a[148:151], a[68:71], v[192:195]\n"
    "ds_read_b64_tr_b16 a[128:129], v242 offset:0\n"
    "v_mfma_f32_16x16x32_bf16 v[196:199], a[144:147], a[72:75], v[196:199]\n"
    "s_or_b64 vcc, vcc, s[78:79]\n"
    "s_cmp_eq_u64 vcc, -1\n"
    "v_mfma_f32_16x16x32_bf16 v[196:199], a[148:151], a[76:79], v[196:199]\n"
    "s_cbranch_scc0 .Lg48_%=\n"
    ".Lg49_%=:\n"
    "ds_read_b64_tr_b16 a[130:131], v242 offset:1024\n"
    "v_mfma_f32_16x16x32_bf16 v[200:203], a[144:147], a[80:83], v[200:203]\n"
    "v_mfma_f32_16x16x32_bf16 v[200:203], a[148:151], a[84:87], v[200:203]\n"
    "ds_read_b64_tr_b16 a[132:133], v242 offset:8192\n"
    "v_mfma_f32_16x16x32_bf16 v[204:207], a[144:147], a[88:91], v[204:207]\n"
    "v_mfma_f32_16x16x32_bf16 v[204:207], a[148:151], a[92:95], v[204:207]\n"
    "ds_read_b64_tr_b16 a[134:135], v242 offset:9216\n"
    "v_mfma_f32_16x16x32_bf16 v[208:211], a[144:147], a[96:99], v[208:211]\n"
    "s_add_u32 m0, s55, 0\n"
    "v_mfma_f32_16x16x32_bf16 v[208:211], a[148:151], a[100:103], v[208:211]\n"
    "buffer_load_dwordx4 v224, s[44:47], s53 offen sc1 lds\n"
    "v_mfma_f32_16x16x32_bf16 v[212:215], a[144:147], a[104:107], v[212:215]\n"
    "s_add_u32 m0, s55, 1024\n"
    "v_mfma_f32_16x16x32_bf16 v[212:215], a[148:151], a[108:111], v[212:215]\n"
    "buffer_load_dwordx4 v225, s[44:47], s53 offen sc1 lds\n"
    "v_mfma_f32_16x16x32_bf16 v[216:219], a[144:147], a[112:115], v[216:219]\n"
    "s_add_u32 m0, s55, 2048\n"
    "v_mfma_f32_16x16x32_bf16 v[216:219], a[148:151], a[116:119], v[216:219]\n"
    "buffer_load_dwordx4 v226, s[44:47], s53 offen sc1 lds\n"
    "v_mfma_f32_16x16x32_bf16 v[220:223], a[144:147], a[120:123], v[220:223]\n"
    "s_add_u32 m0, s55, 3072\n"
    "v_mfma_f32_16x16x32_bf16 v[220:223], a[148:151], a[124:127], v[220:223]\n"
    "buffer_load_dwordx4 v227, s[44:47], s53 offen sc1 lds\n"
    "s_waitcnt lgkmcnt(5)\n"
    "v_mfma_f32_16x16x32_bf16 a[160:163], a[152:155], a[64:67], a[160:163]\n"
    "v_mfma_f32_16x16x32_bf16 a[160:163], a[156:159], a[68:71], a[160:163]\n"
    "ds_read_b64_tr_b16 a[136:137], v243 offset:0\n"
    "v_mfma_f32_16x16x32_bf16 a[164:167], a[152:155], a[72:75], a[164:167]\n"
    "v_mfma_f32_16x16x32_bf16 a[164:167], a[156:159], a[76:79], a[164:167]\n"
    "ds_read_b64_tr_b16 a[138:139], v243 offset:1024\n"
    "v_mfma_f32_16x16x32_bf16 a[168:171], a[152:155], a[80:83], a[168:171]\n"
    "v_mfma_f32_16x16x32_bf16 a[168:171], a[156:159], a[84:87], a[168:171]\n"
    "ds_read_b64_tr_b16 a[140:141], v243 offset:8192\n"
    "v_mfma_f32_16x16x32_bf16 a[172:175], a[152:155], a[88:91], a[172:175]\n"
    "v_mfma_f32_16x16x32_bf16 a[172:175], a[156:159], a[92:95], a[172:175]\n"
    "ds_read_b64_tr_b16 a[142:143], v243 offset:9216\n"
    "v_mfma_f32_16x16x32_bf16 a[176:179], a[152:155], a[96:99], a[176:179]\n"
    "s_add_u32 m0, s55, 4096\n"
    "v_mfma_f32_16x16x32_bf16 a[176:179], a[156:159], a[100:103], a[176:179]\n"
    "buffer_load_dwordx4 v228, s[44:47], s53 offen sc1 lds\n"
    "v_mfma_f32_16x16x32_bf16 a[180:183], a[152:155], a[104:107], a[180:183]\n"
    "s_add_u32 m0, s55, 5120\n"
    "v_mfma_f32_16x16x32_bf16 a[180:183], a[156:159], a[108:111], a[180:183]\n"
    "buffer_load_dwordx4 v229, s[44:47], s53 offen sc1 lds\n"
    "v_mfma_f32_16x16x32_bf16 a[184:187], a[152:155], a[112:115], a[184:187]\n"
    "s_add_u32 m0, s55, 6144\n"
    "v_mfma_f32_16x16x32_bf16 a[184:187], a[156:159], a[116:119], a[184:187]\n"
    "buffer_load_dwordx4 v230, s[44:47], s53 offen sc1 lds\n"
    "v_mfma_f32_16x16x32_bf16 a[188:191], a[152:155], a[120:123], a[188:191]\n"
    "s_add_u32 m0, s55, 7168\n"
    "v_mfma_f32_16x16x32_bf16 a[188:191], a[156:159], a[124:127], a[188:191]\n"
    "buffer_load_dwordx4 v231, s[44:47], s53 offen sc1 lds\n"
    "s_add_u32 s52, s52, 1\n"
    "s_sub_u32 s60, s60, 1\n"
    "s_cmp_lg_u32 s60, 0\n"
    "s_cbranch_scc1 .Lg47_%=\n"
    ".Lg46_%=:\n"
    "s_waitcnt lgkmcnt(4)\n"
    "v_mfma_f32_16x16x32_bf16 v[0:3], a[128:131], a[0:3], v[0:3]\n"
    "s_add_u32 s59, s52, 2\n"
    "v_mfma_f32_16x16x32_bf16 v[0:3], a[132:135], a[4:7], v[0:3]\n"
    "ds_read_b64_tr_b16 a[144:145], v242 offset:256\n"
    "v_readlane_b32 s54, v245, s59\n"
    "v_mfma_f32_16x16x32_bf16 v[4:7], a[128:131], a[8:11], v[4:7]\n"
    "v_mfma_f32_16x16x32_bf16 v[4:7], a[132:135], a[12:15], v[4:7]\n"
    "ds_read_b64_tr_b16 a[146:147], v242 offset:1280\n"
    "v_mfma_f32_16x16x32_bf16 v[8:11], a[128:131], a[16:19], v[8:11]\n"
    "v_mfma_f32_16x16x32_bf16 v[8:11], a[132:135], a[20:23], v[8:11]\n"
    "ds_read_b64_tr_b16 a[148:149], v242 offset:8448\n"
    "v_mfma_f32_16x16x32_bf16 v[12:15], a[128:131], a[24:27], v[12:15]\n"
    "v_mfma_f32_16x16x32_bf16 v[12:15], a[132:135], a[28:31], v[12:15]\n"
    "ds_read_b64_tr_b16 a[150:151], v242 offset:9472\n"
    "v_mfma_f32_16x16x32_bf16 v[16:19], a[128:131], a[32:35], v[16:19]\n"
    "s_add_u32 m0, s58, 0\n"
    "v_mfma_f32_16x16x32_bf16 v[16:19], a[132:135], a[36:39], v[16:19]\n"
    "buffer_load_dwordx4 v232, s[40:43], s54 offen lds\n"
    "v_mfma_f32_16x16x32_bf16 v[20:23], a[128:131], a[40:43], v[20:23]\n"
    "v_mfma_f32_16x16x32_bf16 v[20:23], a[132:135], a[44:47], v[20:23]\n"
    "ds_read_b128 a[64:67], v240 offset:32768\n"
    "v_mfma_f32_16x16x32_bf16 v[24:27], a[128:131], a[48:51], v[24:27]\n"
    "s_add_u32 m0, s58, 1024\n"
    "v_mfma_f32_16x16x32_bf16 v[24:27], a[132:135], a[52:55], v[24:27]\n"
    "buffer_load_dwordx4 v233, s[40:43], s54 offen lds\n"
    "v_mfma_f32_16x16x32_bf16 v[28:31], a[128:131], a[56:59], v[28:31]\n"
    "v_mfma_f32_16x16x32_bf16 v[28:31], a[132:135], a[60:63], v[28:31]\n"
    "ds_read_b128 a[68:71], v241 offset:32768\n"
    "s_waitcnt lgkmcnt(6)\n"
    "v_mfma_f32_16x16x32_bf16 v[32:35], a[136:139], a[0:3], v[32:35]\n"
    "v_mfma_f32_16x16x32_bf16 v[32:35], a[140:143], a[4:7], v[32:35]\n"
    "ds_read_b64_tr_b16 a[152:153], v243 offset:256\n"
    "v_mfma_f32_16x16x32_bf16 v[36:39], a[136:139], a[8:11], v[36:39]\n"
    "v_mfma_f32_16x16x32_bf16 v[36:39], a[140:143], a[12:15], v[36:39]\n"
    "ds_read_b64_tr_b16 a[154:155], v243 offset:1280\n"
    "v_mfma_f32_16x16x32_bf16 v[40:43], a[136:139], a[16:19], v[40:43]\n"
    "v_mfma_f32_16x16x32_bf16 v[40:43], a[140:143], a[20:23], v[40:43]\n"
    "ds_read_b64_tr_b16 a[156:157], v243 offset:8448\n"
    "v_mfma_f32_16x16x32_bf16 v[44:47], a[136:139], a[24:27], v[44:47]\n"
    "v_mfma_f32_16x16x32_bf16 v[44:47], a[140:143], a[28:31], v[44:47]\n"
    "ds_read_b64_tr_b16 a[158:159], v243 offset:9472\n"
    "v_mfma_f32_16x16x32_bf16 v[48:51], a[136:139], a[32:35], v[48:51]\n"
    "s_add_u32 m0, s58, 4096\n"
    "v_mfma_f32_16x16x32_bf16 v[48:51], a[140:143], a[36:39], v[48:51]\n"
    "buffer_load_dwordx4 v234, s[40:43], s54 offen lds\n"
    "v_mfma_f32_16x16x32_bf16 v[52:55], a[136:139], a[40:43], v[52:55]\n"
    "v_mfma_f32_16x16x32_bf16 v[52:55], a[140:143], a[44:47], v[52:55]\n"
    "ds_read_b128 a[72:75], v240 offset:34816\n"
    "v_mfma_f32_16x16x32_bf16 v[56:59], a[136:139], a[48:51], v[56:59]\n"
    "s_add_u32 m0, s58, 5120\n"
    "v_mfma_f32_16x16x32_bf16 v[56:59], a[140:143], a[52:55], v[56:59]\n"
    "buffer_load_dwordx4 v235, s[40:43], s54 offen lds\n"
    "v_mfma_f32_16x16x32_bf16 v[60:63], a[136:139], a[56:59], v[60:63]\n"
    "v_mfma_f32_16x16x32_bf16 v[60:63], a[140:143], a[60:63], v[60:63]\n"
    "ds_read_b128 a[76:79], v241 offset:34816\n"
    "s_waitcnt lgkmcnt(8)\n"
    "v_mfma_f32_16x16x32_bf16 v[64:67], a[144:147], a[0:3], v[64:67]\n"
    "v_mfma_f32_16x16x32_bf16 v[64:67], a[148:151], a[4:7], v[64:67]\n"
    "ds_read_b64_tr_b16 a[128:129], v242 offset:512\n"
    "v_mfma_f32_16x16x32_bf16 v[68:71], a[144:147], a[8:11], v[68:71]\n"
    "v_mfma_f32_16x16x32_bf16 v[68:71], a[148:151], a[12:15], v[68:71]\n"
    "ds_read_b64_tr_b16 a[130:131], v242 offset:1536\n"
    "v_mfma_f32_16x16x32_bf16 v[72:75], a[144:147], a[16:19], v[72:75]\n"
    "v_mfma_f32_16x16x32_bf16 v[72:75], a[148:151], a[20:23], v[72:75]\n"
    "ds_read_b64_tr_b16 a[132:133], v242 offset:8704\n"
    "v_mfma_f32_16x16x32_bf16 v[76:79], a[144:147], a[24:27], v[76:79]\n"
    "v_mfma_f32_16x16x32_bf16 v[76:79], a[148:151], a[28:31], v[76:79]\n"
    "ds_read_b64_tr_b16 a[134:135], v242 offset:9728\n"
    "v_mfma_f32_16x16x32_bf16 v[80:83], a[144:147], a[32:35], v[80:83]\n"
    "s_add_u32 m0, s58, 8192\n"
    "v_mfma_f32_16x16x32_bf16 v[80:83], a[148:151], a[36:39], v[80:83]\n"
    "buffer_load_dwordx4 v236, s[40:43], s54 offen lds\n"
    "v_mfma_f32_16x16x32_bf16 v[84:87], a[144:147], a[40:43], v[84:87]\n"
    "v_mfma_f32_16x16x32_bf16 v[84:87], a[148:151], a[44:47], v[84:87]\n"
    "ds_read_b128 a[80:83], v240 offset:36864\n"
    "v_mfma_f32_16x16x32_bf16 v[88:91], a[144:147], a[48:51], v[88:91]\n"
    "s_add_u32 m0, s58, 9216\n"
    "v_mfma_f32_16x16x32_bf16 v[88:91], a[148:151], a[52:55], v[88:91]\n"
    "buffer_load_dwordx4 v237, s[40:43], s54 offen lds\n"
    "v_mfma_f32_16x16x32_bf16 v[92:95], a[144:147], a[56:59], v[92:95]\n"
    "v_mfma_f32_16x16x32_bf16 v[92:95], a[148:151], a[60:63], v[92:95]\n"
    "ds_read_b128 a[84:87], v241 offset:36864\n"
    "s_waitcnt lgkmcnt(8)\n"
    "v_mfma_f32_16x16x32_bf16 v[96:99], a[152:155], a[0:3], v[96:99]\n"
    "v_mfma_f32_16x16x32_bf16 v[96:99], a[156:159], a[4:7], v[96:99]\n"
    "ds_read_b64_tr_b16 a[136:137], v243 offset:512\n"
    "v_mfma_f32_16x16x32_bf16 v[100:103], a[152:155], a[8:11], v[100:103]\n"
    "v_mfma_f32_16x16x32_bf16 v[100:103], a[156:159], a[12:15], v[100:103]\n"
    "ds_read_b64_tr_b16 a[138:139], v243 offset:1536\n"
    "v_mfma_f32_16x16x32_bf16 v[104:107], a[152:155], a[16:19], v[104:107]\n"
    "v_mfma_f32_16x16x32_bf16 v[104:107], a[156:159], a[20:23], v[104:107]\n"
    "ds_read_b64_tr_b16 a[140:141], v243 offset:8704\n"
    "v_mfma_f32_16x16x32_bf16 v[108:111], a[152:155], a[24:27], v[108:111]\n"
    "v_mfma_f32_16x16x32_bf16 v[108:111], a[156:159], a[28:31], v[108:111]\n"
    "ds_read_b64_tr_b16 a[142:143], v243 offset:9728\n"
    "v_mfma_f32_16x16x32_bf16 v[112:115], a[152:155], a[32:35], v[112:115]\n"
    "s_add_u32 m0, s58, 12288\n"
    "v_mfma_f32_16x16x32_bf16 v[112:115], a[156:159], a[36:39], v[112:115]\n"
    "buffer_load_dwordx4 v238, s[40:43], s54 offen lds\n"
    "v_mfma_f32_16x16x32_bf16 v[116:119], a[152:155], a[40:43], v[116:119]\n"
    "v_mfma_f32_16x16x32_bf16 v[116:119], a[156:159], a[44:47], v[116:119]\n"
    "ds_read_b128 a[88:91], v240 offset:38912\n"
    "v_mfma_f32_16x16x32_bf16 v[120:123], a[152:155], a[48:51], v[120:123]\n"
    "s_add_u32 m0, s58, 13312\n"
    "v_mfma_f32_16x16x32_bf16 v[120:123], a[156:159], a[52:55], v[120:123]\n"
    "buffer_load_dwordx4 v239, s[40:43], s54 offen lds\n"
    "v_mfma_f32_16x16x32_bf16 v[124:127], a[152:155], a[56:59], v[124:127]\n"
    "v_mfma_f32_16x16x32_bf16 v[124:127], a[156:159], a[60:63], v[124:127]\n"
    "ds_read_b128 a[92:95], v241 offset:38912\n"
    "s_waitcnt lgkmcnt(8)\n"
    "v_mfma_f32_16x16x32_bf16 v[128:131], a[128:131], a[0:3], v[128:131]\n"
    "v_mfma_f32_16x16x32_bf16 v[128:131], a[132:135], a[4:7], v[128:131]\n"
    "ds_read_b64_tr_b16 a[144:145], v242 offset:768\n"
    "v_mfma_f32_16x16x32_bf16 v[132:135], a[128:131], a[8:11], v[132:135]\n"
    "s_add_u32 s58, s58, 32768\n"
    "v_mfma_f32_16x16x32_bf16 v[132:135], a[132:135], a[12:15], v[132:135]\n"
    "ds_read_b64_tr_b16 a[146:147], v242 offset:1792\n"
    "v_mfma_f32_16x16x32_bf16 v[136:139], a[128:131], a[16:19], v[136:139]\n"
    "s_sub_u32 s59, s58, 98304\n"
    "v_mfma_f32_16x16x32_bf16 v[136:139], a[132:135], a[20:23], v[136:139]\n"
    "ds_read_b64_tr_b16 a[148:149], v242 offset:8960\n"
    "v_mfma_f32_16x16x32_bf16 v[140:143], a[128:131], a[24:27], v[140:143]\n"
    "v_readlane_b32 s54, v245, 63\n"
    "s_cmp_ge_u32 s58, s67\n"
    "v_mfma_f32_16x16x32_bf16 v[140:143], a[132:135], a[28:31], v[140:143]\n"
    "ds_read_b64_tr_b16 a[150:151], v242 offset:9984\n"
    "v_mfma_f32_16x16x32_bf16 v[144:147], a[128:131], a[32:35], v[144:147]\n"
    "s_cselect_b32 s58, s59, s58\n"
    "v_mfma_f32_16x16x32_bf16 v[144:147], a[132:135], a[36:39], v[144:147]\n"
    "ds_read_b128 a[96:99], v240 offset:40960\n"
    "v_mfma_f32_16x16x32_bf16 v[148:151], a[128:131], a[40:43], v[148:151]\n"
    "v_mfma_f32_16x16x32_bf16 v[148:151], a[132:135], a[44:47], v[148:151]\n"
    "ds_read_b128 a[100:103], v241 offset:40960\n"
    "v_mfma_f32_16x16x32_bf16 v[152:155], a[128:131], a[48:51], v[152:155]\n"
    "v_mfma_f32_16x16x32_bf16 v[152:155], a[132:135], a[52:55], v[152:155]\n"
    "ds_read_b128 a[104:107], v240 offset:43008\n"
    "v_mfma_f32_16x16x32_bf16 v[156:159], a[128:131], a[56:59], v[156:159]\n"
    "v_mfma_f32_16x16x32_bf16 v[156:159], a[132:135], a[60:63], v[156:159]\n"
    "ds_read_b128 a[108:111], v241 offset:43008\n"
    "s_waitcnt lgkmcnt(10)\n"
    "v_mfma_f32_16x16x32_bf16 v[160:163], a[136:139], a[0:3], v[160:163]\n"
    "v_mfma_f32_16x16x32_bf16 v[160:163], a[140:143], a[4:7], v[160:163]\n"
    "ds_read_b64_tr_b16 a[152:153], v243 offset:768\n"
    "ds_read_b128 a[112:115], v240 offset:45056\n"
    "v_mfma_f32_16x16x32_bf16 v[164:167], a[136:139], a[8:11], v[164:167]\n"
    "s_add_u32 s59, s52, 3\n"
    "v_mfma_f32_16x16x32_bf16 v[164:167], a[140:143], a[12:15], v[164:167]\n"
    "ds_read_b64_tr_b16 a[154:155], v243 offset:1792\n"
    "ds_read_b128 a[116:119], v241 offset:45056\n"
    "v_readlane_b32 s53, v244, s59\n"
    "v_mfma_f32_16x16x32_bf16 v[168:171], a[136:139], a[16:19], v[168:171]\n"
    "v_mfma_f32_16x16x32_bf16 v[168:171], a[140:143], a[20:23], v[168:171]\n"
    "ds_read_b64_tr_b16 a[156:157], v243 offset:8960\n"
    "ds_read_b128 a[120:123], v240 offset:47104\n"
    "v_mfma_f32_16x16x32_bf16 v[172:175], a[136:139], a[24:27], v[172:175]\n"
    "v_mfma_f32_16x16x32_bf16 v[172:175], a[140:143], a[28:31], v[172:175]\n"
    "ds_read_b64_tr_b16 a[158:159], v243 offset:9984\n"
    "ds_read_b128 a[124:127], v241 offset:47104\n"
    "v_mfma_f32_16x16x32_bf16 v[176:179], a[136:139], a[32:35], v[176:179]\n"
    "s_add_u32 s57, s57, 1\n"
    "v_mfma_f32_16x16x32_bf16 v[176:179], a[140:143], a[36:39], v[176:179]\n"
    "s_cmp_eq_u32 s57, 3\n"
    "v_mfma_f32_16x16x32_bf16 v[180:183], a[136:139], a[40:43], v[180:183]\n"
    "s_cselect_b32 s57, 0, s57\n"
    "s_cselect_b32 s59, 4294901760, s68\n"
    "v_mfma_f32_16x16x32_bf16 v[180:183], a[140:143], a[44:47], v[180:183]\n"
    "v_mfma_f32_16x16x32_bf16 v[184:187], a[136:139], a[48:51], v[184:187]\n"
    "v_add_u32 v242, s59, v242\n"
    "v_mfma_f32_16x16x32_bf16 v[184:187], a[140:143], a[52:55], v[184:187]\n"
    "v_add_u32 v243, s59, v243\n"
    "v_mfma_f32_16x16x32_bf16 v[188:191], a[136:139], a[56:59], v[188:191]\n"
    "v_mfma_f32_16x16x32_bf16 v[188:191], a[140:143], a[60:63], v[188:191]\n"
    "s_waitcnt vmcnt(8) lgkmcnt(0)\n"
    "s_barrier\n"
    "v_mfma_f32_16x16x32_bf16 v[192:195], a[144:147], a[0:3], v[192:195]\n"
    "v_mfma_f32_16x16x32_bf16 v[192:195], a[148:151], a[4:7], v[192:195]\n"
    "ds_read_b64_tr_b16 a[128:129], v242 offset:0\n"
    "v_mfma_f32_16x16x32_bf16 v[196:199], a[144:147], a[8:11], v[196:199]\n"
    "v_mfma_f32_16x16x32_bf16 v[196:199], a[148:151], a[12:15], v[196:199]\n"
    "ds_read_b64_tr_b16 a[130:131], v242 offset:1024\n"
    "v_mfma_f32_16x16x32_bf16 v[200:203], a[144:147], a[16:19], v[200:203]\n"
    "v_mfma_f32_16x16x32_bf16 v[200:203], a[148:151], a[20:23], v[200:203]\n"
    "ds_read_b64_tr_b16 a[132:133], v242 offset:8192\n"
    "v_mfma_f32_16x16x32_bf16 v[204:207], a[144:147], a[24:27], v[204:207]\n"
    "v_mfma_f32_16x16x32_bf16 v[204:207], a[148:151], a[28:31], v[204:207]\n"
    "ds_read_b64_tr_b16 a[134:135], v242 offset:9216\n"
    "v_mfma_f32_16x16x32_bf16 v[208:211], a[144:147], a[32:35], v[208:211]\n"
    "s_add_u32 m0, s55, 32768\n"
    "v_mfma_f32_16x16x32_bf16 v[208:211], a[148:151], a[36:39], v[208:211]\n"
    "buffer_load_dwordx4 v224, s[44:47], s53 offen sc1 lds\n"
    "v_mfma_f32_16x16x32_bf16 v[212:215], a[144:147], a[40:43], v[212:215]\n"
    "s_add_u32 m0, s55, 33792\n"
    "v_mfma_f32_16x16x32_bf16 v[212:215], a[148:151], a[44:47], v[212:215]\n"
    "buffer_load_dwordx4 v225, s[44:47], s53 offen sc1 lds\n"
    "v_mfma_f32_16x16x32_bf16 v[216:219], a[144:147], a[48:51], v[216:219]\n"
    "s_add_u32 m0, s55, 34816\n"
    "v_mfma_f32_16x16x32_bf16 v[216:219], a[148:151], a[52:55], v[216:219]\n"
    "buffer_load_dwordx4 v226, s[44:47], s53 offen sc1 lds\n"
    "v_mfma_f32_16x16x32_bf16 v[220:223], a[144:147], a[56:59], v[220:223]\n"
    "s_add_u32 m0, s55, 35840\n"
    "v_mfma_f32_16x16x32_bf16 v[220:223], a[148:151], a[60:63], v[220:223]\n"
    "buffer_load_dwordx4 v227, s[44:47], s53 offen sc1 lds\n"
    "s_waitcnt lgkmcnt(5)\n"
    "v_mfma_f32_16x16x32_bf16 a[160:163], a[152:155], a[0:3], a[160:163]\n"
    "v_mfma_f32_16x16x32_bf16 a[160:163], a[156:159], a[4:7], a[160:163]\n"
    "ds_read_b64_tr_b16 a[136:137], v243 offset:0\n"
    "v_mfma_f32_16x16x32_bf16 a[164:167], a[152:155], a[8:11], a[164:167]\n"
    "v_mfma_f32_16x16x32_bf16 a[164:167], a[156:159], a[12:15], a[164:167]\n"
    "ds_read_b64_tr_b16 a[138:139], v243 offset:1024\n"
    "v_mfma_f32_16x16x32_bf16 a[168:171], a[152:155], a[16:19], a[168:171]\n"
    "v_mfma_f32_16x16x32_bf16 a[168:171], a[156:159], a[20:23], a[168:171]\n"
    "ds_read_b64_tr_b16 a[140:141], v243 offset:8192\n"
    "v_mfma_f32_16x16x32_bf16 a[172:175], a[152:155], a[24:27], a[172:175]\n"
    "v_mfma_f32_16x16x32_bf16 a[172:175], a[156:159], a[28:31], a[172:175]\n"
    "ds_read_b64_tr_b16 a[142:143], v243 offset:9216\n"
    "v_mfma_f32_16x16x32_bf16 a[176:179], a[152:155], a[32:35], a[176:179]\n"
    "s_add_u32 m0, s55, 36864\n"
    "v_mfma_f32_16x16x32_bf16 a[176:179], a[156:159], a[36:39], a[176:179]\n"
    "buffer_load_dwordx4 v228, s[44:47], s53 offen sc1 lds\n"
    "v_mfma_f32_16x16x32_bf16 a[180:183], a[152:155], a[40:43], a[180:183]\n"
    "s_add_u32 m0, s55, 37888\n"
    "v_mfma_f32_16x16x32_bf16 a[180:183], a[156:159], a[44:47], a[180:183]\n"
    "buffer_load_dwordx4 v229, s[44:47], s53 offen sc1 lds\n"
    "v_mfma_f32_16x16x32_bf16 a[184:187], a[152:155], a[48:51], a[184:187]\n"
    "s_add_u32 m0, s55, 38912\n"
    "v_mfma_f32_16x16x32_bf16 a[184:187], a[156:159], a[52:55], a[184:187]\n"
    "buffer_load_dwordx4 v230, s[44:47], s53 offen sc1 lds\n"
    "v_mfma_f32_16x16x32_bf16 a[188:191], a[152:155], a[56:59], a[188:191]\n"
    "s_add_u32 m0, s55, 39936\n"
    "v_mfma_f32_16x16x32_bf16 a[188:191], a[156:159], a[60:63], a[188:191]\n"
    "buffer_load_dwordx4 v231, s[44:47], s53 offen sc1 lds\n"
    "s_add_u32 s52, s52, 1\n"
    "buffer_load_dword v253, off, s[88:91], 0 sc1\n"
    "s_waitcnt lgkmcnt(4)\n"
    "v_mfma_f32_16x16x32_bf16 v[0:3], a[128:131], a[64:67], v[0:3]\n"
    "s_add_u32 m0, s58, 0\n"
    "v_mfma_f32_16x16x32_bf16 v[0:3], a[132:135], a[68:71], v[0:3]\n"
    "ds_read_b64_tr_b16 a[144:145], v242 offset:256\n"
    "buffer_load_dwordx4 v232, s[40:43], s54 offen lds\n"
    "v_mfma_f32_16x16x32_bf16 v[4:7], a[128:131], a[72:75], v[4:7]\n"
    "s_add_u32 m0, s58, 1024\n"
    "v_mfma_f32_16x16x32_bf16 v[4:7], a[132:135], a[76:79], v[4:7]\n"
    "ds_read_b64_tr_b16 a[146:147], v242 offset:1280\n"
    "buffer_load_dwordx4 v233, s[40:43], s54 offen lds\n"
    "v_mfma_f32_16x16x32_bf16 v[8:11], a[128:131], a[80:83], v[8:11]\n"
    "s_add_u32 m0, s58, 4096\n"
    "v_mfma_f32_16x16x32_bf16 v[8:11], a[132:135], a[84:87], v[8:11]\n"
    "ds_read_b64_tr_b16 a[148:149], v242 offset:8448\n"
    "buffer_load_dwordx4 v234, s[40:43], s54 offen lds\n"
    "v_mfma_f32_16x16x32_bf16 v[12:15], a[128:131], a[88:91], v[12:15]\n"
    "s_add_u32 m0, s58, 5120\n"
    "v_mfma_f32_16x16x32_bf16 v[12:15], a[132:135], a[92:95], v[12:15]\n"
    "ds_read_b64_tr_b16 a[150:151], v242 offset:9472\n"
    "buffer_load_dwordx4 v235, s[40:43], s54 offen lds\n"
    "v_mfma_f32_16x16x32_bf16 v[16:19], a[128:131], a[96:99], v[16:19]\n"
    "s_add_u32 m0, s58, 8192\n"
    "v_mfma_f32_16x16x32_bf16 v[16:19], a[132:135], a[100:103], v[16:19]\n"
    "buffer_load_dwordx4 v236, s[40:43], s54 offen lds\n"
    "v_mfma_f32_16x16x32_bf16 v[20:23], a[128:131], a[104:107], v[20:23]\n"
    "s_add_u32 m0, s58, 9216\n"
    "v_mfma_f32_16x16x32_bf16 v[20:23], a[132:135], a[108:111], v[20:23]\n"
    "ds_read_b128 a[0:3], v240 offset:0\n"
    "buffer_load_dwordx4 v237, s[40:43], s54 offen lds\n"
    "v_mfma_f32_16x16x32_bf16 v[24:27], a[128:131], a[112:115], v[24:27]\n"
    "s_add_u32 m0, s58, 12288\n"
    "v_mfma_f32_16x16x32_bf16 v[24:27], a[132:135], a[116:119], v[24:27]\n"
    "buffer_load_dwordx4 v238, s[40:43], s54 offen lds\n"
    "v_mfma_f32_16x16x32_bf16 v[28:31], a[128:131], a[120:123], v[28:31]\n"
    "s_add_u32 m0, s58, 13312\n"
    "v_mfma_f32_16x16x32_bf16 v[28:31], a[132:135], a[124:127], v[28:31]\n"
    "ds_read_b128 a[4:7], v241 offset:0\n"
    "buffer_load_dwordx4 v239, s[40:43], s54 offen lds\n"
    "s_waitcnt lgkmcnt(6)\n"
    "v_mfma_f32_16x16x32_bf16 v[32:35], a[136:139], a[64:67], v[32:35]\n"
    "v_mfma_f32_16x16x32_bf16 v[32:35], a[140:143], a[68:71], v[32:35]\n"
    "ds_read_b64_tr_b16 a[152:153], v243 offset:256\n"
    "v_mfma_f32_16x16x32_bf16 v[36:39], a[136:139], a[72:75], v[36:39]\n"
    "v_mfma_f32_16x16x32_bf16 v[36:39], a[140:143], a[76:79], v[36:39]\n"
    "ds_read_b64_tr_b16 a[154:155], v243 offset:1280\n"
    "v_mfma_f32_16x16x32_bf16 v[40:43], a[136:139], a[80:83], v[40:43]\n"
    "v_mfma_f32_16x16x32_bf16 v[40:43], a[140:143], a[84:87], v[40:43]\n"
    "ds_read_b64_tr_b16 a[156:157], v243 offset:8448\n"
    "v_mfma_f32_16x16x32_bf16 v[44:47], a[136:139], a[88:91], v[44:47]\n"
    "v_mfma_f32_16x16x32_bf16 v[44:47], a[140:143], a[92:95], v[44:47]\n"
    "ds_read_b64_tr_b16 a[158:159], v243 offset:9472\n"
    "v_mfma_f32_16x16x32_bf16 v[48:51], a[136:139], a[96:99], v[48:51]\n"
    "v_mfma_f32_16x16x32_bf16 v[48:51], a[140:143], a[100:103], v[48:51]\n"
    "v_mfma_f32_16x16x32_bf16 v[52:55], a[136:139], a[104:107], v[52:55]\n"
    "v_mfma_f32_16x16x32_bf16 v[52:55], a[140:143], a[108:111], v[52:55]\n"
    "ds_read_b128 a[8:11], v240 offset:2048\n"
    "v_mfma_f32_16x16x32_bf16 v[56:59], a[136:139], a[112:115], v[56:59]\n"
    "v_mfma_f32_16x16x32_bf16 v[56:59], a[140:143], a[116:119], v[56:59]\n"
    "v_mfma_f32_16x16x32_bf16 v[60:63], a[136:139], a[120:123], v[60:63]\n"
    "v_mfma_f32_16x16x32_bf16 v[60:63], a[140:143], a[124:127], v[60:63]\n"
    "ds_read_b128 a[12:15], v241 offset:2048\n"
    "s_waitcnt lgkmcnt(8)\n"
    "v_mfma_f32_16x16x32_bf16 v[64:67], a[144:147], a[64:67], v[64:67]\n"
    "v_mfma_f32_16x16x32_bf16 v[64:67], a[148:151], a[68:71], v[64:67]\n"
    "ds_read_b64_tr_b16 a[128:129], v242 offset:512\n"
    "v_mfma_f32_16x16x32_bf16 v[68:71], a[144:147], a[72:75], v[68:71]\n"
    "v_mfma_f32_16x16x32_bf16 v[68:71], a[148:151], a[76:79], v[68:71]\n"
    "ds_read_b64_tr_b16 a[130:131], v242 offset:1536\n"
    "v_mfma_f32_16x16x32_bf16 v[72:75], a[144:147], a[80:83], v[72:75]\n"
    "v_mfma_f32_16x16x32_bf16 v[72:75], a[148:151], a[84:87], v[72:75]\n"
    "ds_read_b64_tr_b16 a[132:133], v242 offset:8704\n"
    "v_mfma_f32_16x16x32_bf16 v[76:79], a[144:147], a[88:91], v[76:79]\n"
    "v_mfma_f32_16x16x32_bf16 v[76:79], a[148:151], a[92:95], v[76:79]\n"
    "ds_read_b64_tr_b16 a[134:135], v242 offset:9728\n"
    "v_mfma_f32_16x16x32_bf16 v[80:83], a[144:147], a[96:99], v[80:83]\n"
    "s_mov_b32 s44, 0x100000\n"
    "v_mfma_f32_16x16x32_bf16 v[80:83], a[148:151], a[100:103], v[80:83]\n"
    "v_mfma_f32_16x16x32_bf16 v[84:87], a[144:147], a[104:107], v[84:87]\n"
    "s_mov_b32 s45, 0x200000\n"
    "v_mfma_f32_16x16x32_bf16 v[84:87], a[148:151], a[108:111], v[84:87]\n"
    "ds_read_b128 a[16:19], v240 offset:4096\n"
    "v_mfma_f32_16x16x32_bf16 v[88:91], a[144:147], a[112:115], v[88:91]\n"
    "s_mov_b32 s46, 0x300000\n"
    "v_mfma_f32_16x16x32_bf16 v[88:91], a[148:151], a[116:119], v[88:91]\n"
    "v_mfma_f32_16x16x32_bf16 v[92:95], a[144:147], a[120:123], v[92:95]\n"
    "s_mov_b32 s47, 0x400000\n"
    "v_mfma_f32_16x16x32_bf16 v[92:95], a[148:151], a[124:127], v[92:95]\n"
    "ds_read_b128 a[20:23], v241 offset:4096\n"
    "s_waitcnt lgkmcnt(8)\n"
    "v_mfma_f32_16x16x32_bf16 v[96:99], a[152:155], a[64:67], v[96:99]\n"
    "s_mov_b32 s48, 0x500000\n"
    "v_mfma_f32_16x16x32_bf16 v[96:99], a[156:159], a[68:71], v[96:99]\n"
    "ds_read_b64_tr_b16 a[136:137], v243 offset:512\n"
    "v_mfma_f32_16x16x32_bf16 v[100:103], a[152:155], a[72:75], v[100:103]\n"
    "s_mov_b32 s49, 0x600000\n"
    "v_mfma_f32_16x16x32_bf16 v[100:103], a[156:159], a[76:79], v[100:103]\n"
    "ds_read_b64_tr_b16 a[138:139], v243 offset:1536\n"
    "v_mfma_f32_16x16x32_bf16 v[104:107], a[152:155], a[80:83], v[104:107]\n"
    "s_mov_b32 s50, 0x700000\n"
    "v_mfma_f32_16x16x32_bf16 v[104:107], a[156:159], a[84:87], v[104:107]\n"
    "ds_read_b64_tr_b16 a[140:141], v243 offset:8704\n"
    "v_mfma_f32_16x16x32_bf16 v[108:111], a[152:155], a[88:91], v[108:111]\n"
    "v_add_u32 v246, 524288, v248\n"
    "v_mfma_f32_16x16x32_bf16 v[108:111], a[156:159], a[92:95], v[108:111]\n"
    "ds_read_b64_tr_b16 a[142:143], v243 offset:9728\n"
    "v_mfma_f32_16x16x32_bf16 v[112:115], a[152:155], a[96:99], v[112:115]\n"
    "v_mfma_f32_16x16x32_bf16 v[112:115], a[156:159], a[100:103], v[112:115]\n"
    "v_mfma_f32_16x16x32_bf16 v[116:119], a[152:155], a[104:107], v[116:119]\n"
    "v_mfma_f32_16x16x32_bf16 v[116:119], a[156:159], a[108:111], v[116:119]\n"
    "ds_read_b128 a[24:27], v240 offset:6144\n"
    "v_mfma_f32_16x16x32_bf16 v[120:123], a[152:155], a[112:115], v[120:123]\n"
    "v_mfma_f32_16x16x32_bf16 v[120:123], a[156:159], a[116:119], v[120:123]\n"
    "v_mfma_f32_16x16x32_bf16 v[124:127], a[152:155], a[120:123], v[124:127]\n"
    "v_mfma_f32_16x16x32_bf16 v[124:127], a[156:159], a[124:127], v[124:127]\n"
    "ds_read_b128 a[28:31], v241 offset:6144\n"
    "s_waitcnt lgkmcnt(8)\n"
    "v_mfma_f32_16x16x32_bf16 v[128:131], a[128:131], a[64:67], v[128:131]\n"
    "v_mfma_f32_16x16x32_bf16 v[128:131], a[132:135], a[68:71], v[128:131]\n"
    "ds_read_b64_tr_b16 a[144:145], v242 offset:768\n"
    "v_mfma_f32_16x16x32_bf16 v[132:135], a[128:131], a[72:75], v[132:135]\n"
    "s_add_u32 s58, s58, 32768\n"
    "v_mfma_f32_16x16x32_bf16 v[132:135], a[132:135], a[76:79], v[132:135]\n"
    "ds_read_b64_tr_b16 a[146:147], v242 offset:1792\n"
    "v_mfma_f32_16x16x32_bf16 v[136:139], a[128:131], a[80:83], v[136:139]\n"
    "s_sub_u32 s59, s58, 98304\n"
    "v_mfma_f32_16x16x32_bf16 v[136:139], a[132:135], a[84:87], v[136:139]\n"
    "ds_read_b64_tr_b16 a[148:149], v242 offset:8960\n"
    "v_mfma_f32_16x16x32_bf16 v[140:143], a[128:131], a[88:91], v[140:143]\n"
    "s_cmp_ge_u32 s58, s67\n"
    "v_mfma_f32_16x16x32_bf16 v[140:143], a[132:135], a[92:95], v[140:143]\n"
    "ds_read_b64_tr_b16 a[150:151], v242 offset:9984\n"
    "v_mfma_f32_16x16x32_bf16 v[144:147], a[128:131], a[96:99], v[144:147]\n"
    "s_cselect_b32 s58, s59, s58\n"
    "v_mfma_f32_16x16x32_bf16 v[144:147], a[132:135], a[100:103], v[144:147]\n"
    "ds_read_b128 a[32:35], v240 offset:8192\n"
    "v_mfma_f32_16x16x32_bf16 v[148:151], a[128:131], a[104:107], v[148:151]\n"
    "v_mfma_f32_16x16x32_bf16 v[148:151], a[132:135], a[108:111], v[148:151]\n"
    "ds_read_b128 a[36:39], v241 offset:8192\n"
    "v_mfma_f32_16x16x32_bf16 v[152:155], a[128:131], a[112:115], v[152:155]\n"
    "v_mfma_f32_16x16x32_bf16 v[152:155], a[132:135], a[116:119], v[152:155]\n"
    "ds_read_b128 a[40:43], v240 offset:10240\n"
    "v_mfma_f32_16x16x32_bf16 v[156:159], a[128:131], a[120:123], v[156:159]\n"
    "v_mfma_f32_16x16x32_bf16 v[156:159], a[132:135], a[124:127], v[156:159]\n"
    "ds_read_b128 a[44:47], v241 offset:10240\n"
    "s_waitcnt lgkmcnt(10)\n"
    "v_mfma_f32_16x16x32_bf16 v[160:163], a[136:139], a[64:67], v[160:163]\n"
    "v_mfma_f32_16x16x32_bf16 v[160:163], a[140:143], a[68:71], v[160:163]\n"
    "ds_read_b64_tr_b16 a[152:153], v243 offset:768\n"
    "ds_read_b128 a[48:51], v240 offset:12288\n"
    "v_mfma_f32_16x16x32_bf16 v[164:167], a[136:139], a[72:75], v[164:167]\n"
    "v_mfma_f32_16x16x32_bf16 v[164:167], a[140:143], a[76:79], v[164:167]\n"
    "ds_read_b64_tr_b16 a[154:155], v243 offset:1792\n"
    "ds_read_b128 a[52:55], v241 offset:12288\n"
    "v_mfma_f32_16x16x32_bf16 v[168:171], a[136:139], a[80:83], v[168:171]\n"
    "v_mfma_f32_16x16x32_bf16 v[168:171], a[140:143], a[84:87], v[168:171]\n"
    "ds_read_b64_tr_b16 a[156:157], v243 offset:8960\n"
    "ds_read_b128 a[56:59], v240 offset:14336\n"
    "v_mfma_f32_16x16x32_bf16 v[172:175], a[136:139], a[88:91], v[172:175]\n"
    "s_memrealtime s[34:35]\n"
    "v_mfma_f32_16x16x32_bf16 v[172:175], a[140:143], a[92:95], v[172:175]\n"
    "ds_read_b64_tr_b16 a[158:159], v243 offset:9984\n"
    "ds_read_b128 a[60:63], v241 offset:14336\n"
    "v_mfma_f32_16x16x32_bf16 v[176:179], a[136:139], a[96:99], v[176:179]\n"
    "s_add_u32 s57, s57, 1\n"
    "v_mfma_f32_16x16x32_bf16 v[176:179], a[140:143], a[100:103], v[176:179]\n"
    "s_cmp_eq_u32 s57, 3\n"
    "v_mfma_f32_16x16x32_bf16 v[180:183], a[136:139], a[104:107], v[180:183]\n"
    "s_cselect_b32 s57, 0, s57\n"
    "s_cselect_b32 s59, 4294901760, s68\n"
    "v_mfma_f32_16x16x32_bf16 v[180:183], a[140:143], a[108:111], v[180:183]\n"
    "v_mfma_f32_16x16x32_bf16 v[184:187], a[136:139], a[112:115], v[184:187]\n"
    "v_add_u32 v242, s59, v242\n"
    "v_mfma_f32_16x16x32_bf16 v[184:187], a[140:143], a[116:119], v[184:187]\n"
    "v_add_u32 v243, s59, v243\n"
    "v_mfma_f32_16x16x32_bf16 v[188:191], a[136:139], a[120:123], v[188:191]\n"
    "v_add_u32 v247, 4294901760, v242\n"
    "v_mfma_f32_16x16x32_bf16 v[188:191], a[140:143], a[124:127], v[188:191]\n"
    "v_add_u32 v252, 4294901760, v243\n"
    "s_waitcnt vmcnt(8) lgkmcnt(0)\n"
    "s_barrier\n"
    "v_mfma_f32_16x16x32_bf16 v[192:195], a[144:147], a[64:67], v[192:195]\n"
    "v_mfma_f32_16x16x32_bf16 v[192:195], a[148:151], a[68:71], v[192:195]\n"
    "ds_read_b64_tr_b16 a[128:129], v242 offset:0\n"
    "v_mfma_f32_16x16x32_bf16 v[196:199], a[144:147], a[72:75], v[196:199]\n"
    "v_mov_b32 v255, s34\n"
    "v_mfma_f32_16x16x32_bf16 v[196:199], a[148:151], a[76:79], v[196:199]\n"
    "ds_read_b64_tr_b16 a[130:131], v242 offset:1024\n"
    "v_mfma_f32_16x16x32_bf16 v[200:203], a[144:147], a[80:83], v[200:203]\n"
    "v_mfma_f32_16x16x32_bf16 v[200:203], a[148:151], a[84:87], v[200:203]\n"
    "ds_read_b64_tr_b16 a[132:133], v242 offset:8192\n"
    "v_mfma_f32_16x16x32_bf16 v[204:207], a[144:147], a[88:91], v[204:207]\n"
    "v_mfma_f32_16x16x32_bf16 v[204:207], a[148:151], a[92:95], v[204:207]\n"
    "ds_read_b64_tr_b16 a[134:135], v242 offset:9216\n"
    "v_mfma_f32_16x16x32_bf16 v[208:211], a[144:147], a[96:99], v[208:211]\n"
    "v_mfma_f32_16x16x32_bf16 v[208:211], a[148:151], a[100:103], v[208:211]\n"
    "ds_read_b128 a[192:195], v240 offset:32768\n"
    "v_mfma_f32_16x16x32_bf16 v[212:215], a[144:147], a[104:107], v[212:215]\n"
    "v_mfma_f32_16x16x32_bf16 v[212:215], a[148:151], a[108:111], v[212:215]\n"
    "ds_read_b128 a[196:199], v241 offset:32768\n"
    "v_mfma_f32_16x16x32_bf16 v[216:219], a[144:147], a[112:115], v[216:219]\n"
    "v_mfma_f32_16x16x32_bf16 v[216:219], a[148:151], a[116:119], v[216:219]\n"
    "ds_read_b128 a[200:203], v240 offset:34816\n"
    "v_mfma_f32_16x16x32_bf16 v[220:223], a[144:147], a[120:123], v[220:223]\n"
    "v_mfma_f32_16x16x32_bf16 v[220:223], a[148:151], a[124:127], v[220:223]\n"
    "ds_read_b128 a[204:207], v241 offset:34816\n"
    "s_waitcnt lgkmcnt(9)\n"
    "v_mfma_f32_16x16x32_bf16 a[160:163], a[152:155], a[64:67], a[160:163]\n"
    "v_mfma_f32_16x16x32_bf16 a[160:163], a[156:159], a[68:71], a[160:163]\n"
    "ds_read_b64_tr_b16 a[136:137], v243 offset:0\n"
    "v_mfma_f32_16x16x32_bf16 a[164:167], a[152:155], a[72:75], a[164:167]\n"
    "v_mfma_f32_16x16x32_bf16 a[164:167], a[156:159], a[76:79], a[164:167]\n"
    "ds_read_b64_tr_b16 a[138:139], v243 offset:1024\n"
    "v_mfma_f32_16x16x32_bf16 a[168:171], a[152:155], a[80:83], a[168:171]\n"
    "v_mfma_f32_16x16x32_bf16 a[168:171], a[156:159], a[84:87], a[168:171]\n"
    "ds_read_b64_tr_b16 a[140:141], v243 offset:8192\n"
    "v_mfma_f32_16x16x32_bf16 a[172:175], a[152:155], a[88:91], a[172:175]\n"
    "v_mfma_f32_16x16x32_bf16 a[172:175], a[156:159], a[92:95], a[172:175]\n"
    "ds_read_b64_tr_b16 a[142:143], v243 offset:9216\n"
    "v_mfma_f32_16x16x32_bf16 a[176:179], a[152:155], a[96:99], a[176:179]\n"
    "v_mfma_f32_16x16x32_bf16 a[176:179], a[156:159], a[100:103], a[176:179]\n"
    "ds_read_b128 a[208:211], v240 offset:36864\n"
    "v_mfma_f32_16x16x32_bf16 a[180:183], a[152:155], a[104:107], a[180:183]\n"
    "v_mfma_f32_16x16x32_bf16 a[180:183], a[156:159], a[108:111], a[180:183]\n"
    "ds_read_b128 a[212:215], v241 offset:36864\n"
    "v_mfma_f32_16x16x32_bf16 a[184:187], a[152:155], a[112:115], a[184:187]\n"
    "v_mfma_f32_16x16x32_bf16 a[184:187], a[156:159], a[116:119], a[184:187]\n"
    "ds_read_b128 a[216:219], v240 offset:38912\n"
    "v_mfma_f32_16x16x32_bf16 a[188:191], a[152:155], a[120:123], a[188:191]\n"
    "v_mfma_f32_16x16x32_bf16 a[188:191], a[156:159], a[124:127], a[188:191]\n"
    "ds_read_b128 a[220:223], v241 offset:38912\n"
    "s_add_u32 s52, s52, 1\n"
    "s_waitcnt vmcnt(0)\n"
    "s_barrier\n"
    "v_readfirstlane_b32 s59, v253\n"
    "s_cmp_eq_u32 s59, s63\n"
    "s_cbranch_scc1 .Lg51_%=\n"
    "s_waitcnt lgkmcnt(12)\n"
    "v_mfma_f32_16x16x32_bf16 v[0:3], a[128:131], a[0:3], v[0:3]\n"
    "buffer_store_dwordx2 v[254:255], v251, s[88:91], s69 offen sc1\n"
    "v_mfma_f32_16x16x32_bf16 v[0:3], a[132:135], a[4:7], v[0:3]\n"
    "ds_read_b64_tr_b16 a[144:145], v247 offset:0\n"
    "v_mfma_f32_16x16x32_bf16 v[4:7], a[128:131], a[8:11], v[4:7]\n"
    "v_mfma_f32_16x16x32_bf16 v[4:7], a[132:135], a[12:15], v[4:7]\n"
    "ds_read_b64_tr_b16 a[146:147], v247 offset:1024\n"
    "v_mfma_f32_16x16x32_bf16 v[8:11], a[128:131], a[16:19], v[8:11]\n"
    "v_mfma_f32_16x16x32_bf16 v[8:11], a[132:135], a[20:23], v[8:11]\n"
    "ds_read_b64_tr_b16 a[148:149], v247 offset:8192\n"
    "v_mfma_f32_16x16x32_bf16 v[12:15], a[128:131], a[24:27], v[12:15]\n"
    "v_mfma_f32_16x16x32_bf16 v[12:15], a[132:135], a[28:31], v[12:15]\n"
    "ds_read_b64_tr_b16 a[150:151], v247 offset:9216\n"
    "v_mfma_f32_16x16x32_bf16 v[16:19], a[128:131], a[32:35], v[16:19]\n"
    "v_mfma_f32_16x16x32_bf16 v[16:19], a[132:135], a[36:39], v[16:19]\n"
    "ds_read_b128 a[224:227], v240 offset:40960\n"
    "v_mfma_f32_16x16x32_bf16 v[20:23], a[128:131], a[40:43], v[20:23]\n"
    "v_mfma_f32_16x16x32_bf16 v[20:23], a[132:135], a[44:47], v[20:23]\n"
    "ds_read_b128 a[228:231], v241 offset:40960\n"
    "v_mfma_f32_16x16x32_bf16 v[24:27], a[128:131], a[48:51], v[24:27]\n"
    "v_mfma_f32_16x16x32_bf16 v[24:27], a[132:135], a[52:55], v[24:27]\n"
    "ds_read_b128 a[232:235], v240 offset:43008\n"
    "v_mfma_f32_16x16x32_bf16 v[28:31], a[128:131], a[56:59], v[28:31]\n"
    "v_mfma_f32_16x16x32_bf16 v[28:31], a[132:135], a[60:63], v[28:31]\n"
    "ds_read_b128 a[236:239], v241 offset:43008\n"
    "s_waitcnt lgkmcnt(12)\n"
    "v_mfma_f32_16x16x32_bf16 v[32:35], a[136:139], a[0:3], v[32:35]\n"
    "v_mfma_f32_16x16x32_bf16 v[32:35], a[140:143], a[4:7], v[32:35]\n"
    "ds_read_b64_tr_b16 a[152:153], v252 offset:0\n"
    "v_mfma_f32_16x16x32_bf16 v[36:39], a[136:139], a[8:11], v[36:39]\n"
    "v_mfma_f32_16x16x32_bf16 v[36:39], a[140:143], a[12:15], v[36:39]\n"
    "ds_read_b64_tr_b16 a[154:155], v252 offset:1024\n"
    "v_mfma_f32_16x16x32_bf16 v[40:43], a[136:139], a[16:19], v[40:43]\n"
    "v_mfma_f32_16x16x32_bf16 v[40:43], a[140:143], a[20:23], v[40:43]\n"
    "ds_read_b64_tr_b16 a[156:157], v252 offset:8192\n"
    "v_mfma_f32_16x16x32_bf16 v[44:47], a[136:139], a[24:27], v[44:47]\n"
    "v_mfma_f32_16x16x32_bf16 v[44:47], a[140:143], a[28:31], v[44:47]\n"
    "ds_read_b64_tr_b16 a[158:159], v252 offset:9216\n"
    "v_mfma_f32_16x16x32_bf16 v[48:51], a[136:139], a[32:35], v[48:51]\n"
    "v_mfma_f32_16x16x32_bf16 v[48:51], a[140:143], a[36:39], v[48:51]\n"
    "ds_read_b128 a[240:243], v240 offset:45056\n"
    "v_mfma_f32_16x16x32_bf16 v[52:55], a[136:139], a[40:43], v[52:55]\n"
    "v_mfma_f32_16x16x32_bf16 v[52:55], a[140:143], a[44:47], v[52:55]\n"
    "ds_read_b128 a[244:247], v241 offset:45056\n"
    "v_mfma_f32_16x16x32_bf16 v[56:59], a[136:139], a[48:51], v[56:59]\n"
    "v_mfma_f32_16x16x32_bf16 v[56:59], a[140:143], a[52:55], v[56:59]\n"
    "ds_read_b128 a[248:251], v240 offset:47104\n"
    "v_mfma_f32_16x16x32_bf16 v[60:63], a[136:139], a[56:59], v[60:63]\n"
    "v_mfma_f32_16x16x32_bf16 v[60:63], a[140:143], a[60:63], v[60:63]\n"
    "ds_read_b128 a[252:255], v241 offset:47104\n"
    "s_waitcnt lgkmcnt(12)\n"
    "v_mfma_f32_16x16x32_bf16 v[0:3], a[144:147], a[192:195], v[0:3]\n"
    "v_mfma_f32_16x16x32_bf16 v[0:3], a[148:151], a[196:199], v[0:3]\n"
    "ds_read_b64_tr_b16 a[128:129], v242 offset:256\n"
    "v_mfma_f32_16x16x32_bf16 v[4:7], a[144:147], a[200:203], v[4:7]\n"
    "v_mfma_f32_16x16x32_bf16 v[4:7], a[148:151], a[204:207], v[4:7]\n"
    "ds_read_b64_tr_b16 a[130:131], v242 offset:1280\n"
    "v_mfma_f32_16x16x32_bf16 v[8:11], a[144:147], a[208:211], v[8:11]\n"
    "v_mfma_f32_16x16x32_bf16 v[8:11], a[148:151], a[212:215], v[8:11]\n"
    "ds_read_b64_tr_b16 a[132:133], v242 offset:8448\n"
    "v_mfma_f32_16x16x32_bf16 v[12:15], a[144:147], a[216:219], v[12:15]\n"
    "v_mfma_f32_16x16x32_bf16 v[12:15], a[148:151], a[220:223], v[12:15]\n"
    "ds_read_b64_tr_b16 a[134:135], v242 offset:9472\n"
    "s_waitcnt lgkmcnt(12)\n"
    "v_mfma_f32_16x16x32_bf16 v[16:19], a[144:147], a[224:227], v[16:19]\n"
    "v_mfma_f32_16x16x32_bf16 v[16:19], a[148:151], a[228:231], v[16:19]\n"
    "s_mov_b32 s40, 0xbfb8aa3b\n"
    "v_mfma_f32_16x16x32_bf16 v[20:23], a[144:147], a[232:235], v[20:23]\n"
    "v_mfma_f32_16x16x32_bf16 v[20:23], a[148:151], a[236:239], v[20:23]\n"
    "s_mov_b32 s41, s40\n"
    "s_waitcnt lgkmcnt(4)\n"
    "v_mfma_f32_16x16x32_bf16 v[24:27], a[144:147], a[240:243], v[24:27]\n"
    "v_mfma_f32_16x16x32_bf16 v[24:27], a[148:151], a[244:247], v[24:27]\n"
    "s_mov_b32 s42, 0x6abcede5\n"
    "v_mfma_f32_16x16x32_bf16 v[28:31], a[144:147], a[248:251], v[28:31]\n"
    "v_mfma_f32_16x16x32_bf16 v[28:31], a[148:151], a[252:255], v[28:31]\n"
    "s_mov_b32 s43, s42\n"
    "s_waitcnt lgkmcnt(8)\n"
    "v_mfma_f32_16x16x32_bf16 v[32:35], a[152:155], a[192:195], v[32:35]\n"
    "v_mfma_f32_16x16x32_bf16 v[32:35], a[156:159], a[196:199], v[32:35]\n"
    "ds_read_b64_tr_b16 a[136:137], v243 offset:256\n"
    "v_mov_b32 v240, 0xc2ad1f97\n"
    "v_mfma_f32_16x16x32_bf16 v[36:39], a[152:155], a[200:203], v[36:39]\n"
    "v_mfma_f32_16x16x32_bf16 v[36:39], a[156:159], a[204:207], v[36:39]\n"
    "ds_read_b64_tr_b16 a[138:139], v243 offset:1280\n"
    "v_mov_b32 v241, v240\n"
    "v_mfma_f32_16x16x32_bf16 v[40:43], a[152:155], a[208:211], v[40:43]\n"
    "v_mfma_f32_16x16x32_bf16 v[40:43], a[156:159], a[212:215], v[40:43]\n"
    "ds_read_b64_tr_b16 a[140:141], v243 offset:8448\n"
    "v_mov_b32 v244, 1.0\n"
    "v_mfma_f32_16x16x32_bf16 v[44:47], a[152:155], a[216:219], v[44:47]\n"
    "v_pk_fma_f32 v[224:225], v[0:1], s[40:41], v[240:241]\n"
    "v_pk_fma_f32 v[226:227], v[2:3], s[40:41], v[240:241]\n"
    "v_mfma_f32_16x16x32_bf16 v[44:47], a[156:159], a[220:223], v[44:47]\n"
    "ds_read_b64_tr_b16 a[142:143], v243 offset:9472\n"
    "v_mov_b32 v245, 1.0\n"
    "v_mfma_f32_16x16x32_bf16 v[48:51], a[152:155], a[224:227], v[48:51]\n"
    "v_mfma_f32_16x16x32_bf16 v[48:51], a[156:159], a[228:231], v[48:51]\n"
    "v_mfma_f32_16x16x32_bf16 v[52:55], a[152:155], a[232:235], v[52:55]\n"
    "v_mfma_f32_16x16x32_bf16 v[52:55], a[156:159], a[236:239], v[52:55]\n"
    "v_mfma_f32_16x16x32_bf16 v[56:59], a[152:155], a[240:243], v[56:59]\n"
    "v_mfma_f32_16x16x32_bf16 v[56:59], a[156:159], a[244:247], v[56:59]\n"
    "v_mfma_f32_16x16x32_bf16 v[60:63], a[152:155], a[248:251], v[60:63]\n"
    "v_mfma_f32_16x16x32_bf16 v[60:63], a[156:159], a[252:255], v[60:63]\n"
    "s_waitcnt lgkmcnt(4)\n"
    "v_mfma_f32_16x16x32_bf16 v[64:67], a[128:131], a[0:3], v[64:67]\n"
    "v_mfma_f32_16x16x32_bf16 v[64:67], a[132:135], a[4:7], v[64:67]\n"
    "ds_read_b64_tr_b16 a[144:145], v247 offset:256\n"
    "v_mfma_f32_16x16x32_bf16 v[68:71], a[128:131], a[8:11], v[68:71]\n"
    "v_mfma_f32_16x16x32_bf16 v[68:71], a[132:135], a[12:15], v[68:71]\n"
    "ds_read_b64_tr_b16 a[146:147], v247 offset:1280\n"
    "v_mfma_f32_16x16x32_bf16 v[72:75], a[128:131], a[16:19], v[72:75]\n"
    "v_mfma_f32_16x16x32_bf16 v[72:75], a[132:135], a[20:23], v[72:75]\n"
    "ds_read_b64_tr_b16 a[148:149], v247 offset:8448\n"
    "v_mfma_f32_16x16x32_bf16 v[76:79], a[128:131], a[24:27], v[76:79]\n"
    "v_pk_fma_f32 v[228:229], v[32:33], s[40:41], v[240:241]\n"
    "v_pk_fma_f32 v[230:231], v[34:35], s[40:41], v[240:241]\n"
    "v_mfma_f32_16x16x32_bf16 v[76:79], a[132:135], a[28:31], v[76:79]\n"
    "ds_read_b64_tr_b16 a[150:151], v247 offset:9472\n"
    "v_mfma_f32_16x16x32_bf16 v[80:83], a[128:131], a[32:35], v[80:83]\n"
    "v_exp_f32_e64 v224, v224 clamp\n"
    "v_mfma_f32_16x16x32_bf16 v[80:83], a[132:135], a[36:39], v[80:83]\n"
    "v_exp_f32_e64 v225, v225 clamp\n"
    "v_mfma_f32_16x16x32_bf16 v[84:87], a[128:131], a[40:43], v[84:87]\n"
    "v_exp_f32_e64 v226, v226 clamp\n"
    "v_mfma_f32_16x16x32_bf16 v[84:87], a[132:135], a[44:47], v[84:87]\n"
    "v_exp_f32_e64 v227, v227 clamp\n"
    "v_mfma_f32_16x16x32_bf16 v[88:91], a[128:131], a[48:51], v[88:91]\n"
    "v_exp_f32_e64 v228, v228 clamp\n"
    "v_mfma_f32_16x16x32_bf16 v[88:91], a[132:135], a[52:55], v[88:91]\n"
    "v_exp_f32_e64 v229, v229 clamp\n"
    "v_mfma_f32_16x16x32_bf16 v[92:95], a[128:131], a[56:59], v[92:95]\n"
    "v_exp_f32_e64 v230, v230 clamp\n"
    "v_mfma_f32_16x16x32_bf16 v[92:95], a[132:135], a[60:63], v[92:95]\n"
    "v_exp_f32_e64 v231, v231 clamp\n"
    "s_waitcnt lgkmcnt(4)\n"
    "v_mfma_f32_16x16x32_bf16 v[96:99], a[136:139], a[0:3], v[96:99]\n"
    "v_pk_fma_f32 v[224:225], v[224:225], s[42:43], v[244:245]\n"
    "v_pk_fma_f32 v[226:227], v[226:227], s[42:43], v[244:245]\n"
    "v_mfma_f32_16x16x32_bf16 v[96:99], a[140:143], a[4:7], v[96:99]\n"
    "ds_read_b64_tr_b16 a[152:153], v252 offset:256\n"
    "v_pk_fma_f32 v[228:229], v[228:229], s[42:43], v[244:245]\n"
    "v_mfma_f32_16x16x32_bf16 v[100:103], a[136:139], a[8:11], v[100:103]\n"
    "v_pk_fma_f32 v[230:231], v[230:231], s[42:43], v[244:245]\n"
    "v_mfma_f32_16x16x32_bf16 v[100:103], a[140:143], a[12:15], v[100:103]\n"
    "ds_read_b64_tr_b16 a[154:155], v252 offset:1280\n"
    "v_mfma_f32_16x16x32_bf16 v[104:107], a[136:139], a[16:19], v[104:107]\n"
    "v_rcp_f32 v224, v224\n"
    "v_mfma_f32_16x16x32_bf16 v[104:107], a[140:143], a[20:23], v[104:107]\n"
    "ds_read_b64_tr_b16 a[156:157], v252 offset:8448\n"
    "v_mfma_f32_16x16x32_bf16 v[108:111], a[136:139], a[24:27], v[108:111]\n"
    "v_rcp_f32 v225, v225\n"
    "v_mfma_f32_16x16x32_bf16 v[108:111], a[140:143], a[28:31], v[108:111]\n"
    "ds_read_b64_tr_b16 a[158:159], v252 offset:9472\n"
    "v_mfma_f32_16x16x32_bf16 v[112:115], a[136:139], a[32:35], v[112:115]\n"
    "v_rcp_f32 v226, v226\n"
    "v_mfma_f32_16x16x32_bf16 v[112:115], a[140:143], a[36:39], v[112:115]\n"
    "v_rcp_f32 v227, v227\n"
    "v_mfma_f32_16x16x32_bf16 v[116:119], a[136:139], a[40:43], v[116:119]\n"
    "v_rcp_f32 v228, v228\n"
    "v_mfma_f32_16x16x32_bf16 v[116:119], a[140:143], a[44:47], v[116:119]\n"
    "v_rcp_f32 v229, v229\n"
    "v_mfma_f32_16x16x32_bf16 v[120:123], a[136:139], a[48:51], v[120:123]\n"
    "v_rcp_f32 v230, v230\n"
    "v_mfma_f32_16x16x32_bf16 v[120:123], a[140:143], a[52:55], v[120:123]\n"
    "v_rcp_f32 v231, v231\n"
    "v_mfma_f32_16x16x32_bf16 v[124:127], a[136:139], a[56:59], v[124:127]\n"
    "v_pk_mul_f32 v[0:1], v[0:1], v[224:225]\n"
    "v_pk_mul_f32 v[2:3], v[2:3], v[226:227]\n"
    "v_mfma_f32_16x16x32_bf16 v[124:127], a[140:143], a[60:63], v[124:127]\n"
    "v_pk_mul_f32 v[32:33], v[32:33], v[228:229]\n"
    "v_pk_mul_f32 v[34:35], v[34:35], v[230:231]\n"
    "s_waitcnt lgkmcnt(4)\n"
    "v_mfma_f32_16x16x32_bf16 v[64:67], a[144:147], a[192:195], v[64:67]\n"
    "v_cvt_pk_bf16_f32 v0, v0, v1\n"
    "v_cvt_pk_bf16_f32 v1, v2, v3\n"
    "v_mfma_f32_16x16x32_bf16 v[64:67], a[148:151], a[196:199], v[64:67]\n"
    "ds_read_b64_tr_b16 a[128:129], v242 offset:512\n"
    "v_cvt_pk_bf16_f32 v2, v32, v33\n"
    "v_mfma_f32_16x16x32_bf16 v[68:71], a[144:147], a[200:203], v[68:71]\n"
    "v_cvt_pk_bf16_f32 v3, v34, v35\n"
    "v_pk_fma_f32 v[232:233], v[4:5], s[40:41], v[240:241]\n"
    "v_mfma_f32_16x16x32_bf16 v[68:71], a[148:151], a[204:207], v[68:71]\n"
    "ds_read_b64_tr_b16 a[130:131], v242 offset:1536\n"
    "v_pk_fma_f32 v[234:235], v[6:7], s[40:41], v[240:241]\n"
    "v_mfma_f32_16x16x32_bf16 v[72:75], a[144:147], a[208:211], v[72:75]\n"
    "v_pk_fma_f32 v[236:237], v[36:37], s[40:41], v[240:241]\n"
    "v_pk_fma_f32 v[238:239], v[38:39], s[40:41], v[240:241]\n"
    "v_mfma_f32_16x16x32_bf16 v[72:75], a[148:151], a[212:215], v[72:75]\n"
    "ds_read_b64_tr_b16 a[132:133], v242 offset:8704\n"
    "v_permlane16_swap_b32 v0, v2\n"
    "v_mfma_f32_16x16x32_bf16 v[76:79], a[144:147], a[216:219], v[76:79]\n"
    "v_permlane16_swap_b32 v1, v3\n"
    "ds_write_b128 v249, v[0:3] offset:0\n"
    "v_mfma_f32_16x16x32_bf16 v[76:79], a[148:151], a[220:223], v[76:79]\n"
    "ds_read_b64_tr_b16 a[134:135], v242 offset:9728\n"
    "v_mfma_f32_16x16x32_bf16 v[80:83], a[144:147], a[224:227], v[80:83]\n"
    "v_exp_f32_e64 v232, v232 clamp\n"
    "v_mfma_f32_16x16x32_bf16 v[80:83], a[148:151], a[228:231], v[80:83]\n"
    "v_exp_f32_e64 v233, v233 clamp\n"
    "v_mfma_f32_16x16x32_bf16 v[84:87], a[144:147], a[232:235], v[84:87]\n"
    "v_exp_f32_e64 v234, v234 clamp\n"
    "v_mfma_f32_16x16x32_bf16 v[84:87], a[148:151], a[236:239], v[84:87]\n"
    "v_exp_f32_e64 v235, v235 clamp\n"
    "v_mfma_f32_16x16x32_bf16 v[88:91], a[144:147], a[240:243], v[88:91]\n"
    "v_exp_f32_e64 v236, v236 clamp\n"
    "v_mfma_f32_16x16x32_bf16 v[88:91], a[148:151], a[244:247], v[88:91]\n"
    "v_exp_f32_e64 v237, v237 clamp\n"
    "v_mfma_f32_16x16x32_bf16 v[92:95], a[144:147], a[248:251], v[92:95]\n"
    "v_exp_f32_e64 v238, v238 clamp\n"
    "v_mfma_f32_16x16x32_bf16 v[92:95], a[148:151], a[252:255], v[92:95]\n"
    "v_exp_f32_e64 v239, v239 clamp\n"
    "s_waitcnt lgkmcnt(5)\n"
    "v_mfma_f32_16x16x32_bf16 v[96:99], a[152:155], a[192:195], v[96:99]\n"
    "v_pk_fma_f32 v[232:233], v[232:233], s[42:43], v[244:245]\n"
    "v_pk_fma_f32 v[234:235], v[234:235], s[42:43], v[244:245]\n"
    "v_mfma_f32_16x16x32_bf16 v[96:99], a[156:159], a[196:199], v[96:99]\n"
    "ds_read_b64_tr_b16 a[136:137], v243 offset:512\n"
    "v_pk_fma_f32 v[236:237], v[236:237], s[42:43], v[244:245]\n"
    "v_mfma_f32_16x16x32_bf16 v[100:103], a[152:155], a[200:203], v[100:103]\n"
    "v_pk_fma_f32 v[238:239], v[238:239], s[42:43], v[244:245]\n"
    "v_mfma_f32_16x16x32_bf16 v[100:103], a[156:159], a[204:207], v[100:103]\n"
    "ds_read_b64_tr_b16 a[138:139], v243 offset:1536\n"
    "v_mfma_f32_16x16x32_bf16 v[104:107], a[152:155], a[208:211], v[104:107]\n"
    "v_rcp_f32 v232, v232\n"
    "v_mfma_f32_16x16x32_bf16 v[104:107], a[156:159], a[212:215], v[104:107]\n"
    "ds_read_b64_tr_b16 a[140:141], v243 offset:8704\n"
    "v_mfma_f32_16x16x32_bf16 v[108:111], a[152:155], a[216:219], v[108:111]\n"
    "v_rcp_f32 v233, v233\n"
    "v_mfma_f32_16x16x32_bf16 v[108:111], a[156:159], a[220:223], v[108:111]\n"
    "ds_read_b64_tr_b16 a[142:143], v243 offset:9728\n"
    "v_mfma_f32_16x16x32_bf16 v[112:115], a[152:155], a[224:227], v[112:115]\n"
    "v_rcp_f32 v234, v234\n"
    "v_mfma_f32_16x16x32_bf16 v[112:115], a[156:159], a[228:231], v[112:115]\n"
    "v_rcp_f32 v235, v235\n"
    "v_mfma_f32_16x16x32_bf16 v[116:119], a[152:155], a[232:235], v[116:119]\n"
    "v_rcp_f32 v236, v236\n"
    "v_mfma_f32_16x16x32_bf16 v[116:119], a[156:159], a[236:239], v[116:119]\n"
    "v_rcp_f32 v237, v237\n"
    "v_mfma_f32_16x16x32_bf16 v[120:123], a[152:155], a[240:243], v[120:123]\n"
    "v_rcp_f32 v238, v238\n"
    "v_mfma_f32_16x16x32_bf16 v[120:123], a[156:159], a[244:247], v[120:123]\n"
    "v_rcp_f32 v239, v239\n"
    "v_mfma_f32_16x16x32_bf16 v[124:127], a[152:155], a[248:251], v[124:127]\n"
    "v_pk_mul_f32 v[4:5], v[4:5], v[232:233]\n"
    "v_pk_mul_f32 v[6:7], v[6:7], v[234:235]\n"
    "v_mfma_f32_16x16x32_bf16 v[124:127], a[156:159], a[252:255], v[124:127]\n"
    "v_pk_mul_f32 v[36:37], v[36:37], v[236:237]\n"
    "v_pk_mul_f32 v[38:39], v[38:39], v[238:239]\n"
    "s_waitcnt lgkmcnt(4)\n"
    "v_mfma_f32_16x16x32_bf16 v[128:131], a[128:131], a[0:3], v[128:131]\n"
    "v_cvt_pk_bf16_f32 v4, v4, v5\n"
    "v_cvt_pk_bf16_f32 v5, v6, v7\n"
    "v_mfma_f32_16x16x32_bf16 v[128:131], a[132:135], a[4:7], v[128:131]\n"
    "ds_read_b64_tr_b16 a[144:145], v247 offset:512\n"
    "v_cvt_pk_bf16_f32 v6, v36, v37\n"
    "v_mfma_f32_16x16x32_bf16 v[132:135], a[128:131], a[8:11], v[132:135]\n"
    "v_cvt_pk_bf16_f32 v7, v38, v39\n"
    "v_pk_fma_f32 v[224:225], v[64:65], s[40:41], v[240:241]\n"
    "v_mfma_f32_16x16x32_bf16 v[132:135], a[132:135], a[12:15], v[132:135]\n"
    "ds_read_b64_tr_b16 a[146:147], v247 offset:1536\n"
    "v_pk_fma_f32 v[226:227], v[66:67], s[40:41], v[240:241]\n"
    "v_mfma_f32_16x16x32_bf16 v[136:139], a[128:131], a[16:19], v[136:139]\n"
    "v_mfma_f32_16x16x32_bf16 v[136:139], a[132:135], a[20:23], v[136:139]\n"
    "ds_read_b64_tr_b16 a[148:149], v247 offset:8704\n"
    "v_mfma_f32_16x16x32_bf16 v[140:143], a[128:131], a[24:27], v[140:143]\n"
    "v_pk_fma_f32 v[228:229], v[96:97], s[40:41], v[240:241]\n"
    "v_pk_fma_f32 v[230:231], v[98:99], s[40:41], v[240:241]\n"
    "v_mfma_f32_16x16x32_bf16 v[140:143], a[132:135], a[28:31], v[140:143]\n"
    "ds_read_b64_tr_b16 a[150:151], v247 offset:9728\n"
    "v_permlane16_swap_b32 v4, v6\n"
    "v_mfma_f32_16x16x32_bf16 v[144:147], a[128:131], a[32:35], v[144:147]\n"
    "v_permlane16_swap_b32 v5, v7\n"
    "ds_write_b128 v249, v[4:7] offset:2304\n"
    "v_mfma_f32_16x16x32_bf16 v[144:147], a[132:135], a[36:39], v[144:147]\n"
    "v_exp_f32_e64 v224, v224 clamp\n"
    "v_mfma_f32_16x16x32_bf16 v[148:151], a[128:131], a[40:43], v[148:151]\n"
    "v_exp_f32_e64 v225, v225 clamp\n"
    "v_mfma_f32_16x16x32_bf16 v[148:151], a[132:135], a[44:47], v[148:151]\n"
    "v_exp_f32_e64 v226, v226 clamp\n"
    "v_mfma_f32_16x16x32_bf16 v[152:155], a[128:131], a[48:51], v[152:155]\n"
    "v_exp_f32_e64 v227, v227 clamp\n"
    "v_mfma_f32_16x16x32_bf16 v[152:155], a[132:135], a[52:55], v[152:155]\n"
    "v_exp_f32_e64 v228, v228 clamp\n"
    "v_mfma_f32_16x16x32_bf16 v[156:159], a[128:131], a[56:59], v[156:159]\n"
    "v_exp_f32_e64 v229, v229 clamp\n"
    "v_mfma_f32_16x16x32_bf16 v[156:159], a[132:135], a[60:63], v[156:159]\n"
    "v_exp_f32_e64 v230, v230 clamp\n"
    "s_waitcnt lgkmcnt(5)\n"
    "v_mfma_f32_16x16x32_bf16 v[160:163], a[136:139], a[0:3], v[160:163]\n"
    "v_exp_f32_e64 v231, v231 clamp\n"
    "v_mfma_f32_16x16x32_bf16 v[160:163], a[140:143], a[4:7], v[160:163]\n"
    "ds_read_b64_tr_b16 a[152:153], v252 offset:512\n"
    "v_pk_fma_f32 v[224:225], v[224:225], s[42:43], v[244:245]\n"
    "v_mfma_f32_16x16x32_bf16 v[164:167], a[136:139], a[8:11], v[164:167]\n"
    "v_pk_fma_f32 v[226:227], v[226:227], s[42:43], v[244:245]\n"
    "v_pk_fma_f32 v[228:229], v[228:229], s[42:43], v[244:245]\n"
    "v_mfma_f32_16x16x32_bf16 v[164:167], a[140:143], a[12:15], v[164:167]\n"
    "ds_read_b64_tr_b16 a[154:155], v252 offset:1536\n"
    "v_pk_fma_f32 v[230:231], v[230:231], s[42:43], v[244:245]\n"
    "v_mfma_f32_16x16x32_bf16 v[168:171], a[136:139], a[16:19], v[168:171]\n"
    "v_rcp_f32 v224, v224\n"
    "v_mfma_f32_16x16x32_bf16 v[168:171], a[140:143], a[20:23], v[168:171]\n"
    "ds_read_b64_tr_b16 a[156:157], v252 offset:8704\n"
    "v_mfma_f32_16x16x32_bf16 v[172:175], a[136:139], a[24:27], v[172:175]\n"
    "v_rcp_f32 v225, v225\n"
    "v_mfma_f32_16x16x32_bf16 v[172:175], a[140:143], a[28:31], v[172:175]\n"
    "ds_read_b64_tr_b16 a[158:159], v252 offset:9728\n"
    "v_mfma_f32_16x16x32_bf16 v[176:179], a[136:139], a[32:35], v[176:179]\n"
    "v_rcp_f32 v226, v226\n"
    "v_mfma_f32_16x16x32_bf16 v[176:179], a[140:143], a[36:39], v[176:179]\n"
    "v_rcp_f32 v227, v227\n"
    "v_mfma_f32_16x16x32_bf16 v[180:183], a[136:139], a[40:43], v[180:183]\n"
    "v_rcp_f32 v228, v228\n"
    "v_mfma_f32_16x16x32_bf16 v[180:183], a[140:143], a[44:47], v[180:183]\n"
    "v_rcp_f32 v229, v229\n"
    "v_mfma_f32_16x16x32_bf16 v[184:187], a[136:139], a[48:51], v[184:187]\n"
    "v_rcp_f32 v230, v230\n"
    "v_mfma_f32_16x16x32_bf16 v[184:187], a[140:143], a[52:55], v[184:187]\n"
    "v_rcp_f32 v231, v231\n"
    "v_mfma_f32_16x16x32_bf16 v[188:191], a[136:139], a[56:59], v[188:191]\n"
    "v_pk_mul_f32 v[64:65], v[64:65], v[224:225]\n"
    "v_pk_mul_f32 v[66:67], v[66:67], v[226:227]\n"
    "v_mfma_f32_16x16x32_bf16 v[188:191], a[140:143], a[60:63], v[188:191]\n"
    "v_pk_mul_f32 v[96:97], v[96:97], v[228:229]\n"
    "v_pk_mul_f32 v[98:99], v[98:99], v[230:231]\n"
    "s_waitcnt lgkmcnt(5)\n"
    "v_mfma_f32_16x16x32_bf16 v[128:131], a[144:147], a[192:195], v[128:131]\n"
    "v_cvt_pk_bf16_f32 v64, v64, v65\n"
    "v_cvt_pk_bf16_f32 v65, v66, v67\n"
    "v_mfma_f32_16x16x32_bf16 v[128:131], a[148:151], a[196:199], v[128:131]\n"
    "ds_read_b64_tr_b16 a[128:129], v242 offset:768\n"
    "v_cvt_pk_bf16_f32 v66, v96, v97\n"
    "v_mfma_f32_16x16x32_bf16 v[132:135], a[144:147], a[200:203], v[132:135]\n"
    "v_cvt_pk_bf16_f32 v67, v98, v99\n"
    "v_pk_fma_f32 v[232:233], v[68:69], s[40:41], v[240:241]\n"
    "v_mfma_f32_16x16x32_bf16 v[132:135], a[148:151], a[204:207], v[132:135]\n"
    "ds_read_b64_tr_b16 a[130:131], v242 offset:1792\n"
    "v_pk_fma_f32 v[234:235], v[70:71], s[40:41], v[240:241]\n"
    "v_mfma_f32_16x16x32_bf16 v[136:139], a[144:147], a[208:211], v[136:139]\n"
    "v_pk_fma_f32 v[236:237], v[100:101], s[40:41], v[240:241]\n"
    "v_pk_fma_f32 v[238:239], v[102:103], s[40:41], v[240:241]\n"
    "v_mfma_f32_16x16x32_bf16 v[136:139], a[148:151], a[212:215], v[136:139]\n"
    "ds_read_b64_tr_b16 a[132:133], v242 offset:8960\n"
    "v_permlane16_swap_b32 v64, v66\n"
    "v_mfma_f32_16x16x32_bf16 v[140:143], a[144:147], a[216:219], v[140:143]\n"
    "v_permlane16_swap_b32 v65, v67\n"
    "ds_write_b128 v249, v[64:67] offset:64\n"
    "v_mfma_f32_16x16x32_bf16 v[140:143], a[148:151], a[220:223], v[140:143]\n"
    "ds_read_b64_tr_b16 a[134:135], v242 offset:9984\n"
    "ds_read_b128 v[0:3], v250 offset:0\n"
    "v_mfma_f32_16x16x32_bf16 v[144:147], a[144:147], a[224:227], v[144:147]\n"
    "ds_read_b128 v[64:67], v250 offset:1152\n"
    "v_mfma_f32_16x16x32_bf16 v[144:147], a[148:151], a[228:231], v[144:147]\n"
    "v_exp_f32_e64 v232, v232 clamp\n"
    "v_mfma_f32_16x16x32_bf16 v[148:151], a[144:147], a[232:235], v[148:151]\n"
    "v_exp_f32_e64 v233, v233 clamp\n"
    "v_mfma_f32_16x16x32_bf16 v[148:151], a[148:151], a[236:239], v[148:151]\n"
    "v_exp_f32_e64 v234, v234 clamp\n"
    "v_mfma_f32_16x16x32_bf16 v[152:155], a[144:147], a[240:243], v[152:155]\n"
    "v_exp_f32_e64 v235, v235 clamp\n"
    "v_mfma_f32_16x16x32_bf16 v[152:155], a[148:151], a[244:247], v[152:155]\n"
    "v_exp_f32_e64 v236, v236 clamp\n"
    "v_mfma_f32_16x16x32_bf16 v[156:159], a[144:147], a[248:251], v[156:159]\n"
    "v_exp_f32_e64 v237, v237 clamp\n"
    "v_mfma_f32_16x16x32_bf16 v[156:159], a[148:151], a[252:255], v[156:159]\n"
    "v_exp_f32_e64 v238, v238 clamp\n"
    "s_waitcnt lgkmcnt(7)\n"
    "v_mfma_f32_16x16x32_bf16 v[160:163], a[152:155], a[192:195], v[160:163]\n"
    "v_exp_f32_e64 v239, v239 clamp\n"
    "v_mfma_f32_16x16x32_bf16 v[160:163], a[156:159], a[196:199], v[160:163]\n"
    "ds_read_b64_tr_b16 a[136:137], v243 offset:768\n"
    "v_pk_fma_f32 v[232:233], v[232:233], s[42:43], v[244:245]\n"
    "v_mfma_f32_16x16x32_bf16 v[164:167], a[152:155], a[200:203], v[164:167]\n"
    "v_pk_fma_f32 v[234:235], v[234:235], s[42:43], v[244:245]\n"
    "v_pk_fma_f32 v[236:237], v[236:237], s[42:43], v[244:245]\n"
    "v_mfma_f32_16x16x32_bf16 v[164:167], a[156:159], a[204:207], v[164:167]\n"
    "ds_read_b64_tr_b16 a[138:139], v243 offset:1792\n"
    "v_pk_fma_f32 v[238:239], v[238:239], s[42:43], v[244:245]\n"
    "v_mfma_f32_16x16x32_bf16 v[168:171], a[152:155], a[208:211], v[168:171]\n"
    "v_rcp_f32 v232, v232\n"
    "v_mfma_f32_16x16x32_bf16 v[168:171], a[156:159], a[212:215], v[168:171]\n"
    "ds_read_b64_tr_b16 a[140:141], v243 offset:8960\n"
    "v_mfma_f32_16x16x32_bf16 v[172:175], a[152:155], a[216:219], v[172:175]\n"
    "v_rcp_f32 v233, v233\n"
    "v_mfma_f32_16x16x32_bf16 v[172:175], a[156:159], a[220:223], v[172:175]\n"
    "ds_read_b64_tr_b16 a[142:143], v243 offset:9984\n"
    "v_mfma_f32_16x16x32_bf16 v[176:179], a[152:155], a[224:227], v[176:179]\n"
    "v_rcp_f32 v234, v234\n"
    "v_mfma_f32_16x16x32_bf16 v[176:179], a[156:159], a[228:231], v[176:179]\n"
    "v_rcp_f32 v235, v235\n"
    "v_mfma_f32_16x16x32_bf16 v[180:183], a[152:155], a[232:235], v[180:183]\n"
    "v_rcp_f32 v236, v236\n"
    "v_mfma_f32_16x16x32_bf16 v[180:183], a[156:159], a[236:239], v[180:183]\n"
    "v_rcp_f32 v237, v237\n"
    "v_mfma_f32_16x16x32_bf16 v[184:187], a[152:155], a[240:243], v[184:187]\n"
    "v_rcp_f32 v238, v238\n"
    "v_mfma_f32_16x16x32_bf16 v[184:187], a[156:159], a[244:247], v[184:187]\n"
    "v_rcp_f32 v239, v239\n"
    "v_mfma_f32_16x16x32_bf16 v[188:191], a[152:155], a[248:251], v[188:191]\n"
    "s_waitcnt lgkmcnt(4)\n"
    "buffer_store_dwordx4 v[0:3], v248, s[96:99], 0 offen offset:0 sc0 sc1 nt\n"
    "v_mfma_f32_16x16x32_bf16 v[188:191], a[156:159], a[252:255], v[188:191]\n"
    "buffer_store_dwordx4 v[64:67], v246, s[96:99], 0 offen offset:0 sc0 sc1 nt\n"
    "v_pk_mul_f32 v[68:69], v[68:69], v[232:233]\n"
    "s_waitcnt lgkmcnt(6)\n"
    "v_mfma_f32_16x16x32_bf16 v[192:195], a[128:131], a[0:3], v[192:195]\n"
    "v_pk_mul_f32 v[70:71], v[70:71], v[234:235]\n"
    "v_pk_mul_f32 v[100:101], v[100:101], v[236:237]\n"
    "v_mfma_f32_16x16x32_bf16 v[192:195], a[132:135], a[4:7], v[192:195]\n"
    "ds_read_b64_tr_b16 a[144:145], v247 offset:768\n"
    "v_pk_mul_f32 v[102:103], v[102:103], v[238:239]\n"
    "v_mfma_f32_16x16x32_bf16 v[196:199], a[128:131], a[8:11], v[196:199]\n"
    "v_cvt_pk_bf16_f32 v68, v68, v69\n"
    "v_cvt_pk_bf16_f32 v69, v70, v71\n"
    "v_mfma_f32_16x16x32_bf16 v[196:199], a[132:135], a[12:15], v[196:199]\n"
    "ds_read_b64_tr_b16 a[146:147], v247 offset:1792\n"
    "v_cvt_pk_bf16_f32 v70, v100, v101\n"
    "v_mfma_f32_16x16x32_bf16 v[200:203], a[128:131], a[16:19], v[200:203]\n"
    "v_cvt_pk_bf16_f32 v71, v102, v103\n"
    "v_pk_fma_f32 v[224:225], v[8:9], s[40:41], v[240:241]\n"
    "v_mfma_f32_16x16x32_bf16 v[200:203], a[132:135], a[20:23], v[200:203]\n"
    "ds_read_b64_tr_b16 a[148:149], v247 offset:8960\n"
    "v_pk_fma_f32 v[226:227], v[10:11], s[40:41], v[240:241]\n"
    "v_mfma_f32_16x16x32_bf16 v[204:207], a[128:131], a[24:27], v[204:207]\n"
    "v_pk_fma_f32 v[228:229], v[40:41], s[40:41], v[240:241]\n"
    "v_pk_fma_f32 v[230:231], v[42:43], s[40:41], v[240:241]\n"
    "v_mfma_f32_16x16x32_bf16 v[204:207], a[132:135], a[28:31], v[204:207]\n"
    "ds_read_b64_tr_b16 a[150:151], v247 offset:9984\n"
    "v_permlane16_swap_b32 v68, v70\n"
    "v_mfma_f32_16x16x32_bf16 v[208:211], a[128:131], a[32:35], v[208:211]\n"
    "v_permlane16_swap_b32 v69, v71\n"
    "ds_write_b128 v249, v[68:71] offset:2368\n"
    "v_mfma_f32_16x16x32_bf16 v[208:211], a[132:135], a[36:39], v[208:211]\n"
    "ds_read_b128 v[4:7], v250 offset:2304\n"
    "ds_read_b128 v[68:71], v250 offset:3456\n"
    "v_mfma_f32_16x16x32_bf16 v[212:215], a[128:131], a[40:43], v[212:215]\n"
    "v_exp_f32_e64 v224, v224 clamp\n"
    "v_mfma_f32_16x16x32_bf16 v[212:215], a[132:135], a[44:47], v[212:215]\n"
    "v_exp_f32_e64 v225, v225 clamp\n"
    "v_mfma_f32_16x16x32_bf16 v[216:219], a[128:131], a[48:51], v[216:219]\n"
    "v_exp_f32_e64 v226, v226 clamp\n"
    "v_mfma_f32_16x16x32_bf16 v[216:219], a[132:135], a[52:55], v[216:219]\n"
    "v_exp_f32_e64 v227, v227 clamp\n"
    "v_mfma_f32_16x16x32_bf16 v[220:223], a[128:131], a[56:59], v[220:223]\n"
    "v_exp_f32_e64 v228, v228 clamp\n"
    "v_mfma_f32_16x16x32_bf16 v[220:223], a[132:135], a[60:63], v[220:223]\n"
    "v_exp_f32_e64 v229, v229 clamp\n"
    "s_waitcnt lgkmcnt(7)\n"
    "v_mfma_f32_16x16x32_bf16 a[160:163], a[136:139], a[0:3], a[160:163]\n"
    "v_exp_f32_e64 v230, v230 clamp\n"
    "v_mfma_f32_16x16x32_bf16 a[160:163], a[140:143], a[4:7], a[160:163]\n"
    "ds_read_b64_tr_b16 a[152:153], v252 offset:768\n"
    "v_mfma_f32_16x16x32_bf16 a[164:167], a[136:139], a[8:11], a[164:167]\n"
    "v_exp_f32_e64 v231, v231 clamp\n"
    "v_mfma_f32_16x16x32_bf16 a[164:167], a[140:143], a[12:15], a[164:167]\n"
    "ds_read_b64_tr_b16 a[154:155], v252 offset:1792\n"
    "v_pk_fma_f32 v[224:225], v[224:225], s[42:43], v[244:245]\n"
    "v_mfma_f32_16x16x32_bf16 a[168:171], a[136:139], a[16:19], a[168:171]\n"
    "v_pk_fma_f32 v[226:227], v[226:227], s[42:43], v[244:245]\n"
    "v_pk_fma_f32 v[228:229], v[228:229], s[42:43], v[244:245]\n"
    "v_mfma_f32_16x16x32_bf16 a[168:171], a[140:143], a[20:23], a[168:171]\n"
    "ds_read_b64_tr_b16 a[156:157], v252 offset:8960\n"
    "v_pk_fma_f32 v[230:231], v[230:231], s[42:43], v[244:245]\n"
    "v_mfma_f32_16x16x32_bf16 a[172:175], a[136:139], a[24:27], a[172:175]\n"
    "v_rcp_f32 v224, v224\n"
    "v_mfma_f32_16x16x32_bf16 a[172:175], a[140:143], a[28:31], a[172:175]\n"
    "ds_read_b64_tr_b16 a[158:159], v252 offset:9984\n"
    "v_mfma_f32_16x16x32_bf16 a[176:179], a[136:139], a[32:35], a[176:179]\n"
    "v_rcp_f32 v225, v225\n"
    "v_mfma_f32_16x16x32_bf16 a[176:179], a[140:143], a[36:39], a[176:179]\n"
    "v_rcp_f32 v226, v226\n"
    "v_mfma_f32_16x16x32_bf16 a[180:183], a[136:139], a[40:43], a[180:183]\n"
    "v_rcp_f32 v227, v227\n"
    "v_mfma_f32_16x16x32_bf16 a[180:183], a[140:143], a[44:47], a[180:183]\n"
    "v_rcp_f32 v228, v228\n"
    "v_mfma_f32_16x16x32_bf16 a[184:187], a[136:139], a[48:51], a[184:187]\n"
    "v_rcp_f32 v229, v229\n"
    "v_mfma_f32_16x16x32_bf16 a[184:187], a[140:143], a[52:55], a[184:187]\n"
    "v_rcp_f32 v230, v230\n"
    "v_mfma_f32_16x16x32_bf16 a[188:191], a[136:139], a[56:59], a[188:191]\n"
    "v_rcp_f32 v231, v231\n"
    "v_mfma_f32_16x16x32_bf16 a[188:191], a[140:143], a[60:63], a[188:191]\n"
    "s_waitcnt lgkmcnt(4)\n"
    "buffer_store_dwordx4 v[4:7], v248, s[96:99], s44 offen offset:0 sc0 sc1 nt\n"
    "s_waitcnt lgkmcnt(7)\n"
    "v_mfma_f32_16x16x32_bf16 v[192:195], a[144:147], a[192:195], v[192:195]\n"
    "buffer_store_dwordx4 v[68:71], v246, s[96:99], s44 offen offset:0 sc0 sc1 nt\n"
    "v_pk_mul_f32 v[8:9], v[8:9], v[224:225]\n"
    "v_mfma_f32_16x16x32_bf16 v[192:195], a[148:151], a[196:199], v[192:195]\n"
    "v_pk_mul_f32 v[10:11], v[10:11], v[226:227]\n"
    "v_pk_mul_f32 v[40:41], v[40:41], v[228:229]\n"
    "v_mfma_f32_16x16x32_bf16 v[196:199], a[144:147], a[200:203], v[196:199]\n"
    "v_pk_mul_f32 v[42:43], v[42:43], v[230:231]\n"
    "v_cvt_pk_bf16_f32 v8, v8, v9\n"
    "v_mfma_f32_16x16x32_bf16 v[196:199], a[148:151], a[204:207], v[196:199]\n"
    "v_cvt_pk_bf16_f32 v9, v10, v11\n"
    "v_cvt_pk_bf16_f32 v10, v40, v41\n"
    "v_mfma_f32_16x16x32_bf16 v[200:203], a[144:147], a[208:211], v[200:203]\n"
    "v_cvt_pk_bf16_f32 v11, v42, v43\n"
    "v_pk_fma_f32 v[232:233], v[72:73], s[40:41], v[240:241]\n"
    "v_mfma_f32_16x16x32_bf16 v[200:203], a[148:151], a[212:215], v[200:203]\n"
    "v_pk_fma_f32 v[234:235], v[74:75], s[40:41], v[240:241]\n"
    "v_pk_fma_f32 v[236:237], v[104:105], s[40:41], v[240:241]\n"
    "v_mfma_f32_16x16x32_bf16 v[204:207], a[144:147], a[216:219], v[204:207]\n"
    "v_pk_fma_f32 v[238:239], v[106:107], s[40:41], v[240:241]\n"
    "v_permlane16_swap_b32 v8, v10\n"
    "v_mfma_f32_16x16x32_bf16 v[204:207], a[148:151], a[220:223], v[204:207]\n"
    "v_permlane16_swap_b32 v9, v11\n"
    "ds_write_b128 v249, v[8:11] offset:4608\n"
    "v_mfma_f32_16x16x32_bf16 v[208:211], a[144:147], a[224:227], v[208:211]\n"
    "v_exp_f32_e64 v232, v232 clamp\n"
    "v_mfma_f32_16x16x32_bf16 v[208:211], a[148:151], a[228:231], v[208:211]\n"
    "v_exp_f32_e64 v233, v233 clamp\n"
    "v_mfma_f32_16x16x32_bf16 v[212:215], a[144:147], a[232:235], v[212:215]\n"
    "v_exp_f32_e64 v234, v234 clamp\n"
    "v_mfma_f32_16x16x32_bf16 v[212:215], a[148:151], a[236:239], v[212:215]\n"
    "v_exp_f32_e64 v235, v235 clamp\n"
    "v_mfma_f32_16x16x32_bf16 v[216:219], a[144:147], a[240:243], v[216:219]\n"
    "v_exp_f32_e64 v236, v236 clamp\n"
    "v_mfma_f32_16x16x32_bf16 v[216:219], a[148:151], a[244:247], v[216:219]\n"
    "v_exp_f32_e64 v237, v237 clamp\n"
    "v_mfma_f32_16x16x32_bf16 v[220:223], a[144:147], a[248:251], v[220:223]\n"
    "v_exp_f32_e64 v238, v238 clamp\n"
    "v_mfma_f32_16x16x32_bf16 v[220:223], a[148:151], a[252:255], v[220:223]\n"
    "v_exp_f32_e64 v239, v239 clamp\n"
    "s_waitcnt lgkmcnt(1)\n"
    "v_mfma_f32_16x16x32_bf16 a[160:163], a[152:155], a[192:195], a[160:163]\n"
    "v_pk_fma_f32 v[232:233], v[232:233], s[42:43], v[244:245]\n"
    "v_pk_fma_f32 v[234:235], v[234:235], s[42:43], v[244:245]\n"
    "v_mfma_f32_16x16x32_bf16 a[160:163], a[156:159], a[196:199], a[160:163]\n"
    "v_pk_fma_f32 v[236:237], v[236:237], s[42:43], v[244:245]\n"
    "v_pk_fma_f32 v[238:239], v[238:239], s[42:43], v[244:245]\n"
    "v_mfma_f32_16x16x32_bf16 a[164:167], a[152:155], a[200:203], a[164:167]\n"
    "v_rcp_f32 v232, v232\n"
    "v_mfma_f32_16x16x32_bf16 a[164:167], a[156:159], a[204:207], a[164:167]\n"
    "v_rcp_f32 v233, v233\n"
    "v_mfma_f32_16x16x32_bf16 a[168:171], a[152:155], a[208:211], a[168:171]\n"
    "v_rcp_f32 v234, v234\n"
    "v_mfma_f32_16x16x32_bf16 a[168:171], a[156:159], a[212:215], a[168:171]\n"
    "v_rcp_f32 v235, v235\n"
    "v_mfma_f32_16x16x32_bf16 a[172:175], a[152:155], a[216:219], a[172:175]\n"
    "v_rcp_f32 v236, v236\n"
    "v_mfma_f32_16x16x32_bf16 a[172:175], a[156:159], a[220:223], a[172:175]\n"
    "v_rcp_f32 v237, v237\n"
    "v_mfma_f32_16x16x32_bf16 a[176:179], a[152:155], a[224:227], a[176:179]\n"
    "v_rcp_f32 v238, v238\n"
    "v_mfma_f32_16x16x32_bf16 a[176:179], a[156:159], a[228:231], a[176:179]\n"
    "v_rcp_f32 v239, v239\n"
    "v_mfma_f32_16x16x32_bf16 a[180:183], a[152:155], a[232:235], a[180:183]\n"
    "v_pk_mul_f32 v[72:73], v[72:73], v[232:233]\n"
    "v_pk_mul_f32 v[74:75], v[74:75], v[234:235]\n"
    "v_mfma_f32_16x16x32_bf16 a[180:183], a[156:159], a[236:239], a[180:183]\n"
    "v_pk_mul_f32 v[104:105], v[104:105], v[236:237]\n"
    "v_pk_mul_f32 v[106:107], v[106:107], v[238:239]\n"
    "v_mfma_f32_16x16x32_bf16 a[184:187], a[152:155], a[240:243], a[184:187]\n"
    "v_cvt_pk_bf16_f32 v72, v72, v73\n"
    "v_cvt_pk_bf16_f32 v73, v74, v75\n"
    "v_mfma_f32_16x16x32_bf16 a[184:187], a[156:159], a[244:247], a[184:187]\n"
    "v_cvt_pk_bf16_f32 v74, v104, v105\n"
    "v_cvt_pk_bf16_f32 v75, v106, v107\n"
    "v_mfma_f32_16x16x32_bf16 a[188:191], a[152:155], a[248:251], a[188:191]\n"
    "v_pk_fma_f32 v[224:225], v[12:13], s[40:41], v[240:241]\n"
    "v_pk_fma_f32 v[226:227], v[14:15], s[40:41], v[240:241]\n"
    "v_mfma_f32_16x16x32_bf16 a[188:191], a[156:159], a[252:255], a[188:191]\n"
    "v_pk_fma_f32 v[228:229], v[44:45], s[40:41], v[240:241]\n"
    "v_pk_fma_f32 v[230:231], v[46:47], s[40:41], v[240:241]\n"
    "s_nop 7\n"
    "v_permlane16_swap_b32 v72, v74\n"
    "v_permlane16_swap_b32 v73, v75\n"
    "ds_write_b128 v249, v[72:75] offset:4672\n"
    "ds_read_b128 v[8:11], v250 offset:4608\n"
    "ds_read_b128 v[72:75], v250 offset:5760\n"
    "v_exp_f32_e64 v224, v224 clamp\n"
    "v_exp_f32_e64 v225, v225 clamp\n"
    "v_exp_f32_e64 v226, v226 clamp\n"
    "v_exp_f32_e64 v227, v227 clamp\n"
    "v_exp_f32_e64 v228, v228 clamp\n"
    "v_exp_f32_e64 v229, v229 clamp\n"
    "v_exp_f32_e64 v230, v230 clamp\n"
    "v_exp_f32_e64 v231, v231 clamp\n"
    "v_pk_fma_f32 v[224:225], v[224:225], s[42:43], v[244:245]\n"
    "v_pk_fma_f32 v[226:227], v[226:227], s[42:43], v[244:245]\n"
    "v_pk_fma_f32 v[228:229], v[228:229], s[42:43], v[244:245]\n"
    "v_pk_fma_f32 v[230:231], v[230:231], s[42:43], v[244:245]\n"
    "v_rcp_f32 v224, v224\n"
    "v_rcp_f32 v225, v225\n"
    "v_rcp_f32 v226, v226\n"
    "v_rcp_f32 v227, v227\n"
    "v_rcp_f32 v228, v228\n"
    "v_rcp_f32 v229, v229\n"
    "v_rcp_f32 v230, v230\n"
    "v_rcp_f32 v231, v231\n"
    "s_waitcnt lgkmcnt(0)\n"
    "buffer_store_dwordx4 v[8:11], v248, s[96:99], s45 offen offset:0 sc0 sc1 nt\n"
    "buffer_store_dwordx4 v[72:75], v246, s[96:99], s45 offen offset:0 sc0 sc1 nt\n"
    "v_pk_mul_f32 v[12:13], v[12:13], v[224:225]\n"
    "v_pk_mul_f32 v[14:15], v[14:15], v[226:227]\n"
    "v_pk_mul_f32 v[44:45], v[44:45], v[228:229]\n"
    "v_pk_mul_f32 v[46:47], v[46:47], v[230:231]\n"
    "v_cvt_pk_bf16_f32 v12, v12, v13\n"
    "v_cvt_pk_bf16_f32 v13, v14, v15\n"
    "v_cvt_pk_bf16_f32 v14, v44, v45\n"
    "v_cvt_pk_bf16_f32 v15, v46, v47\n"
    "v_pk_fma_f32 v[232:233], v[76:77], s[40:41], v[240:241]\n"
    "v_pk_fma_f32 v[234:235], v[78:79], s[40:41], v[240:241]\n"
    "v_pk_fma_f32 v[236:237], v[108:109], s[40:41], v[240:241]\n"
    "v_pk_fma_f32 v[238:239], v[110:111], s[40:41], v[240:241]\n"
    "v_permlane16_swap_b32 v12, v14\n"
    "v_permlane16_swap_b32 v13, v15\n"
    "ds_write_b128 v249, v[12:15] offset:6912\n"
    "v_exp_f32_e64 v232, v232 clamp\n"
    "v_exp_f32_e64 v233, v233 clamp\n"
    "v_exp_f32_e64 v234, v234 clamp\n"
    "v_exp_f32_e64 v235, v235 clamp\n"
    "v_exp_f32_e64 v236, v236 clamp\n"
    "v_exp_f32_e64 v237, v237 clamp\n"
    "v_exp_f32_e64 v238, v238 clamp\n"
    "v_exp_f32_e64 v239, v239 clamp\n"
    "v_pk_fma_f32 v[232:233], v[232:233], s[42:43], v[244:245]\n"
    "v_pk_fma_f32 v[234:235], v[234:235], s[42:43], v[244:245]\n"
    "v_pk_fma_f32 v[236:237], v[236:237], s[42:43], v[244:245]\n"
    "v_pk_fma_f32 v[238:239], v[238:239], s[42:43], v[244:245]\n"
    "v_rcp_f32 v232, v232\n"
    "v_rcp_f32 v233, v233\n"
    "v_rcp_f32 v234, v234\n"
    "v_rcp_f32 v235, v235\n"
    "v_rcp_f32 v236, v236\n"
    "v_rcp_f32 v237, v237\n"
    "v_rcp_f32 v238, v238\n"
    "v_rcp_f32 v239, v239\n"
    "v_pk_mul_f32 v[76:77], v[76:77], v[232:233]\n"
    "v_pk_mul_f32 v[78:79], v[78:79], v[234:235]\n"
    "v_pk_mul_f32 v[108:109], v[108:109], v[236:237]\n"
    "v_pk_mul_f32 v[110:111], v[110:111], v[238:239]\n"
    "v_cvt_pk_bf16_f32 v76, v76, v77\n"
    "v_cvt_pk_bf16_f32 v77, v78, v79\n"
    "v_cvt_pk_bf16_f32 v78, v108, v109\n"
    "v_cvt_pk_bf16_f32 v79, v110, v111\n"
    "v_pk_fma_f32 v[224:225], v[16:17], s[40:41], v[240:241]\n"
    "v_pk_fma_f32 v[226:227], v[18:19], s[40:41], v[240:241]\n"
    "v_pk_fma_f32 v[228:229], v[48:49], s[40:41], v[240:241]\n"
    "v_pk_fma_f32 v[230:231], v[50:51], s[40:41], v[240:241]\n"
    "v_permlane16_swap_b32 v76, v78\n"
    "v_permlane16_swap_b32 v77, v79\n"
    "ds_write_b128 v249, v[76:79] offset:6976\n"
    "ds_read_b128 v[12:15], v250 offset:6912\n"
    "ds_read_b128 v[76:79], v250 offset:8064\n"
    "v_exp_f32_e64 v224, v224 clamp\n"
    "v_exp_f32_e64 v225, v225 clamp\n"
    "v_exp_f32_e64 v226, v226 clamp\n"
    "v_exp_f32_e64 v227, v227 clamp\n"
    "v_exp_f32_e64 v228, v228 clamp\n"
    "v_exp_f32_e64 v229, v229 clamp\n"
    "v_exp_f32_e64 v230, v230 clamp\n"
    "v_exp_f32_e64 v231, v231 clamp\n"
    "v_pk_fma_f32 v[224:225], v[224:225], s[42:43], v[244:245]\n"
    "v_pk_fma_f32 v[226:227], v[226:227], s[42:43], v[244:245]\n"
    "v_pk_fma_f32 v[228:229], v[228:229], s[42:43], v[244:245]\n"
    "v_pk_fma_f32 v[230:231], v[230:231], s[42:43], v[244:245]\n"
    "v_rcp_f32 v224, v224\n"
    "v_rcp_f32 v225, v225\n"
    "v_rcp_f32 v226, v226\n"
    "v_rcp_f32 v227, v227\n"
    "v_rcp_f32 v228, v228\n"
    "v_rcp_f32 v229, v229\n"
    "v_rcp_f32 v230, v230\n"
    "v_rcp_f32 v231, v231\n"
    "s_waitcnt lgkmcnt(0)\n"
    "buffer_store_dwordx4 v[12:15], v248, s[96:99], s46 offen offset:0 sc0 sc1 nt\n"
    "buffer_store_dwordx4 v[76:79], v246, s[96:99], s46 offen offset:0 sc0 sc1 nt\n"
    "v_pk_mul_f32 v[16:17], v[16:17], v[224:225]\n"
    "v_pk_mul_f32 v[18:19], v[18:19], v[226:227]\n"
    "v_pk_mul_f32 v[48:49], v[48:49], v[228:229]\n"
    "v_pk_mul_f32 v[50:51], v[50:51], v[230:231]\n"
    "v_cvt_pk_bf16_f32 v16, v16, v17\n"
    "v_cvt_pk_bf16_f32 v17, v18, v19\n"
    "v_cvt_pk_bf16_f32 v18, v48, v49\n"
    "v_cvt_pk_bf16_f32 v19, v50, v51\n"
    "v_pk_fma_f32 v[232:233], v[80:81], s[40:41], v[240:241]\n"
    "v_pk_fma_f32 v[234:235], v[82:83], s[40:41], v[240:241]\n"
    "v_pk_fma_f32 v[236:237], v[112:113], s[40:41], v[240:241]\n"
    "v_pk_fma_f32 v[238:239], v[114:115], s[40:41], v[240:241]\n"
    "v_permlane16_swap_b32 v16, v18\n"
    "v_permlane16_swap_b32 v17, v19\n"
    "ds_write_b128 v249, v[16:19] offset:0\n"
    "v_exp_f32_e64 v232, v232 clamp\n"
    "v_exp_f32_e64 v233, v233 clamp\n"
    "v_exp_f32_e64 v234, v234 clamp\n"
    "v_exp_f32_e64 v235, v235 clamp\n"
    "v_exp_f32_e64 v236, v236 clamp\n"
    "v_exp_f32_e64 v237, v237 clamp\n"
    "v_exp_f32_e64 v238, v238 clamp\n"
    "v_exp_f32_e64 v239, v239 clamp\n"
    "v_pk_fma_f32 v[232:233], v[232:233], s[42:43], v[244:245]\n"
    "v_pk_fma_f32 v[234:235], v[234:235], s[42:43], v[244:245]\n"
    "v_pk_fma_f32 v[236:237], v[236:237], s[42:43], v[244:245]\n"
    "v_pk_fma_f32 v[238:239], v[238:239], s[42:43], v[244:245]\n"
    "v_rcp_f32 v232, v232\n"
    "v_rcp_f32 v233, v233\n"
    "v_rcp_f32 v234, v234\n"
    "v_rcp_f32 v235, v235\n"
    "v_rcp_f32 v236, v236\n"
    "v_rcp_f32 v237, v237\n"
    "v_rcp_f32 v238, v238\n"
    "v_rcp_f32 v239, v239\n"
    "v_pk_mul_f32 v[80:81], v[80:81], v[232:233]\n"
    "v_pk_mul_f32 v[82:83], v[82:83], v[234:235]\n"
    "v_pk_mul_f32 v[112:113], v[112:113], v[236:237]\n"
    "v_pk_mul_f32 v[114:115], v[114:115], v[238:239]\n"
    "v_cvt_pk_bf16_f32 v80, v80, v81\n"
    "v_cvt_pk_bf16_f32 v81, v82, v83\n"
    "v_cvt_pk_bf16_f32 v82, v112, v113\n"
    "v_cvt_pk_bf16_f32 v83, v114, v115\n"
    "v_pk_fma_f32 v[224:225], v[20:21], s[40:41], v[240:241]\n"
    "v_pk_fma_f32 v[226:227], v[22:23], s[40:41], v[240:241]\n"
    "v_pk_fma_f32 v[228:229], v[52:53], s[40:41], v[240:241]\n"
    "v_pk_fma_f32 v[230:231], v[54:55], s[40:41], v[240:241]\n"
    "v_permlane16_swap_b32 v80, v82\n"
    "v_permlane16_swap_b32 v81, v83\n"
    "ds_write_b128 v249, v[80:83] offset:64\n"
    "ds_read_b128 v[16:19], v250 offset:0\n"
    "ds_read_b128 v[80:83], v250 offset:1152\n"
    "v_exp_f32_e64 v224, v224 clamp\n"
    "v_exp_f32_e64 v225, v225 clamp\n"
    "v_exp_f32_e64 v226, v226 clamp\n"
    "v_exp_f32_e64 v227, v227 clamp\n"
    "v_exp_f32_e64 v228, v228 clamp\n"
    "v_exp_f32_e64 v229, v229 clamp\n"
    "v_exp_f32_e64 v230, v230 clamp\n"
    "v_exp_f32_e64 v231, v231 clamp\n"
    "v_pk_fma_f32 v[224:225], v[224:225], s[42:43], v[244:245]\n"
    "v_pk_fma_f32 v[226:227], v[226:227], s[42:43], v[244:245]\n"
    "v_pk_fma_f32 v[228:229], v[228:229], s[42:43], v[244:245]\n"
    "v_pk_fma_f32 v[230:231], v[230:231], s[42:43], v[244:245]\n"
    "v_rcp_f32 v224, v224\n"
    "v_rcp_f32 v225, v225\n"
    "v_rcp_f32 v226, v226\n"
    "v_rcp_f32 v227, v227\n"
    "v_rcp_f32 v228, v228\n"
    "v_rcp_f32 v229, v229\n"
    "v_rcp_f32 v230, v230\n"
    "v_rcp_f32 v231, v231\n"
    "s_waitcnt lgkmcnt(0)\n"
    "buffer_store_dwordx4 v[16:19], v248, s[96:99], s47 offen offset:0 sc0 sc1 nt\n"
    "buffer_store_dwordx4 v[80:83], v246, s[96:99], s47 offen offset:0 sc0 sc1 nt\n"
    "v_pk_mul_f32 v[20:21], v[20:21], v[224:225]\n"
    "v_pk_mul_f32 v[22:23], v[22:23], v[226:227]\n"
    "v_pk_mul_f32 v[52:53], v[52:53], v[228:229]\n"
    "v_pk_mul_f32 v[54:55], v[54:55], v[230:231]\n"
    "v_cvt_pk_bf16_f32 v20, v20, v21\n"
    "v_cvt_pk_bf16_f32 v21, v22, v23\n"
    "v_cvt_pk_bf16_f32 v22, v52, v53\n"
    "v_cvt_pk_bf16_f32 v23, v54, v55\n"
    "v_pk_fma_f32 v[232:233], v[84:85], s[40:41], v[240:241]\n"
    "v_pk_fma_f32 v[234:235], v[86:87], s[40:41], v[240:241]\n"
    "v_pk_fma_f32 v[236:237], v[116:117], s[40:41], v[240:241]\n"
    "v_pk_fma_f32 v[238:239], v[118:119], s[40:41], v[240:241]\n"
    "v_permlane16_swap_b32 v20, v22\n"
    "v_permlane16_swap_b32 v21, v23\n"
    "ds_write_b128 v249, v[20:23] offset:2304\n"
    "v_exp_f32_e64 v232, v232 clamp\n"
    "v_exp_f32_e64 v233, v233 clamp\n"
    "v_exp_f32_e64 v234, v234 clamp\n"
    "v_exp_f32_e64 v235, v235 clamp\n"
    "v_exp_f32_e64 v236, v236 clamp\n"
    "v_exp_f32_e64 v237, v237 clamp\n"
    "v_exp_f32_e64 v238, v238 clamp\n"
    "v_exp_f32_e64 v239, v239 clamp\n"
    "v_pk_fma_f32 v[232:233], v[232:233], s[42:43], v[244:245]\n"
    "v_pk_fma_f32 v[234:235], v[234:235], s[42:43], v[244:245]\n"
    "v_pk_fma_f32 v[236:237], v[236:237], s[42:43], v[244:245]\n"
    "v_pk_fma_f32 v[238:239], v[238:239], s[42:43], v[244:245]\n"
    "v_rcp_f32 v232, v232\n"
    "v_rcp_f32 v233, v233\n"
    "v_rcp_f32 v234, v234\n"
    "v_rcp_f32 v235, v235\n"
    "v_rcp_f32 v236, v236\n"
    "v_rcp_f32 v237, v237\n"
    "v_rcp_f32 v238, v238\n"
    "v_rcp_f32 v239, v239\n"
    "v_pk_mul_f32 v[84:85], v[84:85], v[232:233]\n"
    "v_pk_mul_f32 v[86:87], v[86:87], v[234:235]\n"
    "v_pk_mul_f32 v[116:117], v[116:117], v[236:237]\n"
    "v_pk_mul_f32 v[118:119], v[118:119], v[238:239]\n"
    "v_cvt_pk_bf16_f32 v84, v84, v85\n"
    "v_cvt_pk_bf16_f32 v85, v86, v87\n"
    "v_cvt_pk_bf16_f32 v86, v116, v117\n"
    "v_cvt_pk_bf16_f32 v87, v118, v119\n"
    "v_pk_fma_f32 v[224:225], v[24:25], s[40:41], v[240:241]\n"
    "v_pk_fma_f32 v[226:227], v[26:27], s[40:41], v[240:241]\n"
    "v_pk_fma_f32 v[228:229], v[56:57], s[40:41], v[240:241]\n"
    "v_pk_fma_f32 v[230:231], v[58:59], s[40:41], v[240:241]\n"
    "v_permlane16_swap_b32 v84, v86\n"
    "v_permlane16_swap_b32 v85, v87\n"
    "ds_write_b128 v249, v[84:87] offset:2368\n"
    "ds_read_b128 v[20:23], v250 offset:2304\n"
    "ds_read_b128 v[84:87], v250 offset:3456\n"
    "v_exp_f32_e64 v224, v224 clamp\n"
    "v_exp_f32_e64 v225, v225 clamp\n"
    "v_exp_f32_e64 v226, v226 clamp\n"
    "v_exp_f32_e64 v227, v227 clamp\n"
    "v_exp_f32_e64 v228, v228 clamp\n"
    "v_exp_f32_e64 v229, v229 clamp\n"
    "v_exp_f32_e64 v230, v230 clamp\n"
    "v_exp_f32_e64 v231, v231 clamp\n"
    "v_pk_fma_f32 v[224:225], v[224:225], s[42:43], v[244:245]\n"
    "v_pk_fma_f32 v[226:227], v[226:227], s[42:43], v[244:245]\n"
    "v_pk_fma_f32 v[228:229], v[228:229], s[42:43], v[244:245]\n"
    "v_pk_fma_f32 v[230:231], v[230:231], s[42:43], v[244:245]\n"
    "v_rcp_f32 v224, v224\n"
    "v_rcp_f32 v225, v225\n"
    "v_rcp_f32 v226, v226\n"
    "v_rcp_f32 v227, v227\n"
    "v_rcp_f32 v228, v228\n"
    "v_rcp_f32 v229, v229\n"
    "v_rcp_f32 v230, v230\n"
    "v_rcp_f32 v231, v231\n"
    "s_waitcnt lgkmcnt(0)\n"
    "buffer_store_dwordx4 v[20:23], v248, s[96:99], s48 offen offset:0 sc0 sc1 nt\n"
    "buffer_store_dwordx4 v[84:87], v246, s[96:99], s48 offen offset:0 sc0 sc1 nt\n"
    "v_pk_mul_f32 v[24:25], v[24:25], v[224:225]\n"
    "v_pk_mul_f32 v[26:27], v[26:27], v[226:227]\n"
    "v_pk_mul_f32 v[56:57], v[56:57], v[228:229]\n"
    "v_pk_mul_f32 v[58:59], v[58:59], v[230:231]\n"
    "v_cvt_pk_bf16_f32 v24, v24, v25\n"
    "v_cvt_pk_bf16_f32 v25, v26, v27\n"
    "v_cvt_pk_bf16_f32 v26, v56, v57\n"
    "v_cvt_pk_bf16_f32 v27, v58, v59\n"
    "v_pk_fma_f32 v[232:233], v[88:89], s[40:41], v[240:241]\n"
    "v_pk_fma_f32 v[234:235], v[90:91], s[40:41], v[240:241]\n"
    "v_pk_fma_f32 v[236:237], v[120:121], s[40:41], v[240:241]\n"
    "v_pk_fma_f32 v[238:239], v[122:123], s[40:41], v[240:241]\n"
    "v_permlane16_swap_b32 v24, v26\n"
    "v_permlane16_swap_b32 v25, v27\n"
    "ds_write_b128 v249, v[24:27] offset:4608\n"
    "v_exp_f32_e64 v232, v232 clamp\n"
    "v_exp_f32_e64 v233, v233 clamp\n"
    "v_exp_f32_e64 v234, v234 clamp\n"
    "v_exp_f32_e64 v235, v235 clamp\n"
    "v_exp_f32_e64 v236, v236 clamp\n"
    "v_exp_f32_e64 v237, v237 clamp\n"
    "v_exp_f32_e64 v238, v238 clamp\n"
    "v_exp_f32_e64 v239, v239 clamp\n"
    "v_pk_fma_f32 v[232:233], v[232:233], s[42:43], v[244:245]\n"
    "v_pk_fma_f32 v[234:235], v[234:235], s[42:43], v[244:245]\n"
    "v_pk_fma_f32 v[236:237], v[236:237], s[42:43], v[244:245]\n"
    "v_pk_fma_f32 v[238:239], v[238:239], s[42:43], v[244:245]\n"
    "v_rcp_f32 v232, v232\n"
    "v_rcp_f32 v233, v233\n"
    "v_rcp_f32 v234, v234\n"
    "v_rcp_f32 v235, v235\n"
    "v_rcp_f32 v236, v236\n"
    "v_rcp_f32 v237, v237\n"
    "v_rcp_f32 v238, v238\n"
    "v_rcp_f32 v239, v239\n"
    "v_pk_mul_f32 v[88:89], v[88:89], v[232:233]\n"
    "v_pk_mul_f32 v[90:91], v[90:91], v[234:235]\n"
    "v_pk_mul_f32 v[120:121], v[120:121], v[236:237]\n"
    "v_pk_mul_f32 v[122:123], v[122:123], v[238:239]\n"
    "v_cvt_pk_bf16_f32 v88, v88, v89\n"
    "v_cvt_pk_bf16_f32 v89, v90, v91\n"
    "v_cvt_pk_bf16_f32 v90, v120, v121\n"
    "v_cvt_pk_bf16_f32 v91, v122, v123\n"
    "v_pk_fma_f32 v[224:225], v[28:29], s[40:41], v[240:241]\n"
    "v_pk_fma_f32 v[226:227], v[30:31], s[40:41], v[240:241]\n"
    "v_pk_fma_f32 v[228:229], v[60:61], s[40:41], v[240:241]\n"
    "v_pk_fma_f32 v[230:231], v[62:63], s[40:41], v[240:241]\n"
    "v_permlane16_swap_b32 v88, v90\n"
    "v_permlane16_swap_b32 v89, v91\n"
    "ds_write_b128 v249, v[88:91] offset:4672\n"
    "ds_read_b128 v[24:27], v250 offset:4608\n"
    "ds_read_b128 v[88:91], v250 offset:5760\n"
    "v_exp_f32_e64 v224, v224 clamp\n"
    "v_exp_f32_e64 v225, v225 clamp\n"
    "v_exp_f32_e64 v226, v226 clamp\n"
    "v_exp_f32_e64 v227, v227 clamp\n"
    "v_exp_f32_e64 v228, v228 clamp\n"
    "v_exp_f32_e64 v229, v229 clamp\n"
    "v_exp_f32_e64 v230, v230 clamp\n"
    "v_exp_f32_e64 v231, v231 clamp\n"
    "v_pk_fma_f32 v[224:225], v[224:225], s[42:43], v[244:245]\n"
    "v_pk_fma_f32 v[226:227], v[226:227], s[42:43], v[244:245]\n"
    "v_pk_fma_f32 v[228:229], v[228:229], s[42:43], v[244:245]\n"
    "v_pk_fma_f32 v[230:231], v[230:231], s[42:43], v[244:245]\n"
    "v_rcp_f32 v224, v224\n"
    "v_rcp_f32 v225, v225\n"
    "v_rcp_f32 v226, v226\n"
    "v_rcp_f32 v227, v227\n"
    "v_rcp_f32 v228, v228\n"
    "v_rcp_f32 v229, v229\n"
    "v_rcp_f32 v230, v230\n"
    "v_rcp_f32 v231, v231\n"
    "s_waitcnt lgkmcnt(0)\n"
    "buffer_store_dwordx4 v[24:27], v248, s[96:99], s49 offen offset:0 sc0 sc1 nt\n"
    "buffer_store_dwordx4 v[88:91], v246, s[96:99], s49 offen offset:0 sc0 sc1 nt\n"
    "v_pk_mul_f32 v[28:29], v[28:29], v[224:225]\n"
    "v_pk_mul_f32 v[30:31], v[30:31], v[226:227]\n"
    "v_pk_mul_f32 v[60:61], v[60:61], v[228:229]\n"
    "v_pk_mul_f32 v[62:63], v[62:63], v[230:231]\n"
    "v_cvt_pk_bf16_f32 v28, v28, v29\n"
    "v_cvt_pk_bf16_f32 v29, v30, v31\n"
    "v_cvt_pk_bf16_f32 v30, v60, v61\n"
    "v_cvt_pk_bf16_f32 v31, v62, v63\n"
    "v_pk_fma_f32 v[232:233], v[92:93], s[40:41], v[240:241]\n"
    "v_pk_fma_f32 v[234:235], v[94:95], s[40:41], v[240:241]\n"
    "v_pk_fma_f32 v[236:237], v[124:125], s[40:41], v[240:241]\n"
    "v_pk_fma_f32 v[238:239], v[126:127], s[40:41], v[240:241]\n"
    "v_permlane16_swap_b32 v28, v30\n"
    "v_permlane16_swap_b32 v29, v31\n"
    "ds_write_b128 v249, v[28:31] offset:6912\n"
    "v_exp_f32_e64 v232, v232 clamp\n"
    "v_exp_f32_e64 v233, v233 clamp\n"
    "v_exp_f32_e64 v234, v234 clamp\n"
    "v_exp_f32_e64 v235, v235 clamp\n"
    "v_exp_f32_e64 v236, v236 clamp\n"
    "v_exp_f32_e64 v237, v237 clamp\n"
    "v_exp_f32_e64 v238, v238 clamp\n"
    "v_exp_f32_e64 v239, v239 clamp\n"
    "v_pk_fma_f32 v[232:233], v[232:233], s[42:43], v[244:245]\n"
    "v_pk_fma_f32 v[234:235], v[234:235], s[42:43], v[244:245]\n"
    "v_pk_fma_f32 v[236:237], v[236:237], s[42:43], v[244:245]\n"
    "v_pk_fma_f32 v[238:239], v[238:239], s[42:43], v[244:245]\n"
    "v_rcp_f32 v232, v232\n"
    "v_rcp_f32 v233, v233\n"
    "v_rcp_f32 v234, v234\n"
    "v_rcp_f32 v235, v235\n"
    "v_rcp_f32 v236, v236\n"
    "v_rcp_f32 v237, v237\n"
    "v_rcp_f32 v238, v238\n"
    "v_rcp_f32 v239, v239\n"
    "v_pk_mul_f32 v[92:93], v[92:93], v[232:233]\n"
    "v_pk_mul_f32 v[94:95], v[94:95], v[234:235]\n"
    "v_pk_mul_f32 v[124:125], v[124:125], v[236:237]\n"
    "v_pk_mul_f32 v[126:127], v[126:127], v[238:239]\n"
    "v_cvt_pk_bf16_f32 v92, v92, v93\n"
    "v_cvt_pk_bf16_f32 v93, v94, v95\n"
    "v_cvt_pk_bf16_f32 v94, v124, v125\n"
    "v_cvt_pk_bf16_f32 v95, v126, v127\n"
    "v_pk_fma_f32 v[224:225], v[128:129], s[40:41], v[240:241]\n"
    "v_pk_fma_f32 v[226:227], v[130:131], s[40:41], v[240:241]\n"
    "v_pk_fma_f32 v[228:229], v[160:161], s[40:41], v[240:241]\n"
    "v_pk_fma_f32 v[230:231], v[162:163], s[40:41], v[240:241]\n"
    "v_permlane16_swap_b32 v92, v94\n"
    "v_permlane16_swap_b32 v93, v95\n"
    "ds_write_b128 v249, v[92:95] offset:6976\n"
    "ds_read_b128 v[28:31], v250 offset:6912\n"
    "ds_read_b128 v[92:95], v250 offset:8064\n"
    "v_exp_f32_e64 v224, v224 clamp\n"
    "v_exp_f32_e64 v225, v225 clamp\n"
    "v_exp_f32_e64 v226, v226 clamp\n"
    "v_exp_f32_e64 v227, v227 clamp\n"
    "v_exp_f32_e64 v228, v228 clamp\n"
    "v_exp_f32_e64 v229, v229 clamp\n"
    "v_exp_f32_e64 v230, v230 clamp\n"
    "v_exp_f32_e64 v231, v231 clamp\n"
    "v_pk_fma_f32 v[224:225], v[224:225], s[42:43], v[244:245]\n"
    "v_pk_fma_f32 v[226:227], v[226:227], s[42:43], v[244:245]\n"
    "v_pk_fma_f32 v[228:229], v[228:229], s[42:43], v[244:245]\n"
    "v_pk_fma_f32 v[230:231], v[230:231], s[42:43], v[244:245]\n"
    "v_rcp_f32 v224, v224\n"
    "v_rcp_f32 v225, v225\n"
    "v_rcp_f32 v226, v226\n"
    "v_rcp_f32 v227, v227\n"
    "v_rcp_f32 v228, v228\n"
    "v_rcp_f32 v229, v229\n"
    "v_rcp_f32 v230, v230\n"
    "v_rcp_f32 v231, v231\n"
    "s_waitcnt lgkmcnt(0)\n"
    "buffer_store_dwordx4 v[28:31], v248, s[96:99], s50 offen offset:0 sc0 sc1 nt\n"
    "buffer_store_dwordx4 v[92:95], v246, s[96:99], s50 offen offset:0 sc0 sc1 nt\n"
    "v_pk_mul_f32 v[128:129], v[128:129], v[224:225]\n"
    "v_pk_mul_f32 v[130:131], v[130:131], v[226:227]\n"
    "v_pk_mul_f32 v[160:161], v[160:161], v[228:229]\n"
    "v_pk_mul_f32 v[162:163], v[162:163], v[230:231]\n"
    "v_cvt_pk_bf16_f32 v128, v128, v129\n"
    "v_cvt_pk_bf16_f32 v129, v130, v131\n"
    "v_cvt_pk_bf16_f32 v130, v160, v161\n"
    "v_cvt_pk_bf16_f32 v131, v162, v163\n"
    "v_accvgpr_read_b32 v32, a160\n"
    "v_accvgpr_read_b32 v33, a161\n"
    "v_accvgpr_read_b32 v34, a162\n"
    "v_accvgpr_read_b32 v35, a163\n"
    "v_pk_fma_f32 v[232:233], v[192:193], s[40:41], v[240:241]\n"
    "v_pk_fma_f32 v[234:235], v[194:195], s[40:41], v[240:241]\n"
    "v_pk_fma_f32 v[236:237], v[32:33], s[40:41], v[240:241]\n"
    "v_pk_fma_f32 v[238:239], v[34:35], s[40:41], v[240:241]\n"
    "v_permlane16_swap_b32 v128, v130\n"
    "v_permlane16_swap_b32 v129, v131\n"
    "ds_write_b128 v249, v[128:131] offset:0\n"
    "v_exp_f32_e64 v232, v232 clamp\n"
    "v_exp_f32_e64 v233, v233 clamp\n"
    "v_exp_f32_e64 v234, v234 clamp\n"
    "v_exp_f32_e64 v235, v235 clamp\n"
    "v_exp_f32_e64 v236, v236 clamp\n"
    "v_exp_f32_e64 v237, v237 clamp\n"
    "v_exp_f32_e64 v238, v238 clamp\n"
    "v_exp_f32_e64 v239, v239 clamp\n"
    "v_pk_fma_f32 v[232:233], v[232:233], s[42:43], v[244:245]\n"
    "v_pk_fma_f32 v[234:235], v[234:235], s[42:43], v[244:245]\n"
    "v_pk_fma_f32 v[236:237], v[236:237], s[42:43], v[244:245]\n"
    "v_pk_fma_f32 v[238:239], v[238:239], s[42:43], v[244:245]\n"
    "v_rcp_f32 v232, v232\n"
    "v_rcp_f32 v233, v233\n"
    "v_rcp_f32 v234, v234\n"
    "v_rcp_f32 v235, v235\n"
    "v_rcp_f32 v236, v236\n"
    "v_rcp_f32 v237, v237\n"
    "v_rcp_f32 v238, v238\n"
    "v_rcp_f32 v239, v239\n"
    "v_pk_mul_f32 v[192:193], v[192:193], v[232:233]\n"
    "v_pk_mul_f32 v[194:195], v[194:195], v[234:235]\n"
    "v_pk_mul_f32 v[32:33], v[32:33], v[236:237]\n"
    "v_pk_mul_f32 v[34:35], v[34:35], v[238:239]\n"
    "v_cvt_pk_bf16_f32 v192, v192, v193\n"
    "v_cvt_pk_bf16_f32 v193, v194, v195\n"
    "v_cvt_pk_bf16_f32 v194, v32, v33\n"
    "v_cvt_pk_bf16_f32 v195, v34, v35\n"
    "v_pk_fma_f32 v[224:225], v[132:133], s[40:41], v[240:241]\n"
    "v_pk_fma_f32 v[226:227], v[134:135], s[40:41], v[240:241]\n"
    "v_pk_fma_f32 v[228:229], v[164:165], s[40:41], v[240:241]\n"
    "v_pk_fma_f32 v[230:231], v[166:167], s[40:41], v[240:241]\n"
    "v_permlane16_swap_b32 v192, v194\n"
    "v_permlane16_swap_b32 v193, v195\n"
    "ds_write_b128 v249, v[192:195] offset:64\n"
    "ds_read_b128 v[128:131], v250 offset:0\n"
    "ds_read_b128 v[192:195], v250 offset:1152\n"
    "v_exp_f32_e64 v224, v224 clamp\n"
    "v_exp_f32_e64 v225, v225 clamp\n"
    "v_exp_f32_e64 v226, v226 clamp\n"
    "v_exp_f32_e64 v227, v227 clamp\n"
    "v_exp_f32_e64 v228, v228 clamp\n"
    "v_exp_f32_e64 v229, v229 clamp\n"
    "v_exp_f32_e64 v230, v230 clamp\n"
    "v_exp_f32_e64 v231, v231 clamp\n"
    "v_pk_fma_f32 v[224:225], v[224:225], s[42:43], v[244:245]\n"
    "v_pk_fma_f32 v[226:227], v[226:227], s[42:43], v[244:245]\n"
    "v_pk_fma_f32 v[228:229], v[228:229], s[42:43], v[244:245]\n"
    "v_pk_fma_f32 v[230:231], v[230:231], s[42:43], v[244:245]\n"
    "v_rcp_f32 v224, v224\n"
    "v_rcp_f32 v225, v225\n"
    "v_rcp_f32 v226, v226\n"
    "v_rcp_f32 v227, v227\n"
    "v_rcp_f32 v228, v228\n"
    "v_rcp_f32 v229, v229\n"
    "v_rcp_f32 v230, v230\n"
    "v_rcp_f32 v231, v231\n"
    "s_waitcnt lgkmcnt(0)\n"
    "buffer_store_dwordx4 v[128:131], v248, s[96:99], 0 offen offset:128 sc0 sc1 nt\n"
    "buffer_store_dwordx4 v[192:195], v246, s[96:99], 0 offen offset:128 sc0 sc1 nt\n"
    "v_pk_mul_f32 v[132:133], v[132:133], v[224:225]\n"
    "v_pk_mul_f32 v[134:135], v[134:135], v[226:227]\n"
    "v_pk_mul_f32 v[164:165], v[164:165], v[228:229]\n"
    "v_pk_mul_f32 v[166:167], v[166:167], v[230:231]\n"
    "v_cvt_pk_bf16_f32 v132, v132, v133\n"
    "v_cvt_pk_bf16_f32 v133, v134, v135\n"
    "v_cvt_pk_bf16_f32 v134, v164, v165\n"
    "v_cvt_pk_bf16_f32 v135, v166, v167\n"
    "v_accvgpr_read_b32 v36, a164\n"
    "v_accvgpr_read_b32 v37, a165\n"
    "v_accvgpr_read_b32 v38, a166\n"
    "v_accvgpr_read_b32 v39, a167\n"
    "v_pk_fma_f32 v[232:233], v[196:197], s[40:41], v[240:241]\n"
    "v_pk_fma_f32 v[234:235], v[198:199], s[40:41], v[240:241]\n"
    "v_pk_fma_f32 v[236:237], v[36:37], s[40:41], v[240:241]\n"
    "v_pk_fma_f32 v[238:239], v[38:39], s[40:41], v[240:241]\n"
    "v_permlane16_swap_b32 v132, v134\n"
    "v_permlane16_swap_b32 v133, v135\n"
    "ds_write_b128 v249, v[132:135] offset:2304\n"
    "v_exp_f32_e64 v232, v232 clamp\n"
    "v_exp_f32_e64 v233, v233 clamp\n"
    "v_exp_f32_e64 v234, v234 clamp\n"
    "v_exp_f32_e64 v235, v235 clamp\n"
    "v_exp_f32_e64 v236, v236 clamp\n"
    "v_exp_f32_e64 v237, v237 clamp\n"
    "v_exp_f32_e64 v238, v238 clamp\n"
    "v_exp_f32_e64 v239, v239 clamp\n"
    "v_pk_fma_f32 v[232:233], v[232:233], s[42:43], v[244:245]\n"
    "v_pk_fma_f32 v[234:235], v[234:235], s[42:43], v[244:245]\n"
    "v_pk_fma_f32 v[236:237], v[236:237], s[42:43], v[244:245]\n"
    "v_pk_fma_f32 v[238:239], v[238:239], s[42:43], v[244:245]\n"
    "v_rcp_f32 v232, v232\n"
    "v_rcp_f32 v233, v233\n"
    "v_rcp_f32 v234, v234\n"
    "v_rcp_f32 v235, v235\n"
    "v_rcp_f32 v236, v236\n"
    "v_rcp_f32 v237, v237\n"
    "v_rcp_f32 v238, v238\n"
    "v_rcp_f32 v239, v239\n"
    "v_pk_mul_f32 v[196:197], v[196:197], v[232:233]\n"
    "v_pk_mul_f32 v[198:199], v[198:199], v[234:235]\n"
    "v_pk_mul_f32 v[36:37], v[36:37], v[236:237]\n"
    "v_pk_mul_f32 v[38:39], v[38:39], v[238:239]\n"
    "v_cvt_pk_bf16_f32 v196, v196, v197\n"
    "v_cvt_pk_bf16_f32 v197, v198, v199\n"
    "v_cvt_pk_bf16_f32 v198, v36, v37\n"
    "v_cvt_pk_bf16_f32 v199, v38, v39\n"
    "v_pk_fma_f32 v[224:225], v[136:137], s[40:41], v[240:241]\n"
    "v_pk_fma_f32 v[226:227], v[138:139], s[40:41], v[240:241]\n"
    "v_pk_fma_f32 v[228:229], v[168:169], s[40:41], v[240:241]\n"
    "v_pk_fma_f32 v[230:231], v[170:171], s[40:41], v[240:241]\n"
    "v_permlane16_swap_b32 v196, v198\n"
    "v_permlane16_swap_b32 v197, v199\n"
    "ds_write_b128 v249, v[196:199] offset:2368\n"
    "ds_read_b128 v[132:135], v250 offset:2304\n"
    "ds_read_b128 v[196:199], v250 offset:3456\n"
    "v_exp_f32_e64 v224, v224 clamp\n"
    "v_exp_f32_e64 v225, v225 clamp\n"
    "v_exp_f32_e64 v226, v226 clamp\n"
    "v_exp_f32_e64 v227, v227 clamp\n"
    "v_exp_f32_e64 v228, v228 clamp\n"
    "v_exp_f32_e64 v229, v229 clamp\n"
    "v_exp_f32_e64 v230, v230 clamp\n"
    "v_exp_f32_e64 v231, v231 clamp\n"
    "v_pk_fma_f32 v[224:225], v[224:225], s[42:43], v[244:245]\n"
    "v_pk_fma_f32 v[226:227], v[226:227], s[42:43], v[244:245]\n"
    "v_pk_fma_f32 v[228:229], v[228:229], s[42:43], v[244:245]\n"
    "v_pk_fma_f32 v[230:231], v[230:231], s[42:43], v[244:245]\n"
    "v_rcp_f32 v224, v224\n"
    "v_rcp_f32 v225, v225\n"
    "v_rcp_f32 v226, v226\n"
    "v_rcp_f32 v227, v227\n"
    "v_rcp_f32 v228, v228\n"
    "v_rcp_f32 v229, v229\n"
    "v_rcp_f32 v230, v230\n"
    "v_rcp_f32 v231, v231\n"
    "s_waitcnt lgkmcnt(0)\n"
    "buffer_store_dwordx4 v[132:135], v248, s[96:99], s44 offen offset:128 sc0 sc1 nt\n"
    "buffer_store_dwordx4 v[196:199], v246, s[96:99], s44 offen offset:128 sc0 sc1 nt\n"
    "v_pk_mul_f32 v[136:137], v[136:137], v[224:225]\n"
    "v_pk_mul_f32 v[138:139], v[138:139], v[226:227]\n"
    "v_pk_mul_f32 v[168:169], v[168:169], v[228:229]\n"
    "v_pk_mul_f32 v[170:171], v[170:171], v[230:231]\n"
    "v_cvt_pk_bf16_f32 v136, v136, v137\n"
    "v_cvt_pk_bf16_f32 v137, v138, v139\n"
    "v_cvt_pk_bf16_f32 v138, v168, v169\n"
    "v_cvt_pk_bf16_f32 v139, v170, v171\n"
    "v_accvgpr_read_b32 v40, a168\n"
    "v_accvgpr_read_b32 v41, a169\n"
    "v_accvgpr_read_b32 v42, a170\n"
    "v_accvgpr_read_b32 v43, a171\n"
    "v_pk_fma_f32 v[232:233], v[200:201], s[40:41], v[240:241]\n"
    "v_pk_fma_f32 v[234:235], v[202:203], s[40:41], v[240:241]\n"
    "v_pk_fma_f32 v[236:237], v[40:41], s[40:41], v[240:241]\n"
    "v_pk_fma_f32 v[238:239], v[42:43], s[40:41], v[240:241]\n"
    "v_permlane16_swap_b32 v136, v138\n"
    "v_permlane16_swap_b32 v137, v139\n"
    "ds_write_b128 v249, v[136:139] offset:4608\n"
    "v_exp_f32_e64 v232, v232 clamp\n"
    "v_exp_f32_e64 v233, v233 clamp\n"
    "v_exp_f32_e64 v234, v234 clamp\n"
    "v_exp_f32_e64 v235, v235 clamp\n"
    "v_exp_f32_e64 v236, v236 clamp\n"
    "v_exp_f32_e64 v237, v237 clamp\n"
    "v_exp_f32_e64 v238, v238 clamp\n"
    "v_exp_f32_e64 v239, v239 clamp\n"
    "v_pk_fma_f32 v[232:233], v[232:233], s[42:43], v[244:245]\n"
    "v_pk_fma_f32 v[234:235], v[234:235], s[42:43], v[244:245]\n"
    "v_pk_fma_f32 v[236:237], v[236:237], s[42:43], v[244:245]\n"
    "v_pk_fma_f32 v[238:239], v[238:239], s[42:43], v[244:245]\n"
    "v_rcp_f32 v232, v232\n"
    "v_rcp_f32 v233, v233\n"
    "v_rcp_f32 v234, v234\n"
    "v_rcp_f32 v235, v235\n"
    "v_rcp_f32 v236, v236\n"
    "v_rcp_f32 v237, v237\n"
    "v_rcp_f32 v238, v238\n"
    "v_rcp_f32 v239, v239\n"
    "v_pk_mul_f32 v[200:201], v[200:201], v[232:233]\n"
    "v_pk_mul_f32 v[202:203], v[202:203], v[234:235]\n"
    "v_pk_mul_f32 v[40:41], v[40:41], v[236:237]\n"
    "v_pk_mul_f32 v[42:43], v[42:43], v[238:239]\n"
    "v_cvt_pk_bf16_f32 v200, v200, v201\n"
    "v_cvt_pk_bf16_f32 v201, v202, v203\n"
    "v_cvt_pk_bf16_f32 v202, v40, v41\n"
    "v_cvt_pk_bf16_f32 v203, v42, v43\n"
    "v_pk_fma_f32 v[224:225], v[140:141], s[40:41], v[240:241]\n"
    "v_pk_fma_f32 v[226:227], v[142:143], s[40:41], v[240:241]\n"
    "v_pk_fma_f32 v[228:229], v[172:173], s[40:41], v[240:241]\n"
    "v_pk_fma_f32 v[230:231], v[174:175], s[40:41], v[240:241]\n"
    "v_permlane16_swap_b32 v200, v202\n"
    "v_permlane16_swap_b32 v201, v203\n"
    "ds_write_b128 v249, v[200:203] offset:4672\n"
    "ds_read_b128 v[136:139], v250 offset:4608\n"
    "ds_read_b128 v[200:203], v250 offset:5760\n"
    "v_exp_f32_e64 v224, v224 clamp\n"
    "v_exp_f32_e64 v225, v225 clamp\n"
    "v_exp_f32_e64 v226, v226 clamp\n"
    "v_exp_f32_e64 v227, v227 clamp\n"
    "v_exp_f32_e64 v228, v228 clamp\n"
    "v_exp_f32_e64 v229, v229 clamp\n"
    "v_exp_f32_e64 v230, v230 clamp\n"
    "v_exp_f32_e64 v231, v231 clamp\n"
    "v_pk_fma_f32 v[224:225], v[224:225], s[42:43], v[244:245]\n"
    "v_pk_fma_f32 v[226:227], v[226:227], s[42:43], v[244:245]\n"
    "v_pk_fma_f32 v[228:229], v[228:229], s[42:43], v[244:245]\n"
    "v_pk_fma_f32 v[230:231], v[230:231], s[42:43], v[244:245]\n"
    "v_rcp_f32 v224, v224\n"
    "v_rcp_f32 v225, v225\n"
    "v_rcp_f32 v226, v226\n"
    "v_rcp_f32 v227, v227\n"
    "v_rcp_f32 v228, v228\n"
    "v_rcp_f32 v229, v229\n"
    "v_rcp_f32 v230, v230\n"
    "v_rcp_f32 v231, v231\n"
    "s_waitcnt lgkmcnt(0)\n"
    "buffer_store_dwordx4 v[136:139], v248, s[96:99], s45 offen offset:128 sc0 sc1 nt\n"
    "buffer_store_dwordx4 v[200:203], v246, s[96:99], s45 offen offset:128 sc0 sc1 nt\n"
    "v_pk_mul_f32 v[140:141], v[140:141], v[224:225]\n"
    "v_pk_mul_f32 v[142:143], v[142:143], v[226:227]\n"
    "v_pk_mul_f32 v[172:173], v[172:173], v[228:229]\n"
    "v_pk_mul_f32 v[174:175], v[174:175], v[230:231]\n"
    "v_cvt_pk_bf16_f32 v140, v140, v141\n"
    "v_cvt_pk_bf16_f32 v141, v142, v143\n"
    "v_cvt_pk_bf16_f32 v142, v172, v173\n"
    "v_cvt_pk_bf16_f32 v143, v174, v175\n"
    "v_accvgpr_read_b32 v44, a172\n"
    "v_accvgpr_read_b32 v45, a173\n"
    "v_accvgpr_read_b32 v46, a174\n"
    "v_accvgpr_read_b32 v47, a175\n"
    "v_pk_fma_f32 v[232:233], v[204:205], s[40:41], v[240:241]\n"
    "v_pk_fma_f32 v[234:235], v[206:207], s[40:41], v[240:241]\n"
    "v_pk_fma_f32 v[236:237], v[44:45], s[40:41], v[240:241]\n"
    "v_pk_fma_f32 v[238:239], v[46:47], s[40:41], v[240:241]\n"
    "v_permlane16_swap_b32 v140, v142\n"
    "v_permlane16_swap_b32 v141, v143\n"
    "ds_write_b128 v249, v[140:143] offset:6912\n"
    "v_exp_f32_e64 v232, v232 clamp\n"
    "v_exp_f32_e64 v233, v233 clamp\n"
    "v_exp_f32_e64 v234, v234 clamp\n"
    "v_exp_f32_e64 v235, v235 clamp\n"
    "v_exp_f32_e64 v236, v236 clamp\n"
    "v_exp_f32_e64 v237, v237 clamp\n"
    "v_exp_f32_e64 v238, v238 clamp\n"
    "v_exp_f32_e64 v239, v239 clamp\n"
    "v_pk_fma_f32 v[232:233], v[232:233], s[42:43], v[244:245]\n"
    "v_pk_fma_f32 v[234:235], v[234:235], s[42:43], v[244:245]\n"
    "v_pk_fma_f32 v[236:237], v[236:237], s[42:43], v[244:245]\n"
    "v_pk_fma_f32 v[238:239], v[238:239], s[42:43], v[244:245]\n"
    "v_rcp_f32 v232, v232\n"
    "v_rcp_f32 v233, v233\n"
    "v_rcp_f32 v234, v234\n"
    "v_rcp_f32 v235, v235\n"
    "v_rcp_f32 v236, v236\n"
    "v_rcp_f32 v237, v237\n"
    "v_rcp_f32 v238, v238\n"
    "v_rcp_f32 v239, v239\n"
    "v_pk_mul_f32 v[204:205], v[204:205], v[232:233]\n"
    "v_pk_mul_f32 v[206:207], v[206:207], v[234:235]\n"
    "v_pk_mul_f32 v[44:45], v[44:45], v[236:237]\n"
    "v_pk_mul_f32 v[46:47], v[46:47], v[238:239]\n"
    "v_cvt_pk_bf16_f32 v204, v204, v205\n"
    "v_cvt_pk_bf16_f32 v205, v206, v207\n"
    "v_cvt_pk_bf16_f32 v206, v44, v45\n"
    "v_cvt_pk_bf16_f32 v207, v46, v47\n"
    "v_pk_fma_f32 v[224:225], v[144:145], s[40:41], v[240:241]\n"
    "v_pk_fma_f32 v[226:227], v[146:147], s[40:41], v[240:241]\n"
    "v_pk_fma_f32 v[228:229], v[176:177], s[40:41], v[240:241]\n"
    "v_pk_fma_f32 v[230:231], v[178:179], s[40:41], v[240:241]\n"
    "v_permlane16_swap_b32 v204, v206\n"
    "v_permlane16_swap_b32 v205, v207\n"
    "ds_write_b128 v249, v[204:207] offset:6976\n"
    "ds_read_b128 v[140:143], v250 offset:6912\n"
    "ds_read_b128 v[204:207], v250 offset:8064\n"
    "v_exp_f32_e64 v224, v224 clamp\n"
    "v_exp_f32_e64 v225, v225 clamp\n"
    "v_exp_f32_e64 v226, v226 clamp\n"
    "v_exp_f32_e64 v227, v227 clamp\n"
    "v_exp_f32_e64 v228, v228 clamp\n"
    "v_exp_f32_e64 v229, v229 clamp\n"
    "v_exp_f32_e64 v230, v230 clamp\n"
    "v_exp_f32_e64 v231, v231 clamp\n"
    "v_pk_fma_f32 v[224:225], v[224:225], s[42:43], v[244:245]\n"
    "v_pk_fma_f32 v[226:227], v[226:227], s[42:43], v[244:245]\n"
    "v_pk_fma_f32 v[228:229], v[228:229], s[42:43], v[244:245]\n"
    "v_pk_fma_f32 v[230:231], v[230:231], s[42:43], v[244:245]\n"
    "v_rcp_f32 v224, v224\n"
    "v_rcp_f32 v225, v225\n"
    "v_rcp_f32 v226, v226\n"
    "v_rcp_f32 v227, v227\n"
    "v_rcp_f32 v228, v228\n"
    "v_rcp_f32 v229, v229\n"
    "v_rcp_f32 v230, v230\n"
    "v_rcp_f32 v231, v231\n"
    "s_waitcnt lgkmcnt(0)\n"
    "buffer_store_dwordx4 v[140:143], v248, s[96:99], s46 offen offset:128 sc0 sc1 nt\n"
    "buffer_store_dwordx4 v[204:207], v246, s[96:99], s46 offen offset:128 sc0 sc1 nt\n"
    "v_pk_mul_f32 v[144:145], v[144:145], v[224:225]\n"
    "v_pk_mul_f32 v[146:147], v[146:147], v[226:227]\n"
    "v_pk_mul_f32 v[176:177], v[176:177], v[228:229]\n"
    "v_pk_mul_f32 v[178:179], v[178:179], v[230:231]\n"
    "v_cvt_pk_bf16_f32 v144, v144, v145\n"
    "v_cvt_pk_bf16_f32 v145, v146, v147\n"
    "v_cvt_pk_bf16_f32 v146, v176, v177\n"
    "v_cvt_pk_bf16_f32 v147, v178, v179\n"
    "v_accvgpr_read_b32 v48, a176\n"
    "v_accvgpr_read_b32 v49, a177\n"
    "v_accvgpr_read_b32 v50, a178\n"
    "v_accvgpr_read_b32 v51, a179\n"
    "v_pk_fma_f32 v[232:233], v[208:209], s[40:41], v[240:241]\n"
    "v_pk_fma_f32 v[234:235], v[210:211], s[40:41], v[240:241]\n"
    "v_pk_fma_f32 v[236:237], v[48:49], s[40:41], v[240:241]\n"
    "v_pk_fma_f32 v[238:239], v[50:51], s[40:41], v[240:241]\n"
    "v_permlane16_swap_b32 v144, v146\n"
    "v_permlane16_swap_b32 v145, v147\n"
    "ds_write_b128 v249, v[144:147] offset:0\n"
    "v_exp_f32_e64 v232, v232 clamp\n"
    "v_exp_f32_e64 v233, v233 clamp\n"
    "v_exp_f32_e64 v234, v234 clamp\n"
    "v_exp_f32_e64 v235, v235 clamp\n"
    "v_exp_f32_e64 v236, v236 clamp\n"
    "v_exp_f32_e64 v237, v237 clamp\n"
    "v_exp_f32_e64 v238, v238 clamp\n"
    "v_exp_f32_e64 v239, v239 clamp\n"
    "v_pk_fma_f32 v[232:233], v[232:233], s[42:43], v[244:245]\n"
    "v_pk_fma_f32 v[234:235], v[234:235], s[42:43], v[244:245]\n"
    "v_pk_fma_f32 v[236:237], v[236:237], s[42:43], v[244:245]\n"
    "v_pk_fma_f32 v[238:239], v[238:239], s[42:43], v[244:245]\n"
    "v_rcp_f32 v232, v232\n"
    "v_rcp_f32 v233, v233\n"
    "v_rcp_f32 v234, v234\n"
    "v_rcp_f32 v235, v235\n"
    "v_rcp_f32 v236, v236\n"
    "v_rcp_f32 v237, v237\n"
    "v_rcp_f32 v238, v238\n"
    "v_rcp_f32 v239, v239\n"
    "v_pk_mul_f32 v[208:209], v[208:209], v[232:233]\n"
    "v_pk_mul_f32 v[210:211], v[210:211], v[234:235]\n"
    "v_pk_mul_f32 v[48:49], v[48:49], v[236:237]\n"
    "v_pk_mul_f32 v[50:51], v[50:51], v[238:239]\n"
    "v_cvt_pk_bf16_f32 v208, v208, v209\n"
    "v_cvt_pk_bf16_f32 v209, v210, v211\n"
    "v_cvt_pk_bf16_f32 v210, v48, v49\n"
    "v_cvt_pk_bf16_f32 v211, v50, v51\n"
    "v_pk_fma_f32 v[224:225], v[148:149], s[40:41], v[240:241]\n"
    "v_pk_fma_f32 v[226:227], v[150:151], s[40:41], v[240:241]\n"
    "v_pk_fma_f32 v[228:229], v[180:181], s[40:41], v[240:241]\n"
    "v_pk_fma_f32 v[230:231], v[182:183], s[40:41], v[240:241]\n"
    "v_permlane16_swap_b32 v208, v210\n"
    "v_permlane16_swap_b32 v209, v211\n"
    "ds_write_b128 v249, v[208:211] offset:64\n"
    "ds_read_b128 v[144:147], v250 offset:0\n"
    "ds_read_b128 v[208:211], v250 offset:1152\n"
    "v_exp_f32_e64 v224, v224 clamp\n"
    "v_exp_f32_e64 v225, v225 clamp\n"
    "v_exp_f32_e64 v226, v226 clamp\n"
    "v_exp_f32_e64 v227, v227 clamp\n"
    "v_exp_f32_e64 v228, v228 clamp\n"
    "v_exp_f32_e64 v229, v229 clamp\n"
    "v_exp_f32_e64 v230, v230 clamp\n"
    "v_exp_f32_e64 v231, v231 clamp\n"
    "v_pk_fma_f32 v[224:225], v[224:225], s[42:43], v[244:245]\n"
    "v_pk_fma_f32 v[226:227], v[226:227], s[42:43], v[244:245]\n"
    "v_pk_fma_f32 v[228:229], v[228:229], s[42:43], v[244:245]\n"
    "v_pk_fma_f32 v[230:231], v[230:231], s[42:43], v[244:245]\n"
    "v_rcp_f32 v224, v224\n"
    "v_rcp_f32 v225, v225\n"
    "v_rcp_f32 v226, v226\n"
    "v_rcp_f32 v227, v227\n"
    "v_rcp_f32 v228, v228\n"
    "v_rcp_f32 v229, v229\n"
    "v_rcp_f32 v230, v230\n"
    "v_rcp_f32 v231, v231\n"
    "s_waitcnt lgkmcnt(0)\n"
    "buffer_store_dwordx4 v[144:147], v248, s[96:99], s47 offen offset:128 sc0 sc1 nt\n"
    "buffer_store_dwordx4 v[208:211], v246, s[96:99], s47 offen offset:128 sc0 sc1 nt\n"
    "v_pk_mul_f32 v[148:149], v[148:149], v[224:225]\n"
    "v_pk_mul_f32 v[150:151], v[150:151], v[226:227]\n"
    "v_pk_mul_f32 v[180:181], v[180:181], v[228:229]\n"
    "v_pk_mul_f32 v[182:183], v[182:183], v[230:231]\n"
    "v_cvt_pk_bf16_f32 v148, v148, v149\n"
    "v_cvt_pk_bf16_f32 v149, v150, v151\n"
    "v_cvt_pk_bf16_f32 v150, v180, v181\n"
    "v_cvt_pk_bf16_f32 v151, v182, v183\n"
    "v_accvgpr_read_b32 v52, a180\n"
    "v_accvgpr_read_b32 v53, a181\n"
    "v_accvgpr_read_b32 v54, a182\n"
    "v_accvgpr_read_b32 v55, a183\n"
    "v_pk_fma_f32 v[232:233], v[212:213], s[40:41], v[240:241]\n"
    "v_pk_fma_f32 v[234:235], v[214:215], s[40:41], v[240:241]\n"
    "v_pk_fma_f32 v[236:237], v[52:53], s[40:41], v[240:241]\n"
    "v_pk_fma_f32 v[238:239], v[54:55], s[40:41], v[240:241]\n"
    "v_permlane16_swap_b32 v148, v150\n"
    "v_permlane16_swap_b32 v149, v151\n"
    "ds_write_b128 v249, v[148:151] offset:2304\n"
    "v_exp_f32_e64 v232, v232 clamp\n"
    "v_exp_f32_e64 v233, v233 clamp\n"
    "v_exp_f32_e64 v234, v234 clamp\n"
    "v_exp_f32_e64 v235, v235 clamp\n"
    "v_exp_f32_e64 v236, v236 clamp\n"
    "v_exp_f32_e64 v237, v237 clamp\n"
    "v_exp_f32_e64 v238, v238 clamp\n"
    "v_exp_f32_e64 v239, v239 clamp\n"
    "v_pk_fma_f32 v[232:233], v[232:233], s[42:43], v[244:245]\n"
    "v_pk_fma_f32 v[234:235], v[234:235], s[42:43], v[244:245]\n"
    "v_pk_fma_f32 v[236:237], v[236:237], s[42:43], v[244:245]\n"
    "v_pk_fma_f32 v[238:239], v[238:239], s[42:43], v[244:245]\n"
    "v_rcp_f32 v232, v232\n"
    "v_rcp_f32 v233, v233\n"
    "v_rcp_f32 v234, v234\n"
    "v_rcp_f32 v235, v235\n"
    "v_rcp_f32 v236, v236\n"
    "v_rcp_f32 v237, v237\n"
    "v_rcp_f32 v238, v238\n"
    "v_rcp_f32 v239, v239\n"
    "v_pk_mul_f32 v[212:213], v[212:213], v[232:233]\n"
    "v_pk_mul_f32 v[214:215], v[214:215], v[234:235]\n"
    "v_pk_mul_f32 v[52:53], v[52:53], v[236:237]\n"
    "v_pk_mul_f32 v[54:55], v[54:55], v[238:239]\n"
    "v_cvt_pk_bf16_f32 v212, v212, v213\n"
    "v_cvt_pk_bf16_f32 v213, v214, v215\n"
    "v_cvt_pk_bf16_f32 v214, v52, v53\n"
    "v_cvt_pk_bf16_f32 v215, v54, v55\n"
    "v_pk_fma_f32 v[224:225], v[152:153], s[40:41], v[240:241]\n"
    "v_pk_fma_f32 v[226:227], v[154:155], s[40:41], v[240:241]\n"
    "v_pk_fma_f32 v[228:229], v[184:185], s[40:41], v[240:241]\n"
    "v_pk_fma_f32 v[230:231], v[186:187], s[40:41], v[240:241]\n"
    "v_permlane16_swap_b32 v212, v214\n"
    "v_permlane16_swap_b32 v213, v215\n"
    "ds_write_b128 v249, v[212:215] offset:2368\n"
    "ds_read_b128 v[148:151], v250 offset:2304\n"
    "ds_read_b128 v[212:215], v250 offset:3456\n"
    "v_exp_f32_e64 v224, v224 clamp\n"
    "v_exp_f32_e64 v225, v225 clamp\n"
    "v_exp_f32_e64 v226, v226 clamp\n"
    "v_exp_f32_e64 v227, v227 clamp\n"
    "v_exp_f32_e64 v228, v228 clamp\n"
    "v_exp_f32_e64 v229, v229 clamp\n"
    "v_exp_f32_e64 v230, v230 clamp\n"
    "v_exp_f32_e64 v231, v231 clamp\n"
    "v_pk_fma_f32 v[224:225], v[224:225], s[42:43], v[244:245]\n"
    "v_pk_fma_f32 v[226:227], v[226:227], s[42:43], v[244:245]\n"
    "v_pk_fma_f32 v[228:229], v[228:229], s[42:43], v[244:245]\n"
    "v_pk_fma_f32 v[230:231], v[230:231], s[42:43], v[244:245]\n"
    "v_rcp_f32 v224, v224\n"
    "v_rcp_f32 v225, v225\n"
    "v_rcp_f32 v226, v226\n"
    "v_rcp_f32 v227, v227\n"
    "v_rcp_f32 v228, v228\n"
    "v_rcp_f32 v229, v229\n"
    "v_rcp_f32 v230, v230\n"
    "v_rcp_f32 v231, v231\n"
    "s_waitcnt lgkmcnt(0)\n"
    "buffer_store_dwordx4 v[148:151], v248, s[96:99], s48 offen offset:128 sc0 sc1 nt\n"
    "buffer_store_dwordx4 v[212:215], v246, s[96:99], s48 offen offset:128 sc0 sc1 nt\n"
    "v_pk_mul_f32 v[152:153], v[152:153], v[224:225]\n"
    "v_pk_mul_f32 v[154:155], v[154:155], v[226:227]\n"
    "v_pk_mul_f32 v[184:185], v[184:185], v[228:229]\n"
    "v_pk_mul_f32 v[186:187], v[186:187], v[230:231]\n"
    "v_cvt_pk_bf16_f32 v152, v152, v153\n"
    "v_cvt_pk_bf16_f32 v153, v154, v155\n"
    "v_cvt_pk_bf16_f32 v154, v184, v185\n"
    "v_cvt_pk_bf16_f32 v155, v186, v187\n"
    "v_accvgpr_read_b32 v56, a184\n"
    "v_accvgpr_read_b32 v57, a185\n"
    "v_accvgpr_read_b32 v58, a186\n"
    "v_accvgpr_read_b32 v59, a187\n"
    "v_pk_fma_f32 v[232:233], v[216:217], s[40:41], v[240:241]\n"
    "v_pk_fma_f32 v[234:235], v[218:219], s[40:41], v[240:241]\n"
    "v_pk_fma_f32 v[236:237], v[56:57], s[40:41], v[240:241]\n"
    "v_pk_fma_f32 v[238:239], v[58:59], s[40:41], v[240:241]\n"
    "v_permlane16_swap_b32 v152, v154\n"
    "v_permlane16_swap_b32 v153, v155\n"
    "ds_write_b128 v249, v[152:155] offset:4608\n"
    "v_exp_f32_e64 v232, v232 clamp\n"
    "v_exp_f32_e64 v233, v233 clamp\n"
    "v_exp_f32_e64 v234, v234 clamp\n"
    "v_exp_f32_e64 v235, v235 clamp\n"
    "v_exp_f32_e64 v236, v236 clamp\n"
    "v_exp_f32_e64 v237, v237 clamp\n"
    "v_exp_f32_e64 v238, v238 clamp\n"
    "v_exp_f32_e64 v239, v239 clamp\n"
    "v_pk_fma_f32 v[232:233], v[232:233], s[42:43], v[244:245]\n"
    "v_pk_fma_f32 v[234:235], v[234:235], s[42:43], v[244:245]\n"
    "v_pk_fma_f32 v[236:237], v[236:237], s[42:43], v[244:245]\n"
    "v_pk_fma_f32 v[238:239], v[238:239], s[42:43], v[244:245]\n"
    "v_rcp_f32 v232, v232\n"
    "v_rcp_f32 v233, v233\n"
    "v_rcp_f32 v234, v234\n"
    "v_rcp_f32 v235, v235\n"
    "v_rcp_f32 v236, v236\n"
    "v_rcp_f32 v237, v237\n"
    "v_rcp_f32 v238, v238\n"
    "v_rcp_f32 v239, v239\n"
    "v_pk_mul_f32 v[216:217], v[216:217], v[232:233]\n"
    "v_pk_mul_f32 v[218:219], v[218:219], v[234:235]\n"
    "v_pk_mul_f32 v[56:57], v[56:57], v[236:237]\n"
    "v_pk_mul_f32 v[58:59], v[58:59], v[238:239]\n"
    "v_cvt_pk_bf16_f32 v216, v216, v217\n"
    "v_cvt_pk_bf16_f32 v217, v218, v219\n"
    "v_cvt_pk_bf16_f32 v218, v56, v57\n"
    "v_cvt_pk_bf16_f32 v219, v58, v59\n"
    "v_pk_fma_f32 v[224:225], v[156:157], s[40:41], v[240:241]\n"
    "v_pk_fma_f32 v[226:227], v[158:159], s[40:41], v[240:241]\n"
    "v_pk_fma_f32 v[228:229], v[188:189], s[40:41], v[240:241]\n"
    "v_pk_fma_f32 v[230:231], v[190:191], s[40:41], v[240:241]\n"
    "v_permlane16_swap_b32 v216, v218\n"
    "v_permlane16_swap_b32 v217, v219\n"
    "ds_write_b128 v249, v[216:219] offset:4672\n"
    "ds_read_b128 v[152:155], v250 offset:4608\n"
    "ds_read_b128 v[216:219], v250 offset:5760\n"
    "v_exp_f32_e64 v224, v224 clamp\n"
    "v_exp_f32_e64 v225, v225 clamp\n"
    "v_exp_f32_e64 v226, v226 clamp\n"
    "v_exp_f32_e64 v227, v227 clamp\n"
    "v_exp_f32_e64 v228, v228 clamp\n"
    "v_exp_f32_e64 v229, v229 clamp\n"
    "v_exp_f32_e64 v230, v230 clamp\n"
    "v_exp_f32_e64 v231, v231 clamp\n"
    "v_pk_fma_f32 v[224:225], v[224:225], s[42:43], v[244:245]\n"
    "v_pk_fma_f32 v[226:227], v[226:227], s[42:43], v[244:245]\n"
    "v_pk_fma_f32 v[228:229], v[228:229], s[42:43], v[244:245]\n"
    "v_pk_fma_f32 v[230:231], v[230:231], s[42:43], v[244:245]\n"
    "v_rcp_f32 v224, v224\n"
    "v_rcp_f32 v225, v225\n"
    "v_rcp_f32 v226, v226\n"
    "v_rcp_f32 v227, v227\n"
    "v_rcp_f32 v228, v228\n"
    "v_rcp_f32 v229, v229\n"
    "v_rcp_f32 v230, v230\n"
    "v_rcp_f32 v231, v231\n"
    "s_waitcnt lgkmcnt(0)\n"
    "buffer_store_dwordx4 v[152:155], v248, s[96:99], s49 offen offset:128 sc0 sc1 nt\n"
    "buffer_store_dwordx4 v[216:219], v246, s[96:99], s49 offen offset:128 sc0 sc1 nt\n"
    "v_pk_mul_f32 v[156:157], v[156:157], v[224:225]\n"
    "v_pk_mul_f32 v[158:159], v[158:159], v[226:227]\n"
    "v_pk_mul_f32 v[188:189], v[188:189], v[228:229]\n"
    "v_pk_mul_f32 v[190:191], v[190:191], v[230:231]\n"
    "v_cvt_pk_bf16_f32 v156, v156, v157\n"
    "v_cvt_pk_bf16_f32 v157, v158, v159\n"
    "v_cvt_pk_bf16_f32 v158, v188, v189\n"
    "v_cvt_pk_bf16_f32 v159, v190, v191\n"
    "v_accvgpr_read_b32 v60, a188\n"
    "v_accvgpr_read_b32 v61, a189\n"
    "v_accvgpr_read_b32 v62, a190\n"
    "v_accvgpr_read_b32 v63, a191\n"
    "v_pk_fma_f32 v[232:233], v[220:221], s[40:41], v[240:241]\n"
    "v_pk_fma_f32 v[234:235], v[222:223], s[40:41], v[240:241]\n"
    "v_pk_fma_f32 v[236:237], v[60:61], s[40:41], v[240:241]\n"
    "v_pk_fma_f32 v[238:239], v[62:63], s[40:41], v[240:241]\n"
    "v_permlane16_swap_b32 v156, v158\n"
    "v_permlane16_swap_b32 v157, v159\n"
    "ds_write_b128 v249, v[156:159] offset:6912\n"
    "v_exp_f32_e64 v232, v232 clamp\n"
    "v_exp_f32_e64 v233, v233 clamp\n"
    "v_exp_f32_e64 v234, v234 clamp\n"
    "v_exp_f32_e64 v235, v235 clamp\n"
    "v_exp_f32_e64 v236, v236 clamp\n"
    "v_exp_f32_e64 v237, v237 clamp\n"
    "v_exp_f32_e64 v238, v238 clamp\n"
    "v_exp_f32_e64 v239, v239 clamp\n"
    "v_pk_fma_f32 v[232:233], v[232:233], s[42:43], v[244:245]\n"
    "v_pk_fma_f32 v[234:235], v[234:235], s[42:43], v[244:245]\n"
    "v_pk_fma_f32 v[236:237], v[236:237], s[42:43], v[244:245]\n"
    "v_pk_fma_f32 v[238:239], v[238:239], s[42:43], v[244:245]\n"
    "v_rcp_f32 v232, v232\n"
    "v_rcp_f32 v233, v233\n"
    "v_rcp_f32 v234, v234\n"
    "v_rcp_f32 v235, v235\n"
    "v_rcp_f32 v236, v236\n"
    "v_rcp_f32 v237, v237\n"
    "v_rcp_f32 v238, v238\n"
    "v_rcp_f32 v239, v239\n"
    "v_pk_mul_f32 v[220:221], v[220:221], v[232:233]\n"
    "v_pk_mul_f32 v[222:223], v[222:223], v[234:235]\n"
    "v_pk_mul_f32 v[60:61], v[60:61], v[236:237]\n"
    "v_pk_mul_f32 v[62:63], v[62:63], v[238:239]\n"
    "v_cvt_pk_bf16_f32 v220, v220, v221\n"
    "v_cvt_pk_bf16_f32 v221, v222, v223\n"
    "v_cvt_pk_bf16_f32 v222, v60, v61\n"
    "v_cvt_pk_bf16_f32 v223, v62, v63\n"
    "s_nop 1\n"
    "v_permlane16_swap_b32 v220, v222\n"
    "v_permlane16_swap_b32 v221, v223\n"
    "ds_write_b128 v249, v[220:223] offset:6976\n"
    "ds_read_b128 v[156:159], v250 offset:6912\n"
    "ds_read_b128 v[220:223], v250 offset:8064\n"
    "s_waitcnt lgkmcnt(0)\n"
    "buffer_store_dwordx4 v[156:159], v248, s[96:99], s50 offen offset:128 sc0 sc1 nt\n"
    "buffer_store_dwordx4 v[220:223], v246, s[96:99], s50 offen offset:128 sc0 sc1 nt\n"
    "s_endpgm\n"
    ".Lg35_%=:\n"
    "s_waitcnt lgkmcnt(4)\n"
    "v_mfma_f32_16x16x32_bf16 v[0:3], a[128:131], a[0:3], 0\n"
    "buffer_load_dwordx4 a[192:195], v224, s[80:83], s64 offen offset:0\n"
    "s_add_u32 s59, s52, 2\n"
    "v_mfma_f32_16x16x32_bf16 v[0:3], a[132:135], a[4:7], v[0:3]\n"
    "ds_read_b64_tr_b16 a[144:145], v242 offset:256\n"
    "buffer_load_dwordx4 a[196:199], v225, s[80:83], s64 offen offset:0\n"
    "v_readlane_b32 s54, v245, s59\n"
    "v_mfma_f32_16x16x32_bf16 v[4:7], a[128:131], a[8:11], 0\n"
    "buffer_load_dwordx4 a[200:203], v226, s[80:83], s64 offen offset:0\n"
    "v_mfma_f32_16x16x32_bf16 v[4:7], a[132:135], a[12:15], v[4:7]\n"
    "ds_read_b64_tr_b16 a[146:147], v242 offset:1280\n"
    "buffer_load_dwordx4 a[204:207], v227, s[80:83], s64 offen offset:0\n"
    "v_mfma_f32_16x16x32_bf16 v[8:11], a[128:131], a[16:19], 0\n"
    "buffer_load_dwordx4 a[208:211], v228, s[80:83], s64 offen offset:0\n"
    "v_mfma_f32_16x16x32_bf16 v[8:11], a[132:135], a[20:23], v[8:11]\n"
    "ds_read_b64_tr_b16 a[148:149], v242 offset:8448\n"
    "buffer_load_dwordx4 a[212:215], v229, s[80:83], s64 offen offset:0\n"
    "v_mfma_f32_16x16x32_bf16 v[12:15], a[128:131], a[24:27], 0\n"
    "buffer_load_dwordx4 a[216:219], v230, s[80:83], s64 offen offset:0\n"
    "v_mfma_f32_16x16x32_bf16 v[12:15], a[132:135], a[28:31], v[12:15]\n"
    "ds_read_b64_tr_b16 a[150:151], v242 offset:9472\n"
    "buffer_load_dwordx4 a[220:223], v231, s[80:83], s64 offen offset:0\n"
    "v_mfma_f32_16x16x32_bf16 v[16:19], a[128:131], a[32:35], 0\n"
    "s_add_u32 m0, s58, 0\n"
    "v_mfma_f32_16x16x32_bf16 v[16:19], a[132:135], a[36:39], v[16:19]\n"
    "buffer_load_dwordx4 v232, s[40:43], s54 offen nt lds\n"
    "v_mfma_f32_16x16x32_bf16 v[20:23], a[128:131], a[40:43], 0\n"
    "v_mfma_f32_16x16x32_bf16 v[20:23], a[132:135], a[44:47], v[20:23]\n"
    "ds_read_b128 a[64:67], v240 offset:32768\n"
    "v_mfma_f32_16x16x32_bf16 v[24:27], a[128:131], a[48:51], 0\n"
    "s_add_u32 m0, s58, 1024\n"
    "v_mfma_f32_16x16x32_bf16 v[24:27], a[132:135], a[52:55], v[24:27]\n"
    "buffer_load_dwordx4 v233, s[40:43], s54 offen nt lds\n"
    "v_mfma_f32_16x16x32_bf16 v[28:31], a[128:131], a[56:59], 0\n"
    "v_mfma_f32_16x16x32_bf16 v[28:31], a[132:135], a[60:63], v[28:31]\n"
    "ds_read_b128 a[68:71], v241 offset:32768\n"
    "s_waitcnt lgkmcnt(6)\n"
    "v_mfma_f32_16x16x32_bf16 v[32:35], a[136:139], a[0:3], 0\n"
    "v_mfma_f32_16x16x32_bf16 v[32:35], a[140:143], a[4:7], v[32:35]\n"
    "ds_read_b64_tr_b16 a[152:153], v243 offset:256\n"
    "v_mfma_f32_16x16x32_bf16 v[36:39], a[136:139], a[8:11], 0\n"
    "v_mfma_f32_16x16x32_bf16 v[36:39], a[140:143], a[12:15], v[36:39]\n"
    "ds_read_b64_tr_b16 a[154:155], v243 offset:1280\n"
    "v_mfma_f32_16x16x32_bf16 v[40:43], a[136:139], a[16:19], 0\n"
    "v_mfma_f32_16x16x32_bf16 v[40:43], a[140:143], a[20:23], v[40:43]\n"
    "ds_read_b64_tr_b16 a[156:157], v243 offset:8448\n"
    "v_mfma_f32_16x16x32_bf16 v[44:47], a[136:139], a[24:27], 0\n"
    "v_mfma_f32_16x16x32_bf16 v[44:47], a[140:143], a[28:31], v[44:47]\n"
    "ds_read_b64_tr_b16 a[158:159], v243 offset:9472\n"
    "v_mfma_f32_16x16x32_bf16 v[48:51], a[136:139], a[32:35], 0\n"
    "s_add_u32 m0, s58, 4096\n"
    "v_mfma_f32_16x16x32_bf16 v[48:51], a[140:143], a[36:39], v[48:51]\n"
    "buffer_load_dwordx4 v234, s[40:43], s54 offen nt lds\n"
    "v_mfma_f32_16x16x32_bf16 v[52:55], a[136:139], a[40:43], 0\n"
    "v_mfma_f32_16x16x32_bf16 v[52:55], a[140:143], a[44:47], v[52:55]\n"
    "ds_read_b128 a[72:75], v240 offset:34816\n"
    "v_mfma_f32_16x16x32_bf16 v[56:59], a[136:139], a[48:51], 0\n"
    "s_add_u32 m0, s58, 5120\n"
    "v_mfma_f32_16x16x32_bf16 v[56:59], a[140:143], a[52:55], v[56:59]\n"
    "buffer_load_dwordx4 v235, s[40:43], s54 offen nt lds\n"
    "v_mfma_f32_16x16x32_bf16 v[60:63], a[136:139], a[56:59], 0\n"
    "v_mfma_f32_16x16x32_bf16 v[60:63], a[140:143], a[60:63], v[60:63]\n"
    "ds_read_b128 a[76:79], v241 offset:34816\n"
    "s_waitcnt lgkmcnt(8)\n"
    "v_mfma_f32_16x16x32_bf16 v[64:67], a[144:147], a[0:3], 0\n"
    "v_mfma_f32_16x16x32_bf16 v[64:67], a[148:151], a[4:7], v[64:67]\n"
    "ds_read_b64_tr_b16 a[128:129], v242 offset:512\n"
    "v_mfma_f32_16x16x32_bf16 v[68:71], a[144:147], a[8:11], 0\n"
    "v_mfma_f32_16x16x32_bf16 v[68:71], a[148:151], a[12:15], v[68:71]\n"
    "ds_read_b64_tr_b16 a[130:131], v242 offset:1536\n"
    "v_mfma_f32_16x16x32_bf16 v[72:75], a[144:147], a[16:19], 0\n"
    "v_mfma_f32_16x16x32_bf16 v[72:75], a[148:151], a[20:23], v[72:75]\n"
    "ds_read_b64_tr_b16 a[132:133], v242 offset:8704\n"
    "v_mfma_f32_16x16x32_bf16 v[76:79], a[144:147], a[24:27], 0\n"
    "v_mfma_f32_16x16x32_bf16 v[76:79], a[148:151], a[28:31], v[76:79]\n"
    "ds_read_b64_tr_b16 a[134:135], v242 offset:9728\n"
    "v_mfma_f32_16x16x32_bf16 v[80:83], a[144:147], a[32:35], 0\n"
    "s_add_u32 m0, s58, 8192\n"
    "v_mfma_f32_16x16x32_bf16 v[80:83], a[148:151], a[36:39], v[80:83]\n"
    "buffer_load_dwordx4 v236, s[40:43], s54 offen nt lds\n"
    "v_mfma_f32_16x16x32_bf16 v[84:87], a[144:147], a[40:43], 0\n"
    "v_mfma_f32_16x16x32_bf16 v[84:87], a[148:151], a[44:47], v[84:87]\n"
    "ds_read_b128 a[80:83], v240 offset:36864\n"
    "v_mfma_f32_16x16x32_bf16 v[88:91], a[144:147], a[48:51], 0\n"
    "s_add_u32 m0, s58, 9216\n"
    "v_mfma_f32_16x16x32_bf16 v[88:91], a[148:151], a[52:55], v[88:91]\n"
    "buffer_load_dwordx4 v237, s[40:43], s54 offen nt lds\n"
    "v_mfma_f32_16x16x32_bf16 v[92:95], a[144:147], a[56:59], 0\n"
    "v_mfma_f32_16x16x32_bf16 v[92:95], a[148:151], a[60:63], v[92:95]\n"
    "ds_read_b128 a[84:87], v241 offset:36864\n"
    "s_waitcnt lgkmcnt(8)\n"
    "v_mfma_f32_16x16x32_bf16 v[96:99], a[152:155], a[0:3], 0\n"
    "v_mfma_f32_16x16x32_bf16 v[96:99], a[156:159], a[4:7], v[96:99]\n"
    "ds_read_b64_tr_b16 a[136:137], v243 offset:512\n"
    "v_mfma_f32_16x16x32_bf16 v[100:103], a[152:155], a[8:11], 0\n"
    "v_mfma_f32_16x16x32_bf16 v[100:103], a[156:159], a[12:15], v[100:103]\n"
    "ds_read_b64_tr_b16 a[138:139], v243 offset:1536\n"
    "s_waitcnt vmcnt(6)\n"
    "v_mfma_f32_16x16x32_bf16 v[104:107], a[152:155], a[16:19], 0\n"
    "v_mfma_f32_16x16x32_bf16 v[104:107], a[156:159], a[20:23], v[104:107]\n"
    "ds_read_b64_tr_b16 a[140:141], v243 offset:8704\n"
    "buffer_store_dwordx4 a[192:195], v224, s[84:87], s65 offen offset:0 sc0 sc1\n"
    "v_mfma_f32_16x16x32_bf16 v[108:111], a[152:155], a[24:27], 0\n"
    "v_mfma_f32_16x16x32_bf16 v[108:111], a[156:159], a[28:31], v[108:111]\n"
    "ds_read_b64_tr_b16 a[142:143], v243 offset:9728\n"
    "v_mfma_f32_16x16x32_bf16 v[112:115], a[152:155], a[32:35], 0\n"
    "s_add_u32 m0, s58, 12288\n"
    "v_mfma_f32_16x16x32_bf16 v[112:115], a[156:159], a[36:39], v[112:115]\n"
    "buffer_load_dwordx4 v238, s[40:43], s54 offen nt lds\n"
    "v_mfma_f32_16x16x32_bf16 v[116:119], a[152:155], a[40:43], 0\n"
    "v_mfma_f32_16x16x32_bf16 v[116:119], a[156:159], a[44:47], v[116:119]\n"
    "ds_read_b128 a[88:91], v240 offset:38912\n"
    "v_mfma_f32_16x16x32_bf16 v[120:123], a[152:155], a[48:51], 0\n"
    "s_add_u32 m0, s58, 13312\n"
    "v_mfma_f32_16x16x32_bf16 v[120:123], a[156:159], a[52:55], v[120:123]\n"
    "buffer_store_dwordx4 a[196:199], v225, s[84:87], s65 offen offset:0 sc0 sc1\n"
    "buffer_load_dwordx4 v239, s[40:43], s54 offen nt lds\n"
    "v_mfma_f32_16x16x32_bf16 v[124:127], a[152:155], a[56:59], 0\n"
    "v_mfma_f32_16x16x32_bf16 v[124:127], a[156:159], a[60:63], v[124:127]\n"
    "ds_read_b128 a[92:95], v241 offset:38912\n"
    "s_waitcnt lgkmcnt(8)\n"
    "v_mfma_f32_16x16x32_bf16 v[128:131], a[128:131], a[0:3], 0\n"
    "v_mfma_f32_16x16x32_bf16 v[128:131], a[132:135], a[4:7], v[128:131]\n"
    "ds_read_b64_tr_b16 a[144:145], v242 offset:768\n"
    "v_mfma_f32_16x16x32_bf16 v[132:135], a[128:131], a[8:11], 0\n"
    "s_add_u32 s58, s58, 32768\n"
    "v_mfma_f32_16x16x32_bf16 v[132:135], a[132:135], a[12:15], v[132:135]\n"
    "ds_read_b64_tr_b16 a[146:147], v242 offset:1792\n"
    "v_mfma_f32_16x16x32_bf16 v[136:139], a[128:131], a[16:19], 0\n"
    "s_sub_u32 s59, s58, 98304\n"
    "v_mfma_f32_16x16x32_bf16 v[136:139], a[132:135], a[20:23], v[136:139]\n"
    "ds_read_b64_tr_b16 a[148:149], v242 offset:8960\n"
    "buffer_store_dwordx4 a[200:203], v226, s[84:87], s65 offen offset:0 sc0 sc1\n"
    "v_mfma_f32_16x16x32_bf16 v[140:143], a[128:131], a[24:27], 0\n"
    "s_cmp_ge_u32 s58, s67\n"
    "v_mfma_f32_16x16x32_bf16 v[140:143], a[132:135], a[28:31], v[140:143]\n"
    "ds_read_b64_tr_b16 a[150:151], v242 offset:9984\n"
    "v_mfma_f32_16x16x32_bf16 v[144:147], a[128:131], a[32:35], 0\n"
    "s_cselect_b32 s58, s59, s58\n"
    "v_mfma_f32_16x16x32_bf16 v[144:147], a[132:135], a[36:39], v[144:147]\n"
    "ds_read_b128 a[96:99], v240 offset:40960\n"
    "v_mfma_f32_16x16x32_bf16 v[148:151], a[128:131], a[40:43], 0\n"
    "v_mfma_f32_16x16x32_bf16 v[148:151], a[132:135], a[44:47], v[148:151]\n"
    "ds_read_b128 a[100:103], v241 offset:40960\n"
    "v_mfma_f32_16x16x32_bf16 v[152:155], a[128:131], a[48:51], 0\n"
    "v_mfma_f32_16x16x32_bf16 v[152:155], a[132:135], a[52:55], v[152:155]\n"
    "ds_read_b128 a[104:107], v240 offset:43008\n"
    "buffer_store_dwordx4 a[204:207], v227, s[84:87], s65 offen offset:0 sc0 sc1\n"
    "v_mfma_f32_16x16x32_bf16 v[156:159], a[128:131], a[56:59], 0\n"
    "v_mfma_f32_16x16x32_bf16 v[156:159], a[132:135], a[60:63], v[156:159]\n"
    "ds_read_b128 a[108:111], v241 offset:43008\n"
    "s_waitcnt lgkmcnt(10)\n"
    "v_mfma_f32_16x16x32_bf16 v[160:163], a[136:139], a[0:3], 0\n"
    "v_mfma_f32_16x16x32_bf16 v[160:163], a[140:143], a[4:7], v[160:163]\n"
    "ds_read_b64_tr_b16 a[152:153], v243 offset:768\n"
    "ds_read_b128 a[112:115], v240 offset:45056\n"
    "v_mfma_f32_16x16x32_bf16 v[164:167], a[136:139], a[8:11], 0\n"
    "s_add_u32 s59, s52, 3\n"
    "v_mfma_f32_16x16x32_bf16 v[164:167], a[140:143], a[12:15], v[164:167]\n"
    "ds_read_b64_tr_b16 a[154:155], v243 offset:1792\n"
    "ds_read_b128 a[116:119], v241 offset:45056\n"
    "v_readlane_b32 s53, v244, s59\n"
    "v_mfma_f32_16x16x32_bf16 v[168:171], a[136:139], a[16:19], 0\n"
    "v_mfma_f32_16x16x32_bf16 v[168:171], a[140:143], a[20:23], v[168:171]\n"
    "ds_read_b64_tr_b16 a[156:157], v243 offset:8960\n"
    "ds_read_b128 a[120:123], v240 offset:47104\n"
    "buffer_store_dwordx4 a[208:211], v228, s[84:87], s65 offen offset:0 sc0 sc1\n"
    "v_mfma_f32_16x16x32_bf16 v[172:175], a[136:139], a[24:27], 0\n"
    "v_mfma_f32_16x16x32_bf16 v[172:175], a[140:143], a[28:31], v[172:175]\n"
    "ds_read_b64_tr_b16 a[158:159], v243 offset:9984\n"
    "ds_read_b128 a[124:127], v241 offset:47104\n"
    "v_mfma_f32_16x16x32_bf16 v[176:179], a[136:139], a[32:35], 0\n"
    "s_add_u32 s57, s57, 1\n"
    "v_mfma_f32_16x16x32_bf16 v[176:179], a[140:143], a[36:39], v[176:179]\n"
    "s_cmp_eq_u32 s57, 3\n"
    "v_mfma_f32_16x16x32_bf16 v[180:183], a[136:139], a[40:43], 0\n"
    "s_cselect_b32 s57, 0, s57\n"
    "s_cselect_b32 s59, 4294901760, s68\n"
    "v_mfma_f32_16x16x32_bf16 v[180:183], a[140:143], a[44:47], v[180:183]\n"
    "v_mfma_f32_16x16x32_bf16 v[184:187], a[136:139], a[48:51], 0\n"
    "v_add_u32 v242, s59, v242\n"
    "v_mfma_f32_16x16x32_bf16 v[184:187], a[140:143], a[52:55], v[184:187]\n"
    "buffer_store_dwordx4 a[212:215], v229, s[84:87], s65 offen offset:0 sc0 sc1\n"
    "v_add_u32 v243, s59, v243\n"
    "v_mfma_f32_16x16x32_bf16 v[188:191], a[136:139], a[56:59], 0\n"
    "v_mfma_f32_16x16x32_bf16 v[188:191], a[140:143], a[60:63], v[188:191]\n"
    "s_waitcnt vmcnt(14) lgkmcnt(0)\n"
    "s_barrier\n"
    "v_mfma_f32_16x16x32_bf16 v[192:195], a[144:147], a[0:3], 0\n"
    "v_mfma_f32_16x16x32_bf16 v[192:195], a[148:151], a[4:7], v[192:195]\n"
    "ds_read_b64_tr_b16 a[128:129], v242 offset:0\n"
    "v_mfma_f32_16x16x32_bf16 v[196:199], a[144:147], a[8:11], 0\n"
    "v_mfma_f32_16x16x32_bf16 v[196:199], a[148:151], a[12:15], v[196:199]\n"
    "ds_read_b64_tr_b16 a[130:131], v242 offset:1024\n"
    "v_mfma_f32_16x16x32_bf16 v[200:203], a[144:147], a[16:19], 0\n"
    "v_mfma_f32_16x16x32_bf16 v[200:203], a[148:151], a[20:23], v[200:203]\n"
    "ds_read_b64_tr_b16 a[132:133], v242 offset:8192\n"
    "v_mfma_f32_16x16x32_bf16 v[204:207], a[144:147], a[24:27], 0\n"
    "v_mfma_f32_16x16x32_bf16 v[204:207], a[148:151], a[28:31], v[204:207]\n"
    "buffer_store_dwordx4 a[216:219], v230, s[84:87], s65 offen offset:0 sc0 sc1\n"
    "ds_read_b64_tr_b16 a[134:135], v242 offset:9216\n"
    "v_mfma_f32_16x16x32_bf16 v[208:211], a[144:147], a[32:35], 0\n"
    "s_add_u32 m0, s55, 32768\n"
    "v_mfma_f32_16x16x32_bf16 v[208:211], a[148:151], a[36:39], v[208:211]\n"
    "buffer_load_dwordx4 v224, s[44:47], s53 offen sc1 lds\n"
    "v_mfma_f32_16x16x32_bf16 v[212:215], a[144:147], a[40:43], 0\n"
    "s_add_u32 m0, s55, 33792\n"
    "v_mfma_f32_16x16x32_bf16 v[212:215], a[148:151], a[44:47], v[212:215]\n"
    "buffer_load_dwordx4 v225, s[44:47], s53 offen sc1 lds\n"
    "v_mfma_f32_16x16x32_bf16 v[216:219], a[144:147], a[48:51], 0\n"
    "s_add_u32 m0, s55, 34816\n"
    "v_mfma_f32_16x16x32_bf16 v[216:219], a[148:151], a[52:55], v[216:219]\n"
    "buffer_load_dwordx4 v226, s[44:47], s53 offen sc1 lds\n"
    "v_mfma_f32_16x16x32_bf16 v[220:223], a[144:147], a[56:59], 0\n"
    "s_add_u32 m0, s55, 35840\n"
    "v_mfma_f32_16x16x32_bf16 v[220:223], a[148:151], a[60:63], v[220:223]\n"
    "buffer_load_dwordx4 v227, s[44:47], s53 offen sc1 lds\n"
    "s_waitcnt lgkmcnt(5)\n"
    "v_mfma_f32_16x16x32_bf16 a[160:163], a[152:155], a[0:3], 0\n"
    "v_mfma_f32_16x16x32_bf16 a[160:163], a[156:159], a[4:7], a[160:163]\n"
    "ds_read_b64_tr_b16 a[136:137], v243 offset:0\n"
    "v_mfma_f32_16x16x32_bf16 a[164:167], a[152:155], a[8:11], 0\n"
    "v_mfma_f32_16x16x32_bf16 a[164:167], a[156:159], a[12:15], a[164:167]\n"
    "ds_read_b64_tr_b16 a[138:139], v243 offset:1024\n"
    "v_mfma_f32_16x16x32_bf16 a[168:171], a[152:155], a[16:19], 0\n"
    "v_mfma_f32_16x16x32_bf16 a[168:171], a[156:159], a[20:23], a[168:171]\n"
    "buffer_store_dwordx4 a[220:223], v231, s[84:87], s65 offen offset:0 sc0 sc1\n"
    "ds_read_b64_tr_b16 a[140:141], v243 offset:8192\n"
    "v_mfma_f32_16x16x32_bf16 a[172:175], a[152:155], a[24:27], 0\n"
    "v_mfma_f32_16x16x32_bf16 a[172:175], a[156:159], a[28:31], a[172:175]\n"
    "ds_read_b64_tr_b16 a[142:143], v243 offset:9216\n"
    "v_mfma_f32_16x16x32_bf16 a[176:179], a[152:155], a[32:35], 0\n"
    "s_add_u32 m0, s55, 36864\n"
    "v_mfma_f32_16x16x32_bf16 a[176:179], a[156:159], a[36:39], a[176:179]\n"
    "buffer_load_dwordx4 v228, s[44:47], s53 offen sc1 lds\n"
    "v_mfma_f32_16x16x32_bf16 a[180:183], a[152:155], a[40:43], 0\n"
    "s_add_u32 m0, s55, 37888\n"
    "v_mfma_f32_16x16x32_bf16 a[180:183], a[156:159], a[44:47], a[180:183]\n"
    "buffer_load_dwordx4 v229, s[44:47], s53 offen sc1 lds\n"
    "v_mfma_f32_16x16x32_bf16 a[184:187], a[152:155], a[48:51], 0\n"
    "s_add_u32 m0, s55, 38912\n"
    "v_mfma_f32_16x16x32_bf16 a[184:187], a[156:159], a[52:55], a[184:187]\n"
    "buffer_load_dwordx4 v230, s[44:47], s53 offen sc1 lds\n"
    "v_mfma_f32_16x16x32_bf16 a[188:191], a[152:155], a[56:59], 0\n"
    "s_add_u32 m0, s55, 39936\n"
    "v_mfma_f32_16x16x32_bf16 a[188:191], a[156:159], a[60:63], a[188:191]\n"
    "buffer_load_dwordx4 v231, s[44:47], s53 offen sc1 lds\n"
    "s_add_u32 s52, s52, 1\n"
    "s_branch .Lg36_%=\n"
    ".Lg37_%=:\n"
    "s_memrealtime s[92:93]\n"
    "s_waitcnt lgkmcnt(0)\n"
    ".Lg39_%=:\n"
    "buffer_load_dword v252, v246, s[88:91], 0 offen sc0 sc1\n"
    "buffer_load_dword v253, off, s[88:91], 0 sc1\n"
    "s_memrealtime s[94:95]\n"
    "s_waitcnt vmcnt(0) lgkmcnt(0)\n"
    "v_cmp_le_u32 vcc, s63, v252\n"
    "s_or_b64 vcc, vcc, s[78:79]\n"
    "s_cmp_eq_u64 vcc, -1\n"
    "s_cbranch_scc1 .Lg38_%=\n"
    "v_readfirstlane_b32 s59, v253\n"
    "s_cmp_eq_u32 s59, s63\n"
    "s_cbranch_scc1 .Lg38_%=\n"
    "s_sleep 4\n"
    "s_sub_u32 s94, s94, s92\n"
    "s_cmp_lt_u32 s94, 50000000\n"
    "s_cbranch_scc1 .Lg39_%=\n"
    "buffer_store_dword v254, v251, s[88:91], 0 offen sc1\n"
    "s_waitcnt vmcnt(0)\n"
    "s_branch .Lg38_%=\n"
    ".Lg41_%=:\n"
    "s_waitcnt lgkmcnt(4)\n"
    "v_mfma_f32_16x16x32_bf16 v[0:3], a[128:131], a[0:3], v[0:3]\n"
    "buffer_load_dwordx4 a[192:195], v224, s[80:83], s64 offen offset:0\n"
    "s_add_u32 s59, s52, 2\n"
    "v_mfma_f32_16x16x32_bf16 v[0:3], a[132:135], a[4:7], v[0:3]\n"
    "ds_read_b64_tr_b16 a[144:145], v242 offset:256\n"
    "buffer_load_dwordx4 a[196:199], v225, s[80:83], s64 offen offset:0\n"
    "v_readlane_b32 s54, v245, s59\n"
    "v_mfma_f32_16x16x32_bf16 v[4:7], a[128:131], a[8:11], v[4:7]\n"
    "buffer_load_dwordx4 a[200:203], v226, s[80:83], s64 offen offset:0\n"
    "v_mfma_f32_16x16x32_bf16 v[4:7], a[132:135], a[12:15], v[4:7]\n"
    "ds_read_b64_tr_b16 a[146:147], v242 offset:1280\n"
    "buffer_load_dwordx4 a[204:207], v227, s[80:83], s64 offen offset:0\n"
    "v_mfma_f32_16x16x32_bf16 v[8:11], a[128:131], a[16:19], v[8:11]\n"
    "buffer_load_dwordx4 a[208:211], v228, s[80:83], s64 offen offset:0\n"
    "v_mfma_f32_16x16x32_bf16 v[8:11], a[132:135], a[20:23], v[8:11]\n"
    "ds_read_b64_tr_b16 a[148:149], v242 offset:8448\n"
    "buffer_load_dwordx4 a[212:215], v229, s[80:83], s64 offen offset:0\n"
    "v_mfma_f32_16x16x32_bf16 v[12:15], a[128:131], a[24:27], v[12:15]\n"
    "buffer_load_dwordx4 a[216:219], v230, s[80:83], s64 offen offset:0\n"
    "v_mfma_f32_16x16x32_bf16 v[12:15], a[132:135], a[28:31], v[12:15]\n"
    "ds_read_b64_tr_b16 a[150:151], v242 offset:9472\n"
    "buffer_load_dwordx4 a[220:223], v231, s[80:83], s64 offen offset:0\n"
    "v_mfma_f32_16x16x32_bf16 v[16:19], a[128:131], a[32:35], v[16:19]\n"
    "s_add_u32 m0, s58, 0\n"
    "v_mfma_f32_16x16x32_bf16 v[16:19], a[132:135], a[36:39], v[16:19]\n"
    "buffer_load_dwordx4 v232, s[40:43], s54 offen lds\n"
    "v_mfma_f32_16x16x32_bf16 v[20:23], a[128:131], a[40:43], v[20:23]\n"
    "v_mfma_f32_16x16x32_bf16 v[20:23], a[132:135], a[44:47], v[20:23]\n"
    "ds_read_b128 a[64:67], v240 offset:32768\n"
    "v_mfma_f32_16x16x32_bf16 v[24:27], a[128:131], a[48:51], v[24:27]\n"
    "s_add_u32 m0, s58, 1024\n"
    "v_mfma_f32_16x16x32_bf16 v[24:27], a[132:135], a[52:55], v[24:27]\n"
    "buffer_load_dwordx4 v233, s[40:43], s54 offen lds\n"
    "v_mfma_f32_16x16x32_bf16 v[28:31], a[128:131], a[56:59], v[28:31]\n"
    "v_mfma_f32_16x16x32_bf16 v[28:31], a[132:135], a[60:63], v[28:31]\n"
    "ds_read_b128 a[68:71], v241 offset:32768\n"
    "s_waitcnt lgkmcnt(6)\n"
    "v_mfma_f32_16x16x32_bf16 v[32:35], a[136:139], a[0:3], v[32:35]\n"
    "v_mfma_f32_16x16x32_bf16 v[32:35], a[140:143], a[4:7], v[32:35]\n"
    "ds_read_b64_tr_b16 a[152:153], v243 offset:256\n"
    "v_mfma_f32_16x16x32_bf16 v[36:39], a[136:139], a[8:11], v[36:39]\n"
    "v_mfma_f32_16x16x32_bf16 v[36:39], a[140:143], a[12:15], v[36:39]\n"
    "ds_read_b64_tr_b16 a[154:155], v243 offset:1280\n"
    "v_mfma_f32_16x16x32_bf16 v[40:43], a[136:139], a[16:19], v[40:43]\n"
    "v_mfma_f32_16x16x32_bf16 v[40:43], a[140:143], a[20:23], v[40:43]\n"
    "ds_read_b64_tr_b16 a[156:157], v243 offset:8448\n"
    "v_mfma_f32_16x16x32_bf16 v[44:47], a[136:139], a[24:27], v[44:47]\n"
    "v_mfma_f32_16x16x32_bf16 v[44:47], a[140:143], a[28:31], v[44:47]\n"
    "ds_read_b64_tr_b16 a[158:159], v243 offset:9472\n"
    "v_mfma_f32_16x16x32_bf16 v[48:51], a[136:139], a[32:35], v[48:51]\n"
    "s_add_u32 m0, s58, 4096\n"
    "v_mfma_f32_16x16x32_bf16 v[48:51], a[140:143], a[36:39], v[48:51]\n"
    "buffer_load_dwordx4 v234, s[40:43], s54 offen lds\n"
    "v_mfma_f32_16x16x32_bf16 v[52:55], a[136:139], a[40:43], v[52:55]\n"
    "v_mfma_f32_16x16x32_bf16 v[52:55], a[140:143], a[44:47], v[52:55]\n"
    "ds_read_b128 a[72:75], v240 offset:34816\n"
    "v_mfma_f32_16x16x32_bf16 v[56:59], a[136:139], a[48:51], v[56:59]\n"
    "s_add_u32 m0, s58, 5120\n"
    "v_mfma_f32_16x16x32_bf16 v[56:59], a[140:143], a[52:55], v[56:59]\n"
    "buffer_load_dwordx4 v235, s[40:43], s54 offen lds\n"
    "v_mfma_f32_16x16x32_bf16 v[60:63], a[136:139], a[56:59], v[60:63]\n"
    "v_mfma_f32_16x16x32_bf16 v[60:63], a[140:143], a[60:63], v[60:63]\n"
    "ds_read_b128 a[76:79], v241 offset:34816\n"
    "s_waitcnt lgkmcnt(8)\n"
    "v_mfma_f32_16x16x32_bf16 v[64:67], a[144:147], a[0:3], v[64:67]\n"
    "v_mfma_f32_16x16x32_bf16 v[64:67], a[148:151], a[4:7], v[64:67]\n"
    "ds_read_b64_tr_b16 a[128:129], v242 offset:512\n"
    "v_mfma_f32_16x16x32_bf16 v[68:71], a[144:147], a[8:11], v[68:71]\n"
    "v_mfma_f32_16x16x32_bf16 v[68:71], a[148:151], a[12:15], v[68:71]\n"
    "ds_read_b64_tr_b16 a[130:131], v242 offset:1536\n"
    "v_mfma_f32_16x16x32_bf16 v[72:75], a[144:147], a[16:19], v[72:75]\n"
    "v_mfma_f32_16x16x32_bf16 v[72:75], a[148:151], a[20:23], v[72:75]\n"
    "ds_read_b64_tr_b16 a[132:133], v242 offset:8704\n"
    "v_mfma_f32_16x16x32_bf16 v[76:79], a[144:147], a[24:27], v[76:79]\n"
    "v_mfma_f32_16x16x32_bf16 v[76:79], a[148:151], a[28:31], v[76:79]\n"
    "ds_read_b64_tr_b16 a[134:135], v242 offset:9728\n"
    "v_mfma_f32_16x16x32_bf16 v[80:83], a[144:147], a[32:35], v[80:83]\n"
    "s_add_u32 m0, s58, 8192\n"
    "v_mfma_f32_16x16x32_bf16 v[80:83], a[148:151], a[36:39], v[80:83]\n"
    "buffer_load_dwordx4 v236, s[40:43], s54 offen lds\n"
    "v_mfma_f32_16x16x32_bf16 v[84:87], a[144:147], a[40:43], v[84:87]\n"
    "v_mfma_f32_16x16x32_bf16 v[84:87], a[148:151], a[44:47], v[84:87]\n"
    "ds_read_b128 a[80:83], v240 offset:36864\n"
    "v_mfma_f32_16x16x32_bf16 v[88:91], a[144:147], a[48:51], v[88:91]\n"
    "s_add_u32 m0, s58, 9216\n"
    "v_mfma_f32_16x16x32_bf16 v[88:91], a[148:151], a[52:55], v[88:91]\n"
    "buffer_load_dwordx4 v237, s[40:43], s54 offen lds\n"
    "v_mfma_f32_16x16x32_bf16 v[92:95], a[144:147], a[56:59], v[92:95]\n"
    "v_mfma_f32_16x16x32_bf16 v[92:95], a[148:151], a[60:63], v[92:95]\n"
    "ds_read_b128 a[84:87], v241 offset:36864\n"
    "s_waitcnt lgkmcnt(8)\n"
    "v_mfma_f32_16x16x32_bf16 v[96:99], a[152:155], a[0:3], v[96:99]\n"
    "v_mfma_f32_16x16x32_bf16 v[96:99], a[156:159], a[4:7], v[96:99]\n"
    "ds_read_b64_tr_b16 a[136:137], v243 offset:512\n"
    "v_mfma_f32_16x16x32_bf16 v[100:103], a[152:155], a[8:11], v[100:103]\n"
    "v_mfma_f32_16x16x32_bf16 v[100:103], a[156:159], a[12:15], v[100:103]\n"
    "ds_read_b64_tr_b16 a[138:139], v243 offset:1536\n"
    "s_waitcnt vmcnt(6)\n"
    "v_mfma_f32_16x16x32_bf16 v[104:107], a[152:155], a[16:19], v[104:107]\n"
    "v_mfma_f32_16x16x32_bf16 v[104:107], a[156:159], a[20:23], v[104:107]\n"
    "ds_read_b64_tr_b16 a[140:141], v243 offset:8704\n"
    "buffer_store_dwordx4 a[192:195], v224, s[84:87], s65 offen offset:0 sc0 sc1\n"
    "v_mfma_f32_16x16x32_bf16 v[108:111], a[152:155], a[24:27], v[108:111]\n"
    "v_mfma_f32_16x16x32_bf16 v[108:111], a[156:159], a[28:31], v[108:111]\n"
    "ds_read_b64_tr_b16 a[142:143], v243 offset:9728\n"
    "v_mfma_f32_16x16x32_bf16 v[112:115], a[152:155], a[32:35], v[112:115]\n"
    "s_add_u32 m0, s58, 12288\n"
    "v_mfma_f32_16x16x32_bf16 v[112:115], a[156:159], a[36:39], v[112:115]\n"
    "buffer_load_dwordx4 v238, s[40:43], s54 offen lds\n"
    "v_mfma_f32_16x16x32_bf16 v[116:119], a[152:155], a[40:43], v[116:119]\n"
    "v_mfma_f32_16x16x32_bf16 v[116:119], a[156:159], a[44:47], v[116:119]\n"
    "ds_read_b128 a[88:91], v240 offset:38912\n"
    "v_mfma_f32_16x16x32_bf16 v[120:123], a[152:155], a[48:51], v[120:123]\n"
    "s_add_u32 m0, s58, 13312\n"
    "v_mfma_f32_16x16x32_bf16 v[120:123], a[156:159], a[52:55], v[120:123]\n"
    "buffer_store_dwordx4 a[196:199], v225, s[84:87], s65 offen offset:0 sc0 sc1\n"
    "buffer_load_dwordx4 v239, s[40:43], s54 offen lds\n"
    "v_mfma_f32_16x16x32_bf16 v[124:127], a[152:155], a[56:59], v[124:127]\n"
    "v_mfma_f32_16x16x32_bf16 v[124:127], a[156:159], a[60:63], v[124:127]\n"
    "ds_read_b128 a[92:95], v241 offset:38912\n"
    "s_waitcnt lgkmcnt(8)\n"
    "v_mfma_f32_16x16x32_bf16 v[128:131], a[128:131], a[0:3], v[128:131]\n"
    "v_mfma_f32_16x16x32_bf16 v[128:131], a[132:135], a[4:7], v[128:131]\n"
    "ds_read_b64_tr_b16 a[144:145], v242 offset:768\n"
    "v_mfma_f32_16x16x32_bf16 v[132:135], a[128:131], a[8:11], v[132:135]\n"
    "s_add_u32 s58, s58, 32768\n"
    "v_mfma_f32_16x16x32_bf16 v[132:135], a[132:135], a[12:15], v[132:135]\n"
    "ds_read_b64_tr_b16 a[146:147], v242 offset:1792\n"
    "v_mfma_f32_16x16x32_bf16 v[136:139], a[128:131], a[16:19], v[136:139]\n"
    "s_sub_u32 s59, s58, 98304\n"
    "v_mfma_f32_16x16x32_bf16 v[136:139], a[132:135], a[20:23], v[136:139]\n"
    "ds_read_b64_tr_b16 a[148:149], v242 offset:8960\n"
    "buffer_store_dwordx4 a[200:203], v226, s[84:87], s65 offen offset:0 sc0 sc1\n"
    "v_mfma_f32_16x16x32_bf16 v[140:143], a[128:131], a[24:27], v[140:143]\n"
    "s_cmp_ge_u32 s58, s67\n"
    "v_mfma_f32_16x16x32_bf16 v[140:143], a[132:135], a[28:31], v[140:143]\n"
    "ds_read_b64_tr_b16 a[150:151], v242 offset:9984\n"
    "v_mfma_f32_16x16x32_bf16 v[144:147], a[128:131], a[32:35], v[144:147]\n"
    "s_cselect_b32 s58, s59, s58\n"
    "v_mfma_f32_16x16x32_bf16 v[144:147], a[132:135], a[36:39], v[144:147]\n"
    "ds_read_b128 a[96:99], v240 offset:40960\n"
    "v_mfma_f32_16x16x32_bf16 v[148:151], a[128:131], a[40:43], v[148:151]\n"
    "v_mfma_f32_16x16x32_bf16 v[148:151], a[132:135], a[44:47], v[148:151]\n"
    "ds_read_b128 a[100:103], v241 offset:40960\n"
    "v_mfma_f32_16x16x32_bf16 v[152:155], a[128:131], a[48:51], v[152:155]\n"
    "v_mfma_f32_16x16x32_bf16 v[152:155], a[132:135], a[52:55], v[152:155]\n"
    "ds_read_b128 a[104:107], v240 offset:43008\n"
    "buffer_store_dwordx4 a[204:207], v227, s[84:87], s65 offen offset:0 sc0 sc1\n"
    "v_mfma_f32_16x16x32_bf16 v[156:159], a[128:131], a[56:59], v[156:159]\n"
    "v_mfma_f32_16x16x32_bf16 v[156:159], a[132:135], a[60:63], v[156:159]\n"
    "ds_read_b128 a[108:111], v241 offset:43008\n"
    "s_waitcnt lgkmcnt(10)\n"
    "v_mfma_f32_16x16x32_bf16 v[160:163], a[136:139], a[0:3], v[160:163]\n"
    "v_mfma_f32_16x16x32_bf16 v[160:163], a[140:143], a[4:7], v[160:163]\n"
    "ds_read_b64_tr_b16 a[152:153], v243 offset:768\n"
    "ds_read_b128 a[112:115], v240 offset:45056\n"
    "v_mfma_f32_16x16x32_bf16 v[164:167], a[136:139], a[8:11], v[164:167]\n"
    "s_add_u32 s59, s52, 3\n"
    "v_mfma_f32_16x16x32_bf16 v[164:167], a[140:143], a[12:15], v[164:167]\n"
    "ds_read_b64_tr_b16 a[154:155], v243 offset:1792\n"
    "ds_read_b128 a[116:119], v241 offset:45056\n"
    "v_readlane_b32 s53, v244, s59\n"
    "v_mfma_f32_16x16x32_bf16 v[168:171], a[136:139], a[16:19], v[168:171]\n"
    "v_mfma_f32_16x16x32_bf16 v[168:171], a[140:143], a[20:23], v[168:171]\n"
    "ds_read_b64_tr_b16 a[156:157], v243 offset:8960\n"
    "ds_read_b128 a[120:123], v240 offset:47104\n"
    "buffer_store_dwordx4 a[208:211], v228, s[84:87], s65 offen offset:0 sc0 sc1\n"
    "v_mfma_f32_16x16x32_bf16 v[172:175], a[136:139], a[24:27], v[172:175]\n"
    "v_mfma_f32_16x16x32_bf16 v[172:175], a[140:143], a[28:31], v[172:175]\n"
    "ds_read_b64_tr_b16 a[158:159], v243 offset:9984\n"
    "ds_read_b128 a[124:127], v241 offset:47104\n"
    "v_mfma_f32_16x16x32_bf16 v[176:179], a[136:139], a[32:35], v[176:179]\n"
    "s_add_u32 s57, s57, 1\n"
    "v_mfma_f32_16x16x32_bf16 v[176:179], a[140:143], a[36:39], v[176:179]\n"
    "s_cmp_eq_u32 s57, 3\n"
    "v_mfma_f32_16x16x32_bf16 v[180:183], a[136:139], a[40:43], v[180:183]\n"
    "s_cselect_b32 s57, 0, s57\n"
    "s_cselect_b32 s59, 4294901760, s68\n"
    "v_mfma_f32_16x16x32_bf16 v[180:183], a[140:143], a[44:47], v[180:183]\n"
    "v_mfma_f32_16x16x32_bf16 v[184:187], a[136:139], a[48:51], v[184:187]\n"
    "v_add_u32 v242, s59, v242\n"
    "v_mfma_f32_16x16x32_bf16 v[184:187], a[140:143], a[52:55], v[184:187]\n"
    "buffer_store_dwordx4 a[212:215], v229, s[84:87], s65 offen offset:0 sc0 sc1\n"
    "v_add_u32 v243, s59, v243\n"
    "v_mfma_f32_16x16x32_bf16 v[188:191], a[136:139], a[56:59], v[188:191]\n"
    "v_mfma_f32_16x16x32_bf16 v[188:191], a[140:143], a[60:63], v[188:191]\n"
    "s_waitcnt vmcnt(14) lgkmcnt(0)\n"
    "s_barrier\n"
    "v_mfma_f32_16x16x32_bf16 v[192:195], a[144:147], a[0:3], v[192:195]\n"
    "v_mfma_f32_16x16x32_bf16 v[192:195], a[148:151], a[4:7], v[192:195]\n"
    "ds_read_b64_tr_b16 a[128:129], v242 offset:0\n"
    "v_mfma_f32_16x16x32_bf16 v[196:199], a[144:147], a[8:11], v[196:199]\n"
    "v_mfma_f32_16x16x32_bf16 v[196:199], a[148:151], a[12:15], v[196:199]\n"
    "ds_read_b64_tr_b16 a[130:131], v242 offset:1024\n"
    "v_mfma_f32_16x16x32_bf16 v[200:203], a[144:147], a[16:19], v[200:203]\n"
    "v_mfma_f32_16x16x32_bf16 v[200:203], a[148:151], a[20:23], v[200:203]\n"
    "ds_read_b64_tr_b16 a[132:133], v242 offset:8192\n"
    "v_mfma_f32_16x16x32_bf16 v[204:207], a[144:147], a[24:27], v[204:207]\n"
    "v_mfma_f32_16x16x32_bf16 v[204:207], a[148:151], a[28:31], v[204:207]\n"
    "buffer_store_dwordx4 a[216:219], v230, s[84:87], s65 offen offset:0 sc0 sc1\n"
    "ds_read_b64_tr_b16 a[134:135], v242 offset:9216\n"
    "v_mfma_f32_16x16x32_bf16 v[208:211], a[144:147], a[32:35], v[208:211]\n"
    "s_add_u32 m0, s55, 32768\n"
    "v_mfma_f32_16x16x32_bf16 v[208:211], a[148:151], a[36:39], v[208:211]\n"
    "buffer_load_dwordx4 v224, s[44:47], s53 offen sc1 lds\n"
    "v_mfma_f32_16x16x32_bf16 v[212:215], a[144:147], a[40:43], v[212:215]\n"
    "s_add_u32 m0, s55, 33792\n"
    "v_mfma_f32_16x16x32_bf16 v[212:215], a[148:151], a[44:47], v[212:215]\n"
    "buffer_load_dwordx4 v225, s[44:47], s53 offen sc1 lds\n"
    "v_mfma_f32_16x16x32_bf16 v[216:219], a[144:147], a[48:51], v[216:219]\n"
    "s_add_u32 m0, s55, 34816\n"
    "v_mfma_f32_16x16x32_bf16 v[216:219], a[148:151], a[52:55], v[216:219]\n"
    "buffer_load_dwordx4 v226, s[44:47], s53 offen sc1 lds\n"
    "v_mfma_f32_16x16x32_bf16 v[220:223], a[144:147], a[56:59], v[220:223]\n"
    "s_add_u32 m0, s55, 35840\n"
    "v_mfma_f32_16x16x32_bf16 v[220:223], a[148:151], a[60:63], v[220:223]\n"
    "buffer_load_dwordx4 v227, s[44:47], s53 offen sc1 lds\n"
    "s_waitcnt lgkmcnt(5)\n"
    "v_mfma_f32_16x16x32_bf16 a[160:163], a[152:155], a[0:3], a[160:163]\n"
    "v_mfma_f32_16x16x32_bf16 a[160:163], a[156:159], a[4:7], a[160:163]\n"
    "ds_read_b64_tr_b16 a[136:137], v243 offset:0\n"
    "v_mfma_f32_16x16x32_bf16 a[164:167], a[152:155], a[8:11], a[164:167]\n"
    "v_mfma_f32_16x16x32_bf16 a[164:167], a[156:159], a[12:15], a[164:167]\n"
    "ds_read_b64_tr_b16 a[138:139], v243 offset:1024\n"
    "v_mfma_f32_16x16x32_bf16 a[168:171], a[152:155], a[16:19], a[168:171]\n"
    "v_mfma_f32_16x16x32_bf16 a[168:171], a[156:159], a[20:23], a[168:171]\n"
    "buffer_store_dwordx4 a[220:223], v231, s[84:87], s65 offen offset:0 sc0 sc1\n"
    "ds_read_b64_tr_b16 a[140:141], v243 offset:8192\n"
    "v_mfma_f32_16x16x32_bf16 a[172:175], a[152:155], a[24:27], a[172:175]\n"
    "v_mfma_f32_16x16x32_bf16 a[172:175], a[156:159], a[28:31], a[172:175]\n"
    "ds_read_b64_tr_b16 a[142:143], v243 offset:9216\n"
    "v_mfma_f32_16x16x32_bf16 a[176:179], a[152:155], a[32:35], a[176:179]\n"
    "s_add_u32 m0, s55, 36864\n"
    "v_mfma_f32_16x16x32_bf16 a[176:179], a[156:159], a[36:39], a[176:179]\n"
    "buffer_load_dwordx4 v228, s[44:47], s53 offen sc1 lds\n"
    "v_mfma_f32_16x16x32_bf16 a[180:183], a[152:155], a[40:43], a[180:183]\n"
    "s_add_u32 m0, s55, 37888\n"
    "v_mfma_f32_16x16x32_bf16 a[180:183], a[156:159], a[44:47], a[180:183]\n"
    "buffer_load_dwordx4 v229, s[44:47], s53 offen sc1 lds\n"
    "v_mfma_f32_16x16x32_bf16 a[184:187], a[152:155], a[48:51], a[184:187]\n"
    "s_add_u32 m0, s55, 38912\n"
    "v_mfma_f32_16x16x32_bf16 a[184:187], a[156:159], a[52:55], a[184:187]\n"
    "buffer_load_dwordx4 v230, s[44:47], s53 offen sc1 lds\n"
    "v_mfma_f32_16x16x32_bf16 a[188:191], a[152:155], a[56:59], a[188:191]\n"
    "s_add_u32 m0, s55, 39936\n"
    "v_mfma_f32_16x16x32_bf16 a[188:191], a[156:159], a[60:63], a[188:191]\n"
    "buffer_load_dwordx4 v231, s[44:47], s53 offen sc1 lds\n"
    "s_add_u32 s52, s52, 1\n"
    "s_branch .Lg42_%=\n"
    ".Lg43_%=:\n"
    "s_memrealtime s[92:93]\n"
    "s_waitcnt lgkmcnt(0)\n"
    ".Lg45_%=:\n"
    "buffer_load_dword v252, v246, s[88:91], 0 offen sc0 sc1\n"
    "buffer_load_dword v253, off, s[88:91], 0 sc1\n"
    "s_memrealtime s[94:95]\n"
    "s_waitcnt vmcnt(0) lgkmcnt(0)\n"
    "v_cmp_le_u32 vcc, s63, v252\n"
    "s_or_b64 vcc, vcc, s[78:79]\n"
    "s_cmp_eq_u64 vcc, -1\n"
    "s_cbranch_scc1 .Lg44_%=\n"
    "v_readfirstlane_b32 s59, v253\n"
    "s_cmp_eq_u32 s59, s63\n"
    "s_cbranch_scc1 .Lg44_%=\n"
    "s_sleep 4\n"
    "s_sub_u32 s94, s94, s92\n"
    "s_cmp_lt_u32 s94, 50000000\n"
    "s_cbranch_scc1 .Lg45_%=\n"
    "buffer_store_dword v254, v251, s[88:91], 0 offen sc1\n"
    "s_waitcnt vmcnt(0)\n"
    "s_branch .Lg44_%=\n"
    ".Lg48_%=:\n"
    "s_memrealtime s[92:93]\n"
    "s_waitcnt lgkmcnt(0)\n"
    ".Lg50_%=:\n"
    "buffer_load_dword v252, v246, s[88:91], 0 offen sc0 sc1\n"
    "buffer_load_dword v253, off, s[88:91], 0 sc1\n"
    "s_memrealtime s[94:95]\n"
    "s_waitcnt vmcnt(0) lgkmcnt(0)\n"
    "v_cmp_le_u32 vcc, s63, v252\n"
    "s_or_b64 vcc, vcc, s[78:79]\n"
    "s_cmp_eq_u64 vcc, -1\n"
    "s_cbranch_scc1 .Lg49_%=\n"
    "v_readfirstlane_b32 s59, v253\n"
    "s_cmp_eq_u32 s59, s63\n"
    "s_cbranch_scc1 .Lg49_%=\n"
    "s_sleep 4\n"
    "s_sub_u32 s94, s94, s92\n"
    "s_cmp_lt_u32 s94, 50000000\n"
    "s_cbranch_scc1 .Lg50_%=\n"
    "buffer_store_dword v254, v251, s[88:91], 0 offen sc1\n"
    "s_waitcnt vmcnt(0)\n"
    "s_branch .Lg49_%=\n"
    ".Lg51_%=:\n"
    "s_nop 7\n"
    "s_nop 7\n"
    "buffer_store_dwordx2 v[254:255], v251, s[88:91], s69 offen sc1\n"
    "v_mov_b32 v0, 0x7fc07fc0\n"
    "v_mov_b32 v1, 0x7fc07fc0\n"
    "v_mov_b32 v2, 0x7fc07fc0\n"
    "v_mov_b32 v3, 0x7fc07fc0\n"
    "buffer_store_dwordx4 v[0:3], v248, s[96:99], 0 offen offset:0\n"
    "buffer_store_dwordx4 v[0:3], v246, s[96:99], 0 offen offset:0\n"
    "buffer_store_dwordx4 v[0:3], v248, s[96:99], 0 offen offset:128\n"
    "buffer_store_dwordx4 v[0:3], v246, s[96:99], 0 offen offset:128\n"
    "buffer_store_dwordx4 v[0:3], v248, s[96:99], s44 offen offset:0\n"
    "buffer_store_dwordx4 v[0:3], v246, s[96:99], s44 offen offset:0\n"
    "buffer_store_dwordx4 v[0:3], v248, s[96:99], s44 offen offset:128\n"
    "buffer_store_dwordx4 v[0:3], v246, s[96:99], s44 offen offset:128\n"
    "buffer_store_dwordx4 v[0:3], v248, s[96:99], s45 offen offset:0\n"
    "buffer_store_dwordx4 v[0:3], v246, s[96:99], s45 offen offset:0\n"
    "buffer_store_dwordx4 v[0:3], v248, s[96:99], s45 offen offset:128\n"
    "buffer_store_dwordx4 v[0:3], v246, s[96:99], s45 offen offset:128\n"
    "buffer_store_dwordx4 v[0:3], v248, s[96:99], s46 offen offset:0\n"
    "buffer_store_dwordx4 v[0:3], v246, s[96:99], s46 offen offset:0\n"
    "buffer_store_dwordx4 v[0:3], v248, s[96:99], s46 offen offset:128\n"
    "buffer_store_dwordx4 v[0:3], v246, s[96:99], s46 offen offset:128\n"
    "buffer_store_dwordx4 v[0:3], v248, s[96:99], s47 offen offset:0\n"
    "buffer_store_dwordx4 v[0:3], v246, s[96:99], s47 offen offset:0\n"
    "buffer_store_dwordx4 v[0:3], v248, s[96:99], s47 offen offset:128\n"
    "buffer_store_dwordx4 v[0:3], v246, s[96:99], s47 offen offset:128\n"
    "buffer_store_dwordx4 v[0:3], v248, s[96:99], s48 offen offset:0\n"
    "buffer_store_dwordx4 v[0:3], v246, s[96:99], s48 offen offset:0\n"
    "buffer_store_dwordx4 v[0:3], v248, s[96:99], s48 offen offset:128\n"
    "buffer_store_dwordx4 v[0:3], v246, s[96:99], s48 offen offset:128\n"
    "buffer_store_dwordx4 v[0:3], v248, s[96:99], s49 offen offset:0\n"
    "buffer_store_dwordx4 v[0:3], v246, s[96:99], s49 offen offset:0\n"
    "buffer_store_dwordx4 v[0:3], v248, s[96:99], s49 offen offset:128\n"
    "buffer_store_dwordx4 v[0:3], v246, s[96:99], s49 offen offset:128\n"
    "buffer_store_dwordx4 v[0:3], v248, s[96:99], s50 offen offset:0\n"
    "buffer_store_dwordx4 v[0:3], v246, s[96:99], s50 offen offset:0\n"
    "buffer_store_dwordx4 v[0:3], v248, s[96:99], s50 offen offset:128\n"
    "buffer_store_dwordx4 v[0:3], v246, s[96:99], s50 offen offset:128\n"
    "s_waitcnt lgkmcnt(0)\n"
    "s_branch .Lg1_%=\n"
    ".Lg1_%=:\n"
    : "+{s[40:43]}"(srdW), "+{s[44:47]}"(srdA), "+{s[48:51]}"(srdR), "+{s58}"(bwr), "+{s[64:66]}"(offs), "+{s[32:33]}"(pace2), "+{s30}"(capt), "+{s[80:83]}"(srdX), "+{s92}"(tent), "+{s29}"(cred),
      "+{v[224:231]}"(voffA), "+{v[232:239]}"(voffB), "+{v[240:247]}"(misc)
    : "{v[248:250]}"(ev), "{s37}"(pace), "{s38}"(pacen), "{s55}"(m0a), "{s56}"(sendt), "{s69}"(ctroff),
      "{s[84:87]}"(srdD), "{s[88:91]}"(srdF), "{s[96:99]}"(srdO)
    : "memory", "scc", "vcc", "v0","v1","v2","v3","v4","v5","v6","v7","v8","v9","v10","v11","v12","v13","v14","v15","v16","v17","v18","v19","v20","v21","v22","v23","v24","v25","v26","v27","v28","v29","v30","v31","v32","v33","v34","v35","v36","v37","v38","v39","v40","v41","v42","v43","v44","v45","v46","v47","v48","v49","v50","v51","v52","v53","v54","v55","v56","v57","v58","v59","v60","v61","v62","v63","v64","v65","v66","v67","v68","v69","v70","v71","v72","v73","v74","v75","v76","v77","v78","v79","v80","v81","v82","v83","v84","v85","v86","v87","v88","v89","v90","v91","v92","v93","v94","v95","v96","v97","v98","v99","v100","v101","v102","v103","v104","v105","v106","v107","v108","v109","v110","v111","v112","v113","v114","v115","v116","v117","v118","v119","v120","v121","v122","v123","v124","v125","v126","v127","v128","v129","v130","v131","v132","v133","v134","v135","v136","v137","v138","v139","v140","v141","v142","v143","v144","v145","v146","v147","v148","v149","v150","v151","v152","v153","v154","v155","v156","v157","v158","v159","v160","v161","v162","v163","v164","v165","v166","v167","v168","v169","v170","v171","v172","v173","v174","v175","v176","v177","v178","v179","v180","v181","v182","v183","v184","v185","v186","v187","v188","v189","v190","v191","v192","v193","v194","v195","v196","v197","v198","v199","v200","v201","v202","v203","v204","v205","v206","v207","v208","v209","v210","v211","v212","v213","v214","v215","v216","v217","v218","v219","v220","v221","v222","v223","v251","v252","v253","v254","v255","a0","a1","a2","a3","a4","a5","a6","a7","a8","a9","a10","a11","a12","a13","a14","a15","a16","a17","a18","a19","a20","a21","a22","a23","a24","a25","a26","a27","a28","a29","a30","a31","a32","a33","a34","a35","a36","a37","a38","a39","a40","a41","a42","a43","a44","a45","a46","a47","a48","a49","a50","a51","a52","a53","a54","a55","a56","a57","a58","a59","a60","a61","a62","a63","a64","a65","a66","a67","a68","a69","a70","a71","a72","a73","a74","a75","a76","a77","a78","a79","a80","a81","a82","a83","a84","a85","a86","a87","a88","a89","a90","a91","a92","a93","a94","a95","a96","a97","a98","a99","a100","a101","a102","a103","a104","a105","a106","a107","a108","a109","a110","a111","a112","a113","a114","a115","a116","a117","a118","a119","a120","a121","a122","a123","a124","a125","a126","a127","a128","a129","a130","a131","a132","a133","a134","a135","a136","a137","a138","a139","a140","a141","a142","a143","a144","a145","a146","a147","a148","a149","a150","a151","a152","a153","a154","a155","a156","a157","a158","a159","a160","a161","a162","a163","a164","a165","a166","a167","a168","a169","a170","a171","a172","a173","a174","a175","a176","a177","a178","a179","a180","a181","a182","a183","a184","a185","a186","a187","a188","a189","a190","a191","a192","a193","a194","a195","a196","a197","a198","a199","a200","a201","a202","a203","a204","a205","a206","a207","a208","a209","a210","a211","a212","a213","a214","a215","a216","a217","a218","a219","a220","a221","a222","a223","a224","a225","a226","a227","a228","a229","a230","a231","a232","a233","a234","a235","a236","a237","a238","a239","a240","a241","a242","a243","a244","a245","a246","a247","a248","a249","a250","a251","a252","a253","a254","a255","s28","s31","s34","s35","s36","s39","s52","s53","s54","s57","s59","s60","s61","s62","s63","s67","s68","s78","s79","s93","s94","s95");

}

extern "C" void kernel_launch(int rank, int nranks, ncclComm_t comm,
                              hipStream_t stream,
                              void* const* d_in, const int* in_sizes, int n_in,
                              void* d_out, int out_size,
                              const int* in_dtypes, int out_dtype,
                              void* const* d_out_all,
                              void* const* d_ws_all, size_t ws_bytes) {
  if (nranks != NR || (unsigned)rank >= (unsigned)NR || ws_bytes < WS_END || n_in < 2 || in_sizes[0] != 4096 * 512 || in_sizes[1] != 4096 * 32768 || out_size != 512 * 32768) {
    fprintf(stderr, "unexpected configuration\n"); return; }
  static bool init = false;
  if (!init) { HIP_CHECK(hipFuncSetAttribute((const void*)fused, hipFuncAttributeMaxDynamicSharedMemorySize, LDS_BYTES)); init = true; }
  Params p = {};
  p.x = (const char*)d_in[0]; p.W = (const char*)d_in[1]; p.out = (char*)d_out; p.rank = rank;
  p.myws = (char*)d_ws_all[rank];
  for (int i = 0; i < NR - 1; ++i) p.peer[i] = (char*)d_ws_all[(rank + 1 + i) % NR];
  fused<<<256, 256, LDS_BYTES, stream>>>(p);
  HIP_CHECK(hipGetLastError());
}
```
